```python
import numpy as np

kernel_name = "gpu_gemm_ar_m4096_k4096_n16384_e4m3e5m2_dqrelu"


ABSMAX_RTOL = 0.05

COMPUTE_DTYPE = "float32"
EPILOGUE = "dequant_relu"
IN_QUANT = None
OUT_QUANT = None
IN_FP8 = "e4m3e5m2"
IN_DTYPES = {"x": "e4m3", "w_mat": "e5m2",
             "scale_x": "float32", "scale_w": "float32"}
WORLD_SIZE = 8

def _snap_fp8(x, mant_bits, max_abs, min_normal):
    a = np.abs(x.astype(np.float64))
    _m, _e = np.frexp(a)
    q = np.where(a >= min_normal, np.ldexp(1.0, _e - 1 - mant_bits),
                 min_normal * 2.0 ** -mant_bits)
    snapped = np.minimum(np.round(a / q) * q, max_abs)
    return (np.sign(x) * snapped).astype(np.float32)

def setup_inputs():

    x = _snap_fp8(np.random.randn(4096, 4096).astype(np.float32), 3, 448.0, 2.0 ** -6)
    w_mat = _snap_fp8(np.random.randn(4096, 16384).astype(np.float32), 2, 57344.0, 2.0 ** -14)
    scale_x = (0.02 * (1.0 + np.random.rand(1))).astype(np.float32)
    scale_w = (0.02 * (1.0 + np.random.rand(1))).astype(np.float32)
    return {"x": x, "w_mat": w_mat, "scale_x": scale_x, "scale_w": scale_w}

def reference(x, w_mat, scale_x, scale_w):
    acc = (x @ w_mat).astype(np.float32)
    y = (acc * (scale_x * scale_w)).astype(np.float32)
    return np.maximum(y, 0.0).astype(np.float32)
```

```cpp
#include <hip/hip_runtime.h>
#include <rccl/rccl.h>
#include <cstdio>

#define HIP_CHECK(expr) do { hipError_t _e = (expr); if (_e != hipSuccess) { \
    fprintf(stderr, "HIP error %s at %s:%d\n", hipGetErrorString(_e), __FILE__, __LINE__); \
    return; } } while (0)

typedef int v8i __attribute__((ext_vector_type(8)));
typedef int v4i __attribute__((ext_vector_type(4)));
typedef unsigned v4u __attribute__((ext_vector_type(4)));
typedef float f32x4 __attribute__((ext_vector_type(4)));
#define LAS __attribute__((address_space(3)))

constexpr int NR = 8;
constexpr int M_ = 4096, N_ = 16384, KL = 512;
constexpr int NMT = 16, NNT = 64, NKS = 32;
constexpr int HTB = 16384;
constexpr int STAGE_BYTES = 8 * HTB;
constexpr int LDS_BYTES = STAGE_BYTES + 64;
constexpr size_t SLAB = 32768;
constexpr size_t XI_BYTES = (size_t)NMT * NKS * SLAB;
constexpr size_t WI_BYTES = (size_t)NNT * NKS * SLAB;
constexpr size_t IMG_BYTES = XI_BYTES + WI_BYTES;
constexpr size_t IMG_OFF = 1u << 20;
constexpr size_t WS_NEED = IMG_OFF + 2 * IMG_BYTES;
constexpr int CTR_OFF = 0, TMO_OFF = 128, FLAG_OFF = 4096;
#define QPUSH 1
constexpr int Q = QPUSH;
constexpr int NPUSH = NR * Q;
constexpr int GRID = 256;
constexpr int NG = GRID - NPUSH;
constexpr int NPIECE = 20;
constexpr unsigned long long DEADLINE_TICKS = 40000000ull;

struct Params {
  const unsigned char* x;
  const unsigned char* w;
  const float* sx; const float* sw;
  float* out;
  unsigned char* ws[NR];
  int rank;
};

struct Unit { int pm, pn; };

__device__ __forceinline__ v8i cat8(v4i a, v4i b) { return __builtin_shufflevector(a, b, 0, 1, 2, 3, 4, 5, 6, 7); }

__device__ __forceinline__ void tile_of(int L, Unit& u) {
  const int xcd = L & 7, s = L >> 3, j = s >> 5, o = s & 31;
  u.pm = (xcd & 3) * 4 + (o & 3); u.pn = j * 16 + (xcd >> 2) * 8 + (o >> 2);
}
__device__ __forceinline__ bool next_unit(int i, int b, Unit& u) {
  if (b < NPUSH) return false;
  const int L = i * NG + (b - NPUSH);
  if (L >= NMT * NNT) return false;
  tile_of(L, u); return true;
}

__device__ __forceinline__ int fresh_lane() { int l; asm volatile("v_mbcnt_lo_u32_b32 %0, -1, 0\n\tv_mbcnt_hi_u32_b32 %0, -1, %0" : "=v"(l)); return l; }
__device__ __forceinline__ unsigned ld_sys(const unsigned* p) { return __hip_atomic_load(p, __ATOMIC_RELAXED, __HIP_MEMORY_SCOPE_SYSTEM); }

__device__ __forceinline__ int poll_ready(unsigned char* myws, int rank, int j, int want, unsigned call, int& dead) {
  if (dead) return NKS;
  const int L = fresh_lane() & 31, src = (rank + L) & 7, ks = L >> 3;
  const unsigned need = (call - 1u) * NPIECE + (unsigned)(j == 0 ? 2 * ks + 2 : 4 * j + 5 + ks);
  const unsigned* fl = (const unsigned*)(myws + FLAG_OFF) + src * Q * 32;
  unsigned* tmo = (unsigned*)(myws + TMO_OFF);
  const unsigned long long t0 = __builtin_amdgcn_s_memrealtime();
  int ready;
  for (;;) {
    bool ok = true;
#pragma unroll
    for (int q = 0; q < Q; ++q) ok = ok && ((int)(ld_sys(fl + q * 32) - need) >= 0);
    const unsigned b = (unsigned)__ballot(ok);
    ready = (b == 0xffffffffu) ? NKS : __builtin_ctz(~b);
    if (ready >= want) break;
    if (__builtin_amdgcn_readfirstlane(__hip_atomic_load(tmo, __ATOMIC_RELAXED, __HIP_MEMORY_SCOPE_AGENT)) == call ||
        __builtin_amdgcn_s_memrealtime() - t0 > DEADLINE_TICKS) {
      __hip_atomic_store(tmo, call, __ATOMIC_RELAXED, __HIP_MEMORY_SCOPE_AGENT);
      dead = 1; ready = NKS; break;
    }
    __builtin_amdgcn_s_sleep(2);
  }
  __builtin_amdgcn_fence(__ATOMIC_ACQUIRE, "");
  return __builtin_amdgcn_readfirstlane(ready);
}

__device__ __forceinline__ void pusher(const Params& p, unsigned call, int b, int wid, int lane) {
  const int dst = (p.rank + 1 + (b & 7)) & 7, q = b >> 3;
  unsigned char* dws = p.ws[dst];
  unsigned char* XI = dws + IMG_OFF + (size_t)(call & 1u) * IMG_BYTES;
  unsigned char* WI = XI + XI_BYTES;
  unsigned* flag = (unsigned*)(dws + FLAG_OFF) + (p.rank * Q + q) * 32;
  const int wv = q * 8 + wid;
  const int i = lane & 15, g = lane >> 4;
  for (int piece = 0; piece < NPIECE; ++piece) {
    int isx, j, ks;
    if (piece < 8) { isx = !(piece & 1); j = 0; ks = piece >> 1; } else { isx = 0; j = (piece - 4) >> 2; ks = piece & 3; }
    const int kidx = p.rank * 4 + ks;
    if (isx) {
      for (int u0 = wv; u0 < 512; u0 += 8 * Q * 8) {
        v4u v[8];
#pragma unroll
        for (int e = 0; e < 8; ++e) {
          const int u = u0 + e * 8 * Q;
          const int row = (u >> 1) * 16 + i;
          v[e] = *(const v4u*)(p.x + (size_t)row * KL + ks * 128 + g * 32 + (u & 1) * 16);
        }
#pragma unroll
        for (int e = 0; e < 8; ++e) {
          const int u = u0 + e * 8 * Q;
          *(v4u*)(XI + ((size_t)((u >> 5) * NKS + kidx)) * SLAB + (u & 31) * 1024 + lane * 16) = v[e];
        }
      }
    } else {
      const int perm = 4 * (i & 3) + (i >> 2);
      for (int t0 = wv; t0 < 128; t0 += 8 * Q * 2) {
        unsigned R[2][16];
#pragma unroll
        for (int e = 0; e < 2; ++e) {
          const int t = t0 + e * 8 * Q;
          const int h = t & 1;
          const unsigned char* src = p.w + (size_t)(ks * 128 + g * 32 + h * 16) * N_ + (j * 16) * 256 + (t >> 1) * 64 + 4 * perm;
#pragma unroll
          for (int r = 0; r < 16; ++r) R[e][r] = *(const unsigned*)(src + (size_t)r * N_);
        }
#pragma unroll
        for (int e = 0; e < 2; ++e) {
          const int t = t0 + e * 8 * Q;
          const int h = t & 1;
          v4u D[4];
#pragma unroll
          for (int d = 0; d < 4; ++d) {
            const unsigned a = R[e][4 * d], bb = R[e][4 * d + 1], c = R[e][4 * d + 2], f = R[e][4 * d + 3];
            const unsigned t0_ = __builtin_amdgcn_perm(bb, a, 0x05010400u), t1_ = __builtin_amdgcn_perm(bb, a, 0x07030602u);
            const unsigned t2_ = __builtin_amdgcn_perm(f, c, 0x05010400u), t3_ = __builtin_amdgcn_perm(f, c, 0x07030602u);
            D[0][d] = __builtin_amdgcn_perm(t2_, t0_, 0x05040100u); D[1][d] = __builtin_amdgcn_perm(t2_, t0_, 0x07060302u);
            D[2][d] = __builtin_amdgcn_perm(t3_, t1_, 0x05040100u); D[3][d] = __builtin_amdgcn_perm(t3_, t1_, 0x07060302u);
          }
          unsigned char* o = WI + ((size_t)((j * 16 + (t >> 3)) * NKS + kidx)) * SLAB + ((t >> 1) & 3) * 8192 + h * 1024 + lane * 16;
#pragma unroll
          for (int c = 0; c < 4; ++c) *(v4u*)(o + c * 2048) = D[c];
        }
      }
    }
    asm volatile("s_waitcnt vmcnt(0)" ::: "memory");
    __syncthreads();
    if (threadIdx.x == 0) {
      __builtin_amdgcn_fence(__ATOMIC_RELEASE, "");
      asm volatile("s_waitcnt vmcnt(0)" ::: "memory");
      __hip_atomic_store(flag, (call - 1u) * NPIECE + (unsigned)piece + 1u, __ATOMIC_RELAXED, __HIP_MEMORY_SCOPE_SYSTEM);
    }
  }
}

__global__ __launch_bounds__(512, 2) void fused_kernel(const Params p) {
  extern __shared__ __attribute__((aligned(16))) unsigned char smem[];
  LAS unsigned char* lds = (LAS unsigned char*)smem;
  const int tid = threadIdx.x, wid = __builtin_amdgcn_readfirstlane(tid >> 6), lane = tid & 63, wr = wid >> 2, wc = wid & 3;
  unsigned char* myws = p.ws[p.rank];
  if (tid == 0) {
    const unsigned t = __hip_atomic_fetch_add((unsigned*)(myws + CTR_OFF), 1u, __ATOMIC_RELAXED, __HIP_MEMORY_SCOPE_AGENT);
    *(LAS unsigned*)(lds + STAGE_BYTES) = t / (unsigned)GRID + 1u;
  }
  __syncthreads();
  const unsigned call = __builtin_amdgcn_readfirstlane(*(LAS unsigned*)(lds + STAGE_BYTES));
  const int b = blockIdx.x;
  if (b < NPUSH) { pusher(p, call, b, wid, lane); return; }

  const unsigned char* XI = myws + IMG_OFF + (size_t)(call & 1u) * IMG_BYTES;
  const unsigned char* WI = XI + XI_BYTES;
  const int rank = p.rank;
  const unsigned voff = (unsigned)tid * 16u;
  const unsigned ldsw = (unsigned)wid * 1024u;
  const int aoff = wr * 8192 + lane * 16, boff = wc * 4096 + lane * 16;
#define SA(b_, h) (((b_) * 2 + (h)) * HTB)
#define SB(b_, h) ((4 + (b_) * 2 + (h)) * HTB)
#define STAGE(bufoff, gbase) do { _Pragma("unroll") for (int _i = 0; _i < 2; ++_i) \
    __builtin_amdgcn_global_load_lds((const unsigned*)((const char*)(gbase) + voff + _i * 8192), (LAS unsigned*)(lds + (bufoff) + ldsw + _i * 8192), 16, 0, 0); } while (0)
#define LDA(dst, b_, h) do { _Pragma("unroll") for (int m = 0; m < 4; ++m) { \
    v4i lo = *(const LAS v4i*)(lds + SA(b_, h) + aoff + m * 2048); v4i hi = *(const LAS v4i*)(lds + SA(b_, h) + aoff + m * 2048 + 1024); dst[m] = cat8(lo, hi); } } while (0)
#define LDB(dst, b_, h) do { _Pragma("unroll") for (int n = 0; n < 2; ++n) { \
    v4i lo = *(const LAS v4i*)(lds + SB(b_, h) + boff + n * 2048); v4i hi = *(const LAS v4i*)(lds + SB(b_, h) + boff + n * 2048 + 1024); dst[n] = cat8(lo, hi); } } while (0)
#define MMA(ai, bj, At, Bt) do { __builtin_amdgcn_s_setprio(1); _Pragma("unroll") for (int m = 0; m < 4; ++m) _Pragma("unroll") for (int n = 0; n < 2; ++n) \
    acc[ai][bj][m][n] = __builtin_amdgcn_mfma_scale_f32_16x16x128_f8f6f4(At[m], Bt[n], acc[ai][bj][m][n], 1, 0, 0, 0, 0, 0); __builtin_amdgcn_s_setprio(0); } while (0)
#define WAIT_V(n) asm volatile("s_waitcnt vmcnt(" #n ")" ::: "memory")
#define WAIT_L(n) asm volatile("s_waitcnt lgkmcnt(" #n ")" ::: "memory")
#define BAR __builtin_amdgcn_s_barrier()
#define SCHED __builtin_amdgcn_sched_barrier(0)
#define KOFF(i) ((size_t)(((((rank + (i)) & 7) << 2) | ((i) >> 3))) * SLAB)
  Unit cur, nxt; int ui = 0;
  if (!next_unit(0, b, cur)) return;
  const float scale = __builtin_bit_cast(float, __builtin_amdgcn_readfirstlane(__builtin_bit_cast(int, p.sx[0] * p.sw[0])));
  int dead = 0;
  f32x4 acc[2][2][4][2];
#pragma unroll
  for (int a = 0; a < 2; ++a)
#pragma unroll
    for (int c = 0; c < 2; ++c)
#pragma unroll
      for (int m = 0; m < 4; ++m)
#pragma unroll
        for (int n = 0; n < 2; ++n) acc[a][c][m][n] = (f32x4){0.f, 0.f, 0.f, 0.f};
  v8i At[4], B0[2], B1[2];
  const size_t hstep = HTB, tstep = SLAB * NKS;
  const int nt = NKS;
  const char* cA = (const char*)WI + (size_t)cur.pn * tstep; const char* cB = (const char*)XI + (size_t)cur.pm * tstep;
  int ready = poll_ready(myws, rank, cur.pn >> 4, 2, call, dead);
  {
    const size_t k0 = KOFF(0), k1 = KOFF(1);
    STAGE(SB(0, 0), cB + k0); STAGE(SA(0, 0), cA + k0); STAGE(SB(0, 1), cB + k0 + hstep); STAGE(SA(0, 1), cA + k0 + hstep);
    if (wr == 1) BAR;
    WAIT_V(4); BAR;
    STAGE(SB(1, 0), cB + k1); STAGE(SA(1, 0), cA + k1); STAGE(SB(1, 1), cB + k1 + hstep);
    WAIT_V(6); BAR;
  }
  for (;;) {
    const bool has_next = next_unit(ui + 1, b, nxt);
    const char* nA = has_next ? (const char*)WI + (size_t)nxt.pn * tstep : cA; const char* nB = has_next ? (const char*)XI + (size_t)nxt.pm * tstep : cB;
    for (int t = 0; t < nt; t += 2) {
      const bool last = (t == nt - 2);
      if (last) { if (has_next) ready = poll_ready(myws, rank, nxt.pn >> 4, 2, call, dead); }
      else if (ready < t + 4) ready = poll_ready(myws, rank, cur.pn >> 4, t + 4, call, dead);
      const int t2 = last ? 0 : t + 2;
      const char* a1 = cA + KOFF(t + 1);
      const char* a2 = (last ? nA : cA) + KOFF(t2); const char* b2 = (last ? nB : cB) + KOFF(t2);
      const char* a3 = (last ? nA : cA) + KOFF(t2 + 1); const char* b3 = (last ? nB : cB) + KOFF(t2 + 1);
      LDB(B0, 0, 0); SCHED; LDA(At, 0, 0); STAGE(SA(1, 1), a1 + hstep);
      WAIT_L(8); BAR; WAIT_L(0); MMA(0, 0, At, B0); BAR; SCHED;
      LDB(B1, 0, 1); STAGE(SB(0, 0), b2);
      BAR; WAIT_L(0); MMA(0, 1, At, B1); BAR;
      LDA(At, 0, 1); STAGE(SA(0, 0), a2);
      BAR; WAIT_L(0); MMA(1, 0, At, B0); BAR; SCHED;
      STAGE(SB(0, 1), b2 + hstep);
      WAIT_V(6); BAR; MMA(1, 1, At, B1); BAR;
      LDB(B0, 1, 0); SCHED; LDA(At, 1, 0); STAGE(SA(0, 1), a2 + hstep);
      WAIT_L(8); BAR; WAIT_L(0); MMA(0, 0, At, B0); BAR; SCHED;
      LDB(B1, 1, 1); STAGE(SB(1, 0), b3);
      BAR; WAIT_L(0); MMA(0, 1, At, B1); BAR;
      LDA(At, 1, 1); STAGE(SA(1, 0), a3);
      BAR; WAIT_L(0); MMA(1, 0, At, B0); BAR; SCHED;
      STAGE(SB(1, 1), b3 + hstep);
      WAIT_V(6); BAR; MMA(1, 1, At, B1); BAR;
    }
    {
      const int ln = fresh_lane(); const int fr = ln & 15, fq = ln >> 4;
      float* base = p.out + (size_t)(cur.pm * 256 + wc * 32 + fr) * N_ + cur.pn * 256 + wr * 64 + 4 * fq;
      const float qn = __builtin_nanf("");
#pragma unroll
      for (int bj = 0; bj < 2; ++bj)
#pragma unroll
        for (int n = 0; n < 2; ++n) {
          float* rowp = base + (size_t)(bj * 128 + n * 16) * N_;
#pragma unroll
          for (int ai = 0; ai < 2; ++ai)
#pragma unroll
            for (int r = 0; r < 4; ++r) {
              f32x4 v;
#pragma unroll
              for (int c = 0; c < 4; ++c) v[c] = dead ? qn : fmaxf(acc[ai][bj][c][n][r] * scale, 0.f);
              *(f32x4*)(rowp + ai * 128 + r * 16) = v;
            }
        }
    }
    if (!has_next) break;
#pragma unroll
    for (int a = 0; a < 2; ++a)
#pragma unroll
      for (int c = 0; c < 2; ++c)
#pragma unroll
        for (int m = 0; m < 4; ++m)
#pragma unroll
          for (int n = 0; n < 2; ++n) acc[a][c][m][n] = (f32x4){0.f, 0.f, 0.f, 0.f};
    cur = nxt; cA = nA; cB = nB; ++ui;
  }
  WAIT_V(0);
  if (wr == 0) BAR;
}

extern "C" void kernel_launch(int rank, int nranks, ncclComm_t comm,
                              hipStream_t stream,
                              void* const* d_in, const int* in_sizes, int n_in,
                              void* d_out, int out_size,
                              const int* in_dtypes, int out_dtype,
                              void* const* d_out_all,
                              void* const* d_ws_all, size_t ws_bytes) {
  if (nranks != NR || (unsigned)rank >= (unsigned)NR || ws_bytes < WS_NEED || n_in < 4) {
    fprintf(stderr, "kernel_launch: unexpected configuration\n"); return;
  }
  static bool attr_done[64] = {};
  int dev = 0; HIP_CHECK(hipGetDevice(&dev));
  if (!attr_done[dev & 63]) {
    HIP_CHECK(hipFuncSetAttribute((const void*)fused_kernel, hipFuncAttributeMaxDynamicSharedMemorySize, LDS_BYTES));
    attr_done[dev & 63] = true;
  }
  Params p;
  p.x = (const unsigned char*)d_in[0]; p.w = (const unsigned char*)d_in[1];
  p.sx = (const float*)d_in[2]; p.sw = (const float*)d_in[3];
  p.out = (float*)d_out;
  for (int i = 0; i < NR; ++i) p.ws[i] = (unsigned char*)d_ws_all[i];
  p.rank = rank;
  fused_kernel<<<GRID, 512, LDS_BYTES, stream>>>(p);
}
```

```cpp
#include <hip/hip_runtime.h>
#include <rccl/rccl.h>
#include <cstdio>

#define HIP_CHECK(expr) do { hipError_t _e = (expr); if (_e != hipSuccess) { \
    fprintf(stderr, "HIP error %s at %s:%d\n", hipGetErrorString(_e), __FILE__, __LINE__); \
    return; } } while (0)

typedef int v8i __attribute__((ext_vector_type(8)));
typedef int v4i __attribute__((ext_vector_type(4)));
typedef unsigned v4u __attribute__((ext_vector_type(4)));
typedef float f32x4 __attribute__((ext_vector_type(4)));
#define LAS __attribute__((address_space(3)))

constexpr int NR = 8;
constexpr int M_ = 4096, N_ = 16384, KL = 512;
constexpr int NMT = 16, NNT = 64, NKS = 32;
constexpr int HTB = 16384;
constexpr int STAGE_BYTES = 8 * HTB;
constexpr int LDS_BYTES = STAGE_BYTES + 256;
constexpr size_t SLAB = 32768;
constexpr size_t XI_BYTES = (size_t)NMT * NKS * SLAB;
constexpr size_t WI_BYTES = (size_t)NNT * NKS * SLAB;
constexpr size_t IMG_BYTES = XI_BYTES + WI_BYTES;
constexpr size_t IMG_OFF = 1u << 20;
constexpr size_t WS_NEED = IMG_OFF + 2 * IMG_BYTES;
constexpr int CTR_OFF = 0, TMO_OFF = 128, TICK_OFF = 256, FLAG_OFF = 4096, DBG_OFF = 65536;
constexpr int NPUSH = 16;
constexpr int GRID = 256;
constexpr int NG = GRID - NPUSH;
constexpr int NSLAB = 320;
constexpr int TICKS_PER_CALL = NSLAB + 2 * NPUSH;
constexpr unsigned long long DEADLINE_TICKS = 40000000ull;

struct Params {
  const unsigned char* x;
  const unsigned char* w;
  const float* sx; const float* sw;
  float* out;
  unsigned char* ws[NR];
  int rank;
};

struct Unit { int pm, pn; };

#define TS(ws_, b_, slot) do {} while (0)
__device__ __forceinline__ v8i cat8(v4i a, v4i b) { return __builtin_shufflevector(a, b, 0, 1, 2, 3, 4, 5, 6, 7); }

__device__ __forceinline__ int slabX(int m, int ks) { return m < 8 ? ks * 16 + m : 64 + ks * 19 + (m - 8); }
__device__ __forceinline__ int slabW(int n, int ks) {
  if (n < 8) return ks * 16 + 8 + n;
  if (n < 19) return 64 + ks * 19 + n;
  const int c = (n - 19) / 15, o = (n - 19) - c * 15;
  return 140 + c * 60 + ks * 15 + o;
}
__device__ __forceinline__ void slab_of(int s, int& isx, int& t, int& ks) {
  if (s < 64) { ks = s >> 4; const int o = s & 15; isx = o < 8; t = o & 7; }
  else if (s < 140) { const int u = s - 64; ks = u / 19; const int o = u - ks * 19; isx = o < 8; t = isx ? 8 + o : o; }
  else { const int u = s - 140, c = u / 60, v = u - c * 60; ks = v / 15; isx = 0; t = 19 + c * 15 + (v - ks * 15); }
}
__device__ __forceinline__ bool next_unit(int i, int g, Unit& u) {
  const int x = g & 7, r = g >> 3;
  const int k = i + (r < 8 ? 0 : 1);
  if (k == 0) { u.pm = (x & 3) * 2 + (r & 1); u.pn = (x >> 2) * 4 + (r >> 1); return true; }
  if (k == 1) {
    int idx = 30 * x + r;
    if (idx < 88) { const int band = idx >= 44, o = idx - band * 44; u.pm = band * 4 + (o & 3); u.pn = 8 + (o >> 2); }
    else { idx -= 88; const int band = idx >= 76, o = idx - band * 76; u.pm = 8 + band * 4 + (o & 3); u.pn = o >> 2; }
    return true;
  }
  if (k > 4) return false;
  const int o = (x >> 2) * 30 + r;
  u.pm = (x & 3) * 4 + (o & 3); u.pn = 19 + (k - 2) * 15 + (o >> 2);
  return true;
}

__device__ __forceinline__ int fresh_lane() { int l; asm volatile("v_mbcnt_lo_u32_b32 %0, -1, 0\n\tv_mbcnt_hi_u32_b32 %0, -1, %0" : "=v"(l)); return l; }
__device__ __forceinline__ unsigned ld_sys(const unsigned* p) { return __hip_atomic_load(p, __ATOMIC_RELAXED, __HIP_MEMORY_SCOPE_SYSTEM); }

__device__ __forceinline__ int wait_ready(unsigned char* myws, int rank, int wid, int ui, const Unit& u, int want, unsigned call,
                                          LAS unsigned* lready, int& dead) {
  if (dead) return NKS;
  const unsigned long long t0 = __builtin_amdgcn_s_memrealtime();
  int ready;
  if (wid == 0) {
    const int ln = fresh_lane(), i = ln & 31, src = (rank + i) & 7, ks = i >> 3;
    const int s = (ln >> 5) ? slabW(u.pn, ks) : slabX(u.pm, ks);
    const unsigned need = call;
    const unsigned* fl = (const unsigned*)(myws + FLAG_OFF) + src * NSLAB + s;
    unsigned* tmo = (unsigned*)(myws + TMO_OFF);
    for (;;) {
      const unsigned long long bm = __ballot((int)(ld_sys(fl) - need) >= 0);
      const unsigned m = (unsigned)bm & (unsigned)(bm >> 32);
      ready = (m == 0xffffffffu) ? NKS : __builtin_ctz(~m);
      if (ready >= want) break;
      if (__builtin_amdgcn_readfirstlane(__hip_atomic_load(tmo, __ATOMIC_RELAXED, __HIP_MEMORY_SCOPE_AGENT)) == call ||
          __builtin_amdgcn_s_memrealtime() - t0 > DEADLINE_TICKS) {
        __hip_atomic_store(tmo, call, __ATOMIC_RELAXED, __HIP_MEMORY_SCOPE_AGENT);
        dead = 1; ready = NKS; break;
      }
      __builtin_amdgcn_s_sleep(4);
    }
    __builtin_amdgcn_fence(__ATOMIC_ACQUIRE, "");
    asm volatile("s_waitcnt vmcnt(0)" ::: "memory");
    __hip_atomic_store(lready, (unsigned)(ui * 64 + ready) | ((unsigned)dead << 31), __ATOMIC_RELAXED, __HIP_MEMORY_SCOPE_WORKGROUP);
  } else {
    for (;;) {
      const unsigned v = __builtin_amdgcn_readfirstlane(__hip_atomic_load(lready, __ATOMIC_RELAXED, __HIP_MEMORY_SCOPE_WORKGROUP));
      if (v >> 31) { dead = 1; ready = NKS; break; }
      ready = (int)v - ui * 64;
      if (ready >= want) break;
      if (__builtin_amdgcn_s_memrealtime() - t0 > 2 * DEADLINE_TICKS) { dead = 1; ready = NKS; break; }
      __builtin_amdgcn_s_sleep(1);
    }
    asm volatile("" ::: "memory");
  }
  return __builtin_amdgcn_readfirstlane(ready);
}

__device__ __forceinline__ void st16_sys(unsigned char* p, v4u v) {
  asm volatile("global_store_dwordx4 %0, %1, off sc0 sc1\n\ts_nop 1" :: "v"(p), "v"(v) : "memory");
}
__device__ __forceinline__ unsigned ld4(const unsigned char* p) { unsigned v; asm volatile("global_load_dword %0, %1, off" : "=v"(v) : "v"(p) : "memory"); return v; }
__device__ __forceinline__ void st4_sys(unsigned* p, unsigned v) { asm volatile("global_store_dword %0, %1, off sc0 sc1" :: "v"(p), "v"(v) : "memory"); }
__device__ __forceinline__ unsigned ticket(unsigned* p) {
  unsigned v = 0;
  if (threadIdx.x == 0) asm volatile("global_atomic_add %0, %1, %2, off sc0" : "=v"(v) : "v"(p), "v"(1u) : "memory");
  return v;
}
#define TIE16(R) "+v"(R[0]), "+v"(R[1]), "+v"(R[2]), "+v"(R[3]), "+v"(R[4]), "+v"(R[5]), "+v"(R[6]), "+v"(R[7]), \
                 "+v"(R[8]), "+v"(R[9]), "+v"(R[10]), "+v"(R[11]), "+v"(R[12]), "+v"(R[13]), "+v"(R[14]), "+v"(R[15])
#define PBAR asm volatile("s_waitcnt lgkmcnt(0)\n\ts_barrier" ::: "memory")
__device__ __forceinline__ void load_job(const Params& p, int s, int wid, int lane, unsigned (&R)[16]) {
  int isx, t, ks; slab_of(s, isx, t, ks);
  const int i = lane & 15, g = lane >> 4, perm = 4 * (i & 3) + (i >> 2);
  const unsigned char* src = isx ? p.x + (size_t)(t * 256 + wid * 32 + i) * KL + ks * 128 + g * 32
                                 : p.w + (size_t)(ks * 128 + g * 32 + (wid & 1) * 16) * N_ + t * 256 + (wid >> 1) * 64 + 4 * perm;
  const unsigned s1 = isx ? 4 : N_, s2 = isx ? 16 : 4 * N_, s3 = isx ? 16 * KL : 8 * N_;
#pragma unroll
  for (int r = 0; r < 16; ++r) R[r] = ld4(src + (r & 3) * s1 + ((r >> 2) & 1) * s2 + (r >> 3) * s3);
}
__device__ __forceinline__ void publish(const Params& p, unsigned* fbase, unsigned call, int s) {
  if (threadIdx.x < NR) st4_sys(fbase + s, call);
}
__device__ __forceinline__ void pusher(const Params& p, unsigned call, int q, int wid, int lane, volatile LAS unsigned* lslot) {
  const size_t img = IMG_OFF + (size_t)(call & 1u) * IMG_BYTES;
  unsigned* tick = (unsigned*)(p.ws[p.rank] + TICK_OFF);
  const unsigned tbase = (call - 1u) * TICKS_PER_CALL;
  unsigned* fbase = (unsigned*)(p.ws[(p.rank + 1 + lane) & 7] + FLAG_OFF) + p.rank * NSLAB;
  asm volatile("" : "+v"(fbase));
  unsigned ta = ticket(tick), tb = ticket(tick);
  asm volatile("s_waitcnt vmcnt(0)" : "+v"(ta), "+v"(tb) :: "memory");
  if (threadIdx.x == 0) { lslot[2] = ta - tbase; lslot[3] = tb - tbase; }
  PBAR;
  int s = (int)__builtin_amdgcn_readfirstlane(lslot[2]);
  int sn = (int)__builtin_amdgcn_readfirstlane(lslot[3]);
  if (s >= NSLAB) return;
  unsigned R[16];
  load_job(p, s, wid, lane, R);
  asm volatile("s_waitcnt vmcnt(0)" : TIE16(R) :: "memory");
  for (int it = 0; s < NSLAB; ++it) {
    int isx, t, ks; slab_of(s, isx, t, ks);
    const int kidx = p.rank * 4 + ks;
    v4u D[4]; size_t off; unsigned cstep;
    if (isx) {
#pragma unroll
      for (int e = 0; e < 4; ++e) D[e] = (v4u){R[4 * e], R[4 * e + 1], R[4 * e + 2], R[4 * e + 3]};
      off = img + ((size_t)(t * NKS + kidx)) * SLAB + wid * 4096 + lane * 16; cstep = 1024;
    } else {
#pragma unroll
      for (int d = 0; d < 4; ++d) {
        const unsigned a = R[4 * d], bb = R[4 * d + 1], c = R[4 * d + 2], f = R[4 * d + 3];
        const unsigned t0_ = __builtin_amdgcn_perm(bb, a, 0x05010400u), t1_ = __builtin_amdgcn_perm(bb, a, 0x07030602u);
        const unsigned t2_ = __builtin_amdgcn_perm(f, c, 0x05010400u), t3_ = __builtin_amdgcn_perm(f, c, 0x07030602u);
        D[0][d] = __builtin_amdgcn_perm(t2_, t0_, 0x05040100u); D[1][d] = __builtin_amdgcn_perm(t2_, t0_, 0x07060302u);
        D[2][d] = __builtin_amdgcn_perm(t3_, t1_, 0x05040100u); D[3][d] = __builtin_amdgcn_perm(t3_, t1_, 0x07060302u);
      }
      off = img + XI_BYTES + ((size_t)(t * NKS + kidx)) * SLAB + (wid >> 1) * 8192 + (wid & 1) * 1024 + lane * 16; cstep = 2048;
    }
    asm volatile("" : "+v"(D[0]), "+v"(D[1]), "+v"(D[2]), "+v"(D[3]));
    load_job(p, sn < NSLAB ? sn : s, wid, lane, R);
    unsigned tn = ticket(tick);
#pragma unroll
    for (int d = 0; d < NR; ++d) {
      unsigned char* o = p.ws[(p.rank + 1 + d) & 7] + off;
#pragma unroll
      for (int c = 0; c < 4; ++c) st16_sys(o + c * cstep, D[c]);
    }
    asm volatile("s_waitcnt vmcnt(0)" : TIE16(R), "+v"(tn) :: "memory");
    if (threadIdx.x == 0) lslot[it & 1] = tn - tbase;
    PBAR;
    publish(p, fbase, call, s);
    s = sn; sn = (int)__builtin_amdgcn_readfirstlane(lslot[it & 1]);
  }
}

__global__ __launch_bounds__(512, 2) void fused_kernel(const Params p) {
  extern __shared__ __attribute__((aligned(16))) unsigned char smem[];
  LAS unsigned char* lds = (LAS unsigned char*)smem;
  const int tid = threadIdx.x, wid = __builtin_amdgcn_readfirstlane(tid >> 6), lane = tid & 63, wr = wid >> 2, wc = wid & 3;
  unsigned char* myws = p.ws[p.rank];
  if (tid == 0) {
    const unsigned t = __hip_atomic_fetch_add((unsigned*)(myws + CTR_OFF), 1u, __ATOMIC_RELAXED, __HIP_MEMORY_SCOPE_AGENT);
    *(LAS unsigned*)(lds + STAGE_BYTES) = t / (unsigned)GRID + 1u;
  }
  if (tid >= 64 && tid < 64 + 32) *(LAS unsigned*)(lds + STAGE_BYTES + 64 + (tid - 64) * 4) = 0u;
  __syncthreads();
  const unsigned call = __builtin_amdgcn_readfirstlane(*(LAS unsigned*)(lds + STAGE_BYTES));
  const int b = blockIdx.x;
  LAS unsigned* lctl = (LAS unsigned*)(lds + STAGE_BYTES + 64);
  TS(myws, b, 0);
  if (b < NPUSH) { pusher(p, call, b, wid, lane, lctl); return; }
  const int g = b - NPUSH;

  const unsigned char* XI = myws + IMG_OFF + (size_t)(call & 1u) * IMG_BYTES;
  const unsigned char* WI = XI + XI_BYTES;
  const int rank = p.rank;
  const unsigned voff = (unsigned)tid * 16u;
  const unsigned ldsw = (unsigned)wid * 1024u;
  const int aoff = wr * 8192 + lane * 16, boff = wc * 4096 + lane * 16;
#define SA(b_, h) (((b_) * 2 + (h)) * HTB)
#define SB(b_, h) ((4 + (b_) * 2 + (h)) * HTB)
#define STAGE(bufoff, gbase) do { _Pragma("unroll") for (int _i = 0; _i < 2; ++_i) \
    __builtin_amdgcn_global_load_lds((const unsigned*)((const char*)(gbase) + voff + _i * 8192), (LAS unsigned*)(lds + (bufoff) + ldsw + _i * 8192), 16, 0, 0); } while (0)
#define LDA(dst, b_, h) do { _Pragma("unroll") for (int m = 0; m < 4; ++m) { \
    v4i lo = *(const LAS v4i*)(lds + SA(b_, h) + aoff + m * 2048); v4i hi = *(const LAS v4i*)(lds + SA(b_, h) + aoff + m * 2048 + 1024); dst[m] = cat8(lo, hi); } } while (0)
#define LDB(dst, b_, h) do { _Pragma("unroll") for (int n = 0; n < 2; ++n) { \
    v4i lo = *(const LAS v4i*)(lds + SB(b_, h) + boff + n * 2048); v4i hi = *(const LAS v4i*)(lds + SB(b_, h) + boff + n * 2048 + 1024); dst[n] = cat8(lo, hi); } } while (0)
#define MMA(ai, bj, At, Bt) do { __builtin_amdgcn_s_setprio(1); _Pragma("unroll") for (int m = 0; m < 4; ++m) _Pragma("unroll") for (int n = 0; n < 2; ++n) \
    acc[ai][bj][m][n] = __builtin_amdgcn_mfma_scale_f32_16x16x128_f8f6f4(At[m], Bt[n], acc[ai][bj][m][n], 1, 0, 0, 0, 0, 0); __builtin_amdgcn_s_setprio(0); } while (0)
#define WAIT_V(n) asm volatile("s_waitcnt vmcnt(" #n ")" ::: "memory")
#define WAIT_L(n) asm volatile("s_waitcnt lgkmcnt(" #n ")" ::: "memory")
#define BAR __builtin_amdgcn_s_barrier()
#define SCHED __builtin_amdgcn_sched_barrier(0)
#define KOFF(i) ((size_t)(((((rank + (i)) & 7) << 2) | ((i) >> 3))) * SLAB)
  Unit cur, nxt; int ui = 0;
  if (!next_unit(0, g, cur)) return;
  const float scale = __builtin_bit_cast(float, __builtin_amdgcn_readfirstlane(__builtin_bit_cast(int, p.sx[0] * p.sw[0])));
  int dead = 0;
  f32x4 acc[2][2][4][2];
#pragma unroll
  for (int a = 0; a < 2; ++a)
#pragma unroll
    for (int c = 0; c < 2; ++c)
#pragma unroll
      for (int m = 0; m < 4; ++m)
#pragma unroll
        for (int n = 0; n < 2; ++n) acc[a][c][m][n] = (f32x4){0.f, 0.f, 0.f, 0.f};
  v8i At[4], B0[2], B1[2];
  const size_t hstep = HTB, tstep = SLAB * NKS;
  const int nt = NKS;
  const char* cA = (const char*)WI + (size_t)cur.pn * tstep; const char* cB = (const char*)XI + (size_t)cur.pm * tstep;
  int ready = wait_ready(myws, rank, wid, 0, cur, 2, call, lctl, dead);
  {
    const size_t k0 = KOFF(0), k1 = KOFF(1);
    STAGE(SB(0, 0), cB + k0); STAGE(SA(0, 0), cA + k0); STAGE(SB(0, 1), cB + k0 + hstep); STAGE(SA(0, 1), cA + k0 + hstep);
    if (wr == 1) BAR;
    WAIT_V(4); BAR;
    STAGE(SB(1, 0), cB + k1); STAGE(SA(1, 0), cA + k1); STAGE(SB(1, 1), cB + k1 + hstep);
    WAIT_V(6); BAR;
  }
  for (;;) {
    const bool has_next = next_unit(ui + 1, g, nxt);
    const char* nA = has_next ? (const char*)WI + (size_t)nxt.pn * tstep : cA; const char* nB = has_next ? (const char*)XI + (size_t)nxt.pm * tstep : cB;
    for (int t = 0; t < nt; t += 2) {
      const bool last = (t == nt - 2);
      if (last) { if (has_next) ready = wait_ready(myws, rank, wid, ui + 1, nxt, 2, call, lctl, dead); }
      else if (ready < t + 4) ready = wait_ready(myws, rank, wid, ui, cur, t + 4, call, lctl, dead);
      const int t2 = last ? 0 : t + 2;
      const char* a1 = cA + KOFF(t + 1);
      const char* a2 = (last ? nA : cA) + KOFF(t2); const char* b2 = (last ? nB : cB) + KOFF(t2);
      const char* a3 = (last ? nA : cA) + KOFF(t2 + 1); const char* b3 = (last ? nB : cB) + KOFF(t2 + 1);
      LDB(B0, 0, 0); SCHED; LDA(At, 0, 0); STAGE(SA(1, 1), a1 + hstep);
      WAIT_L(8); BAR; WAIT_L(0); MMA(0, 0, At, B0); BAR; SCHED;
      LDB(B1, 0, 1); STAGE(SB(0, 0), b2);
      BAR; WAIT_L(0); MMA(0, 1, At, B1); BAR;
      LDA(At, 0, 1); STAGE(SA(0, 0), a2);
      BAR; WAIT_L(0); MMA(1, 0, At, B0); BAR; SCHED;
      STAGE(SB(0, 1), b2 + hstep);
      WAIT_V(6); BAR; MMA(1, 1, At, B1); BAR;
      LDB(B0, 1, 0); SCHED; LDA(At, 1, 0); STAGE(SA(0, 1), a2 + hstep);
      WAIT_L(8); BAR; WAIT_L(0); MMA(0, 0, At, B0); BAR; SCHED;
      LDB(B1, 1, 1); STAGE(SB(1, 0), b3);
      BAR; WAIT_L(0); MMA(0, 1, At, B1); BAR;
      LDA(At, 1, 1); STAGE(SA(1, 0), a3);
      BAR; WAIT_L(0); MMA(1, 0, At, B0); BAR; SCHED;
      STAGE(SB(1, 1), b3 + hstep);
      WAIT_V(6); BAR; MMA(1, 1, At, B1); BAR;
    }
    TS(myws, b, 1 + 2 * ui);
    {
      const int ln = fresh_lane(); const int fr = ln & 15, fq = ln >> 4;
      float* base = p.out + (size_t)(cur.pm * 256 + wc * 32 + fr) * N_ + cur.pn * 256 + wr * 64 + 4 * fq;
      const float qn = __builtin_nanf("");
#pragma unroll
      for (int bj = 0; bj < 2; ++bj)
#pragma unroll
        for (int n = 0; n < 2; ++n) {
          float* rowp = base + (size_t)(bj * 128 + n * 16) * N_;
#pragma unroll
          for (int ai = 0; ai < 2; ++ai)
#pragma unroll
            for (int r = 0; r < 4; ++r) {
              f32x4 v;
#pragma unroll
              for (int c = 0; c < 4; ++c) v[c] = dead ? qn : fmaxf(acc[ai][bj][c][n][r] * scale, 0.f);
              *(f32x4*)(rowp + ai * 128 + r * 16) = v;
            }
        }
    }
    TS(myws, b, 2 + 2 * ui);
    if (!has_next) break;
#pragma unroll
    for (int a = 0; a < 2; ++a)
#pragma unroll
      for (int c = 0; c < 2; ++c)
#pragma unroll
        for (int m = 0; m < 4; ++m)
#pragma unroll
          for (int n = 0; n < 2; ++n) acc[a][c][m][n] = (f32x4){0.f, 0.f, 0.f, 0.f};
    cur = nxt; cA = nA; cB = nB; ++ui;
  }
  WAIT_V(0);
  if (wr == 0) BAR;
}

extern "C" void kernel_launch(int rank, int nranks, ncclComm_t comm,
                              hipStream_t stream,
                              void* const* d_in, const int* in_sizes, int n_in,
                              void* d_out, int out_size,
                              const int* in_dtypes, int out_dtype,
                              void* const* d_out_all,
                              void* const* d_ws_all, size_t ws_bytes) {
  if (nranks != NR || (unsigned)rank >= (unsigned)NR || ws_bytes < WS_NEED || n_in < 4) {
    fprintf(stderr, "kernel_launch: unexpected configuration\n"); return;
  }
  static bool attr_done[64] = {};
  int dev = 0; HIP_CHECK(hipGetDevice(&dev));
  if (!attr_done[dev & 63]) {
    HIP_CHECK(hipFuncSetAttribute((const void*)fused_kernel, hipFuncAttributeMaxDynamicSharedMemorySize, LDS_BYTES));
    attr_done[dev & 63] = true;
  }
  Params p;
  p.x = (const unsigned char*)d_in[0]; p.w = (const unsigned char*)d_in[1];
  p.sx = (const float*)d_in[2]; p.sw = (const float*)d_in[3];
  p.out = (float*)d_out;
  for (int i = 0; i < NR; ++i) p.ws[i] = (unsigned char*)d_ws_all[i];
  p.rank = rank;
  fused_kernel<<<GRID, 512, LDS_BYTES, stream>>>(p);
}
```

```cpp
#include <hip/hip_runtime.h>
#include <rccl/rccl.h>
#include <cstdio>

#define HIP_CHECK(expr) do { hipError_t _e = (expr); if (_e != hipSuccess) { \
    fprintf(stderr, "HIP error %s at %s:%d\n", hipGetErrorString(_e), __FILE__, __LINE__); \
    return; } } while (0)

typedef int v8i __attribute__((ext_vector_type(8)));
typedef int v4i __attribute__((ext_vector_type(4)));
typedef unsigned v4u __attribute__((ext_vector_type(4)));
typedef float f32x4 __attribute__((ext_vector_type(4)));
#define LAS __attribute__((address_space(3)))

constexpr int NR = 8;
constexpr int M_ = 4096, N_ = 16384, KL = 512;
constexpr int NMT = 16, NNT = 64, NKS = 32;
constexpr int HTB = 16384;
constexpr int STAGE_BYTES = 8 * HTB;
constexpr int LDS_BYTES = STAGE_BYTES + 256;
constexpr size_t SLAB = 32768;
constexpr size_t XI_BYTES = (size_t)NMT * NKS * SLAB;
constexpr size_t WI_BYTES = (size_t)NNT * NKS * SLAB;
constexpr size_t IMG_BYTES = XI_BYTES + WI_BYTES;
constexpr size_t IMG_OFF = 1u << 20;
constexpr size_t WS_NEED = IMG_OFF + 2 * IMG_BYTES;
constexpr int CTR_OFF = 0, TMO_OFF = 128, FLAG_OFF = 4096, SFLAG_OFF = 20480, DBG_OFF = 65536;
constexpr int NCOMM = 32;
#define PPP 4
constexpr int NPULL = 7 * PPP;
constexpr int GRID = 256;
constexpr int NG = GRID - NCOMM;
constexpr int NSLAB = 320;
constexpr unsigned long long DEADLINE_TICKS = 40000000ull;

struct Params {
  const unsigned char* x;
  const unsigned char* w;
  const float* sx; const float* sw;
  float* out;
  unsigned char* ws[NR];
  int rank;
};

struct Unit { int pm, pn; };

#define TS(ws_, b_, slot) do {} while (0)
__device__ __forceinline__ v8i cat8(v4i a, v4i b) { return __builtin_shufflevector(a, b, 0, 1, 2, 3, 4, 5, 6, 7); }

__device__ __forceinline__ int slabX(int m, int ks) { return m < 8 ? ks * 24 + m : 96 + ks * 14 + (m - 8); }
__device__ __forceinline__ int slabW(int n, int ks) {
  if (n < 16) return ks * 24 + 8 + n;
  if (n < 22) return 96 + ks * 14 + (n - 8);
  const int c = (n - 22) / 14, o = (n - 22) - c * 14;
  return 152 + c * 56 + ks * 14 + o;
}
__device__ __forceinline__ void slab_of(int s, int& isx, int& t, int& ks) {
  if (s < 96) { ks = s / 24; const int o = s - ks * 24; isx = o < 8; t = isx ? o : o - 8; }
  else if (s < 152) { const int u = s - 96; ks = u / 14; const int o = u - ks * 14; isx = o < 8; t = o + 8; }
  else { const int u = s - 152, c = u / 56, v = u - c * 56; ks = v / 14; isx = 0; t = 22 + c * 14 + (v - ks * 14); }
}
__device__ __forceinline__ size_t slab_off(int s, int src) {
  int isx, t, ks; slab_of(s, isx, t, ks);
  return (isx ? (size_t)0 : XI_BYTES) + (size_t)(t * NKS + src * 4 + ks) * SLAB;
}
__device__ __forceinline__ bool next_unit(int i, int g, Unit& u) {
  const int x = g & 7, r = g >> 3;
  const int k = i + (r < 16 ? 0 : 1);
  if (k == 0) { u.pm = (x & 1) * 4 + (r & 3); u.pn = (x >> 1) * 4 + (r >> 2); return true; }
  if (k == 1) {
    int idx = 28 * x + r;
    if (idx < 128) { const int band = idx >> 6, o = idx & 63; u.pm = 8 + band * 4 + (o & 3); u.pn = o >> 2; }
    else { idx -= 128; const int band = idx / 24, o = idx - band * 24; u.pm = band * 4 + (o & 3); u.pn = 16 + (o >> 2); }
    return true;
  }
  if (k > 4) return false;
  u.pm = (x & 3) * 4 + (r & 3); u.pn = 22 + (k - 2) * 14 + (x >> 2) * 7 + (r >> 2);
  return true;
}

__device__ __forceinline__ int fresh_lane() { int l; asm volatile("v_mbcnt_lo_u32_b32 %0, -1, 0\n\tv_mbcnt_hi_u32_b32 %0, -1, %0" : "=v"(l)); return l; }
__device__ __forceinline__ unsigned ld_sys(const unsigned* p) { return __hip_atomic_load(p, __ATOMIC_RELAXED, __HIP_MEMORY_SCOPE_SYSTEM); }

__device__ __forceinline__ int wait_ready(unsigned char* myws, int rank, int wid, int ui, const Unit& u, int want, unsigned call,
                                          LAS unsigned* lready, int& dead) {
  if (dead) return NKS;
  const unsigned long long t0 = __builtin_amdgcn_s_memrealtime();
  int ready;
  if (wid == 0) {
    const int ln = fresh_lane(), i = ln & 31, src = (rank + i) & 7, ks = i >> 3;
    const int s = (ln >> 5) ? slabW(u.pn, ks) : slabX(u.pm, ks);
    const unsigned need = call;
    const unsigned* fl = (const unsigned*)(myws + FLAG_OFF) + src * NSLAB + s;
    unsigned* tmo = (unsigned*)(myws + TMO_OFF);
    for (;;) {
      const unsigned long long bm = __ballot((int)(ld_sys(fl) - need) >= 0);
      const unsigned m = (unsigned)bm & (unsigned)(bm >> 32);
      ready = (m == 0xffffffffu) ? NKS : __builtin_ctz(~m);
      if (ready >= want) break;
      if (__builtin_amdgcn_readfirstlane(__hip_atomic_load(tmo, __ATOMIC_RELAXED, __HIP_MEMORY_SCOPE_AGENT)) == call ||
          __builtin_amdgcn_s_memrealtime() - t0 > DEADLINE_TICKS) {
        __hip_atomic_store(tmo, call, __ATOMIC_RELAXED, __HIP_MEMORY_SCOPE_AGENT);
        dead = 1; ready = NKS; break;
      }
      __builtin_amdgcn_s_sleep(4);
    }
    __builtin_amdgcn_fence(__ATOMIC_ACQUIRE, "");
    asm volatile("s_waitcnt vmcnt(0)" ::: "memory");
    __hip_atomic_store(lready, (unsigned)(ui * 64 + ready) | ((unsigned)dead << 31), __ATOMIC_RELAXED, __HIP_MEMORY_SCOPE_WORKGROUP);
  } else {
    for (;;) {
      const unsigned v = __builtin_amdgcn_readfirstlane(__hip_atomic_load(lready, __ATOMIC_RELAXED, __HIP_MEMORY_SCOPE_WORKGROUP));
      if (v >> 31) { dead = 1; ready = NKS; break; }
      ready = (int)v - ui * 64;
      if (ready >= want) break;
      if (__builtin_amdgcn_s_memrealtime() - t0 > 2 * DEADLINE_TICKS) { dead = 1; ready = NKS; break; }
      __builtin_amdgcn_s_sleep(1);
    }
    asm volatile("" ::: "memory");
  }
  return __builtin_amdgcn_readfirstlane(ready);
}

__device__ __forceinline__ void st16_sys(unsigned char* p, v4u v) {
  asm volatile("global_store_dwordx4 %0, %1, off sc0 sc1\n\ts_nop 1" :: "v"(p), "v"(v) : "memory");
}
__device__ __forceinline__ unsigned ld4(const unsigned char* p) { unsigned v; asm volatile("global_load_dword %0, %1, off" : "=v"(v) : "v"(p) : "memory"); return v; }
__device__ __forceinline__ void st4_sys(unsigned* p, unsigned v) { asm volatile("global_store_dword %0, %1, off sc0 sc1" :: "v"(p), "v"(v) : "memory"); }
#define TIE16(R) "+v"(R[0]), "+v"(R[1]), "+v"(R[2]), "+v"(R[3]), "+v"(R[4]), "+v"(R[5]), "+v"(R[6]), "+v"(R[7]), \
                 "+v"(R[8]), "+v"(R[9]), "+v"(R[10]), "+v"(R[11]), "+v"(R[12]), "+v"(R[13]), "+v"(R[14]), "+v"(R[15])
#define PBAR asm volatile("s_waitcnt lgkmcnt(0)\n\ts_barrier" ::: "memory")
__device__ __forceinline__ void load_job(const Params& p, int s, int wid, int lane, unsigned (&R)[16]) {
  int isx, t, ks; slab_of(s, isx, t, ks);
  const int i = lane & 15, g = lane >> 4, perm = 4 * (i & 3) + (i >> 2);
  const unsigned char* src = isx ? p.x + (size_t)(t * 256 + wid * 32 + i) * KL + ks * 128 + g * 32
                                 : p.w + (size_t)(ks * 128 + g * 32 + (wid & 1) * 16) * N_ + t * 256 + (wid >> 1) * 64 + 4 * perm;
  const unsigned s1 = isx ? 4 : N_, s2 = isx ? 16 : 4 * N_, s3 = isx ? 16 * KL : 8 * N_;
#pragma unroll
  for (int r = 0; r < 16; ++r) R[r] = ld4(src + (r & 3) * s1 + ((r >> 2) & 1) * s2 + (r >> 3) * s3);
}
__device__ __forceinline__ void stage_slab(const Params& p, unsigned call, int s, int wid, int lane) {
  const size_t img = IMG_OFF + (size_t)(call & 1u) * IMG_BYTES;
  unsigned R[16];
  load_job(p, s, wid, lane, R);
  asm volatile("s_waitcnt vmcnt(0)" : TIE16(R) :: "memory");
  int isx, t, ks; slab_of(s, isx, t, ks);
  v4u D[4]; unsigned off, cstep;
  if (isx) {
#pragma unroll
    for (int e = 0; e < 4; ++e) D[e] = (v4u){R[4 * e], R[4 * e + 1], R[4 * e + 2], R[4 * e + 3]};
    off = wid * 4096 + lane * 16; cstep = 1024;
  } else {
#pragma unroll
    for (int d = 0; d < 4; ++d) {
      const unsigned a = R[4 * d], bb = R[4 * d + 1], c = R[4 * d + 2], f = R[4 * d + 3];
      const unsigned t0_ = __builtin_amdgcn_perm(bb, a, 0x05010400u), t1_ = __builtin_amdgcn_perm(bb, a, 0x07030602u);
      const unsigned t2_ = __builtin_amdgcn_perm(f, c, 0x05010400u), t3_ = __builtin_amdgcn_perm(f, c, 0x07030602u);
      D[0][d] = __builtin_amdgcn_perm(t2_, t0_, 0x05040100u); D[1][d] = __builtin_amdgcn_perm(t2_, t0_, 0x07060302u);
      D[2][d] = __builtin_amdgcn_perm(t3_, t1_, 0x05040100u); D[3][d] = __builtin_amdgcn_perm(t3_, t1_, 0x07060302u);
    }
    off = (wid >> 1) * 8192 + (wid & 1) * 1024 + lane * 16; cstep = 2048;
  }
  unsigned char* o = p.ws[p.rank] + img + slab_off(s, p.rank) + off;
#pragma unroll
  for (int c = 0; c < 4; ++c) st16_sys(o + c * cstep, D[c]);
  asm volatile("s_waitcnt vmcnt(0)" ::: "memory");
  PBAR;
  if (threadIdx.x < NR) {
    const int d = (p.rank + threadIdx.x) & 7;
    st4_sys((unsigned*)(p.ws[d] + (threadIdx.x ? SFLAG_OFF : FLAG_OFF)) + p.rank * NSLAB + s, call);
  }
}
__device__ __forceinline__ bool wait_staged(unsigned char* myws, int peer, int s0, int n, unsigned call, int lane) {
  const unsigned* fl = (const unsigned*)(myws + SFLAG_OFF) + peer * NSLAB + s0 + lane;
  unsigned* tmo = (unsigned*)(myws + TMO_OFF);
  const unsigned long long t0 = __builtin_amdgcn_s_memrealtime();
  for (;;) {
    int ok = 1;
    for (int i = 0; i < n; ++i) ok &= (int)(ld_sys(fl + i * 64) - call) >= 0;
    if (__ballot(ok) == ~0ull) break;
    if (__builtin_amdgcn_readfirstlane(__hip_atomic_load(tmo, __ATOMIC_RELAXED, __HIP_MEMORY_SCOPE_AGENT)) == call ||
        __builtin_amdgcn_s_memrealtime() - t0 > DEADLINE_TICKS) {
      __hip_atomic_store(tmo, call, __ATOMIC_RELAXED, __HIP_MEMORY_SCOPE_AGENT);
      return false;
    }
    __builtin_amdgcn_s_sleep(2);
  }
  asm volatile("s_waitcnt vmcnt(0)" ::: "memory");
  return true;
}
#define PSTEP(r, D_, so, S_, lo) asm volatile("s_waitcnt vmcnt(2)\n\tglobal_store_dwordx4 %1, %0, %2 offset:" #so " sc0 sc1\n\t" \
    "global_load_dwordx4 %0, %1, %3 offset:" #lo " sc0 sc1" : "+v"(r) : "v"(vo), "s"(D_), "s"(S_) : "memory")
__device__ __forceinline__ void slab_done(unsigned char* myws, LAS unsigned* cnt, unsigned* flag, unsigned call, int lane, int q, int s) {
  unsigned old = 0;
  if (lane == 0) old = __hip_atomic_fetch_add(cnt, 1u, __ATOMIC_RELAXED, __HIP_MEMORY_SCOPE_WORKGROUP);
  if (__builtin_amdgcn_readfirstlane(old) == 7u && lane == 0) {
    st4_sys(flag + s, call);
  }
}
__device__ __forceinline__ void puller(const Params& p, unsigned call, int q, int wid, int lane, LAS unsigned* lcnt) {
  const int peer = (p.rank + 1 + q % 7) & 7, j = q / 7;
  unsigned char* myws = p.ws[p.rank];
  const size_t img = IMG_OFF + (size_t)(call & 1u) * IMG_BYTES;
  const unsigned vo = wid * 4096 + lane * 16;
  const unsigned char* src = p.ws[peer] + img;
  unsigned char* dst = myws + img;
  unsigned* flag = (unsigned*)(myws + FLAG_OFF) + peer * NSLAB;
  if (!wait_staged(myws, peer, 0, 1, call, lane)) return;
  size_t off = slab_off(j, peer);
  v4u ra, rb;
  asm volatile("global_load_dwordx4 %0, %2, %3 sc0 sc1\n\tglobal_load_dwordx4 %1, %2, %3 offset:1024 sc0 sc1\n\ts_waitcnt vmcnt(1)"
               : "=&v"(ra), "=&v"(rb) : "v"(vo), "s"(src + off) : "memory");
  int it = 0;
  for (int s = j; s < NSLAB; s += PPP, ++it) {
    const int sn = s + PPP < NSLAB ? s + PPP : s;
    if (s < 64 && sn >= 64) { if (!wait_staged(myws, peer, 64, 4, call, lane)) return; }
    const size_t offn = slab_off(sn, peer);
    unsigned char* d = dst + off; const unsigned char* sc = src + off; const unsigned char* sx = src + offn;
    PSTEP(ra, d, 0, sc, 2048);
    PSTEP(rb, d, 1024, sc, 3072);
    if (it) slab_done(myws, lcnt + (it - 1), flag, call, lane, q, s - PPP);
    PSTEP(ra, d, 2048, sx, 0);
    PSTEP(rb, d, 3072, sx, 1024);
    off = offn;
  }
  asm volatile("s_waitcnt vmcnt(0)" : "+v"(ra), "+v"(rb) :: "memory");
  slab_done(myws, lcnt + (it - 1), flag, call, lane, q, j + (it - 1) * PPP);
}

__global__ __launch_bounds__(512, 2) void fused_kernel(const Params p) {
  extern __shared__ __attribute__((aligned(16))) unsigned char smem[];
  LAS unsigned char* lds = (LAS unsigned char*)smem;
  const int tid = threadIdx.x, wid = __builtin_amdgcn_readfirstlane(tid >> 6), lane = tid & 63, wr = wid >> 2, wc = wid & 3;
  unsigned char* myws = p.ws[p.rank];
  if (tid == 0) {
    const unsigned t = __hip_atomic_fetch_add((unsigned*)(myws + CTR_OFF), 1u, __ATOMIC_RELAXED, __HIP_MEMORY_SCOPE_AGENT);
    *(LAS unsigned*)(lds + STAGE_BYTES) = t / (unsigned)GRID + 1u;
  }
  if (tid >= 64 && tid < 64 + 32) *(LAS unsigned*)(lds + STAGE_BYTES + 64 + (tid - 64) * 4) = 0u;
  if (tid >= 128) *(LAS unsigned*)(lds + (tid - 128) * 4) = 0u;
  __syncthreads();
  const unsigned call = __builtin_amdgcn_readfirstlane(*(LAS unsigned*)(lds + STAGE_BYTES));
  const int b = blockIdx.x;
  LAS unsigned* lctl = (LAS unsigned*)(lds + STAGE_BYTES + 64);
  TS(myws, b, 0);
  stage_slab(p, call, b, wid, lane);
  if (b < NCOMM) { if (b < NPULL) puller(p, call, b, wid, lane, (LAS unsigned*)lds); return; }
  const int g = b - NCOMM;
  if (g >= 128 && g < 192) stage_slab(p, call, 128 + g, wid, lane);

  const unsigned char* XI = myws + IMG_OFF + (size_t)(call & 1u) * IMG_BYTES;
  const unsigned char* WI = XI + XI_BYTES;
  const int rank = p.rank;
  const unsigned voff = (unsigned)tid * 16u;
  const unsigned ldsw = (unsigned)wid * 1024u;
  const int aoff = wr * 8192 + lane * 16, boff = wc * 4096 + lane * 16;
#define SA(b_, h) (((b_) * 2 + (h)) * HTB)
#define SB(b_, h) ((4 + (b_) * 2 + (h)) * HTB)
#define STAGE(bufoff, gbase) do { _Pragma("unroll") for (int _i = 0; _i < 2; ++_i) \
    __builtin_amdgcn_global_load_lds((const unsigned*)((const char*)(gbase) + voff + _i * 8192), (LAS unsigned*)(lds + (bufoff) + ldsw + _i * 8192), 16, 0, 0); } while (0)
#define LDA(dst, b_, h) do { _Pragma("unroll") for (int m = 0; m < 4; ++m) { \
    v4i lo = *(const LAS v4i*)(lds + SA(b_, h) + aoff + m * 2048); v4i hi = *(const LAS v4i*)(lds + SA(b_, h) + aoff + m * 2048 + 1024); dst[m] = cat8(lo, hi); } } while (0)
#define LDB(dst, b_, h) do { _Pragma("unroll") for (int n = 0; n < 2; ++n) { \
    v4i lo = *(const LAS v4i*)(lds + SB(b_, h) + boff + n * 2048); v4i hi = *(const LAS v4i*)(lds + SB(b_, h) + boff + n * 2048 + 1024); dst[n] = cat8(lo, hi); } } while (0)
#define MMA(ai, bj, At, Bt) do { __builtin_amdgcn_s_setprio(1); _Pragma("unroll") for (int m = 0; m < 4; ++m) _Pragma("unroll") for (int n = 0; n < 2; ++n) \
    acc[ai][bj][m][n] = __builtin_amdgcn_mfma_scale_f32_16x16x128_f8f6f4(At[m], Bt[n], acc[ai][bj][m][n], 1, 0, 0, 0, 0, 0); __builtin_amdgcn_s_setprio(0); } while (0)
#define WAIT_V(n) asm volatile("s_waitcnt vmcnt(" #n ")" ::: "memory")
#define WAIT_L(n) asm volatile("s_waitcnt lgkmcnt(" #n ")" ::: "memory")
#define BAR __builtin_amdgcn_s_barrier()
#define SCHED __builtin_amdgcn_sched_barrier(0)
#define KOFF(i) ((size_t)(((((rank + (i)) & 7) << 2) | ((i) >> 3))) * SLAB)
  Unit cur, nxt; int ui = 0;
  if (!next_unit(0, g, cur)) return;
  const float scale = __builtin_bit_cast(float, __builtin_amdgcn_readfirstlane(__builtin_bit_cast(int, p.sx[0] * p.sw[0])));
  int dead = 0;
  f32x4 acc[2][2][4][2];
#pragma unroll
  for (int a = 0; a < 2; ++a)
#pragma unroll
    for (int c = 0; c < 2; ++c)
#pragma unroll
      for (int m = 0; m < 4; ++m)
#pragma unroll
        for (int n = 0; n < 2; ++n) acc[a][c][m][n] = (f32x4){0.f, 0.f, 0.f, 0.f};
  v8i At[4], B0[2], B1[2];
  const size_t hstep = HTB, tstep = SLAB * NKS;
  const int nt = NKS;
  const char* cA = (const char*)WI + (size_t)cur.pn * tstep; const char* cB = (const char*)XI + (size_t)cur.pm * tstep;
  int ready = wait_ready(myws, rank, wid, 0, cur, 2, call, lctl, dead);
  {
    const size_t k0 = KOFF(0), k1 = KOFF(1);
    STAGE(SB(0, 0), cB + k0); STAGE(SA(0, 0), cA + k0); STAGE(SB(0, 1), cB + k0 + hstep); STAGE(SA(0, 1), cA + k0 + hstep);
    if (wr == 1) BAR;
    WAIT_V(4); BAR;
    STAGE(SB(1, 0), cB + k1); STAGE(SA(1, 0), cA + k1); STAGE(SB(1, 1), cB + k1 + hstep);
    WAIT_V(6); BAR;
  }
  for (;;) {
    const bool has_next = next_unit(ui + 1, g, nxt);
    const char* nA = has_next ? (const char*)WI + (size_t)nxt.pn * tstep : cA; const char* nB = has_next ? (const char*)XI + (size_t)nxt.pm * tstep : cB;
    for (int t = 0; t < nt; t += 2) {
      const bool last = (t == nt - 2);
      if (last) { if (has_next) ready = wait_ready(myws, rank, wid, ui + 1, nxt, 2, call, lctl, dead); }
      else if (ready < t + 4) ready = wait_ready(myws, rank, wid, ui, cur, t + 4, call, lctl, dead);
      const int t2 = last ? 0 : t + 2;
      const char* a1 = cA + KOFF(t + 1);
      const char* a2 = (last ? nA : cA) + KOFF(t2); const char* b2 = (last ? nB : cB) + KOFF(t2);
      const char* a3 = (last ? nA : cA) + KOFF(t2 + 1); const char* b3 = (last ? nB : cB) + KOFF(t2 + 1);
      LDB(B0, 0, 0); SCHED; LDA(At, 0, 0); STAGE(SA(1, 1), a1 + hstep);
      WAIT_L(8); BAR; WAIT_L(0); MMA(0, 0, At, B0); BAR; SCHED;
      LDB(B1, 0, 1); STAGE(SB(0, 0), b2);
      BAR; WAIT_L(0); MMA(0, 1, At, B1); BAR;
      LDA(At, 0, 1); STAGE(SA(0, 0), a2);
      BAR; WAIT_L(0); MMA(1, 0, At, B0); BAR; SCHED;
      STAGE(SB(0, 1), b2 + hstep);
      WAIT_V(6); BAR; MMA(1, 1, At, B1); BAR;
      LDB(B0, 1, 0); SCHED; LDA(At, 1, 0); STAGE(SA(0, 1), a2 + hstep);
      WAIT_L(8); BAR; WAIT_L(0); MMA(0, 0, At, B0); BAR; SCHED;
      LDB(B1, 1, 1); STAGE(SB(1, 0), b3);
      BAR; WAIT_L(0); MMA(0, 1, At, B1); BAR;
      LDA(At, 1, 1); STAGE(SA(1, 0), a3);
      BAR; WAIT_L(0); MMA(1, 0, At, B0); BAR; SCHED;
      STAGE(SB(1, 1), b3 + hstep);
      WAIT_V(6); BAR; MMA(1, 1, At, B1); BAR;
    }
    TS(myws, b, 1 + 2 * ui);
    {
      const int ln = fresh_lane(); const int fr = ln & 15, fq = ln >> 4;
      float* base = p.out + (size_t)(cur.pm * 256 + wc * 32 + fr) * N_ + cur.pn * 256 + wr * 64 + 4 * fq;
      const float qn = __builtin_nanf("");
#pragma unroll
      for (int bj = 0; bj < 2; ++bj)
#pragma unroll
        for (int n = 0; n < 2; ++n) {
          float* rowp = base + (size_t)(bj * 128 + n * 16) * N_;
#pragma unroll
          for (int ai = 0; ai < 2; ++ai)
#pragma unroll
            for (int r = 0; r < 4; ++r) {
              f32x4 v;
#pragma unroll
              for (int c = 0; c < 4; ++c) v[c] = dead ? qn : fmaxf(acc[ai][bj][c][n][r] * scale, 0.f);
              *(f32x4*)(rowp + ai * 128 + r * 16) = v;
            }
        }
    }
    TS(myws, b, 2 + 2 * ui);
    if (!has_next) break;
#pragma unroll
    for (int a = 0; a < 2; ++a)
#pragma unroll
      for (int c = 0; c < 2; ++c)
#pragma unroll
        for (int m = 0; m < 4; ++m)
#pragma unroll
          for (int n = 0; n < 2; ++n) acc[a][c][m][n] = (f32x4){0.f, 0.f, 0.f, 0.f};
    cur = nxt; cA = nA; cB = nB; ++ui;
  }
  WAIT_V(0);
  if (wr == 0) BAR;
}

extern "C" void kernel_launch(int rank, int nranks, ncclComm_t comm,
                              hipStream_t stream,
                              void* const* d_in, const int* in_sizes, int n_in,
                              void* d_out, int out_size,
                              const int* in_dtypes, int out_dtype,
                              void* const* d_out_all,
                              void* const* d_ws_all, size_t ws_bytes) {
  if (nranks != NR || (unsigned)rank >= (unsigned)NR || ws_bytes < WS_NEED || n_in < 4) {
    fprintf(stderr, "kernel_launch: unexpected configuration\n"); return;
  }
  static bool attr_done[64] = {};
  int dev = 0; HIP_CHECK(hipGetDevice(&dev));
  if (!attr_done[dev & 63]) {
    HIP_CHECK(hipFuncSetAttribute((const void*)fused_kernel, hipFuncAttributeMaxDynamicSharedMemorySize, LDS_BYTES));
    attr_done[dev & 63] = true;
  }
  Params p;
  p.x = (const unsigned char*)d_in[0]; p.w = (const unsigned char*)d_in[1];
  p.sx = (const float*)d_in[2]; p.sw = (const float*)d_in[3];
  p.out = (float*)d_out;
  for (int i = 0; i < NR; ++i) p.ws[i] = (unsigned char*)d_ws_all[i];
  p.rank = rank;
  fused_kernel<<<GRID, 512, LDS_BYTES, stream>>>(p);
}
```

```cpp
#include <hip/hip_runtime.h>
#include <rccl/rccl.h>
#include <cstdio>
#include <type_traits>

#define HIP_CHECK(expr) do { hipError_t _e = (expr); if (_e != hipSuccess) { \
    fprintf(stderr, "HIP error %s at %s:%d\n", hipGetErrorString(_e), __FILE__, __LINE__); \
    return; } } while (0)

typedef int v8i __attribute__((ext_vector_type(8)));
typedef int v4i __attribute__((ext_vector_type(4)));
typedef unsigned v4u __attribute__((ext_vector_type(4)));
typedef float f32x4 __attribute__((ext_vector_type(4)));
#define LAS __attribute__((address_space(3)))

constexpr int NR = 8;
constexpr int M_ = 4096, N_ = 16384, KL = 512;
constexpr int NKS = 32;
constexpr int HTB = 16384;
constexpr int STAGE_BYTES = 8 * HTB;
constexpr unsigned LCTL = STAGE_BYTES, LTAB = STAGE_BYTES + 256, LCP = STAGE_BYTES + 1024;
constexpr int LDS_BYTES = STAGE_BYTES + 2048;
constexpr unsigned SLAB = 32768;
constexpr unsigned IMG_BYTES = 2560u * SLAB;
constexpr unsigned IMG_OFF = 1u << 20;
constexpr size_t WS_NEED = (size_t)IMG_OFF + 2 * (size_t)IMG_BYTES;
constexpr unsigned CTR_OFF = 0, TMO_OFF = 128, DONE_OFF = 4096, OFLAG_OFF = 8192, FLAG_OFF = 65536, FLAG_BYTES = 2560u * 64u, DBG_OFF = 524288;
constexpr int GRID = 256;
constexpr int NPUSH = 16;
constexpr int NSH = 10, SHC = 59;
constexpr unsigned DEADV = 0x7fffffffu;
constexpr unsigned long long DEADLINE_TICKS = 90000000ull;
#define WPT 2

struct Params {
  const unsigned char* x;
  const unsigned char* w;
  const float* sx; const float* sw;
  unsigned char* ws[NR];
  unsigned char* out[NR];
  int rank;
};

#define TS(slot) do {} while (0)
#define TV(slot, v) do {} while (0)
__device__ __forceinline__ v8i cat8(v4i a, v4i b) { return __builtin_shufflevector(a, b, 0, 1, 2, 3, 4, 5, 6, 7); }
#define WAIT_V(n) asm volatile("s_waitcnt vmcnt(" #n ")" ::: "memory")
#define WAIT_L(n) asm volatile("s_waitcnt lgkmcnt(" #n ")" ::: "memory")
#define BAR __builtin_amdgcn_s_barrier()
#define SCHED __builtin_amdgcn_sched_barrier(0)
#define TIE16(R) "+v"(R[0]), "+v"(R[1]), "+v"(R[2]), "+v"(R[3]), "+v"(R[4]), "+v"(R[5]), "+v"(R[6]), "+v"(R[7]), \
                 "+v"(R[8]), "+v"(R[9]), "+v"(R[10]), "+v"(R[11]), "+v"(R[12]), "+v"(R[13]), "+v"(R[14]), "+v"(R[15])

__device__ __forceinline__ int fresh_lane() { int l; asm volatile("v_mbcnt_lo_u32_b32 %0, -1, 0\n\tv_mbcnt_hi_u32_b32 %0, -1, %0" : "=v"(l)); return l; }

__device__ __forceinline__ void load_job(const Params& p, int tile, int ks, int jw, int lane, unsigned (&R)[16]) {
  const int i = lane & 15, g = lane >> 4, perm = 4 * (i & 3) + (i >> 2);
  const bool isx = tile < 16, iso = tile >= 80;
  const unsigned vo = isx ? i * KL + g * 32 : iso ? lane * 16 : g * 32 * N_ + 4 * perm;
  const unsigned char* src = isx ? p.x + (size_t)(tile * 256 + (jw >> 1) * 64) * KL + ks * 128 + (jw & 1) * 16
                           : iso ? p.out[p.rank] + (unsigned)ks
                                 : p.w + (size_t)(ks * 128 + (jw & 1) * 16) * N_ + (tile - 16) * 256 + (jw >> 1) * 64;
  const unsigned s1 = isx || iso ? 4 : N_, s2 = isx ? 16 * KL : 4 * N_;
#pragma unroll
  for (int r = 0; r < 16; ++r) asm volatile("global_load_dword %0, %1, %2" : "=v"(R[r]) : "v"(vo), "s"(src + (r & 3) * s1 + (r >> 2) * s2) : "memory");
}
__device__ __forceinline__ void retile(bool plain, const unsigned (&R)[16], v4u (&D)[4]) {
  if (plain) {
#pragma unroll
    for (int e = 0; e < 4; ++e) D[e] = (v4u){R[4 * e], R[4 * e + 1], R[4 * e + 2], R[4 * e + 3]};
  } else {
#pragma unroll
    for (int d = 0; d < 4; ++d) {
      const unsigned a = R[4 * d], bb = R[4 * d + 1], c = R[4 * d + 2], f = R[4 * d + 3];
      const unsigned t0_ = __builtin_amdgcn_perm(bb, a, 0x05010400u), t1_ = __builtin_amdgcn_perm(bb, a, 0x07030602u);
      const unsigned t2_ = __builtin_amdgcn_perm(f, c, 0x05010400u), t3_ = __builtin_amdgcn_perm(f, c, 0x07030602u);
      D[0][d] = __builtin_amdgcn_perm(t2_, t0_, 0x05040100u); D[1][d] = __builtin_amdgcn_perm(t2_, t0_, 0x07060302u);
      D[2][d] = __builtin_amdgcn_perm(t3_, t1_, 0x05040100u); D[3][d] = __builtin_amdgcn_perm(t3_, t1_, 0x07060302u);
    }
  }
}
__device__ __forceinline__ void store_job(unsigned char* ws, unsigned vo, const v4u (&D)[4]) {
  asm volatile("global_store_dwordx4 %0, %1, %5 offset:-4096 sc0 sc1\n\tglobal_store_dwordx4 %0, %2, %5 offset:-2048 sc0 sc1\n\t"
               "global_store_dwordx4 %0, %3, %5 sc0 sc1\n\tglobal_store_dwordx4 %0, %4, %5 offset:2048 sc0 sc1\n\ts_nop 1"
               :: "v"(vo), "v"(D[0]), "v"(D[1]), "v"(D[2]), "v"(D[3]), "s"(ws) : "memory");
}
__device__ __forceinline__ void store_flag1(unsigned char* ws, unsigned fo, unsigned v) {
  asm volatile("s_mov_b64 exec, 1\n\tglobal_store_dword %0, %1, %2 sc0 sc1\n\ts_mov_b64 exec, -1" :: "v"(fo), "v"(v), "s"(ws) : "memory");
}
__device__ __forceinline__ void store_flag7(unsigned long long fa, unsigned v) {
  asm volatile("s_mov_b64 exec, 0x7f\n\tglobal_store_dword %0, %1, off sc0 sc1\n\ts_mov_b64 exec, -1" :: "v"(fa), "v"(v) : "memory");
}
__device__ __forceinline__ unsigned job_off(int jw, int lane) { return (jw >> 1) * 8192 + (jw & 1) * 1024 + lane * 16; }

__device__ __forceinline__ void stream_slab(int j, int snd, int& tile, int& ks) {
  if (j < 124) { ks = j / 31; int q = j - ks * 31 + 4 * snd; if (q >= 31) q -= 31; tile = (q & 1) * 16 + (q >> 1); }
  else if (j < 200) { const int u = j - 124; ks = u / 19; int q = u - ks * 19 + 2 * snd; if (q >= 19) q -= 19; tile = q < 5 ? 75 + q : 26 + q; }
  else { const int u = j - 200, gq = u >= 60, v = u - gq * 60; ks = v / 15; int q = v - ks * 15 + 2 * snd; if (q >= 15) q -= 15; tile = 45 + 15 * gq + q; }
}
__device__ __forceinline__ int stream_idx(int tile, int ks, int snd) {
  if (tile < 31) { int q = (tile < 16 ? 2 * tile : 2 * tile - 31) - 4 * snd; if (q < 0) q += 31; return ks * 31 + q; }
  if (tile < 45 || tile >= 75) { int q = (tile < 45 ? tile - 26 : tile - 75) - 2 * snd; if (q < 0) q += 19; return 124 + ks * 19 + q; }
  const int gq = tile >= 60; int q = tile - 45 - 15 * gq - 2 * snd; if (q < 0) q += 15; return 200 + gq * 60 + ks * 15 + q;
}
__device__ __forceinline__ unsigned shared_off(int r, int i) { const int idx = NSH * r + i; return (unsigned)((idx & 15) * 256 * N_ + (SHC + (idx >> 4)) * 256) * 4u; }

#define INITK 0u
#define LAG 1
#define PWIN 4u
#define OST 0
#define PSLEEP 8
__device__ __forceinline__ bool wait_flags(unsigned char* ws, unsigned fo, unsigned call) {
  const unsigned long long t0 = __builtin_amdgcn_s_memrealtime();
  unsigned* tmo = (unsigned*)(ws + TMO_OFF);
  for (;;) {
    unsigned v, tm; const unsigned to = TMO_OFF;
    asm volatile("global_load_dword %0, %2, %4 sc0 sc1\n\tglobal_load_dword %1, %3, %4 sc0 sc1\n\ts_waitcnt vmcnt(0)" : "=&v"(v), "=&v"(tm) : "v"(fo), "v"(to), "s"(ws) : "memory");
    if (__ballot(v != call) == 0ull) return true;
    if (__builtin_amdgcn_readfirstlane(tm) == call || __builtin_amdgcn_s_memrealtime() - t0 > DEADLINE_TICKS) {
      __hip_atomic_store(tmo, call, __ATOMIC_RELAXED, __HIP_MEMORY_SCOPE_AGENT);
      return false;
    }
    __builtin_amdgcn_s_sleep(16);
  }
}
constexpr int NTURN = 8 / WPT;
__device__ __forceinline__ void push_elem(int it, int e, int jw, int rank, int& tile, int& ks) {
  if (it < 20) stream_slab(16 * it + e, rank, tile, ks);
  else { const int o = 16 * (it - 20) + e; tile = 80; ks = (int)(shared_off(rank, o >> 3) + (unsigned)((o & 7) * 32 + jw * 4) * (N_ * 4u)); }
}
__device__ __forceinline__ void pusher(const Params& p, unsigned imgo, unsigned flo, unsigned call, int b, int wid, int lane, unsigned (&R)[16]) {
  unsigned char* const myws = p.ws[p.rank];
  const int tau = wid / WPT, e = tau * 2 * WPT + b / NTURN, jw = (b % NTURN) * WPT + wid % WPT;
  unsigned long long fa = 0; unsigned pm_ = 0;
  unsigned char* wsd[7]; unsigned char* outd[7]; int pr[7];
#pragma unroll
  for (int k = 0; k < 7; ++k) {
    pr[k] = (p.rank + 1 + (wid + b + k) % 7) & 7; wsd[k] = p.ws[pr[k]]; outd[k] = p.out[pr[k]];
    if (lane == k) fa = (unsigned long long)wsd[k];
  }
  const unsigned jo = imgo + job_off(jw, lane), dno = DONE_OFF + (lane < NSH ? lane : 0) * 64u;
  bool ok = true;
  int tile, ks; push_elem(0, e, jw, p.rank, tile, ks);
  v4u D[4];
  retile(tile < 16, R, D);
  for (int i = 0; i < tau; ++i) BAR;
  for (int it = 0; it < 25; ++it) {
    const bool iso = it >= 20;
    const unsigned sid = tile * 32 + p.rank * 4 + ks;
    const unsigned vo = iso ? (unsigned)ks + lane * 16 : jo + sid * SLAB, ps = iso ? N_ * 4u : 2048u;
    pm_ = tile >= 75 && !iso ? (unsigned)(0xC070180E03ull >> ((tile - 75) * 8)) & 255u : 255u;
    unsigned char* dk[7];
#pragma unroll
    for (int k = 0; k < 7; ++k) dk[k] = iso ? outd[k] : (pm_ >> pr[k] & 1u) ? wsd[k] : myws;
#pragma unroll
    for (int c = 0; c < 4; ++c) {
      const unsigned vc = vo + c * ps;
      asm volatile("global_store_dwordx4 %0, %1, %2 sc0 sc1\n\tglobal_store_dwordx4 %0, %1, %3 sc0 sc1\n\tglobal_store_dwordx4 %0, %1, %4 sc0 sc1\n\t"
                   "global_store_dwordx4 %0, %1, %5 sc0 sc1\n\tglobal_store_dwordx4 %0, %1, %6 sc0 sc1\n\tglobal_store_dwordx4 %0, %1, %7 sc0 sc1\n\t"
                   "global_store_dwordx4 %0, %1, %8 sc0 sc1\n\ts_nop 1"
                   :: "v"(vc), "v"(D[c]), "s"(dk[0]), "s"(dk[1]), "s"(dk[2]), "s"(dk[3]), "s"(dk[4]), "s"(dk[5]), "s"(dk[6]) : "memory");
    }
    for (int i = 0; i < LAG; ++i) BAR;
    WAIT_V(0);
    push_elem(it < 24 ? it + 1 : it, e, jw, p.rank, tile, ks);
    load_job(p, tile, ks, jw, lane, R);
    unsigned dv; asm volatile("global_load_dword %0, %1, %2 sc0 sc1" : "=v"(dv) : "v"(dno), "s"(myws) : "memory");
    store_flag7(!iso ? fa + flo + sid * 64u + jw * 4u : it == 24 ? fa + OFLAG_OFF + (p.rank * 16 + b) * 64u + wid * 4u
                                                                  : (unsigned long long)myws + OFLAG_OFF + (p.rank * 16 + b) * 64u + wid * 4u, call);
    for (int i = LAG + 1; i < NTURN; ++i) BAR;
    asm volatile("s_waitcnt vmcnt(1)" : TIE16(R), "+v"(dv) :: "memory");
    retile(tile < 16 || tile >= 80, R, D);
    if (!ok && tile >= 80) {
#pragma unroll
      for (int c = 0; c < 4; ++c) D[c] = (v4u){0x7fc00000u, 0x7fc00000u, 0x7fc00000u, 0x7fc00000u};
    }
    if (it == 18) {
      if (__ballot(dv != call)) ok = wait_flags(myws, dno, call);
      asm volatile("buffer_inv sc0 sc1" ::: "memory");
    }
    BAR;
  }
  for (int i = tau + 1; i < NTURN; ++i) BAR;
  TS(1);
  if (wid == 0) {
    const int l = lane < 56 ? lane : 0, peer = (p.rank + 1 + (l >> 3)) & 7;
    if (!wait_flags(myws, OFLAG_OFF + (peer * 16 + b) * 64u + (l & 7) * 4u, call))
      *(float*)(p.out[p.rank] + shared_off(peer, 0) + (b * 8 + (l & 7)) * 4u) = __builtin_nanf("");
  }
  WAIT_V(0);
  TS(14);
}

__global__ __launch_bounds__(512, 2) void fused_kernel(const Params p) {
  extern __shared__ __attribute__((aligned(16))) unsigned char smem[];
  LAS unsigned char* lds = (LAS unsigned char*)smem;
  const int tid = threadIdx.x, wid = __builtin_amdgcn_readfirstlane(tid >> 6), wc = wid & 3;
  unsigned char* myws = p.ws[p.rank];
  const int b = blockIdx.x, rank = p.rank, g = b - NPUSH;
  const bool push = b < NPUSH;
  LAS unsigned* lctl = (LAS unsigned*)(lds + LCTL);
  const unsigned lbase = (unsigned)(size_t)lds;
  const int xc = b & 7, L = (xc >> 2) * 30 + (g >> 3), cn = L >> 2;
  const int pm = (xc & 3) * 4 + (L & 3), si = (L & 3) * 4 + (xc & 3);
  const bool skip = cn == 0 && si >= NSH;
  const unsigned nvis = skip ? 96u : 128u;
  auto tile_of = [&](int u, int& tm, int& tn) __attribute__((always_inline)) {
    const int v = u + (skip && u > 0), idx = NSH * rank + si;
    const bool sh = v == 1 && cn == 0;
    tm = sh ? idx & 15 : pm; tn = sh ? SHC + (idx >> 4) : v == 0 ? cn : 15 * v - 1 + cn;
  };
  unsigned call, imgo, flo;
  {
    const int lane = tid & 63;
    unsigned R[16], R2[16];
    int t1, k1, j1 = wid;
    if (push) { j1 = (b % NTURN) * WPT + wid % WPT; push_elem(0, wid / WPT * 2 * WPT + b / NTURN, j1, rank, t1, k1); } else { t1 = g >> 2; k1 = g & 3; }
    load_job(p, t1, k1, j1, lane, R);
    const bool two = !push && g < 80;
    if (two) load_job(p, t1 + 60, k1, wid, lane, R2);
    if (tid == 0) {
      const unsigned t = __hip_atomic_fetch_add((unsigned*)(myws + CTR_OFF), 1u, __ATOMIC_RELAXED, __HIP_MEMORY_SCOPE_AGENT);
      lctl[0] = t / (unsigned)GRID + 1u; lctl[1] = INITK; lctl[2] = 0u; lctl[3] = 0u;
    }
    if (wid >= 4 && !push) {
      const int u = wid - 4, l = lane & 31, src = l >> 2, ks = l & 3;
      int tm, tn; tile_of(u, tm, tn);
      const int ax = stream_idx(tm, ks, src), aw = stream_idx(16 + tn, ks, src);
      const int key = src == rank ? ks : ((ax > aw ? ax : aw) + 1) * 32 + l;
      int rk = 0;
#pragma unroll
      for (int s = 0; s < 32; ++s) rk += __builtin_amdgcn_readlane(key, s) < key;
      *(LAS int*)(lds + LTAB + (u * 32 + rk) * 4) = l;
      lctl[8 + 2 * u] = tm; lctl[9 + 2 * u] = 16 + tn;
    }
    asm volatile("s_waitcnt vmcnt(0)" : TIE16(R) :: "memory");
    asm volatile("" : TIE16(R2) :: "memory");
    __syncthreads();
    call = __builtin_amdgcn_readfirstlane(lctl[0]);
    imgo = IMG_OFF + (call & 1u) * IMG_BYTES;
    flo = FLAG_OFF + (call & 1u) * FLAG_BYTES;
    TS(0);
    if (push) {
      pusher(p, imgo, flo, call, b, wid, lane, R);
      return;
    }
    v4u D[4];
    const unsigned s1 = t1 * 32 + rank * 4 + k1;
    retile(t1 < 16, R, D);
    store_job(myws, imgo + s1 * SLAB + job_off(wid, lane) + 4096, D);
    if (two) {
      retile(false, R2, D);
      store_job(myws, imgo + (s1 + 60 * 32) * SLAB + job_off(wid, lane) + 4096, D);
    }
    WAIT_V(0);
    store_flag1(myws, flo + s1 * 64 + wid * 4, call);
    if (two) store_flag1(myws, flo + (s1 + 60 * 32) * 64 + wid * 4, call);
    TS(1);
  }
  const float scale = __builtin_bit_cast(float, __builtin_amdgcn_readfirstlane(__builtin_bit_cast(int, p.sx[0] * p.sw[0])));
  WAIT_V(0);

  auto run = [&](auto WRC) __attribute__((always_inline)) {
    constexpr int wr = decltype(WRC)::value;
    unsigned ppend = 0, pk = INITK, pw = 4u;
    auto poll = [&](bool blocking, unsigned need) __attribute__((always_inline)) {
      if (!blocking) {
        unsigned st; asm volatile("s_getreg_b32 %0, hwreg(HW_REG_IB_STS)" : "=s"(st) :: "memory");
        st = __builtin_amdgcn_readfirstlane(st);
        if ((st & 15u) | ((st >> 18) & 48u)) return;
      }
      WAIT_V(0);
      const unsigned ln = (unsigned)fresh_lane();
      if (ppend) {
        const v4u f = *(const LAS v4u*)(lds + LCP + ln * 16);
        unsigned long long bm = __ballot(f[0] == call && f[1] == call && f[2] == call && f[3] == call);
        bm &= bm >> 1; bm &= bm >> 2;
        bm = ~bm & 0x1111111111111111ull;
        unsigned c = bm ? (unsigned)__builtin_ctzll(bm) >> 2 : 16u;
        c = c < pw ? c : pw;
        pw = c == pw ? 16u : PWIN;
        const unsigned n = pk + c < nvis ? pk + c : nvis;
        if (n > pk) { asm volatile("buffer_inv sc0 sc1" ::: "memory"); pk = n; lctl[1] = n; }
      }
      if (blocking && ppend && pk < need) __builtin_amdgcn_s_sleep(PSLEEP);
      ppend = pk < nvis;
      if (ppend) {
        unsigned e = pk + ((ln >> 2) < pw ? ln >> 2 : 0u); e = e < nvis ? e : nvis - 1;
        const unsigned kk = *(const LAS unsigned*)(lds + LTAB + e * 4), tl = lctl[8 + (e >> 5) * 2 + (ln >> 1 & 1)];
        const unsigned la = lbase + LCP, fo = flo + (tl * 32 + kk) * 64 + (ln & 1) * 16; unsigned char* const w_ = myws;
        asm volatile("s_mov_b32 m0, %0\n\ts_nop 0\n\tglobal_load_lds_dwordx4 %1, %2 sc0 sc1" :: "s"(la), "v"(fo), "s"(w_) : "memory");
      }
      WAIT_L(0);
    };
    auto slow = [&](unsigned need) __attribute__((always_inline)) {
      const unsigned long long t0 = __builtin_amdgcn_s_memrealtime();
      for (;;) {
        if (wr == 1 && wc == 0) poll(true, need); else __builtin_amdgcn_s_sleep(1);
        const unsigned long long el = __builtin_amdgcn_s_memrealtime() - t0;
        if (wr == 1 && wc == 0 && el > 100000ull) {
          unsigned* tmo = (unsigned*)(myws + TMO_OFF);
          if (__builtin_amdgcn_readfirstlane(__hip_atomic_load(tmo, __ATOMIC_RELAXED, __HIP_MEMORY_SCOPE_AGENT)) == call || el > DEADLINE_TICKS) {
            __hip_atomic_store(tmo, call, __ATOMIC_RELAXED, __HIP_MEMORY_SCOPE_AGENT);
            lctl[2] = 1u; lctl[1] = DEADV; pk = nvis; ppend = 0;
          }
          WAIT_V(0);
        }
        if (__builtin_amdgcn_readfirstlane(__hip_atomic_load(lctl + 1, __ATOMIC_RELAXED, __HIP_MEMORY_SCOPE_WORKGROUP)) >= need || el > 2 * DEADLINE_TICKS) break;
      }
      asm volatile("" ::: "memory");
    };

    const unsigned char* XI = myws + imgo;
    const unsigned char* WI = XI + 512u * SLAB;
    const unsigned voff = (unsigned)(wc * 4096 + (tid & 63) * 16);
    const unsigned ldsw = (unsigned)wc * 4096u;
    const int aoff = wr * 8192 + (tid & 63) * 16, boff = wc * 4096 + (tid & 63) * 16;
#define SA(b_, h) (((b_) * 2 + (h)) * HTB)
#define SB(b_, h) ((4 + (b_) * 2 + (h)) * HTB)
#define STG1(bufoff, gbase, o) __builtin_amdgcn_global_load_lds((const unsigned*)((const char*)(gbase) + voff), (LAS unsigned*)(lds + (bufoff) + ldsw), 16, o, 0)
#define STAGE(bufoff, gbase) do { if (wr == 0) { STG1(bufoff, gbase, 0); STG1(bufoff, gbase, 1024); STG1(bufoff, gbase, 2048); STG1(bufoff, gbase, 3072); } } while (0)
#define LDA(dst, b_, h) do { _Pragma("unroll") for (int m = 0; m < 4; ++m) { \
    v4i lo = *(const LAS v4i*)(lds + SA(b_, h) + aoff + m * 2048); v4i hi = *(const LAS v4i*)(lds + SA(b_, h) + aoff + m * 2048 + 1024); dst[m] = cat8(lo, hi); } } while (0)
#define LDB(dst, b_, h) do { _Pragma("unroll") for (int n = 0; n < 2; ++n) { \
    v4i lo = *(const LAS v4i*)(lds + SB(b_, h) + boff + n * 2048); v4i hi = *(const LAS v4i*)(lds + SB(b_, h) + boff + n * 2048 + 1024); dst[n] = cat8(lo, hi); } } while (0)
#define MMA(ai, bj, At, Bt) do { __builtin_amdgcn_s_setprio(1); _Pragma("unroll") for (int m = 0; m < 4; ++m) _Pragma("unroll") for (int n = 0; n < 2; ++n) \
    acc[ai][bj][m][n] = __builtin_amdgcn_mfma_scale_f32_16x16x128_f8f6f4(At[m], Bt[n], acc[ai][bj][m][n], 1, 0, 0, 0, 0, 0); __builtin_amdgcn_s_setprio(0); } while (0)
#define MEMSTEP do { if (wr == 0) WAIT_V(12); else if (wc == 0 && ppend && pk < need + 24u) poll(false, 0u); } while (0)
#define KOFF(i) ((size_t)(unsigned)__builtin_amdgcn_readlane(kv, (i)))
    f32x4 acc[2][2][4][2];
#pragma unroll
    for (int a = 0; a < 2; ++a)
#pragma unroll
      for (int c = 0; c < 2; ++c)
#pragma unroll
        for (int m = 0; m < 4; ++m)
#pragma unroll
          for (int n = 0; n < 2; ++n) acc[a][c][m][n] = (f32x4){0.f, 0.f, 0.f, 0.f};
    v8i At[4], B0[2], B1[2];
    const size_t hstep = HTB, tstep = (size_t)SLAB * NKS;
    const size_t kown = (size_t)(rank * 4) * SLAB;
    int tm = pm, tn = cn;
    const char* cA = (const char*)WI + (size_t)tn * tstep; const char* cB = (const char*)XI + (size_t)tm * tstep;
    slow(4u);
    unsigned known = 4u;
    {
      const size_t k0 = kown, k1 = kown + SLAB;
      STAGE(SB(0, 0), cB + k0); STAGE(SA(0, 0), cA + k0); STAGE(SB(0, 1), cB + k0 + hstep); STAGE(SA(0, 1), cA + k0 + hstep);
      if (wr == 1) BAR;
      if (wr == 0) WAIT_V(8);
      BAR;
      STAGE(SB(1, 0), cB + k1); STAGE(SA(1, 0), cA + k1); STAGE(SB(1, 1), cB + k1 + hstep);
      if (wr == 0) WAIT_V(12);
      BAR;
    }
    for (int ui = 0;; ++ui) {
      const bool has_next = (unsigned)(ui * 32 + 32) < nvis;
      int nm = tm, nn = tn;
      if (has_next) tile_of(ui + 1, nm, nn);
      const char* nA = (const char*)WI + (size_t)nn * tstep; const char* nB = (const char*)XI + (size_t)nm * tstep;
      int kv = 0;
      if (wr == 0) kv = *(const LAS int*)(lds + LTAB + (ui * 32 + (fresh_lane() & 31)) * 4) * (int)SLAB;
      for (int t = 0; t < NKS; t += 2) {
        const bool last = (t == NKS - 2);
        const unsigned need = last && !has_next ? nvis : (unsigned)(ui * 32 + t + (last ? 6 : 4));
        if (known < need) { if (wr == 0) BAR; slow(need); if (wr == 1) BAR; known = need; }
        const char* a1 = cA + KOFF(t + 1);
        const char* a2 = last ? nA + kown : cA + KOFF(t + 2); const char* b2 = last ? nB + kown : cB + KOFF(t + 2);
        const char* a3 = last ? nA + kown + SLAB : cA + KOFF(t + 3); const char* b3 = last ? nB + kown + SLAB : cB + KOFF(t + 3);
        LDB(B0, 0, 0); SCHED; LDA(At, 0, 0); STAGE(SA(1, 1), a1 + hstep);
        WAIT_L(8); BAR; WAIT_L(0); MMA(0, 0, At, B0); BAR; SCHED;
        LDB(B1, 0, 1); STAGE(SB(0, 0), b2);
        BAR; WAIT_L(0); MMA(0, 1, At, B1); BAR;
        LDA(At, 0, 1); STAGE(SA(0, 0), a2);
        BAR; WAIT_L(0); MMA(1, 0, At, B0); BAR; SCHED;
        STAGE(SB(0, 1), b2 + hstep); MEMSTEP;
        BAR; MMA(1, 1, At, B1); BAR;
        LDB(B0, 1, 0); SCHED; LDA(At, 1, 0); STAGE(SA(0, 1), a2 + hstep);
        WAIT_L(8); BAR; WAIT_L(0); MMA(0, 0, At, B0); BAR; SCHED;
        LDB(B1, 1, 1); const unsigned kn = lctl[1]; STAGE(SB(1, 0), b3);
        BAR; WAIT_L(0); MMA(0, 1, At, B1); known = __builtin_amdgcn_readfirstlane(kn); BAR;
        LDA(At, 1, 1); STAGE(SA(1, 0), a3);
        BAR; WAIT_L(0); MMA(1, 0, At, B0); BAR; SCHED;
        STAGE(SB(1, 1), b3 + hstep); MEMSTEP;
        BAR; MMA(1, 1, At, B1); BAR;
      }
      const int dead = __builtin_amdgcn_readfirstlane(lctl[2]);
      TS(2 + 2 * ui);
      auto epilogue = [&](auto SHT) __attribute__((always_inline)) {
        const int ln = fresh_lane(); const int fr = ln & 15, fq = ln >> 4;
        unsigned char* base = p.out[rank] + (size_t)((tm * 256 + wc * 32) * N_ + tn * 256 + wr * 64) * 4;
        const unsigned vo = (unsigned)(fr * N_ + 4 * fq) * 4u;
        const float qnan = __builtin_nanf("");
#pragma unroll
        for (int bj = 0; bj < 2; ++bj)
#pragma unroll
          for (int n = 0; n < 2; ++n) {
            unsigned char* rowp = base + (size_t)(bj * 128 + n * 16) * N_ * 4;
#pragma unroll
            for (int ai = 0; ai < 2; ++ai)
#pragma unroll
              for (int r4 = 0; r4 < 4; ++r4) {
                f32x4 v;
#pragma unroll
                for (int c = 0; c < 4; ++c) v[c] = dead ? qnan : fmaxf(acc[ai][bj][c][n][r4] * scale, 0.f);
                if (decltype(SHT)::value) asm volatile("global_store_dwordx4 %0, %1, %2 offset:%3 sc0 sc1\n\ts_nop 1" :: "v"(vo), "v"(v), "s"(rowp), "n"(ai * 512 + r4 * 64) : "memory");
                else if (OST == 1) __builtin_nontemporal_store(v, (f32x4*)(rowp + vo + ai * 512 + r4 * 64)); else *(f32x4*)(rowp + vo + ai * 512 + r4 * 64) = v;
              }
          }
      };
      if (tn >= SHC) {
        epilogue(std::true_type{});
        WAIT_V(0);
        unsigned o_ = 0;
        if (fresh_lane() == 0) o_ = __hip_atomic_fetch_add(lctl + 3, 1u, __ATOMIC_RELAXED, __HIP_MEMORY_SCOPE_WORKGROUP);
        if (__builtin_amdgcn_readfirstlane(o_) == 7u) store_flag1(myws, DONE_OFF + si * 64, call);
        if (wr == 0) WAIT_V(0);
      } else epilogue(std::false_type{});
      TS(3 + 2 * ui);
      if (!has_next) break;
#pragma unroll
      for (int a = 0; a < 2; ++a)
#pragma unroll
        for (int c = 0; c < 2; ++c)
#pragma unroll
          for (int m = 0; m < 4; ++m)
#pragma unroll
            for (int n = 0; n < 2; ++n) acc[a][c][m][n] = (f32x4){0.f, 0.f, 0.f, 0.f};
      cA = nA; cB = nB; tm = nm; tn = nn;
    }
    if (wr == 0) BAR;
    WAIT_V(0);
    TS(14);
  };
  if (wid < 4) run(std::integral_constant<int, 0>{}); else run(std::integral_constant<int, 1>{});
}
extern "C" void kernel_launch(int rank, int nranks, ncclComm_t comm,
                              hipStream_t stream,
                              void* const* d_in, const int* in_sizes, int n_in,
                              void* d_out, int out_size,
                              const int* in_dtypes, int out_dtype,
                              void* const* d_out_all,
                              void* const* d_ws_all, size_t ws_bytes) {
  if (nranks != NR || (unsigned)rank >= (unsigned)NR || ws_bytes < WS_NEED || n_in < 4) {
    fprintf(stderr, "kernel_launch: unexpected configuration\n"); return;
  }
  static bool attr_done[64] = {};
  int dev = 0; HIP_CHECK(hipGetDevice(&dev));
  if (!attr_done[dev & 63]) {
    HIP_CHECK(hipFuncSetAttribute((const void*)fused_kernel, hipFuncAttributeMaxDynamicSharedMemorySize, LDS_BYTES));
    attr_done[dev & 63] = true;
  }
  Params p;
  p.x = (const unsigned char*)d_in[0]; p.w = (const unsigned char*)d_in[1];
  p.sx = (const float*)d_in[2]; p.sw = (const float*)d_in[3];
  for (int i = 0; i < NR; ++i) { p.ws[i] = (unsigned char*)d_ws_all[i]; p.out[i] = (unsigned char*)(i == rank ? d_out : d_out_all[i]); }
  p.rank = rank;
  fused_kernel<<<GRID, 512, LDS_BYTES, stream>>>(p);
}
```

```cpp
#include <hip/hip_runtime.h>
#include <rccl/rccl.h>
#include <cstdio>
#include <type_traits>

#define HIP_CHECK(expr) do { hipError_t _e = (expr); if (_e != hipSuccess) { \
    fprintf(stderr, "HIP error %s at %s:%d\n", hipGetErrorString(_e), __FILE__, __LINE__); \
    return; } } while (0)

typedef int v8i __attribute__((ext_vector_type(8)));
typedef int v4i __attribute__((ext_vector_type(4)));
typedef unsigned v4u __attribute__((ext_vector_type(4)));
typedef float f32x4 __attribute__((ext_vector_type(4)));
#define LAS __attribute__((address_space(3)))

constexpr int NR = 8;
constexpr int M_ = 4096, N_ = 16384, KL = 512;
constexpr int NKS = 32;
constexpr int HTB = 16384;
constexpr int STAGE_BYTES = 8 * HTB;
constexpr unsigned LCTL = STAGE_BYTES, LTAB = STAGE_BYTES + 256, LCP = STAGE_BYTES + 1024;
constexpr int LDS_BYTES = STAGE_BYTES + 2048;
constexpr unsigned SLAB = 32768;
constexpr unsigned IMG_BYTES = 2560u * SLAB;
constexpr unsigned IMG_OFF = 1u << 20;
constexpr size_t WS_NEED = (size_t)IMG_OFF + 2 * (size_t)IMG_BYTES;
constexpr unsigned CTR_OFF = 0, TMO_OFF = 128, DONE_OFF = 4096, OFLAG_OFF = 8192, FLAG_OFF = 65536, FLAG_BYTES = 2560u * 64u, DBG_OFF = 524288;
constexpr int GRID = 256;
constexpr int NPUSH = 16;
constexpr int NSH = 10, SHC = 59;
constexpr unsigned DEADV = 0x7fffffffu;
constexpr unsigned long long DEADLINE_TICKS = 90000000ull;
#define WPT 2

struct Params {
  const unsigned char* x;
  const unsigned char* w;
  const float* sx; const float* sw;
  unsigned char* ws[NR];
  unsigned char* out[NR];
  int rank;
};

#define TS(slot) do {} while (0)
#define TV(slot, v) do {} while (0)
__device__ __forceinline__ v8i cat8(v4i a, v4i b) { return __builtin_shufflevector(a, b, 0, 1, 2, 3, 4, 5, 6, 7); }
#define WAIT_V(n) asm volatile("s_waitcnt vmcnt(" #n ")" ::: "memory")
#define WAIT_L(n) asm volatile("s_waitcnt lgkmcnt(" #n ")" ::: "memory")
#define BAR __builtin_amdgcn_s_barrier()
#define SCHED __builtin_amdgcn_sched_barrier(0)
#define TIE16(R) "+v"(R[0]), "+v"(R[1]), "+v"(R[2]), "+v"(R[3]), "+v"(R[4]), "+v"(R[5]), "+v"(R[6]), "+v"(R[7]), \
                 "+v"(R[8]), "+v"(R[9]), "+v"(R[10]), "+v"(R[11]), "+v"(R[12]), "+v"(R[13]), "+v"(R[14]), "+v"(R[15])

__device__ __forceinline__ int fresh_lane() { int l; asm volatile("v_mbcnt_lo_u32_b32 %0, -1, 0\n\tv_mbcnt_hi_u32_b32 %0, -1, %0" : "=v"(l)); return l; }

__device__ __forceinline__ void load_job(const Params& p, int tile, int ks, int jw, int lane, unsigned (&R)[16]) {
  const int i = lane & 15, g = lane >> 4, perm = 4 * (i & 3) + (i >> 2);
  const bool isx = tile < 16, iso = tile >= 80;
  const unsigned vo = isx ? i * KL + g * 32 : iso ? lane * 16 : g * 32 * N_ + 4 * perm;
  const unsigned char* src = isx ? p.x + (size_t)(tile * 256 + (jw >> 1) * 64) * KL + ks * 128 + (jw & 1) * 16
                           : iso ? p.out[p.rank] + (unsigned)ks
                                 : p.w + (size_t)(ks * 128 + (jw & 1) * 16) * N_ + (tile - 16) * 256 + (jw >> 1) * 64;
  const unsigned s1 = isx || iso ? 4 : N_, s2 = isx ? 16 * KL : 4 * N_;
#pragma unroll
  for (int r = 0; r < 16; ++r) asm volatile("global_load_dword %0, %1, %2" : "=v"(R[r]) : "v"(vo), "s"(src + (r & 3) * s1 + (r >> 2) * s2) : "memory");
}
__device__ __forceinline__ void retile(bool plain, const unsigned (&R)[16], v4u (&D)[4]) {
  if (plain) {
#pragma unroll
    for (int e = 0; e < 4; ++e) D[e] = (v4u){R[4 * e], R[4 * e + 1], R[4 * e + 2], R[4 * e + 3]};
  } else {
#pragma unroll
    for (int d = 0; d < 4; ++d) {
      const unsigned a = R[4 * d], bb = R[4 * d + 1], c = R[4 * d + 2], f = R[4 * d + 3];
      const unsigned t0_ = __builtin_amdgcn_perm(bb, a, 0x05010400u), t1_ = __builtin_amdgcn_perm(bb, a, 0x07030602u);
      const unsigned t2_ = __builtin_amdgcn_perm(f, c, 0x05010400u), t3_ = __builtin_amdgcn_perm(f, c, 0x07030602u);
      D[0][d] = __builtin_amdgcn_perm(t2_, t0_, 0x05040100u); D[1][d] = __builtin_amdgcn_perm(t2_, t0_, 0x07060302u);
      D[2][d] = __builtin_amdgcn_perm(t3_, t1_, 0x05040100u); D[3][d] = __builtin_amdgcn_perm(t3_, t1_, 0x07060302u);
    }
  }
}
__device__ __forceinline__ void store_job(unsigned char* ws, unsigned vo, const v4u (&D)[4]) {
  asm volatile("global_store_dwordx4 %0, %1, %5 offset:-4096 sc0 sc1\n\tglobal_store_dwordx4 %0, %2, %5 offset:-2048 sc0 sc1\n\t"
               "global_store_dwordx4 %0, %3, %5 sc0 sc1\n\tglobal_store_dwordx4 %0, %4, %5 offset:2048 sc0 sc1\n\ts_nop 1"
               :: "v"(vo), "v"(D[0]), "v"(D[1]), "v"(D[2]), "v"(D[3]), "s"(ws) : "memory");
}
__device__ __forceinline__ void store_flag1(unsigned char* ws, unsigned fo, unsigned v) {
  asm volatile("s_mov_b64 exec, 1\n\tglobal_store_dword %0, %1, %2 sc0 sc1\n\ts_mov_b64 exec, -1" :: "v"(fo), "v"(v), "s"(ws) : "memory");
}
__device__ __forceinline__ void store_flag7(unsigned long long fa, unsigned v) {
  asm volatile("s_mov_b64 exec, 0x7f\n\tglobal_store_dword %0, %1, off sc0 sc1\n\ts_mov_b64 exec, -1" :: "v"(fa), "v"(v) : "memory");
}
__device__ __forceinline__ unsigned job_off(int jw, int lane) { return (jw >> 1) * 8192 + (jw & 1) * 1024 + lane * 16; }

__device__ __forceinline__ void stream_slab(int j, int snd, int& tile, int& ks) {
  if (j < 124) { ks = j / 31; int q = j - ks * 31 + 4 * snd; if (q >= 31) q -= 31; tile = (q & 1) * 16 + (q >> 1); }
  else if (j < 200) { const int u = j - 124; ks = u / 19; int q = u - ks * 19 + 2 * snd; if (q >= 19) q -= 19; tile = q < 5 ? 75 + q : 26 + q; }
  else { const int u = j - 200, gq = u >= 60, v = u - gq * 60; ks = v / 15; int q = v - ks * 15 + 2 * snd; if (q >= 15) q -= 15; tile = 45 + 15 * gq + q; }
}
__device__ __forceinline__ int stream_idx(int tile, int ks, int snd) {
  if (tile < 31) { int q = (tile < 16 ? 2 * tile : 2 * tile - 31) - 4 * snd; if (q < 0) q += 31; return ks * 31 + q; }
  if (tile < 45 || tile >= 75) { int q = (tile < 45 ? tile - 26 : tile - 75) - 2 * snd; if (q < 0) q += 19; return 124 + ks * 19 + q; }
  const int gq = tile >= 60; int q = tile - 45 - 15 * gq - 2 * snd; if (q < 0) q += 15; return 200 + gq * 60 + ks * 15 + q;
}
__device__ __forceinline__ unsigned shared_off(int r, int i) { const int idx = NSH * r + i; return (unsigned)((idx & 15) * 256 * N_ + (SHC + (idx >> 4)) * 256) * 4u; }

#define INITK 0u
#define LAG 1
#define PWIN 4u
#define PTH 8u
#define INV 0
#define LASTWT 0
#define OST 0
#define PSLEEP 8
__device__ __forceinline__ bool wait_flags(unsigned char* ws, unsigned fo, unsigned call) {
  const unsigned long long t0 = __builtin_amdgcn_s_memrealtime();
  unsigned* tmo = (unsigned*)(ws + TMO_OFF);
  for (;;) {
    unsigned v, tm; const unsigned to = TMO_OFF;
    asm volatile("global_load_dword %0, %2, %4 sc0 sc1\n\tglobal_load_dword %1, %3, %4 sc0 sc1\n\ts_waitcnt vmcnt(0)" : "=&v"(v), "=&v"(tm) : "v"(fo), "v"(to), "s"(ws) : "memory");
    if (__ballot(v != call) == 0ull) return true;
    if (__builtin_amdgcn_readfirstlane(tm) == call || __builtin_amdgcn_s_memrealtime() - t0 > DEADLINE_TICKS) {
      __hip_atomic_store(tmo, call, __ATOMIC_RELAXED, __HIP_MEMORY_SCOPE_AGENT);
      return false;
    }
    __builtin_amdgcn_s_sleep(16);
  }
}
constexpr int NTURN = 8 / WPT;
__device__ __forceinline__ void push_elem(int it, int e, int jw, int rank, int& tile, int& ks) {
  if (it < 20) stream_slab(16 * it + e, rank, tile, ks);
  else { const int o = 16 * (it - 20) + e; tile = 80; ks = (int)(shared_off(rank, o >> 3) + (unsigned)((o & 7) * 32 + jw * 4) * (N_ * 4u)); }
}
__device__ __forceinline__ void pusher(const Params& p, unsigned imgo, unsigned flo, unsigned call, int b, int wid, int lane, unsigned (&R)[16]) {
  unsigned char* const myws = p.ws[p.rank];
  const int tau = wid / WPT, e = tau * 2 * WPT + b / NTURN, jw = (b % NTURN) * WPT + wid % WPT;
  unsigned long long fa = 0; unsigned pm_ = 0;
  unsigned char* wsd[7]; unsigned char* outd[7]; int pr[7];
#pragma unroll
  for (int k = 0; k < 7; ++k) {
    pr[k] = (p.rank + 1 + (wid + b + k) % 7) & 7; wsd[k] = p.ws[pr[k]]; outd[k] = p.out[pr[k]];
    if (lane == k) fa = (unsigned long long)wsd[k];
  }
  const unsigned jo = imgo + job_off(jw, lane), dno = DONE_OFF + (lane < NSH ? lane : 0) * 64u;
  bool ok = true;
  int tile, ks; push_elem(0, e, jw, p.rank, tile, ks);
  v4u D[4];
  retile(tile < 16, R, D);
  for (int i = 0; i < tau; ++i) BAR;
  for (int it = 0; it < 25; ++it) {
    const bool iso = it >= 20;
    const unsigned sid = tile * 32 + p.rank * 4 + ks;
    const unsigned vo = iso ? (unsigned)ks + lane * 16 : jo + sid * SLAB, ps = iso ? N_ * 4u : 2048u;
    pm_ = tile >= 75 && !iso ? (unsigned)(0xC070180E03ull >> ((tile - 75) * 8)) & 255u : 255u;
    unsigned char* dk[7];
#pragma unroll
    for (int k = 0; k < 7; ++k) dk[k] = iso ? outd[k] : (pm_ >> pr[k] & 1u) ? wsd[k] : myws;
#pragma unroll
    for (int c = 0; c < 4; ++c) {
      const unsigned vc = vo + c * ps;
      asm volatile("global_store_dwordx4 %0, %1, %2 sc0 sc1\n\tglobal_store_dwordx4 %0, %1, %3 sc0 sc1\n\tglobal_store_dwordx4 %0, %1, %4 sc0 sc1\n\t"
                   "global_store_dwordx4 %0, %1, %5 sc0 sc1\n\tglobal_store_dwordx4 %0, %1, %6 sc0 sc1\n\tglobal_store_dwordx4 %0, %1, %7 sc0 sc1\n\t"
                   "global_store_dwordx4 %0, %1, %8 sc0 sc1\n\ts_nop 1"
                   :: "v"(vc), "v"(D[c]), "s"(dk[0]), "s"(dk[1]), "s"(dk[2]), "s"(dk[3]), "s"(dk[4]), "s"(dk[5]), "s"(dk[6]) : "memory");
    }
    for (int i = 0; i < LAG; ++i) BAR;
    WAIT_V(0);
    push_elem(it < 24 ? it + 1 : it, e, jw, p.rank, tile, ks);
    load_job(p, tile, ks, jw, lane, R);
    unsigned dv; asm volatile("global_load_dword %0, %1, %2 sc0 sc1" : "=v"(dv) : "v"(dno), "s"(myws) : "memory");
    store_flag7(!iso ? fa + flo + sid * 64u + jw * 4u : it == 24 ? fa + OFLAG_OFF + (p.rank * 16 + b) * 64u + wid * 4u
                                                                  : (unsigned long long)myws + OFLAG_OFF + (p.rank * 16 + b) * 64u + wid * 4u, call);
    for (int i = LAG + 1; i < NTURN; ++i) BAR;
    asm volatile("s_waitcnt vmcnt(1)" : TIE16(R), "+v"(dv) :: "memory");
    retile(tile < 16 || tile >= 80, R, D);
    if (!ok && tile >= 80) {
#pragma unroll
      for (int c = 0; c < 4; ++c) D[c] = (v4u){0x7fc00000u, 0x7fc00000u, 0x7fc00000u, 0x7fc00000u};
    }
    if (it == 18) {
      if (__ballot(dv != call)) ok = wait_flags(myws, dno, call);
    }
    BAR;
  }
  for (int i = tau + 1; i < NTURN; ++i) BAR;
  TS(1);
  if (wid == 0) {
    const int l = lane < 56 ? lane : 0, peer = (p.rank + 1 + (l >> 3)) & 7;
    if (!wait_flags(myws, OFLAG_OFF + (peer * 16 + b) * 64u + (l & 7) * 4u, call))
      *(float*)(p.out[p.rank] + shared_off(peer, 0) + (b * 8 + (l & 7)) * 4u) = __builtin_nanf("");
  }
  WAIT_V(0);
  TS(14);
}

__global__ __launch_bounds__(512, 2) void fused_kernel(const Params p) {
  extern __shared__ __attribute__((aligned(16))) unsigned char smem[];
  LAS unsigned char* lds = (LAS unsigned char*)smem;
  const int tid = threadIdx.x, wid = __builtin_amdgcn_readfirstlane(tid >> 6), wc = wid & 3;
  unsigned char* myws = p.ws[p.rank];
  const int b = blockIdx.x, rank = p.rank, g = b - NPUSH;
  const bool push = b < NPUSH;
  LAS unsigned* lctl = (LAS unsigned*)(lds + LCTL);
  const unsigned lbase = (unsigned)(size_t)lds;
  const int xc = b & 7, L = (xc >> 2) * 30 + (g >> 3), cn = L >> 2;
  const int pm = (xc & 3) * 4 + (L & 3), si = (L & 3) * 4 + (xc & 3);
  const bool skip = cn == 0 && si >= NSH;
  const unsigned nvis = skip ? 96u : 128u;
  auto tile_of = [&](int u, int& tm, int& tn) __attribute__((always_inline)) {
    const int v = u + (skip && u > 0), idx = NSH * rank + si;
    const bool sh = v == 1 && cn == 0;
    tm = sh ? idx & 15 : pm; tn = sh ? SHC + (idx >> 4) : v == 0 ? cn : 15 * v - 1 + cn;
  };
  unsigned call, imgo, flo;
  {
    const int lane = tid & 63;
    unsigned R[16], R2[16];
    int t1, k1, j1 = wid;
    if (push) { j1 = (b % NTURN) * WPT + wid % WPT; push_elem(0, wid / WPT * 2 * WPT + b / NTURN, j1, rank, t1, k1); } else { t1 = g >> 2; k1 = g & 3; }
    load_job(p, t1, k1, j1, lane, R);
    const bool two = !push && g < 80;
    if (two) load_job(p, t1 + 60, k1, wid, lane, R2);
    if (tid == 0) {
      const unsigned t = __hip_atomic_fetch_add((unsigned*)(myws + CTR_OFF), 1u, __ATOMIC_RELAXED, __HIP_MEMORY_SCOPE_AGENT);
      lctl[0] = t / (unsigned)GRID + 1u; lctl[1] = INITK; lctl[2] = 0u; lctl[3] = 0u;
    }
    if (wid >= 4 && !push) {
      const int u = wid - 4, l = lane & 31, src = l >> 2, ks = l & 3;
      int tm, tn; tile_of(u, tm, tn);
      const int ax = stream_idx(tm, ks, src), aw = stream_idx(16 + tn, ks, src);
#define CORD 9
      const int key = src == rank ? ks : u >= CORD ? 32 + ks * 8 + ((src - rank) & 7) : ((ax > aw ? ax : aw) + 1) * 32 + l;
      int rk = 0;
#pragma unroll
      for (int s = 0; s < 32; ++s) rk += __builtin_amdgcn_readlane(key, s) < key;
      *(LAS int*)(lds + LTAB + (u * 32 + rk) * 4) = l;
      lctl[8 + 2 * u] = tm; lctl[9 + 2 * u] = 16 + tn;
    }
    asm volatile("s_waitcnt vmcnt(0)" : TIE16(R) :: "memory");
    asm volatile("" : TIE16(R2) :: "memory");
    __syncthreads();
    call = __builtin_amdgcn_readfirstlane(lctl[0]);
    imgo = IMG_OFF + (call & 1u) * IMG_BYTES;
    flo = FLAG_OFF + (call & 1u) * FLAG_BYTES;
    TS(0);
    if (push) {
      pusher(p, imgo, flo, call, b, wid, lane, R);
      return;
    }
    v4u D[4];
    const unsigned s1 = t1 * 32 + rank * 4 + k1;
    retile(t1 < 16, R, D);
    store_job(myws, imgo + s1 * SLAB + job_off(wid, lane) + 4096, D);
    if (two) {
      retile(false, R2, D);
      store_job(myws, imgo + (s1 + 60 * 32) * SLAB + job_off(wid, lane) + 4096, D);
    }
    WAIT_V(0);
    store_flag1(myws, flo + s1 * 64 + wid * 4, call);
    if (two) store_flag1(myws, flo + (s1 + 60 * 32) * 64 + wid * 4, call);
    TS(1);
  }
  const float scale = __builtin_bit_cast(float, __builtin_amdgcn_readfirstlane(__builtin_bit_cast(int, p.sx[0] * p.sw[0])));
  WAIT_V(0);

  auto run = [&](auto WRC) __attribute__((always_inline)) {
    constexpr int wr = decltype(WRC)::value;
    unsigned ppend = 0, pk = INITK, pw = 4u;
    auto peval = [&]() __attribute__((always_inline)) {
      const v4u f = *(const LAS v4u*)(lds + LCP + (unsigned)fresh_lane() * 16);
      unsigned long long bm = __ballot(f[0] == call && f[1] == call && f[2] == call && f[3] == call);
      bm &= bm >> 1; bm &= bm >> 2;
      bm = ~bm & 0x1111111111111111ull;
      unsigned c = bm ? (unsigned)__builtin_ctzll(bm) >> 2 : 16u;
      c = c < pw ? c : pw;
      pw = c == pw ? 16u : PWIN;
      const unsigned n = pk + c < nvis ? pk + c : nvis;
      if (n > pk) {
        pk = n; lctl[1] = n;
      }
      ppend = 0;
    };
    auto pissue = [&]() __attribute__((always_inline)) {
      const unsigned ln = (unsigned)fresh_lane();
      unsigned e = pk + ((ln >> 2) < pw ? ln >> 2 : 0u); e = e < nvis ? e : nvis - 1;
      const unsigned kk = *(const LAS unsigned*)(lds + LTAB + e * 4), tl = lctl[8 + (e >> 5) * 2 + (ln >> 1 & 1)];
      const unsigned la = lbase + LCP, fo = flo + (tl * 32 + kk) * 64 + (ln & 1) * 16; unsigned char* const w_ = myws;
      asm volatile("s_mov_b32 m0, %0\n\ts_nop 0\n\tglobal_load_lds_dwordx4 %1, %2 sc0 sc1" :: "s"(la), "v"(fo), "s"(w_) : "memory");
      ppend = 1;
    };
    auto poll = [&](bool blocking, unsigned need) __attribute__((always_inline)) {
      if (!blocking) {
        if (ppend) {
          unsigned st; asm volatile("s_getreg_b32 %0, hwreg(HW_REG_IB_STS)" : "=s"(st) :: "memory");
          st = __builtin_amdgcn_readfirstlane(st);
          if ((st & 15u) | ((st >> 18) & 48u)) return;
          WAIT_V(0);
          peval();
        } else { pw = 16u; pissue(); }
        return;
      }
      WAIT_V(0);
      if (ppend) { peval(); if (pk < need) __builtin_amdgcn_s_sleep(PSLEEP); }
      if (pk < nvis) pissue();
      WAIT_L(0);
    };
    auto slow = [&](unsigned need) __attribute__((always_inline)) {
      const unsigned long long t0 = __builtin_amdgcn_s_memrealtime();
      for (;;) {
        if (wr == 1 && wc == 0) poll(true, need); else __builtin_amdgcn_s_sleep(1);
        const unsigned long long el = __builtin_amdgcn_s_memrealtime() - t0;
        if (wr == 1 && wc == 0 && el > 100000ull) {
          unsigned* tmo = (unsigned*)(myws + TMO_OFF);
          if (__builtin_amdgcn_readfirstlane(__hip_atomic_load(tmo, __ATOMIC_RELAXED, __HIP_MEMORY_SCOPE_AGENT)) == call || el > DEADLINE_TICKS) {
            __hip_atomic_store(tmo, call, __ATOMIC_RELAXED, __HIP_MEMORY_SCOPE_AGENT);
            lctl[2] = 1u; lctl[1] = DEADV; pk = nvis; ppend = 0;
          }
          WAIT_V(0);
        }
        if (__builtin_amdgcn_readfirstlane(__hip_atomic_load(lctl + 1, __ATOMIC_RELAXED, __HIP_MEMORY_SCOPE_WORKGROUP)) >= need || el > 2 * DEADLINE_TICKS) break;
      }
      asm volatile("" ::: "memory");
    };

    const unsigned char* XI = myws + imgo;
    const unsigned voff = (unsigned)(wc * 4096 + (tid & 63) * 16);
    const unsigned ldsw = (unsigned)wc * 4096u;
    const int aoff = wr * 8192 + (tid & 63) * 16, boff = wc * 4096 + (tid & 63) * 16;
#define SA(b_, h) (((b_) * 2 + (h)) * HTB)
#define SB(b_, h) ((4 + (b_) * 2 + (h)) * HTB)
#define STG1(bufoff, va, o) __builtin_amdgcn_global_load_lds((const unsigned*)(XI + (va)), (LAS unsigned*)(lds + (bufoff) + ldsw), 16, o, 0)
#define STAGE(bufoff, goff) do { if (wr == 0) { const unsigned va_ = voff + (goff); STG1(bufoff, va_, 0); STG1(bufoff, va_, 1024); STG1(bufoff, va_, 2048); STG1(bufoff, va_, 3072); } } while (0)
#define LDA(dst, b_, h) do { _Pragma("unroll") for (int m = 0; m < 4; ++m) { \
    v4i lo = *(const LAS v4i*)(lds + SA(b_, h) + aoff + m * 2048); v4i hi = *(const LAS v4i*)(lds + SA(b_, h) + aoff + m * 2048 + 1024); dst[m] = cat8(lo, hi); } } while (0)
#define LDB(dst, b_, h) do { _Pragma("unroll") for (int n = 0; n < 2; ++n) { \
    v4i lo = *(const LAS v4i*)(lds + SB(b_, h) + boff + n * 2048); v4i hi = *(const LAS v4i*)(lds + SB(b_, h) + boff + n * 2048 + 1024); dst[n] = cat8(lo, hi); } } while (0)
#define MMA(ai, bj, At, Bt) do { __builtin_amdgcn_s_setprio(1); _Pragma("unroll") for (int m = 0; m < 4; ++m) _Pragma("unroll") for (int n = 0; n < 2; ++n) \
    acc[ai][bj][m][n] = __builtin_amdgcn_mfma_scale_f32_16x16x128_f8f6f4(At[m], Bt[n], acc[ai][bj][m][n], 1, 0, 0, 0, 0, 0); __builtin_amdgcn_s_setprio(0); } while (0)
#define MF1(ai, bj, At, Bt, q) acc[ai][bj][(q) >> 1][(q) & 1] = __builtin_amdgcn_mfma_scale_f32_16x16x128_f8f6f4(At[(q) >> 1], Bt[(q) & 1], acc[ai][bj][(q) >> 1][(q) & 1], 1, 0, 0, 0, 0, 0)
#define WV12 WAIT_V(12)
#define MEMSTEP do { if (wr == 0) WV12; else if (wc == 0 && (ppend || (pk < nvis && pk < need + PTH))) poll(false, 0u); } while (0)
#define KOFF(i) ((unsigned)__builtin_amdgcn_readlane(kv, (i)))
    f32x4 acc[2][2][4][2];
#pragma unroll
    for (int a = 0; a < 2; ++a)
#pragma unroll
      for (int c = 0; c < 2; ++c)
#pragma unroll
        for (int m = 0; m < 4; ++m)
#pragma unroll
          for (int n = 0; n < 2; ++n) acc[a][c][m][n] = (f32x4){0.f, 0.f, 0.f, 0.f};
    v8i At[4], B0[2], B1[2];
    constexpr unsigned hstep = HTB, tstep = SLAB * NKS;
    const unsigned kown = (unsigned)(rank * 4) * SLAB;
    int tm = pm, tn = cn;
    unsigned cA = (unsigned)(16 + tn) * tstep, cB = (unsigned)tm * tstep;
    slow(4u);
    unsigned known = 4u, need = 4u;
    {
      const unsigned k0 = kown, k1 = kown + SLAB;
      STAGE(SB(0, 0), cB + k0); STAGE(SA(0, 0), cA + k0); STAGE(SB(0, 1), cB + k0 + hstep); STAGE(SA(0, 1), cA + k0 + hstep);
      if (wr == 1) BAR;
      if (wr == 0) WAIT_V(8);
      BAR;
      STAGE(SB(1, 0), cB + k1); STAGE(SA(1, 0), cA + k1); STAGE(SB(1, 1), cB + k1 + hstep);
      if (wr == 0) WAIT_V(12);
      BAR;
    }
    unsigned a1 = cA + kown + SLAB, a2 = a1 + SLAB, a3 = a2 + SLAB, b2 = cB + kown + 2 * SLAB, b3 = b2 + SLAB;
    for (int ui = 0;; ++ui) {
      const bool has_next = (unsigned)(ui * 32 + 32) < nvis;
      int nm = tm, nn = tn;
      if (has_next) tile_of(ui + 1, nm, nn);
      const unsigned nA = (unsigned)(16 + nn) * tstep, nB = (unsigned)nm * tstep;
      int kv = 0;
      if (wr == 0) { const int ln = fresh_lane(); kv = *(const LAS int*)(lds + LTAB + (ui * 32 + (ln < 32 ? ln : ln & 3)) * 4) * (int)SLAB; }
      for (int t = 0; t < NKS; t += 2) {
        if (known < need) { if (wr == 0) BAR; slow(need); if (wr == 1) BAR; known = need; }
        LDB(B0, 0, 0); SCHED; LDA(At, 0, 0); SCHED; STAGE(SA(1, 1), a1 + hstep);
        WAIT_L(8); BAR; WAIT_L(0); MMA(0, 0, At, B0); BAR; SCHED;
        LDB(B1, 0, 1); SCHED; STAGE(SB(0, 0), b2);
        BAR; WAIT_L(0); MMA(0, 1, At, B1); BAR;
        LDA(At, 0, 1); SCHED; STAGE(SA(0, 0), a2);
        BAR; WAIT_L(0); MMA(1, 0, At, B0); BAR; SCHED;
        STAGE(SB(0, 1), b2 + hstep); MEMSTEP;
        BAR; MMA(1, 1, At, B1); BAR;
        LDB(B0, 1, 0); SCHED; LDA(At, 1, 0); SCHED; STAGE(SA(0, 1), a2 + hstep);
        WAIT_L(8); BAR; WAIT_L(0); MMA(0, 0, At, B0); BAR; SCHED;
        LDB(B1, 1, 1); const unsigned kn = lctl[1]; SCHED; STAGE(SB(1, 0), b3);
        BAR; WAIT_L(0); MMA(0, 1, At, B1); BAR;
        LDA(At, 1, 1); SCHED; STAGE(SA(1, 0), a3);
        BAR; WAIT_L(0); MMA(1, 0, At, B0); BAR; SCHED;
        STAGE(SB(1, 1), b3 + hstep); MEMSTEP;
        BAR; SCHED;
        __builtin_amdgcn_s_setprio(1);
        MF1(1, 1, At, B1, 0); SCHED;
        unsigned o2 = KOFF(t + 4), o3 = KOFF(t + 5); known = __builtin_amdgcn_readfirstlane(kn);
        asm volatile("" : "+s"(o2), "+s"(o3), "+s"(known)); SCHED;
        MF1(1, 1, At, B1, 1); SCHED;
        unsigned bA = t >= NKS - 4 ? nA : cA, bB = t >= NKS - 4 ? nB : cB;
        asm volatile("" : "+s"(bA), "+s"(bB)); SCHED;
        MF1(1, 1, At, B1, 2); SCHED;
        a1 = a3; a2 = bA + o2; b2 = bB + o2; a3 = bA + o3; b3 = bB + o3;
        asm volatile("" : "+s"(a1), "+s"(a2), "+s"(b2), "+s"(a3), "+s"(b3)); SCHED;
        MF1(1, 1, At, B1, 3); SCHED;
        need = need + 2u < nvis ? need + 2u : nvis;
        asm volatile("" : "+s"(need)); SCHED;
        MF1(1, 1, At, B1, 4); MF1(1, 1, At, B1, 5); MF1(1, 1, At, B1, 6); MF1(1, 1, At, B1, 7);
        __builtin_amdgcn_s_setprio(0);
        BAR;
      }
      const int dead = __builtin_amdgcn_readfirstlane(lctl[2]);
      TS(2 + 2 * ui);
      auto epilogue = [&](auto SHT) __attribute__((always_inline)) {
        const int ln = fresh_lane(); const int fr = ln & 15, fq = ln >> 4;
        unsigned char* base = p.out[rank] + (size_t)((tm * 256 + wc * 32) * N_ + tn * 256 + wr * 64) * 4;
        const unsigned vo = (unsigned)(fr * N_ + 4 * fq) * 4u;
        const float qnan = __builtin_nanf("");
#pragma unroll
        for (int bj = 0; bj < 2; ++bj)
#pragma unroll
          for (int n = 0; n < 2; ++n) {
            unsigned char* rowp = base + (size_t)(bj * 128 + n * 16) * N_ * 4;
#pragma unroll
            for (int ai = 0; ai < 2; ++ai)
#pragma unroll
              for (int r4 = 0; r4 < 4; ++r4) {
                f32x4 v;
#pragma unroll
                for (int c = 0; c < 4; ++c) v[c] = dead ? qnan : fmaxf(acc[ai][bj][c][n][r4] * scale, 0.f);
                if (decltype(SHT)::value) asm volatile("global_store_dwordx4 %0, %1, %2 offset:%3 sc0 sc1\n\ts_nop 1" :: "v"(vo), "v"(v), "s"(rowp), "n"(ai * 512 + r4 * 64) : "memory");
                else if (OST == 1) __builtin_nontemporal_store(v, (f32x4*)(rowp + vo + ai * 512 + r4 * 64)); else *(f32x4*)(rowp + vo + ai * 512 + r4 * 64) = v;
              }
          }
      };
      if (tn >= SHC) {
        epilogue(std::true_type{});
        WAIT_V(0);
        unsigned o_ = 0;
        if (fresh_lane() == 0) o_ = __hip_atomic_fetch_add(lctl + 3, 1u, __ATOMIC_RELAXED, __HIP_MEMORY_SCOPE_WORKGROUP);
        if (__builtin_amdgcn_readfirstlane(o_) == 7u) store_flag1(myws, DONE_OFF + si * 64, call);
        if (wr == 0) WAIT_V(0);
      } else if (LASTWT && !has_next) epilogue(std::true_type{}); else epilogue(std::false_type{});
      TS(3 + 2 * ui);
      if (!has_next) break;
#pragma unroll
      for (int a = 0; a < 2; ++a)
#pragma unroll
        for (int c = 0; c < 2; ++c)
#pragma unroll
          for (int m = 0; m < 4; ++m)
#pragma unroll
            for (int n = 0; n < 2; ++n) acc[a][c][m][n] = (f32x4){0.f, 0.f, 0.f, 0.f};
      cA = nA; cB = nB; tm = nm; tn = nn;
    }
    if (wr == 0) BAR;
    WAIT_V(0);
    TS(14);
  };
  if (wid < 4) run(std::integral_constant<int, 0>{}); else run(std::integral_constant<int, 1>{});
}
extern "C" void kernel_launch(int rank, int nranks, ncclComm_t comm,
                              hipStream_t stream,
                              void* const* d_in, const int* in_sizes, int n_in,
                              void* d_out, int out_size,
                              const int* in_dtypes, int out_dtype,
                              void* const* d_out_all,
                              void* const* d_ws_all, size_t ws_bytes) {
  if (nranks != NR || (unsigned)rank >= (unsigned)NR || ws_bytes < WS_NEED || n_in < 4) {
    fprintf(stderr, "kernel_launch: unexpected configuration\n"); return;
  }
  static bool attr_done[64] = {};
  int dev = 0; HIP_CHECK(hipGetDevice(&dev));
  if (!attr_done[dev & 63]) {
    HIP_CHECK(hipFuncSetAttribute((const void*)fused_kernel, hipFuncAttributeMaxDynamicSharedMemorySize, LDS_BYTES));
    attr_done[dev & 63] = true;
  }
  Params p;
  p.x = (const unsigned char*)d_in[0]; p.w = (const unsigned char*)d_in[1];
  p.sx = (const float*)d_in[2]; p.sw = (const float*)d_in[3];
  for (int i = 0; i < NR; ++i) { p.ws[i] = (unsigned char*)d_ws_all[i]; p.out[i] = (unsigned char*)(i == rank ? d_out : d_out_all[i]); }
  p.rank = rank;
  fused_kernel<<<GRID, 512, LDS_BYTES, stream>>>(p);
}
```

```cpp
#include <hip/hip_runtime.h>
#include <rccl/rccl.h>
#include <cstdio>
#include <type_traits>

#define HIP_CHECK(expr) do { hipError_t _e = (expr); if (_e != hipSuccess) { \
    fprintf(stderr, "HIP error %s at %s:%d\n", hipGetErrorString(_e), __FILE__, __LINE__); \
    return; } } while (0)

typedef int v8i __attribute__((ext_vector_type(8)));
typedef int v4i __attribute__((ext_vector_type(4)));
typedef unsigned v4u __attribute__((ext_vector_type(4)));
typedef float f32x4 __attribute__((ext_vector_type(4)));
#define LAS __attribute__((address_space(3)))

constexpr int NR = 8;
constexpr int M_ = 4096, N_ = 16384, KL = 512;
constexpr int NKS = 32;
constexpr int HTB = 16384;
constexpr int STAGE_BYTES = 8 * HTB;
constexpr unsigned LCTL = STAGE_BYTES, LTAB = STAGE_BYTES + 256, LCP = STAGE_BYTES + 1024;
constexpr int LDS_BYTES = STAGE_BYTES + 2048;
constexpr unsigned SLAB = 32768;
constexpr unsigned IMG_BYTES = 2560u * SLAB;
constexpr unsigned IMG_OFF = 1u << 20;
constexpr size_t WS_NEED = (size_t)IMG_OFF + 2 * (size_t)IMG_BYTES;
constexpr unsigned CTR_OFF = 0, TMO_OFF = 128, DONE_OFF = 4096, OFLAG_OFF = 8192, FLAG_OFF = 65536, FLAG_BYTES = 2560u * 64u, DBG_OFF = 524288;
constexpr int GRID = 256;
constexpr int NPUSH = 16;
constexpr int NSH = 10, SHC = 59;
constexpr unsigned DEADV = 0x7fffffffu;
constexpr unsigned long long DEADLINE_TICKS = 90000000ull;
#define WPT 2

struct Params {
  const unsigned char* x;
  const unsigned char* w;
  const float* sx; const float* sw;
  unsigned char* ws[NR];
  unsigned char* out[NR];
  int rank;
};

#define TS(slot) do {} while (0)
#define TV(slot, v) do {} while (0)
__device__ __forceinline__ v8i cat8(v4i a, v4i b) { return __builtin_shufflevector(a, b, 0, 1, 2, 3, 4, 5, 6, 7); }
#define WAIT_V(n) asm volatile("s_waitcnt vmcnt(" #n ")" ::: "memory")
#define WAIT_L(n) asm volatile("s_waitcnt lgkmcnt(" #n ")" ::: "memory")
#define BAR __builtin_amdgcn_s_barrier()
#define SCHED __builtin_amdgcn_sched_barrier(0)
#define TIE16(R) "+v"(R[0]), "+v"(R[1]), "+v"(R[2]), "+v"(R[3]), "+v"(R[4]), "+v"(R[5]), "+v"(R[6]), "+v"(R[7]), \
                 "+v"(R[8]), "+v"(R[9]), "+v"(R[10]), "+v"(R[11]), "+v"(R[12]), "+v"(R[13]), "+v"(R[14]), "+v"(R[15])

__device__ __forceinline__ int fresh_lane() { int l; asm volatile("v_mbcnt_lo_u32_b32 %0, -1, 0\n\tv_mbcnt_hi_u32_b32 %0, -1, %0" : "=v"(l)); return l; }

__device__ __forceinline__ void load_job(const Params& p, int tile, int ks, int jw, int lane, unsigned (&R)[16]) {
  const int i = lane & 15, g = lane >> 4;
  const bool isx = tile < 16, iso = tile >= 80;
  const unsigned vo = isx ? i * KL + g * 32 : iso ? lane * 16 : g * 32 * N_ + 4 * i;
  const unsigned char* src = isx ? p.x + (size_t)(tile * 256 + (jw >> 1) * 64) * KL + ks * 128 + (jw & 1) * 16
                           : iso ? p.out[p.rank] + (unsigned)ks
                                 : p.w + (size_t)(ks * 128 + (jw & 1) * 16) * N_ + (tile - 16) * 256 + (jw >> 1) * 64;
  const unsigned s1 = isx || iso ? 4 : N_, s2 = isx ? 16 * KL : 4 * N_;
#pragma unroll
  for (int r = 0; r < 16; ++r) asm volatile("global_load_dword %0, %1, %2" : "=v"(R[r]) : "v"(vo), "s"(src + (r & 3) * s1 + (r >> 2) * s2) : "memory");
}
__device__ __forceinline__ void retile(bool plain, const unsigned (&R)[16], v4u (&D)[4]) {
  if (plain) {
#pragma unroll
    for (int e = 0; e < 4; ++e) D[e] = (v4u){R[4 * e], R[4 * e + 1], R[4 * e + 2], R[4 * e + 3]};
  } else {
#pragma unroll
    for (int d = 0; d < 4; ++d) {
      const unsigned a = R[4 * d], bb = R[4 * d + 1], c = R[4 * d + 2], f = R[4 * d + 3];
      const unsigned t0_ = __builtin_amdgcn_perm(bb, a, 0x05010400u), t1_ = __builtin_amdgcn_perm(bb, a, 0x07030602u);
      const unsigned t2_ = __builtin_amdgcn_perm(f, c, 0x05010400u), t3_ = __builtin_amdgcn_perm(f, c, 0x07030602u);
      D[0][d] = __builtin_amdgcn_perm(t2_, t0_, 0x05040100u); D[1][d] = __builtin_amdgcn_perm(t2_, t0_, 0x07060302u);
      D[2][d] = __builtin_amdgcn_perm(t3_, t1_, 0x05040100u); D[3][d] = __builtin_amdgcn_perm(t3_, t1_, 0x07060302u);
    }
  }
}
__device__ __forceinline__ void store_job(unsigned char* ws, unsigned vo, const v4u (&D)[4]) {
  asm volatile("global_store_dwordx4 %0, %1, %5 offset:-4096 sc0 sc1\n\tglobal_store_dwordx4 %0, %2, %5 offset:-2048 sc0 sc1\n\t"
               "global_store_dwordx4 %0, %3, %5 sc0 sc1\n\tglobal_store_dwordx4 %0, %4, %5 offset:2048 sc0 sc1\n\ts_nop 1"
               :: "v"(vo), "v"(D[0]), "v"(D[1]), "v"(D[2]), "v"(D[3]), "s"(ws) : "memory");
}
__device__ __forceinline__ void store_flag1(unsigned char* ws, unsigned fo, unsigned v) {
  asm volatile("s_mov_b64 exec, 1\n\tglobal_store_dword %0, %1, %2 sc0 sc1\n\ts_mov_b64 exec, -1" :: "v"(fo), "v"(v), "s"(ws) : "memory");
}
__device__ __forceinline__ void store_flag7(unsigned long long fa, unsigned v) {
  asm volatile("s_mov_b64 exec, 0x7f\n\tglobal_store_dword %0, %1, off sc0 sc1\n\ts_mov_b64 exec, -1" :: "v"(fa), "v"(v) : "memory");
}
__device__ __forceinline__ unsigned job_off(int jw, int lane) { return (jw >> 1) * 8192 + (jw & 1) * 1024 + lane * 16; }

__device__ __forceinline__ void stream_slab(int j, int snd, int& tile, int& ks) {
  if (j < 124) { ks = j / 31; int q = j - ks * 31 + 4 * snd; if (q >= 31) q -= 31; tile = (q & 1) * 16 + (q >> 1); }
  else if (j < 200) { const int u = j - 124; ks = u / 19; int q = u - ks * 19 + 2 * snd; if (q >= 19) q -= 19; tile = q < 5 ? 75 + q : 26 + q; }
  else { const int u = j - 200, gq = u >= 60, v = u - gq * 60; ks = v / 15; int q = v - ks * 15 + 2 * snd; if (q >= 15) q -= 15; tile = 45 + 15 * gq + q; }
}
__device__ __forceinline__ int stream_idx(int tile, int ks, int snd) {
  if (tile < 31) { int q = (tile < 16 ? 2 * tile : 2 * tile - 31) - 4 * snd; if (q < 0) q += 31; return ks * 31 + q; }
  if (tile < 45 || tile >= 75) { int q = (tile < 45 ? tile - 26 : tile - 75) - 2 * snd; if (q < 0) q += 19; return 124 + ks * 19 + q; }
  const int gq = tile >= 60; int q = tile - 45 - 15 * gq - 2 * snd; if (q < 0) q += 15; return 200 + gq * 60 + ks * 15 + q;
}
__device__ __forceinline__ unsigned shared_off(int r, int i) { const int idx = NSH * r + i; return (unsigned)((idx & 15) * 256 * N_ + (SHC + (idx >> 4)) * 256) * 4u; }

#define INITK 0u
#define LAG 1
#define PWIN 4u
#define PTH 8u
#define INV 0
#define LASTWT 0
#define OST 0
#define PSLEEP 8
__device__ __forceinline__ bool wait_flags(unsigned char* ws, unsigned fo, unsigned call) {
  const unsigned long long t0 = __builtin_amdgcn_s_memrealtime();
  unsigned* tmo = (unsigned*)(ws + TMO_OFF);
  for (;;) {
    unsigned v, tm; const unsigned to = TMO_OFF;
    asm volatile("global_load_dword %0, %2, %4 sc0 sc1\n\tglobal_load_dword %1, %3, %4 sc0 sc1\n\ts_waitcnt vmcnt(0)" : "=&v"(v), "=&v"(tm) : "v"(fo), "v"(to), "s"(ws) : "memory");
    if (__ballot(v != call) == 0ull) return true;
    if (__builtin_amdgcn_readfirstlane(tm) == call || __builtin_amdgcn_s_memrealtime() - t0 > DEADLINE_TICKS) {
      __hip_atomic_store(tmo, call, __ATOMIC_RELAXED, __HIP_MEMORY_SCOPE_AGENT);
      return false;
    }
    __builtin_amdgcn_s_sleep(16);
  }
}
constexpr int NTURN = 8 / WPT;
__device__ __forceinline__ void push_elem(int it, int e, int jw, int rank, int& tile, int& ks) {
  if (it < 20) stream_slab(16 * it + e, rank, tile, ks);
  else { const int o = 16 * (it - 20) + e; tile = 80; ks = (int)(shared_off(rank, o >> 3) + (unsigned)((o & 7) * 32 + jw * 4) * (N_ * 4u)); }
}
__device__ __forceinline__ void pusher(const Params& p, unsigned imgo, unsigned flo, unsigned call, int b, int wid, int lane, unsigned (&R)[16]) {
  unsigned char* const myws = p.ws[p.rank];
  const int tau = wid / WPT, e = tau * 2 * WPT + b / NTURN, jw = (b % NTURN) * WPT + wid % WPT;
  unsigned long long fa = 0; unsigned pm_ = 0;
  unsigned char* wsd[7]; unsigned char* outd[7]; int pr[7];
#pragma unroll
  for (int k = 0; k < 7; ++k) {
    pr[k] = (p.rank + 1 + (wid + b + k) % 7) & 7; wsd[k] = p.ws[pr[k]]; outd[k] = p.out[pr[k]];
    if (lane == k) fa = (unsigned long long)wsd[k];
  }
  const unsigned jo = imgo + job_off(jw, lane), dno = DONE_OFF + (lane < NSH ? lane : 0) * 64u;
  bool ok = true;
  int tile, ks; push_elem(0, e, jw, p.rank, tile, ks);
  v4u D[4];
  retile(tile < 16, R, D);
  for (int i = 0; i < tau; ++i) BAR;
  for (int it = 0; it < 25; ++it) {
    const bool iso = it >= 20;
    const unsigned sid = tile * 32 + p.rank * 4 + ks;
    const unsigned vo = iso ? (unsigned)ks + lane * 16 : jo + sid * SLAB, ps = iso ? N_ * 4u : 2048u;
    pm_ = tile >= 75 && !iso ? (unsigned)(0xC070180E03ull >> ((tile - 75) * 8)) & 255u : 255u;
    unsigned char* dk[7];
#pragma unroll
    for (int k = 0; k < 7; ++k) dk[k] = iso ? outd[k] : (pm_ >> pr[k] & 1u) ? wsd[k] : myws;
#pragma unroll
    for (int c = 0; c < 4; ++c) {
      const unsigned vc = vo + c * ps;
      asm volatile("global_store_dwordx4 %0, %1, %2 sc0 sc1\n\tglobal_store_dwordx4 %0, %1, %3 sc0 sc1\n\tglobal_store_dwordx4 %0, %1, %4 sc0 sc1\n\t"
                   "global_store_dwordx4 %0, %1, %5 sc0 sc1\n\tglobal_store_dwordx4 %0, %1, %6 sc0 sc1\n\tglobal_store_dwordx4 %0, %1, %7 sc0 sc1\n\t"
                   "global_store_dwordx4 %0, %1, %8 sc0 sc1\n\ts_nop 1"
                   :: "v"(vc), "v"(D[c]), "s"(dk[0]), "s"(dk[1]), "s"(dk[2]), "s"(dk[3]), "s"(dk[4]), "s"(dk[5]), "s"(dk[6]) : "memory");
    }
    for (int i = 0; i < LAG; ++i) BAR;
    WAIT_V(0);
    push_elem(it < 24 ? it + 1 : it, e, jw, p.rank, tile, ks);
    load_job(p, tile, ks, jw, lane, R);
    unsigned dv; asm volatile("global_load_dword %0, %1, %2 sc0 sc1" : "=v"(dv) : "v"(dno), "s"(myws) : "memory");
    store_flag7(!iso ? fa + flo + sid * 64u + jw * 4u : it == 24 ? fa + OFLAG_OFF + (p.rank * 16 + b) * 64u + wid * 4u
                                                                  : (unsigned long long)myws + OFLAG_OFF + (p.rank * 16 + b) * 64u + wid * 4u, call);
    for (int i = LAG + 1; i < NTURN; ++i) BAR;
    asm volatile("s_waitcnt vmcnt(1)" : TIE16(R), "+v"(dv) :: "memory");
    retile(tile < 16 || tile >= 80, R, D);
    if (!ok && tile >= 80) {
#pragma unroll
      for (int c = 0; c < 4; ++c) D[c] = (v4u){0x7fc00000u, 0x7fc00000u, 0x7fc00000u, 0x7fc00000u};
    }
    if (it == 18) {
      if (__ballot(dv != call)) ok = wait_flags(myws, dno, call);
    }
    BAR;
  }
  for (int i = tau + 1; i < NTURN; ++i) BAR;
  TS(1);
  if (wid == 0) {
    const int l = lane < 56 ? lane : 0, peer = (p.rank + 1 + (l >> 3)) & 7;
    if (!wait_flags(myws, OFLAG_OFF + (peer * 16 + b) * 64u + (l & 7) * 4u, call))
      *(float*)(p.out[p.rank] + shared_off(peer, 0) + (b * 8 + (l & 7)) * 4u) = __builtin_nanf("");
  }
  WAIT_V(0);
  TS(14);
}

__global__ __launch_bounds__(512, 2) void fused_kernel(const Params p) {
  extern __shared__ __attribute__((aligned(16))) unsigned char smem[];
  LAS unsigned char* lds = (LAS unsigned char*)smem;
  const int tid = threadIdx.x, wid = __builtin_amdgcn_readfirstlane(tid >> 6), wc = wid & 3;
  unsigned char* myws = p.ws[p.rank];
  const int b = blockIdx.x, rank = p.rank, g = b - NPUSH;
  const bool push = b < NPUSH;
  LAS unsigned* lctl = (LAS unsigned*)(lds + LCTL);
  const unsigned lbase = (unsigned)(size_t)lds;
  const int xc = b & 7, L = (xc >> 2) * 30 + (g >> 3), cn = L >> 2;
  const int pm = (xc & 3) * 4 + (L & 3), si = (L & 3) * 4 + (xc & 3);
  const bool skip = cn == 0 && si >= NSH;
  const unsigned nvis = skip ? 96u : 128u;
  auto tile_of = [&](int u, int& tm, int& tn) __attribute__((always_inline)) {
    const int v = u + (skip && u > 0), idx = NSH * rank + si;
    const bool sh = v == 1 && cn == 0;
    tm = sh ? idx & 15 : pm; tn = sh ? SHC + (idx >> 4) : v == 0 ? cn : 15 * v - 1 + cn;
  };
  unsigned call, imgo, flo;
  {
    const int lane = tid & 63;
    unsigned R[16], R2[16];
    int t1, k1, j1 = wid;
    if (push) { j1 = (b % NTURN) * WPT + wid % WPT; push_elem(0, wid / WPT * 2 * WPT + b / NTURN, j1, rank, t1, k1); } else { t1 = g >> 2; k1 = g & 3; }
    load_job(p, t1, k1, j1, lane, R);
    const bool two = !push && g < 80;
    if (two) load_job(p, t1 + 60, k1, wid, lane, R2);
    if (tid == 0) {
      const unsigned t = __hip_atomic_fetch_add((unsigned*)(myws + CTR_OFF), 1u, __ATOMIC_RELAXED, __HIP_MEMORY_SCOPE_AGENT);
      lctl[0] = t / (unsigned)GRID + 1u; lctl[1] = INITK; lctl[2] = 0u; lctl[3] = 0u;
    }
    if (wid >= 4 && !push) {
      const int u = wid - 4, l = lane & 31, src = l >> 2, ks = l & 3;
      int tm, tn; tile_of(u, tm, tn);
      const int ax = stream_idx(tm, ks, src), aw = stream_idx(16 + tn, ks, src);
#define CORD 9
      const int key = src == rank ? ks : u >= CORD ? 32 + ks * 8 + ((src - rank) & 7) : ((ax > aw ? ax : aw) + 1) * 32 + l;
      int rk = 0;
#pragma unroll
      for (int s = 0; s < 32; ++s) rk += __builtin_amdgcn_readlane(key, s) < key;
      *(LAS int*)(lds + LTAB + (u * 32 + rk) * 4) = l;
      lctl[8 + 2 * u] = tm; lctl[9 + 2 * u] = 16 + tn;
    }
    asm volatile("s_waitcnt vmcnt(0)" : TIE16(R) :: "memory");
    asm volatile("" : TIE16(R2) :: "memory");
    __syncthreads();
    call = __builtin_amdgcn_readfirstlane(lctl[0]);
    imgo = IMG_OFF + (call & 1u) * IMG_BYTES;
    flo = FLAG_OFF + (call & 1u) * FLAG_BYTES;
    TS(0);
    if (push) {
      pusher(p, imgo, flo, call, b, wid, lane, R);
      return;
    }
    v4u D[4];
    const unsigned s1 = t1 * 32 + rank * 4 + k1;
    retile(t1 < 16, R, D);
    store_job(myws, imgo + s1 * SLAB + job_off(wid, lane) + 4096, D);
    if (two) {
      retile(false, R2, D);
      store_job(myws, imgo + (s1 + 60 * 32) * SLAB + job_off(wid, lane) + 4096, D);
    }
    WAIT_V(0);
    store_flag1(myws, flo + s1 * 64 + wid * 4, call);
    if (two) store_flag1(myws, flo + (s1 + 60 * 32) * 64 + wid * 4, call);
    TS(1);
  }
  const float scale = __builtin_bit_cast(float, __builtin_amdgcn_readfirstlane(__builtin_bit_cast(int, p.sx[0] * p.sw[0])));
  WAIT_V(0);

  auto run = [&](auto WRC) __attribute__((always_inline)) {
    constexpr int wr = decltype(WRC)::value;
    unsigned ppend = 0, pk = INITK, pw = 4u;
    auto peval = [&]() __attribute__((always_inline)) {
      const v4u f = *(const LAS v4u*)(lds + LCP + (unsigned)fresh_lane() * 16);
      unsigned long long bm = __ballot(f[0] == call && f[1] == call && f[2] == call && f[3] == call);
      bm &= bm >> 1; bm &= bm >> 2;
      bm = ~bm & 0x1111111111111111ull;
      unsigned c = bm ? (unsigned)__builtin_ctzll(bm) >> 2 : 16u;
      c = c < pw ? c : pw;
      pw = c == pw ? 16u : PWIN;
      const unsigned n = pk + c < nvis ? pk + c : nvis;
      if (n > pk) {
        pk = n; lctl[1] = n;
      }
      ppend = 0;
    };
    auto pissue = [&]() __attribute__((always_inline)) {
      const unsigned ln = (unsigned)fresh_lane();
      unsigned e = pk + ((ln >> 2) < pw ? ln >> 2 : 0u); e = e < nvis ? e : nvis - 1;
      const unsigned kk = *(const LAS unsigned*)(lds + LTAB + e * 4), tl = lctl[8 + (e >> 5) * 2 + (ln >> 1 & 1)];
      const unsigned la = lbase + LCP, fo = flo + (tl * 32 + kk) * 64 + (ln & 1) * 16; unsigned char* const w_ = myws;
      asm volatile("s_mov_b32 m0, %0\n\ts_nop 0\n\tglobal_load_lds_dwordx4 %1, %2 sc0 sc1" :: "s"(la), "v"(fo), "s"(w_) : "memory");
      ppend = 1;
    };
    auto poll = [&](bool blocking, unsigned need) __attribute__((always_inline)) {
      if (!blocking) {
        if (ppend) {
          unsigned st; asm volatile("s_getreg_b32 %0, hwreg(HW_REG_IB_STS)" : "=s"(st) :: "memory");
          st = __builtin_amdgcn_readfirstlane(st);
          if ((st & 15u) | ((st >> 18) & 48u)) return;
          WAIT_V(0);
          peval();
        } else { pw = 16u; pissue(); }
        return;
      }
      WAIT_V(0);
      if (ppend) { peval(); if (pk < need) __builtin_amdgcn_s_sleep(PSLEEP); }
      if (pk < nvis) pissue();
      WAIT_L(0);
    };
    auto slow = [&](unsigned need) __attribute__((always_inline)) {
      const unsigned long long t0 = __builtin_amdgcn_s_memrealtime();
      for (;;) {
        if (wr == 1 && wc == 0) poll(true, need); else __builtin_amdgcn_s_sleep(1);
        const unsigned long long el = __builtin_amdgcn_s_memrealtime() - t0;
        if (wr == 1 && wc == 0 && el > 100000ull) {
          unsigned* tmo = (unsigned*)(myws + TMO_OFF);
          if (__builtin_amdgcn_readfirstlane(__hip_atomic_load(tmo, __ATOMIC_RELAXED, __HIP_MEMORY_SCOPE_AGENT)) == call || el > DEADLINE_TICKS) {
            __hip_atomic_store(tmo, call, __ATOMIC_RELAXED, __HIP_MEMORY_SCOPE_AGENT);
            lctl[2] = 1u; lctl[1] = DEADV; pk = nvis; ppend = 0;
          }
          WAIT_V(0);
        }
        if (__builtin_amdgcn_readfirstlane(__hip_atomic_load(lctl + 1, __ATOMIC_RELAXED, __HIP_MEMORY_SCOPE_WORKGROUP)) >= need || el > 2 * DEADLINE_TICKS) break;
      }
      asm volatile("" ::: "memory");
    };

    const unsigned char* XI = myws + imgo;
    const unsigned voff = (unsigned)(wc * 4096 + (tid & 63) * 16);
    const unsigned ldsw = (unsigned)wc * 4096u;
    const int aoff = wr * 8192 + (tid & 63) * 16, boff = wc * 4096 + (tid & 63) * 16;
#define SA(b_, h) (((b_) * 2 + (h)) * HTB)
#define SB(b_, h) ((4 + (b_) * 2 + (h)) * HTB)
#define STG1(bufoff, va, o) __builtin_amdgcn_global_load_lds((const unsigned*)(XI + (va)), (LAS unsigned*)(lds + (bufoff) + ldsw), 16, o, 0)
#define STAGE(bufoff, goff) do { if (wr == 0) { const unsigned va_ = voff + (goff); STG1(bufoff, va_, 0); STG1(bufoff, va_, 1024); STG1(bufoff, va_, 2048); STG1(bufoff, va_, 3072); } } while (0)
#define LDA(dst, b_, h) do { _Pragma("unroll") for (int m = 0; m < 4; ++m) { \
    v4i lo = *(const LAS v4i*)(lds + SA(b_, h) + aoff + m * 2048); v4i hi = *(const LAS v4i*)(lds + SA(b_, h) + aoff + m * 2048 + 1024); dst[m] = cat8(lo, hi); } } while (0)
#define LDB(dst, b_, h) do { _Pragma("unroll") for (int n = 0; n < 2; ++n) { \
    v4i lo = *(const LAS v4i*)(lds + SB(b_, h) + boff + n * 2048); v4i hi = *(const LAS v4i*)(lds + SB(b_, h) + boff + n * 2048 + 1024); dst[n] = cat8(lo, hi); } } while (0)
#define MMA(ai, bj, At, Bt) do { __builtin_amdgcn_s_setprio(1); _Pragma("unroll") for (int m = 0; m < 4; ++m) _Pragma("unroll") for (int n = 0; n < 2; ++n) \
    acc[ai][bj][m][n] = __builtin_amdgcn_mfma_scale_f32_16x16x128_f8f6f4(Bt[n], At[m], acc[ai][bj][m][n], 0, 1, 0, 0, 0, 0); __builtin_amdgcn_s_setprio(0); } while (0)
#define MF1(ai, bj, At, Bt, q) acc[ai][bj][(q) >> 1][(q) & 1] = __builtin_amdgcn_mfma_scale_f32_16x16x128_f8f6f4(Bt[(q) & 1], At[(q) >> 1], acc[ai][bj][(q) >> 1][(q) & 1], 0, 1, 0, 0, 0, 0)
#define WV12 WAIT_V(12)
#define MEMSTEP do { if (wr == 0) WV12; else if (wc == 0 && (ppend || (pk < nvis && pk < need + PTH))) poll(false, 0u); } while (0)
#define KOFF(i) ((unsigned)__builtin_amdgcn_readlane(kv, (i)))
    f32x4 acc[2][2][4][2];
#pragma unroll
    for (int a = 0; a < 2; ++a)
#pragma unroll
      for (int c = 0; c < 2; ++c)
#pragma unroll
        for (int m = 0; m < 4; ++m)
#pragma unroll
          for (int n = 0; n < 2; ++n) acc[a][c][m][n] = (f32x4){0.f, 0.f, 0.f, 0.f};
    v8i At[4], B0[2], B1[2];
    constexpr unsigned hstep = HTB, tstep = SLAB * NKS;
    const unsigned kown = (unsigned)(rank * 4) * SLAB;
    int tm = pm, tn = cn;
    unsigned cA = (unsigned)(16 + tn) * tstep, cB = (unsigned)tm * tstep;
    slow(4u);
    unsigned known = 4u, need = 4u;
    {
      const unsigned k0 = kown, k1 = kown + SLAB;
      STAGE(SB(0, 0), cB + k0); STAGE(SA(0, 0), cA + k0); STAGE(SB(0, 1), cB + k0 + hstep); STAGE(SA(0, 1), cA + k0 + hstep);
      if (wr == 1) BAR;
      if (wr == 0) WAIT_V(8);
      BAR;
      STAGE(SB(1, 0), cB + k1); STAGE(SA(1, 0), cA + k1); STAGE(SB(1, 1), cB + k1 + hstep);
      if (wr == 0) WAIT_V(12);
      BAR;
    }
    unsigned a1 = cA + kown + SLAB, a2 = a1 + SLAB, a3 = a2 + SLAB, b2 = cB + kown + 2 * SLAB, b3 = b2 + SLAB;
    for (int ui = 0;; ++ui) {
      const bool has_next = (unsigned)(ui * 32 + 32) < nvis;
      int nm = tm, nn = tn;
      if (has_next) tile_of(ui + 1, nm, nn);
      const unsigned nA = (unsigned)(16 + nn) * tstep, nB = (unsigned)nm * tstep;
      int kv = 0;
      if (wr == 0) { const int ln = fresh_lane(); kv = *(const LAS int*)(lds + LTAB + (ui * 32 + (ln < 32 ? ln : ln & 3)) * 4) * (int)SLAB; }
      for (int t = 0; t < NKS; t += 2) {
        if (known < need) { if (wr == 0) BAR; slow(need); if (wr == 1) BAR; known = need; }
        LDB(B0, 0, 0); SCHED; LDA(At, 0, 0); SCHED; STAGE(SA(1, 1), a1 + hstep);
        WAIT_L(8); BAR; WAIT_L(0); MMA(0, 0, At, B0); BAR; SCHED;
        LDB(B1, 0, 1); SCHED; STAGE(SB(0, 0), b2);
        BAR; WAIT_L(0); MMA(0, 1, At, B1); BAR;
        LDA(At, 0, 1); SCHED; STAGE(SA(0, 0), a2);
        BAR; WAIT_L(0); MMA(1, 0, At, B0); BAR; SCHED;
        STAGE(SB(0, 1), b2 + hstep); MEMSTEP;
        BAR; MMA(1, 1, At, B1); BAR;
        LDB(B0, 1, 0); SCHED; LDA(At, 1, 0); SCHED; STAGE(SA(0, 1), a2 + hstep);
        WAIT_L(8); BAR; WAIT_L(0); MMA(0, 0, At, B0); BAR; SCHED;
        LDB(B1, 1, 1); const unsigned kn = lctl[1]; SCHED; STAGE(SB(1, 0), b3);
        BAR; WAIT_L(0); MMA(0, 1, At, B1); BAR;
        LDA(At, 1, 1); SCHED; STAGE(SA(1, 0), a3);
        BAR; WAIT_L(0); MMA(1, 0, At, B0); BAR; SCHED;
        STAGE(SB(1, 1), b3 + hstep); MEMSTEP;
        BAR; SCHED;
        __builtin_amdgcn_s_setprio(1);
        MF1(1, 1, At, B1, 0); SCHED;
        unsigned o2 = KOFF(t + 4), o3 = KOFF(t + 5); known = __builtin_amdgcn_readfirstlane(kn);
        asm volatile("" : "+s"(o2), "+s"(o3), "+s"(known)); SCHED;
        MF1(1, 1, At, B1, 1); SCHED;
        unsigned bA = t >= NKS - 4 ? nA : cA, bB = t >= NKS - 4 ? nB : cB;
        asm volatile("" : "+s"(bA), "+s"(bB)); SCHED;
        MF1(1, 1, At, B1, 2); SCHED;
        a1 = a3; a2 = bA + o2; b2 = bB + o2; a3 = bA + o3; b3 = bB + o3;
        asm volatile("" : "+s"(a1), "+s"(a2), "+s"(b2), "+s"(a3), "+s"(b3)); SCHED;
        MF1(1, 1, At, B1, 3); SCHED;
        need = need + 2u < nvis ? need + 2u : nvis;
        asm volatile("" : "+s"(need)); SCHED;
        MF1(1, 1, At, B1, 4); MF1(1, 1, At, B1, 5); MF1(1, 1, At, B1, 6); MF1(1, 1, At, B1, 7);
        __builtin_amdgcn_s_setprio(0);
        BAR;
      }
      const int dead = __builtin_amdgcn_readfirstlane(lctl[2]);
      TS(2 + 2 * ui);
      auto epilogue = [&](auto SHT) __attribute__((always_inline)) {
        const int ln = fresh_lane(); const int fr = ln & 15, fq = ln >> 4;
        unsigned char* base = p.out[rank] + (size_t)((tm * 256 + wc * 32) * N_ + tn * 256 + wr * 64) * 4;
        const unsigned vo = (unsigned)(4 * fq * N_ + 4 * fr) * 4u;
        const float qnan = __builtin_nanf("");
#pragma unroll
        for (int bj = 0; bj < 2; ++bj)
#pragma unroll
          for (int n = 0; n < 2; ++n)
#pragma unroll
            for (int r4 = 0; r4 < 4; ++r4) {
              unsigned char* rowp = base + (size_t)(bj * 128 + n * 16 + r4) * N_ * 4;
#pragma unroll
              for (int ai = 0; ai < 2; ++ai) {
                f32x4 v;
#pragma unroll
                for (int c = 0; c < 4; ++c) v[c] = dead ? qnan : fmaxf(acc[ai][bj][c][n][r4] * scale, 0.f);
                if (decltype(SHT)::value) asm volatile("global_store_dwordx4 %0, %1, %2 offset:%3 sc0 sc1\n\ts_nop 1" :: "v"(vo), "v"(v), "s"(rowp), "n"(ai * 512) : "memory");
                else if (OST == 1) __builtin_nontemporal_store(v, (f32x4*)(rowp + vo + ai * 512)); else *(f32x4*)(rowp + vo + ai * 512) = v;
              }
            }
      };
      if (tn >= SHC) {
        epilogue(std::true_type{});
        WAIT_V(0);
        unsigned o_ = 0;
        if (fresh_lane() == 0) o_ = __hip_atomic_fetch_add(lctl + 3, 1u, __ATOMIC_RELAXED, __HIP_MEMORY_SCOPE_WORKGROUP);
        if (__builtin_amdgcn_readfirstlane(o_) == 7u) store_flag1(myws, DONE_OFF + si * 64, call);
        if (wr == 0) WAIT_V(0);
      } else if (LASTWT && !has_next) epilogue(std::true_type{}); else epilogue(std::false_type{});
      TS(3 + 2 * ui);
      if (!has_next) break;
#pragma unroll
      for (int a = 0; a < 2; ++a)
#pragma unroll
        for (int c = 0; c < 2; ++c)
#pragma unroll
          for (int m = 0; m < 4; ++m)
#pragma unroll
            for (int n = 0; n < 2; ++n) acc[a][c][m][n] = (f32x4){0.f, 0.f, 0.f, 0.f};
      cA = nA; cB = nB; tm = nm; tn = nn;
    }
    if (wr == 0) BAR;
    WAIT_V(0);
    TS(14);
  };
  if (wid < 4) run(std::integral_constant<int, 0>{}); else run(std::integral_constant<int, 1>{});
}
extern "C" void kernel_launch(int rank, int nranks, ncclComm_t comm,
                              hipStream_t stream,
                              void* const* d_in, const int* in_sizes, int n_in,
                              void* d_out, int out_size,
                              const int* in_dtypes, int out_dtype,
                              void* const* d_out_all,
                              void* const* d_ws_all, size_t ws_bytes) {
  if (nranks != NR || (unsigned)rank >= (unsigned)NR || ws_bytes < WS_NEED || n_in < 4) {
    fprintf(stderr, "kernel_launch: unexpected configuration\n"); return;
  }
  static bool attr_done[64] = {};
  int dev = 0; HIP_CHECK(hipGetDevice(&dev));
  if (!attr_done[dev & 63]) {
    HIP_CHECK(hipFuncSetAttribute((const void*)fused_kernel, hipFuncAttributeMaxDynamicSharedMemorySize, LDS_BYTES));
    attr_done[dev & 63] = true;
  }
  Params p;
  p.x = (const unsigned char*)d_in[0]; p.w = (const unsigned char*)d_in[1];
  p.sx = (const float*)d_in[2]; p.sw = (const float*)d_in[3];
  for (int i = 0; i < NR; ++i) { p.ws[i] = (unsigned char*)d_ws_all[i]; p.out[i] = (unsigned char*)(i == rank ? d_out : d_out_all[i]); }
  p.rank = rank;
  fused_kernel<<<GRID, 512, LDS_BYTES, stream>>>(p);
}
```

```cpp
#include <hip/hip_runtime.h>
#include <rccl/rccl.h>
#include <cstdio>
#include <type_traits>

#define HIP_CHECK(expr) do { hipError_t _e = (expr); if (_e != hipSuccess) { \
    fprintf(stderr, "HIP error %s at %s:%d\n", hipGetErrorString(_e), __FILE__, __LINE__); \
    return; } } while (0)

typedef int v8i __attribute__((ext_vector_type(8)));
typedef int v4i __attribute__((ext_vector_type(4)));
typedef unsigned v4u __attribute__((ext_vector_type(4)));
typedef float f32x4 __attribute__((ext_vector_type(4)));
#define LAS __attribute__((address_space(3)))

constexpr int NR = 8;
constexpr int M_ = 4096, N_ = 16384, KL = 512;
constexpr int NKS = 32;
constexpr int HTB = 16384;
constexpr int STAGE_BYTES = 8 * HTB;
constexpr unsigned LCTL = STAGE_BYTES, LTAB = STAGE_BYTES + 256, LCP = STAGE_BYTES + 1024;
constexpr int LDS_BYTES = STAGE_BYTES + 2048;
constexpr unsigned SLAB = 32768;
constexpr unsigned IMG_BYTES = 2560u * SLAB;
constexpr unsigned IMG_OFF = 1u << 20;
constexpr unsigned PK_OFF = IMG_OFF + 2 * IMG_BYTES, PK_SLOT = 5120, PK_JOBS = 640;
constexpr size_t WS_NEED = (size_t)PK_OFF + (size_t)NR * PK_JOBS * PK_SLOT;
constexpr unsigned CTR_OFF = 0, TMO_OFF = 128, WCT_OFF = 256, DONE_OFF = 4096, OFLAG_OFF = 8192, FLAG_OFF = 65536, FLAG_BYTES = 2560u * 64u, DBG_OFF = 524288;
constexpr int GRID = 256;
constexpr int NPUSH = 16;
constexpr int NSH = 10, SHC = 59;
constexpr unsigned DEADV = 0x7fffffffu;
constexpr unsigned long long DEADLINE_TICKS = 90000000ull;
#define WPT 2

struct Params {
  const unsigned char* x;
  const unsigned char* w;
  const float* sx; const float* sw;
  unsigned char* ws[NR];
  unsigned char* out[NR];
  int rank;
};

#define TS(slot) do {} while (0)
#define TV(slot, v) do {} while (0)
__device__ __forceinline__ v8i cat8(v4i a, v4i b) { return __builtin_shufflevector(a, b, 0, 1, 2, 3, 4, 5, 6, 7); }
#define WAIT_V(n) asm volatile("s_waitcnt vmcnt(" #n ")" ::: "memory")
#define WAIT_L(n) asm volatile("s_waitcnt lgkmcnt(" #n ")" ::: "memory")
#define BAR __builtin_amdgcn_s_barrier()
#define SCHED __builtin_amdgcn_sched_barrier(0)
#define TIE16(R) "+v"(R[0]), "+v"(R[1]), "+v"(R[2]), "+v"(R[3]), "+v"(R[4]), "+v"(R[5]), "+v"(R[6]), "+v"(R[7]), \
                 "+v"(R[8]), "+v"(R[9]), "+v"(R[10]), "+v"(R[11]), "+v"(R[12]), "+v"(R[13]), "+v"(R[14]), "+v"(R[15])

__device__ __forceinline__ int fresh_lane() { int l; asm volatile("v_mbcnt_lo_u32_b32 %0, -1, 0\n\tv_mbcnt_hi_u32_b32 %0, -1, %0" : "=v"(l)); return l; }

__device__ __forceinline__ void load_job(const Params& p, int tile, int ks, int jw, int lane, unsigned (&R)[16]) {
  const int i = lane & 15, g = lane >> 4;
  const bool isx = tile < 16, iso = tile >= 80;
  const unsigned vo = isx ? i * KL + g * 32 : iso ? lane * 16 : g * 32 * N_ + 4 * i;
  const unsigned char* src = isx ? p.x + (size_t)(tile * 256 + (jw >> 1) * 64) * KL + ks * 128 + (jw & 1) * 16
                           : iso ? p.out[p.rank] + (unsigned)ks
                                 : p.w + (size_t)(ks * 128 + (jw & 1) * 16) * N_ + (tile - 16) * 256 + (jw >> 1) * 64;
  const unsigned s1 = isx || iso ? 4 : N_, s2 = isx ? 16 * KL : 4 * N_;
#pragma unroll
  for (int r = 0; r < 16; ++r) asm volatile("global_load_dword %0, %1, %2" : "=v"(R[r]) : "v"(vo), "s"(src + (r & 3) * s1 + (r >> 2) * s2) : "memory");
}
__device__ __forceinline__ void retile(bool plain, const unsigned (&R)[16], v4u (&D)[4]) {
  if (plain) {
#pragma unroll
    for (int e = 0; e < 4; ++e) D[e] = (v4u){R[4 * e], R[4 * e + 1], R[4 * e + 2], R[4 * e + 3]};
  } else {
#pragma unroll
    for (int d = 0; d < 4; ++d) {
      const unsigned a = R[4 * d], bb = R[4 * d + 1], c = R[4 * d + 2], f = R[4 * d + 3];
      const unsigned t0_ = __builtin_amdgcn_perm(bb, a, 0x05010400u), t1_ = __builtin_amdgcn_perm(bb, a, 0x07030602u);
      const unsigned t2_ = __builtin_amdgcn_perm(f, c, 0x05010400u), t3_ = __builtin_amdgcn_perm(f, c, 0x07030602u);
      D[0][d] = __builtin_amdgcn_perm(t2_, t0_, 0x05040100u); D[1][d] = __builtin_amdgcn_perm(t2_, t0_, 0x07060302u);
      D[2][d] = __builtin_amdgcn_perm(t3_, t1_, 0x05040100u); D[3][d] = __builtin_amdgcn_perm(t3_, t1_, 0x07060302u);
    }
  }
}
__device__ __forceinline__ void store_job(unsigned char* ws, unsigned vo, const v4u (&D)[4]) {
  asm volatile("global_store_dwordx4 %0, %1, %5 offset:-4096 sc0 sc1\n\tglobal_store_dwordx4 %0, %2, %5 offset:-2048 sc0 sc1\n\t"
               "global_store_dwordx4 %0, %3, %5 sc0 sc1\n\tglobal_store_dwordx4 %0, %4, %5 offset:2048 sc0 sc1\n\ts_nop 1"
               :: "v"(vo), "v"(D[0]), "v"(D[1]), "v"(D[2]), "v"(D[3]), "s"(ws) : "memory");
}
__device__ __forceinline__ void store_flag1(unsigned char* ws, unsigned fo, unsigned v) {
  asm volatile("s_mov_b64 exec, 1\n\tglobal_store_dword %0, %1, %2 sc0 sc1\n\ts_mov_b64 exec, -1" :: "v"(fo), "v"(v), "s"(ws) : "memory");
}
__device__ __forceinline__ void store_flag7(unsigned long long fa, unsigned v) {
  asm volatile("s_mov_b64 exec, 0x7f\n\tglobal_store_dword %0, %1, off sc0 sc1\n\ts_mov_b64 exec, -1" :: "v"(fa), "v"(v) : "memory");
}
__device__ __forceinline__ unsigned job_off(int jw, int lane) { return (jw >> 1) * 8192 + (jw & 1) * 1024 + lane * 16; }

__device__ __forceinline__ void stream_slab(int j, int snd, int& tile, int& ks) {
  if (j < 124) { ks = j / 31; int q = j - ks * 31 + 4 * snd; if (q >= 31) q -= 31; tile = (q & 1) * 16 + (q >> 1); }
  else if (j < 200) { const int u = j - 124; ks = u / 19; int q = u - ks * 19 + 2 * snd; if (q >= 19) q -= 19; tile = q < 5 ? 75 + q : 26 + q; }
  else { const int u = j - 200, gq = u >= 60, v = u - gq * 60; ks = v / 15; int q = v - ks * 15 + 2 * snd; if (q >= 15) q -= 15; tile = 45 + 15 * gq + q; }
}
__device__ __forceinline__ int stream_idx(int tile, int ks, int snd) {
  if (tile < 31) { int q = (tile < 16 ? 2 * tile : 2 * tile - 31) - 4 * snd; if (q < 0) q += 31; return ks * 31 + q; }
  if (tile < 45 || tile >= 75) { int q = (tile < 45 ? tile - 26 : tile - 75) - 2 * snd; if (q < 0) q += 19; return 124 + ks * 19 + q; }
  const int gq = tile >= 60; int q = tile - 45 - 15 * gq - 2 * snd; if (q < 0) q += 15; return 200 + gq * 60 + ks * 15 + q;
}
__device__ __forceinline__ unsigned shared_off(int r, int i) { const int idx = NSH * r + i; return (unsigned)((idx & 15) * 256 * N_ + (SHC + (idx >> 4)) * 256) * 4u; }

#define INITK 0u
#define LAG 1
#define PWIN 4u
#define PTH 8u
#define INV 0
#define LASTWT 0
#define OST 0
#define PSLEEP 8
__device__ __forceinline__ bool wait_flags(unsigned char* ws, unsigned fo, unsigned call) {
  const unsigned long long t0 = __builtin_amdgcn_s_memrealtime();
  unsigned* tmo = (unsigned*)(ws + TMO_OFF);
  for (;;) {
    unsigned v, tm; const unsigned to = TMO_OFF;
    asm volatile("global_load_dword %0, %2, %4 sc0 sc1\n\tglobal_load_dword %1, %3, %4 sc0 sc1\n\ts_waitcnt vmcnt(0)" : "=&v"(v), "=&v"(tm) : "v"(fo), "v"(to), "s"(ws) : "memory");
    if (__ballot(v != call) == 0ull) return true;
    if (__builtin_amdgcn_readfirstlane(tm) == call || __builtin_amdgcn_s_memrealtime() - t0 > DEADLINE_TICKS) {
      __hip_atomic_store(tmo, call, __ATOMIC_RELAXED, __HIP_MEMORY_SCOPE_AGENT);
      return false;
    }
    __builtin_amdgcn_s_sleep(16);
  }
}
constexpr int NTURN = 8 / WPT;
#define NPA 16
constexpr int SPT = NPA * WPT / 8, NIN = 320 / NPA, NIT = NIN + NSH * 8 / NPA;
__device__ __forceinline__ void push_elem(int it, int e, int jw, int rank, int& tile, int& ks) {
  if (it < NIN) stream_slab(NPA * it + e, rank, tile, ks);
  else { const int o = NPA * (it - NIN) + e; tile = 80; ks = (int)(shared_off(rank, o >> 3) + (unsigned)((o & 7) * 32 + jw * 4) * (N_ * 4u)); }
}
#define ST7(vc, d, dk) "global_store_dwordx4 " vc ", " d ", %2 sc0 sc1\n\tglobal_store_dwordx4 " vc ", " d ", %3 sc0 sc1\n\tglobal_store_dwordx4 " vc ", " d ", %4 sc0 sc1\n\t" \
                       "global_store_dwordx4 " vc ", " d ", %5 sc0 sc1\n\tglobal_store_dwordx4 " vc ", " d ", %6 sc0 sc1\n\tglobal_store_dwordx4 " vc ", " d ", %7 sc0 sc1\n\t" \
                       "global_store_dwordx4 " vc ", " d ", %8 sc0 sc1\n\t"
__device__ __forceinline__ void pusher(const Params& p, LAS unsigned char* lds, unsigned imgo, unsigned flo, unsigned call, int b, int wid, int lane, unsigned (&R)[16]) {
  unsigned char* const myws = p.ws[p.rank];
  const int tau = wid / WPT, e = tau * SPT + b / NTURN, jw = (b % NTURN) * WPT + wid % WPT;
  unsigned long long fa = 0; unsigned pm_ = 0;
  unsigned char* wsd[7]; int pr[7];
#pragma unroll
  for (int k = 0; k < 7; ++k) {
    pr[k] = (p.rank + 1 + (wid + b + k) % 7) & 7; wsd[k] = p.ws[pr[k]];
    if (lane == k) fa = (unsigned long long)wsd[k];
  }
  const unsigned jo = imgo + job_off(jw, lane), dno = DONE_OFF + (lane < NSH ? lane : 0) * 64u;
  LAS unsigned* const pk = (LAS unsigned*)(lds + wid * PK_SLOT);
  bool ok = true;
  int tile, ks; push_elem(0, e, jw, p.rank, tile, ks);
  v4u D[4], D4 = {0u, 0u, 0u, 0u};
  unsigned nd = 0;
  retile(tile < 16, R, D);
  for (int i = 0; i < tau; ++i) BAR;
  for (int it = 0; it < NIT; ++it) {
    const bool iso = it >= NIN;
    const unsigned sid = tile * 32 + p.rank * 4 + ks, pj = p.rank * PK_JOBS + (NPA * (it - NIN) + e) * 8 + jw;
    if (!iso) {
      pm_ = tile >= 75 ? (unsigned)(0xC070180E03ull >> ((tile - 75) * 8)) & 255u : 255u;
      unsigned char* dk[7];
#pragma unroll
      for (int k = 0; k < 7; ++k) dk[k] = (pm_ >> pr[k] & 1u) ? wsd[k] : myws;
#pragma unroll
      for (int c = 0; c < 4; ++c) {
        const unsigned vc = jo + sid * SLAB + c * 2048u;
        asm volatile(ST7("%0", "%1", dk) "s_nop 1"
                     :: "v"(vc), "v"(D[c]), "s"(dk[0]), "s"(dk[1]), "s"(dk[2]), "s"(dk[3]), "s"(dk[4]), "s"(dk[5]), "s"(dk[6]) : "memory");
      }
    } else {
#pragma unroll
      for (int c = 0; c < 5; ++c) {
        const int nl = (int)(nd >> 2) - c * 64;
        if (nl > 0) {
          const unsigned vc = PK_OFF + pj * PK_SLOT + c * 1024u + lane * 16u;
          const unsigned long long em = nl >= 64 ? ~0ull : (1ull << nl) - 1ull;
          asm volatile("s_mov_b64 exec, %9\n\t" ST7("%0", "%1", wsd) "s_mov_b64 exec, -1\n\ts_nop 1"
                       :: "v"(vc), "v"(c < 4 ? D[c & 3] : D4), "s"(wsd[0]), "s"(wsd[1]), "s"(wsd[2]), "s"(wsd[3]), "s"(wsd[4]), "s"(wsd[5]), "s"(wsd[6]), "s"(em) : "memory");
        }
      }
    }
    for (int i = 0; i < LAG; ++i) BAR;
    WAIT_V(0);
    push_elem(it < NIT - 1 ? it + 1 : it, e, jw, p.rank, tile, ks);
    load_job(p, tile, ks, jw, lane, R);
    unsigned dv; asm volatile("global_load_dword %0, %1, %2 sc0 sc1" : "=v"(dv) : "v"(dno), "s"(myws) : "memory");
    store_flag7(fa + (iso ? OFLAG_OFF + pj * 4u : flo + sid * 64u + jw * 4u), call);
    for (int i = LAG + 1; i < NTURN; ++i) BAR;
    asm volatile("s_waitcnt vmcnt(1)" : TIE16(R), "+v"(dv) :: "memory");
    if (tile < 80) retile(tile < 16, R, D);
    else {
      unsigned cnt = 32u, hdr = 0u;
#pragma unroll
      for (int j = 0; j < 16; ++j) {
        const unsigned v = ok ? R[j] : 0x7fc00000u;
        const unsigned long long m = __ballot(v != 0u);
        const unsigned lo = (unsigned)m, hi = (unsigned)(m >> 32);
        pk[v != 0u ? __builtin_amdgcn_mbcnt_hi(hi, __builtin_amdgcn_mbcnt_lo(lo, cnt)) : 1100u + (unsigned)lane] = v;
        hdr = lane == 2 * j ? lo : lane == 2 * j + 1 ? hi : hdr;
        cnt += (unsigned)__builtin_popcountll(m);
      }
      pk[lane < 32 ? (unsigned)lane : 1100u + (unsigned)lane] = hdr;
      nd = (cnt + 31u) & ~31u;
#pragma unroll
      for (int c = 0; c < 4; ++c) D[c] = *(const LAS v4u*)(pk + c * 256 + lane * 4);
      D4 = *(const LAS v4u*)(pk + 1024 + lane * 4);
    }
    if (it == NIN - 2) {
      if (__ballot(dv != call)) ok = wait_flags(myws, dno, call);
    }
    BAR;
  }
  for (int i = tau + 1; i < NTURN; ++i) BAR;
  TS(1);
}
__device__ __forceinline__ void expand(const Params& p, LAS unsigned char* lds, unsigned call, int wid) {
  unsigned char* const myws = p.ws[p.rank];
  LAS unsigned* lctl = (LAS unsigned*)(lds + LCTL);
  const int lane = fresh_lane();
  const unsigned lbase = (unsigned)(size_t)lds;
  for (unsigned r = 0;; ++r) {
    if (threadIdx.x == 0) lctl[4 + (r & 1u)] = __hip_atomic_fetch_add((unsigned*)(myws + WCT_OFF + (call & 1u) * 128u), 16u, __ATOMIC_RELAXED, __HIP_MEMORY_SCOPE_AGENT);
    __syncthreads();
    const unsigned q0 = __builtin_amdgcn_readfirstlane(lctl[4 + (r & 1u)]);
    if (q0 >= 7u * PK_JOBS) break;
    unsigned pj[2], oo[2];
#pragma unroll
    for (int h = 0; h < 2; ++h) {
      const unsigned q = q0 + h * 8 + wid, jj = q / 7u, s = (p.rank + 1 + (q - jj * 7u)) & 7u;
      pj[h] = s * PK_JOBS + jj; oo[h] = shared_off(s, jj >> 6) + ((jj & 63u) * 4u) * (N_ * 4u) + lane * 16u;
    }
    const bool ok = wait_flags(myws, OFLAG_OFF + (lane < 32 ? pj[0] : pj[1]) * 4u, call);
#pragma unroll
    for (int h = 0; h < 2; ++h) {
      const unsigned la = lbase + (wid * 2 + h) * PK_SLOT, vo = PK_OFF + pj[h] * PK_SLOT + lane * 16u;
      asm volatile("s_mov_b32 m0, %0\n\ts_nop 0\n\tglobal_load_lds_dwordx4 %1, %2 sc0 sc1\n\tglobal_load_lds_dwordx4 %1, %2 offset:1024 sc0 sc1\n\t"
                   "global_load_lds_dwordx4 %1, %2 offset:2048 sc0 sc1" :: "s"(la), "v"(vo), "s"(myws) : "memory");
    }
    WAIT_V(0);
#pragma unroll
    for (int h = 0; h < 2; ++h) {
      const LAS unsigned* pk = (const LAS unsigned*)(lds + (wid * 2 + h) * PK_SLOT);
      const unsigned hdr = pk[lane & 31];
      unsigned pos[16], cnt = 32u; unsigned long long m[16];
#pragma unroll
      for (int j = 0; j < 16; ++j) {
        const unsigned lo = __builtin_amdgcn_readlane(hdr, 2 * j), hi = __builtin_amdgcn_readlane(hdr, 2 * j + 1);
        m[j] = (unsigned long long)hi << 32 | lo;
        pos[j] = __builtin_amdgcn_mbcnt_hi(hi, __builtin_amdgcn_mbcnt_lo(lo, cnt));
        cnt += (unsigned)__builtin_popcountll(m[j]);
      }
      if (cnt > 768u) {
        const unsigned la = lbase + (wid * 2 + h) * PK_SLOT + 3072u, vo = PK_OFF + pj[h] * PK_SLOT + 3072u + lane * 16u;
        asm volatile("s_mov_b32 m0, %0\n\ts_nop 0\n\tglobal_load_lds_dwordx4 %1, %2 sc0 sc1\n\tglobal_load_lds_dwordx4 %1, %2 offset:1024 sc0 sc1\n\ts_waitcnt vmcnt(0)"
                     :: "s"(la), "v"(vo), "s"(myws) : "memory");
      }
#pragma unroll
      for (int rr = 0; rr < 4; ++rr) {
        v4u v;
#pragma unroll
        for (int c = 0; c < 4; ++c) { const unsigned x = pk[pos[4 * rr + c]]; v[c] = !ok ? 0x7fc00000u : (m[4 * rr + c] >> lane & 1ull) ? x : 0u; }
        *(v4u*)(p.out[p.rank] + oo[h] + rr * (N_ * 4u)) = v;
      }
    }
  }
}

__global__ __launch_bounds__(512, 2) void fused_kernel(const Params p) {
  extern __shared__ __attribute__((aligned(16))) unsigned char smem[];
  LAS unsigned char* lds = (LAS unsigned char*)smem;
  const int tid = threadIdx.x, wid = __builtin_amdgcn_readfirstlane(tid >> 6), wc = wid & 3;
  unsigned char* myws = p.ws[p.rank];
  const int b = blockIdx.x, rank = p.rank, g = b - NPUSH;
  const bool push = b < NPUSH;
  LAS unsigned* lctl = (LAS unsigned*)(lds + LCTL);
  const unsigned lbase = (unsigned)(size_t)lds;
  const int xc = b & 7, L = (xc >> 2) * 30 + (g >> 3), cn = L >> 2;
  const int pm = (xc & 3) * 4 + (L & 3), si = (L & 3) * 4 + (xc & 3);
  const bool skip = cn == 0 && si >= NSH;
  const unsigned nvis = skip ? 96u : 128u;
  auto tile_of = [&](int u, int& tm, int& tn) __attribute__((always_inline)) {
    const int v = u + (skip && u > 0), idx = NSH * rank + si;
    const bool sh = v == 1 && cn == 0;
    tm = sh ? idx & 15 : pm; tn = sh ? SHC + (idx >> 4) : v == 0 ? cn : 15 * v - 1 + cn;
  };
  unsigned call, imgo, flo;
  {
    const int lane = tid & 63;
    unsigned R[16], R2[16];
    int t1, k1, j1 = wid;
    if (push) { j1 = (b % NTURN) * WPT + wid % WPT; push_elem(0, wid / WPT * SPT + b / NTURN, j1, rank, t1, k1); } else { t1 = g >> 2; k1 = g & 3; }
    load_job(p, t1, k1, j1, lane, R);
    const bool two = !push && g < 80;
    if (two) load_job(p, t1 + 60, k1, wid, lane, R2);
    if (tid == 0) {
      const unsigned t = __hip_atomic_fetch_add((unsigned*)(myws + CTR_OFF), 1u, __ATOMIC_RELAXED, __HIP_MEMORY_SCOPE_AGENT);
      lctl[0] = t / (unsigned)GRID + 1u; lctl[1] = INITK; lctl[2] = 0u; lctl[3] = 0u;
      if (b == 0) __hip_atomic_store((unsigned*)(myws + WCT_OFF + (t / (unsigned)GRID & 1u) * 128u), 0u, __ATOMIC_RELAXED, __HIP_MEMORY_SCOPE_AGENT);
    }
    if (wid >= 4 && !push) {
      const int u = wid - 4, l = lane & 31, src = l >> 2, ks = l & 3;
      int tm, tn; tile_of(u, tm, tn);
      const int ax = stream_idx(tm, ks, src), aw = stream_idx(16 + tn, ks, src);
#define CORD 9
      const int key = src == rank ? ks : u >= CORD ? 32 + ks * 8 + ((src - rank) & 7) : ((ax > aw ? ax : aw) + 1) * 32 + l;
      int rk = 0;
#pragma unroll
      for (int s = 0; s < 32; ++s) rk += __builtin_amdgcn_readlane(key, s) < key;
      *(LAS int*)(lds + LTAB + (u * 32 + rk) * 4) = l;
      lctl[8 + 2 * u] = tm; lctl[9 + 2 * u] = 16 + tn;
    }
    asm volatile("s_waitcnt vmcnt(0)" : TIE16(R) :: "memory");
    asm volatile("" : TIE16(R2) :: "memory");
    __syncthreads();
    call = __builtin_amdgcn_readfirstlane(lctl[0]);
    imgo = IMG_OFF + (call & 1u) * IMG_BYTES;
    flo = FLAG_OFF + (call & 1u) * FLAG_BYTES;
    TS(0);
    if (push) {
      if (b < NPA) pusher(p, lds, imgo, flo, call, b, wid, lane, R);
      WAIT_V(0);
      expand(p, lds, call, wid);
      TS(14);
      return;
    }
    v4u D[4];
    const unsigned s1 = t1 * 32 + rank * 4 + k1;
    retile(t1 < 16, R, D);
    store_job(myws, imgo + s1 * SLAB + job_off(wid, lane) + 4096, D);
    if (two) {
      retile(false, R2, D);
      store_job(myws, imgo + (s1 + 60 * 32) * SLAB + job_off(wid, lane) + 4096, D);
    }
    WAIT_V(0);
    store_flag1(myws, flo + s1 * 64 + wid * 4, call);
    if (two) store_flag1(myws, flo + (s1 + 60 * 32) * 64 + wid * 4, call);
    TS(1);
  }
  const float scale = __builtin_bit_cast(float, __builtin_amdgcn_readfirstlane(__builtin_bit_cast(int, p.sx[0] * p.sw[0])));
  WAIT_V(0);

  auto run = [&](auto WRC) __attribute__((always_inline)) {
    constexpr int wr = decltype(WRC)::value;
    unsigned ppend = 0, pk = INITK, pw = 4u;
    auto peval = [&]() __attribute__((always_inline)) {
      const v4u f = *(const LAS v4u*)(lds + LCP + (unsigned)fresh_lane() * 16);
      unsigned long long bm = __ballot(f[0] == call && f[1] == call && f[2] == call && f[3] == call);
      bm &= bm >> 1; bm &= bm >> 2;
      bm = ~bm & 0x1111111111111111ull;
      unsigned c = bm ? (unsigned)__builtin_ctzll(bm) >> 2 : 16u;
      c = c < pw ? c : pw;
      pw = c == pw ? 16u : PWIN;
      const unsigned n = pk + c < nvis ? pk + c : nvis;
      if (n > pk) {
        pk = n; lctl[1] = n;
      }
      ppend = 0;
    };
    auto pissue = [&]() __attribute__((always_inline)) {
      const unsigned ln = (unsigned)fresh_lane();
      unsigned e = pk + ((ln >> 2) < pw ? ln >> 2 : 0u); e = e < nvis ? e : nvis - 1;
      const unsigned kk = *(const LAS unsigned*)(lds + LTAB + e * 4), tl = lctl[8 + (e >> 5) * 2 + (ln >> 1 & 1)];
      const unsigned la = lbase + LCP, fo = flo + (tl * 32 + kk) * 64 + (ln & 1) * 16; unsigned char* const w_ = myws;
      asm volatile("s_mov_b32 m0, %0\n\ts_nop 0\n\tglobal_load_lds_dwordx4 %1, %2 sc0 sc1" :: "s"(la), "v"(fo), "s"(w_) : "memory");
      ppend = 1;
    };
    auto poll = [&](bool blocking, unsigned need) __attribute__((always_inline)) {
      if (!blocking) {
        if (ppend) {
          unsigned st; asm volatile("s_getreg_b32 %0, hwreg(HW_REG_IB_STS)" : "=s"(st) :: "memory");
          st = __builtin_amdgcn_readfirstlane(st);
          if ((st & 15u) | ((st >> 18) & 48u)) return;
          WAIT_V(0);
          peval();
        } else { pw = 16u; pissue(); }
        return;
      }
      WAIT_V(0);
      if (ppend) { peval(); if (pk < need) __builtin_amdgcn_s_sleep(PSLEEP); }
      if (pk < nvis) pissue();
      WAIT_L(0);
    };
    auto slow = [&](unsigned need) __attribute__((always_inline)) {
      const unsigned long long t0 = __builtin_amdgcn_s_memrealtime();
      for (;;) {
        if (wr == 1 && wc == 0) poll(true, need); else __builtin_amdgcn_s_sleep(1);
        const unsigned long long el = __builtin_amdgcn_s_memrealtime() - t0;
        if (wr == 1 && wc == 0 && el > 100000ull) {
          unsigned* tmo = (unsigned*)(myws + TMO_OFF);
          if (__builtin_amdgcn_readfirstlane(__hip_atomic_load(tmo, __ATOMIC_RELAXED, __HIP_MEMORY_SCOPE_AGENT)) == call || el > DEADLINE_TICKS) {
            __hip_atomic_store(tmo, call, __ATOMIC_RELAXED, __HIP_MEMORY_SCOPE_AGENT);
            lctl[2] = 1u; lctl[1] = DEADV; pk = nvis; ppend = 0;
          }
          WAIT_V(0);
        }
        if (__builtin_amdgcn_readfirstlane(__hip_atomic_load(lctl + 1, __ATOMIC_RELAXED, __HIP_MEMORY_SCOPE_WORKGROUP)) >= need || el > 2 * DEADLINE_TICKS) break;
      }
      asm volatile("" ::: "memory");
    };

    const unsigned char* XI = myws + imgo;
    const unsigned voff = (unsigned)(wc * 4096 + (tid & 63) * 16);
    const unsigned ldsw = (unsigned)wc * 4096u;
    const int aoff = wr * 8192 + (tid & 63) * 16, boff = wc * 4096 + (tid & 63) * 16;
#define SA(b_, h) (((b_) * 2 + (h)) * HTB)
#define SB(b_, h) ((4 + (b_) * 2 + (h)) * HTB)
#define STG1(bufoff, va, o) __builtin_amdgcn_global_load_lds((const unsigned*)(XI + (va)), (LAS unsigned*)(lds + (bufoff) + ldsw), 16, o, 0)
#define STAGE(bufoff, goff) do { if (wr == 0) { const unsigned va_ = voff + (goff); STG1(bufoff, va_, 0); STG1(bufoff, va_, 1024); STG1(bufoff, va_, 2048); STG1(bufoff, va_, 3072); } } while (0)
#define LDA(dst, b_, h) do { _Pragma("unroll") for (int m = 0; m < 4; ++m) { \
    v4i lo = *(const LAS v4i*)(lds + SA(b_, h) + aoff + m * 2048); v4i hi = *(const LAS v4i*)(lds + SA(b_, h) + aoff + m * 2048 + 1024); dst[m] = cat8(lo, hi); } } while (0)
#define LDB(dst, b_, h) do { _Pragma("unroll") for (int n = 0; n < 2; ++n) { \
    v4i lo = *(const LAS v4i*)(lds + SB(b_, h) + boff + n * 2048); v4i hi = *(const LAS v4i*)(lds + SB(b_, h) + boff + n * 2048 + 1024); dst[n] = cat8(lo, hi); } } while (0)
#define MMA(ai, bj, At, Bt) do { __builtin_amdgcn_s_setprio(1); _Pragma("unroll") for (int m = 0; m < 4; ++m) _Pragma("unroll") for (int n = 0; n < 2; ++n) \
    acc[ai][bj][m][n] = __builtin_amdgcn_mfma_scale_f32_16x16x128_f8f6f4(Bt[n], At[m], acc[ai][bj][m][n], 0, 1, 0, 0, 0, 0); __builtin_amdgcn_s_setprio(0); } while (0)
#define MF1(ai, bj, At, Bt, q) acc[ai][bj][(q) >> 1][(q) & 1] = __builtin_amdgcn_mfma_scale_f32_16x16x128_f8f6f4(Bt[(q) & 1], At[(q) >> 1], acc[ai][bj][(q) >> 1][(q) & 1], 0, 1, 0, 0, 0, 0)
#define WV12 WAIT_V(12)
#define MEMSTEP do { if (wr == 0) WV12; else if (wc == 0 && (ppend || (pk < nvis && pk < need + PTH))) poll(false, 0u); } while (0)
#define KOFF(i) ((unsigned)__builtin_amdgcn_readlane(kv, (i)))
    f32x4 acc[2][2][4][2];
#pragma unroll
    for (int a = 0; a < 2; ++a)
#pragma unroll
      for (int c = 0; c < 2; ++c)
#pragma unroll
        for (int m = 0; m < 4; ++m)
#pragma unroll
          for (int n = 0; n < 2; ++n) acc[a][c][m][n] = (f32x4){0.f, 0.f, 0.f, 0.f};
    v8i At[4], B0[2], B1[2];
    constexpr unsigned hstep = HTB, tstep = SLAB * NKS;
    const unsigned kown = (unsigned)(rank * 4) * SLAB;
    int tm = pm, tn = cn;
    unsigned cA = (unsigned)(16 + tn) * tstep, cB = (unsigned)tm * tstep;
    slow(4u);
    unsigned known = 4u, need = 4u;
    {
      const unsigned k0 = kown, k1 = kown + SLAB;
      STAGE(SB(0, 0), cB + k0); STAGE(SA(0, 0), cA + k0); STAGE(SB(0, 1), cB + k0 + hstep); STAGE(SA(0, 1), cA + k0 + hstep);
      if (wr == 1) BAR;
      if (wr == 0) WAIT_V(8);
      BAR;
      STAGE(SB(1, 0), cB + k1); STAGE(SA(1, 0), cA + k1); STAGE(SB(1, 1), cB + k1 + hstep);
      if (wr == 0) WAIT_V(12);
      BAR;
    }
    unsigned a1 = cA + kown + SLAB, a2 = a1 + SLAB, a3 = a2 + SLAB, b2 = cB + kown + 2 * SLAB, b3 = b2 + SLAB;
    for (int ui = 0;; ++ui) {
      const bool has_next = (unsigned)(ui * 32 + 32) < nvis;
      int nm = tm, nn = tn;
      if (has_next) tile_of(ui + 1, nm, nn);
      const unsigned nA = (unsigned)(16 + nn) * tstep, nB = (unsigned)nm * tstep;
      int kv = 0;
      if (wr == 0) { const int ln = fresh_lane(); kv = *(const LAS int*)(lds + LTAB + (ui * 32 + (ln < 32 ? ln : ln & 3)) * 4) * (int)SLAB; }
      for (int t = 0; t < NKS; t += 2) {
        if (known < need) { if (wr == 0) BAR; slow(need); if (wr == 1) BAR; known = need; }
        LDB(B0, 0, 0); SCHED; LDA(At, 0, 0); SCHED; STAGE(SA(1, 1), a1 + hstep);
        WAIT_L(8); BAR; WAIT_L(0); MMA(0, 0, At, B0); BAR; SCHED;
        LDB(B1, 0, 1); SCHED; STAGE(SB(0, 0), b2);
        BAR; WAIT_L(0); MMA(0, 1, At, B1); BAR;
        LDA(At, 0, 1); SCHED; STAGE(SA(0, 0), a2);
        BAR; WAIT_L(0); MMA(1, 0, At, B0); BAR; SCHED;
        STAGE(SB(0, 1), b2 + hstep); MEMSTEP;
        BAR; MMA(1, 1, At, B1); BAR;
        LDB(B0, 1, 0); SCHED; LDA(At, 1, 0); SCHED; STAGE(SA(0, 1), a2 + hstep);
        WAIT_L(8); BAR; WAIT_L(0); MMA(0, 0, At, B0); BAR; SCHED;
        LDB(B1, 1, 1); const unsigned kn = lctl[1]; SCHED; STAGE(SB(1, 0), b3);
        BAR; WAIT_L(0); MMA(0, 1, At, B1); BAR;
        LDA(At, 1, 1); SCHED; STAGE(SA(1, 0), a3);
        BAR; WAIT_L(0); MMA(1, 0, At, B0); BAR; SCHED;
        STAGE(SB(1, 1), b3 + hstep); MEMSTEP;
        BAR; SCHED;
        __builtin_amdgcn_s_setprio(1);
        MF1(1, 1, At, B1, 0); SCHED;
        unsigned o2 = KOFF(t + 4), o3 = KOFF(t + 5); known = __builtin_amdgcn_readfirstlane(kn);
        asm volatile("" : "+s"(o2), "+s"(o3), "+s"(known)); SCHED;
        MF1(1, 1, At, B1, 1); SCHED;
        unsigned bA = t >= NKS - 4 ? nA : cA, bB = t >= NKS - 4 ? nB : cB;
        asm volatile("" : "+s"(bA), "+s"(bB)); SCHED;
        MF1(1, 1, At, B1, 2); SCHED;
        a1 = a3; a2 = bA + o2; b2 = bB + o2; a3 = bA + o3; b3 = bB + o3;
        asm volatile("" : "+s"(a1), "+s"(a2), "+s"(b2), "+s"(a3), "+s"(b3)); SCHED;
        MF1(1, 1, At, B1, 3); SCHED;
        need = need + 2u < nvis ? need + 2u : nvis;
        asm volatile("" : "+s"(need)); SCHED;
        MF1(1, 1, At, B1, 4); MF1(1, 1, At, B1, 5); MF1(1, 1, At, B1, 6); MF1(1, 1, At, B1, 7);
        __builtin_amdgcn_s_setprio(0);
        BAR;
      }
      const int dead = __builtin_amdgcn_readfirstlane(lctl[2]);
      TS(2 + 2 * ui);
      auto epilogue = [&](auto SHT) __attribute__((always_inline)) {
        const int ln = fresh_lane(); const int fr = ln & 15, fq = ln >> 4;
        unsigned char* base = p.out[rank] + (size_t)((tm * 256 + wc * 32) * N_ + tn * 256 + wr * 64) * 4;
        const unsigned vo = (unsigned)(4 * fq * N_ + 4 * fr) * 4u;
        const float qnan = __builtin_nanf("");
#pragma unroll
        for (int bj = 0; bj < 2; ++bj)
#pragma unroll
          for (int n = 0; n < 2; ++n)
#pragma unroll
            for (int r4 = 0; r4 < 4; ++r4) {
              unsigned char* rowp = base + (size_t)(bj * 128 + n * 16 + r4) * N_ * 4;
#pragma unroll
              for (int ai = 0; ai < 2; ++ai) {
                f32x4 v;
#pragma unroll
                for (int c = 0; c < 4; ++c) v[c] = dead ? qnan : fmaxf(acc[ai][bj][c][n][r4] * scale, 0.f);
                if (decltype(SHT)::value) asm volatile("global_store_dwordx4 %0, %1, %2 offset:%3 sc0 sc1\n\ts_nop 1" :: "v"(vo), "v"(v), "s"(rowp), "n"(ai * 512) : "memory");
                else if (OST == 1) __builtin_nontemporal_store(v, (f32x4*)(rowp + vo + ai * 512)); else *(f32x4*)(rowp + vo + ai * 512) = v;
              }
            }
      };
      if (tn >= SHC) {
        epilogue(std::true_type{});
        WAIT_V(0);
        unsigned o_ = 0;
        if (fresh_lane() == 0) o_ = __hip_atomic_fetch_add(lctl + 3, 1u, __ATOMIC_RELAXED, __HIP_MEMORY_SCOPE_WORKGROUP);
        if (__builtin_amdgcn_readfirstlane(o_) == 7u) store_flag1(myws, DONE_OFF + si * 64, call);
        if (wr == 0) WAIT_V(0);
      } else if (LASTWT && !has_next) epilogue(std::true_type{}); else epilogue(std::false_type{});
      TS(3 + 2 * ui);
      if (!has_next) break;
#pragma unroll
      for (int a = 0; a < 2; ++a)
#pragma unroll
        for (int c = 0; c < 2; ++c)
#pragma unroll
          for (int m = 0; m < 4; ++m)
#pragma unroll
            for (int n = 0; n < 2; ++n) acc[a][c][m][n] = (f32x4){0.f, 0.f, 0.f, 0.f};
      cA = nA; cB = nB; tm = nm; tn = nn;
    }
    if (wr == 0) BAR;
    WAIT_V(0);
  };
  if (wid < 4) run(std::integral_constant<int, 0>{}); else run(std::integral_constant<int, 1>{});
  expand(p, lds, call, wid);
  TS(14);
}
extern "C" void kernel_launch(int rank, int nranks, ncclComm_t comm,
                              hipStream_t stream,
                              void* const* d_in, const int* in_sizes, int n_in,
                              void* d_out, int out_size,
                              const int* in_dtypes, int out_dtype,
                              void* const* d_out_all,
                              void* const* d_ws_all, size_t ws_bytes) {
  if (nranks != NR || (unsigned)rank >= (unsigned)NR || ws_bytes < WS_NEED || n_in < 4) {
    fprintf(stderr, "kernel_launch: unexpected configuration\n"); return;
  }
  static bool attr_done[64] = {};
  int dev = 0; HIP_CHECK(hipGetDevice(&dev));
  if (!attr_done[dev & 63]) {
    HIP_CHECK(hipFuncSetAttribute((const void*)fused_kernel, hipFuncAttributeMaxDynamicSharedMemorySize, LDS_BYTES));
    attr_done[dev & 63] = true;
  }
  Params p;
  p.x = (const unsigned char*)d_in[0]; p.w = (const unsigned char*)d_in[1];
  p.sx = (const float*)d_in[2]; p.sw = (const float*)d_in[3];
  for (int i = 0; i < NR; ++i) { p.ws[i] = (unsigned char*)d_ws_all[i]; p.out[i] = (unsigned char*)(i == rank ? d_out : d_out_all[i]); }
  p.rank = rank;
  fused_kernel<<<GRID, 512, LDS_BYTES, stream>>>(p);
}
```

```cpp
#include <hip/hip_runtime.h>
#include <rccl/rccl.h>
#include <cstdio>
#include <type_traits>

#define HIP_CHECK(expr) do { hipError_t _e = (expr); if (_e != hipSuccess) { \
    fprintf(stderr, "HIP error %s at %s:%d\n", hipGetErrorString(_e), __FILE__, __LINE__); \
    return; } } while (0)

typedef int v8i __attribute__((ext_vector_type(8)));
typedef int v4i __attribute__((ext_vector_type(4)));
typedef unsigned v4u __attribute__((ext_vector_type(4)));
typedef float f32x4 __attribute__((ext_vector_type(4)));
#define LAS __attribute__((address_space(3)))

constexpr int NR = 8;
constexpr int M_ = 4096, N_ = 16384, KL = 512;
constexpr int NKS = 32;
constexpr int HTB = 16384;
constexpr int STAGE_BYTES = 8 * HTB;
constexpr unsigned LCTL = STAGE_BYTES, LTAB = STAGE_BYTES + 256, LCP = STAGE_BYTES + 1024;
constexpr int LDS_BYTES = STAGE_BYTES + 2048;
constexpr unsigned SLAB = 32768;
constexpr unsigned IMG_BYTES = 2560u * SLAB;
constexpr unsigned IMG_OFF = 1u << 20;
constexpr unsigned PK_OFF = IMG_OFF + 2 * IMG_BYTES, PK_SLOT = 5120, PK_JOBS = 640;
constexpr unsigned PK_BLK = 32 * PK_SLOT, NBLK = 7 * PK_JOBS / 32;
constexpr size_t WS_NEED = (size_t)PK_OFF + (size_t)NR * (PK_JOBS / 32) * PK_BLK;
constexpr unsigned CTR_OFF = 0, TMO_OFF = 128, WCT_OFF = 256, DONE_OFF = 4096, OFLAG_OFF = 8192, FLAG_OFF = 65536, FLAG_BYTES = 2560u * 64u, DBG_OFF = 524288;
constexpr int GRID = 256;
constexpr int NPUSH = 16;
constexpr int NSH = 10, SHC = 59;
constexpr unsigned DEADV = 0x7fffffffu;
constexpr unsigned long long DEADLINE_TICKS = 90000000ull;
#define WPT 2

struct Params {
  const unsigned char* x;
  const unsigned char* w;
  const float* sx; const float* sw;
  unsigned char* ws[NR];
  unsigned char* out[NR];
  int rank;
};

#define TS(slot) do {} while (0)
#define TV(slot, v) do {} while (0)
__device__ __forceinline__ v8i cat8(v4i a, v4i b) { return __builtin_shufflevector(a, b, 0, 1, 2, 3, 4, 5, 6, 7); }
#define WAIT_V(n) asm volatile("s_waitcnt vmcnt(" #n ")" ::: "memory")
#define WAIT_L(n) asm volatile("s_waitcnt lgkmcnt(" #n ")" ::: "memory")
#define BAR __builtin_amdgcn_s_barrier()
#define SCHED __builtin_amdgcn_sched_barrier(0)
#define TIE16(R) "+v"(R[0]), "+v"(R[1]), "+v"(R[2]), "+v"(R[3]), "+v"(R[4]), "+v"(R[5]), "+v"(R[6]), "+v"(R[7]), \
                 "+v"(R[8]), "+v"(R[9]), "+v"(R[10]), "+v"(R[11]), "+v"(R[12]), "+v"(R[13]), "+v"(R[14]), "+v"(R[15])
#define TIE4(X) "+v"(X[0]), "+v"(X[1]), "+v"(X[2]), "+v"(X[3])

#define PKM ""
#define IMM ""
#define LDM ""
#define SHM ""
__device__ __forceinline__ int fresh_lane() { int l; asm volatile("v_mbcnt_lo_u32_b32 %0, -1, 0\n\tv_mbcnt_hi_u32_b32 %0, -1, %0" : "=v"(l)); return l; }

__device__ __forceinline__ void load_job(const Params& p, int tile, int ks, int jw, int lane, unsigned (&R)[16], v4u (&X)[4]) {
  const int i = lane & 15, g = lane >> 4;
  const bool isx = tile < 16, iso = tile >= 80;
  unsigned vo = isx ? i * KL + g * 32 : iso ? lane * 16 : g * 32 * N_ + 4 * i;
  const unsigned char* src = isx ? p.x + (size_t)(tile * 256 + (jw >> 1) * 64) * KL + ks * 128 + (jw & 1) * 16
                           : iso ? p.out[p.rank] + (unsigned)ks
                                 : p.w + (size_t)(ks * 128 + (jw & 1) * 16) * N_ + (tile - 16) * 256 + (jw >> 1) * 64;
  const unsigned st = isx ? 16 * KL : iso ? 4 * N_ : N_;
#define LD1(r) "global_load_dword %" #r ", %20, %21" LDM "\n\tv_add_u32 %20, %22, %20\n\t"
#define LD4(r) "global_load_dwordx4 %" #r ", %20, %21" LDM "\n\tv_add_u32 %20, %22, %20\n\t"
  asm volatile("s_cmp_eq_u32 %22, %23\n\ts_cbranch_scc1 LW_%=\n\t" LD4(16) LD4(17) LD4(18) LD4(19) "s_branch LE_%=\n"
               "LW_%=:\n\t" LD1(0) LD1(1) LD1(2) LD1(3) LD1(4) LD1(5) LD1(6) LD1(7) LD1(8) LD1(9) LD1(10) LD1(11) LD1(12) LD1(13) LD1(14) LD1(15) "\n"
               "LE_%=:"
               : "=&v"(R[0]), "=&v"(R[1]), "=&v"(R[2]), "=&v"(R[3]), "=&v"(R[4]), "=&v"(R[5]), "=&v"(R[6]), "=&v"(R[7]),
                 "=&v"(R[8]), "=&v"(R[9]), "=&v"(R[10]), "=&v"(R[11]), "=&v"(R[12]), "=&v"(R[13]), "=&v"(R[14]), "=&v"(R[15]),
                 "=&v"(X[0]), "=&v"(X[1]), "=&v"(X[2]), "=&v"(X[3]), "+v"(vo)
               : "s"(src), "s"(st), "n"(N_) : "memory", "scc");
}
__device__ __forceinline__ void retile(bool plain, const unsigned (&R)[16], const v4u (&X)[4], v4u (&D)[4]) {
  if (plain) {
#pragma unroll
    for (int e = 0; e < 4; ++e) D[e] = X[e];
  } else {
#pragma unroll
    for (int d = 0; d < 4; ++d) {
      const unsigned a = R[4 * d], bb = R[4 * d + 1], c = R[4 * d + 2], f = R[4 * d + 3];
      const unsigned t0_ = __builtin_amdgcn_perm(bb, a, 0x05010400u), t1_ = __builtin_amdgcn_perm(bb, a, 0x07030602u);
      const unsigned t2_ = __builtin_amdgcn_perm(f, c, 0x05010400u), t3_ = __builtin_amdgcn_perm(f, c, 0x07030602u);
      D[0][d] = __builtin_amdgcn_perm(t2_, t0_, 0x05040100u); D[1][d] = __builtin_amdgcn_perm(t2_, t0_, 0x07060302u);
      D[2][d] = __builtin_amdgcn_perm(t3_, t1_, 0x05040100u); D[3][d] = __builtin_amdgcn_perm(t3_, t1_, 0x07060302u);
    }
  }
}
__device__ __forceinline__ void store_job(unsigned char* ws, unsigned vo, const v4u (&D)[4]) {
  asm volatile("global_store_dwordx4 %0, %1, %5 offset:-4096 sc0 sc1\n\tglobal_store_dwordx4 %0, %2, %5 offset:-2048 sc0 sc1\n\t"
               "global_store_dwordx4 %0, %3, %5 sc0 sc1\n\tglobal_store_dwordx4 %0, %4, %5 offset:2048 sc0 sc1\n\ts_nop 1"
               :: "v"(vo), "v"(D[0]), "v"(D[1]), "v"(D[2]), "v"(D[3]), "s"(ws) : "memory");
}
__device__ __forceinline__ void store_flag1(unsigned char* ws, unsigned fo, unsigned v) {
  asm volatile("s_mov_b64 exec, 1\n\tglobal_store_dword %0, %1, %2 sc0 sc1\n\ts_mov_b64 exec, -1" :: "v"(fo), "v"(v), "s"(ws) : "memory");
}
__device__ __forceinline__ void store_flag7(unsigned long long fa, unsigned v) {
  asm volatile("s_mov_b64 exec, 0x7f\n\tglobal_store_dword %0, %1, off sc0 sc1\n\ts_mov_b64 exec, -1" :: "v"(fa), "v"(v) : "memory");
}
__device__ __forceinline__ unsigned job_off(int jw, int lane) { return (jw >> 1) * 8192 + (jw & 1) * 1024 + lane * 16; }

__device__ __forceinline__ void stream_slab(int j, int snd, int& tile, int& ks) {
  if (j < 124) { ks = j / 31; int q = j - ks * 31 + 4 * snd; if (q >= 31) q -= 31; tile = (q & 1) * 16 + (q >> 1); }
  else if (j < 200) { const int u = j - 124; ks = u / 19; int q = u - ks * 19 + 2 * snd; if (q >= 19) q -= 19; tile = q < 5 ? 75 + q : 26 + q; }
  else { const int u = j - 200, gq = u >= 60, v = u - gq * 60; ks = v / 15; int q = v - ks * 15 + 2 * snd; if (q >= 15) q -= 15; tile = 45 + 15 * gq + q; }
}
__device__ __forceinline__ int stream_idx(int tile, int ks, int snd) {
  if (tile < 31) { int q = (tile < 16 ? 2 * tile : 2 * tile - 31) - 4 * snd; if (q < 0) q += 31; return ks * 31 + q; }
  if (tile < 45 || tile >= 75) { int q = (tile < 45 ? tile - 26 : tile - 75) - 2 * snd; if (q < 0) q += 19; return 124 + ks * 19 + q; }
  const int gq = tile >= 60; int q = tile - 45 - 15 * gq - 2 * snd; if (q < 0) q += 15; return 200 + gq * 60 + ks * 15 + q;
}
__device__ __forceinline__ unsigned shared_off(int r, int i) { const int idx = NSH * r + i; return (unsigned)((idx & 15) * 256 * N_ + (SHC + (idx >> 4)) * 256) * 4u; }

#define INITK 0u
#define LAG 1
#define PWIN 4u
#define PTH 8u
#define INV 0
#define LASTWT 0
#define OST 1
#define OSM " nt"
#define PLN 0
#define XST OST
#define XSM " nt"
#define PSLEEP 8
__device__ __forceinline__ bool wait_flags(unsigned char* ws, unsigned fo, unsigned call) {
  const unsigned long long t0 = __builtin_amdgcn_s_memrealtime();
  unsigned* tmo = (unsigned*)(ws + TMO_OFF);
  for (;;) {
    unsigned v, tm; const unsigned to = TMO_OFF;
    asm volatile("global_load_dword %0, %2, %4 sc0 sc1\n\tglobal_load_dword %1, %3, %4 sc0 sc1\n\ts_waitcnt vmcnt(0)" : "=&v"(v), "=&v"(tm) : "v"(fo), "v"(to), "s"(ws) : "memory");
    if (__ballot(v != call) == 0ull) return true;
    if (__builtin_amdgcn_readfirstlane(tm) == call || __builtin_amdgcn_s_memrealtime() - t0 > DEADLINE_TICKS) {
      __hip_atomic_store(tmo, call, __ATOMIC_RELAXED, __HIP_MEMORY_SCOPE_AGENT);
      return false;
    }
    __builtin_amdgcn_s_sleep(16);
  }
}
constexpr int NTURN = 8 / WPT;
#define NPA 16
constexpr int SPT = NPA * WPT / 8, NIN = 320 / NPA;
__device__ __forceinline__ unsigned pk_off(int r, unsigned jj) { return shared_off(r, jj >> 6) + ((jj & 63u) * 4u) * (N_ * 4u); }
#define ST1(vc, d, k, md) "global_store_dwordx4 " vc ", " d ", %" #k " sc0 sc1" md "\n\t"
#define ST7(vc, d, md) ST1(vc, d, 2, md) ST1(vc, d, 3, md) ST1(vc, d, 4, md) ST1(vc, d, 5, md) ST1(vc, d, 6, md) ST1(vc, d, 7, md) ST1(vc, d, 8, md)
__device__ __forceinline__ void pusher(const Params& p, LAS unsigned char* lds, unsigned imgo, unsigned flo, unsigned call, int b, int wid, int lane, unsigned (&R)[16], v4u (&X)[4]) {
  unsigned char* const myws = p.ws[p.rank];
  const int tau = wid / WPT, e = tau * SPT + b / NTURN, jw = (b % NTURN) * WPT + wid % WPT;
  unsigned long long fa = 0;
  unsigned char* wsd[7]; int pr[7];
#pragma unroll
  for (int k = 0; k < 7; ++k) {
    pr[k] = (p.rank + 1 + (wid + b + k) % 7) & 7; wsd[k] = p.ws[pr[k]];
    if (lane == k) fa = (unsigned long long)wsd[k];
  }
  const unsigned jo = imgo + job_off(jw, lane), dno = DONE_OFF + (lane < NSH ? lane : 0) * 64u;
  LAS unsigned* const pk = (LAS unsigned*)(lds + wid * PK_SLOT);
  bool ok = true;
  int tile, ks; stream_slab(e, p.rank, tile, ks);
  unsigned job = e * 8 + jw;
  v4u D[4];
  retile(tile < 16, R, X, D);
  for (int i = 0; i < tau; ++i) BAR;
  for (int it = 0; it < NIN; ++it) {
    const unsigned sid = tile * 32 + p.rank * 4 + ks;
    const unsigned pm_ = tile >= 75 ? (unsigned)(0xC070180E03ull >> ((tile - 75) * 8)) & 255u : 255u;
    unsigned char* dk[7];
#pragma unroll
    for (int k = 0; k < 7; ++k) dk[k] = (pm_ >> pr[k] & 1u) ? wsd[k] : myws;
#pragma unroll
    for (int c = 0; c < 4; ++c) {
      const unsigned vc = jo + sid * SLAB + c * 2048u;
      asm volatile(ST7("%0", "%1", IMM) "s_nop 1"
                   :: "v"(vc), "v"(D[c]), "s"(dk[0]), "s"(dk[1]), "s"(dk[2]), "s"(dk[3]), "s"(dk[4]), "s"(dk[5]), "s"(dk[6]) : "memory");
    }
    const int nb = it < NIN - 1 ? NTURN - 1 : NTURN - 1 - tau;
    for (int i = 0; i < LAG && i < nb; ++i) BAR;
    WAIT_V(0);
    if (it < NIN - 1) stream_slab(NPA * (it + 1) + e, p.rank, tile, ks); else { tile = 80; ks = (int)pk_off(p.rank, job); }
    load_job(p, tile, ks, jw, lane, R, X);
    unsigned dv; asm volatile("global_load_dword %0, %1, %2 sc0 sc1" : "=v"(dv) : "v"(dno), "s"(myws) : "memory");
    store_flag7(fa + flo + sid * 64u + jw * 4u, call);
    for (int i = LAG; i < nb; ++i) BAR;
    asm volatile("s_waitcnt vmcnt(1)" : TIE16(R), TIE4(X), "+v"(dv) :: "memory");
    if (it == NIN - 1) break;
    retile(tile < 16, R, X, D);
    if (it == NIN - 2) {
      if (__ballot(dv != call)) ok = wait_flags(myws, dno, call);
    }
    BAR;
  }
  for (;;) {
    unsigned nx = 1u; const unsigned co = WCT_OFF + 64u + (call & 1u) * 128u;
    asm volatile("s_mov_b64 exec, 1\n\tglobal_atomic_add %0, %1, %0, %2 sc0\n\ts_mov_b64 exec, -1" : "+v"(nx) : "v"(co), "s"(myws) : "memory");
    unsigned cnt = 32u, hdr = 0u;
#pragma unroll
    for (int j = 0; j < 16; ++j) {
      const unsigned v = ok ? X[j >> 2][j & 3] : 0x7fc00000u;
      const unsigned long long m = __ballot(v != 0u);
      const unsigned lo = (unsigned)m, hi = (unsigned)(m >> 32);
      pk[v != 0u ? __builtin_amdgcn_mbcnt_hi(hi, __builtin_amdgcn_mbcnt_lo(lo, cnt)) : 1100u + (unsigned)lane] = v;
      hdr = lane == 2 * j ? lo : lane == 2 * j + 1 ? hi : hdr;
      cnt += (unsigned)__builtin_popcountll(m);
    }
    pk[lane < 32 ? (unsigned)lane : 1100u + (unsigned)lane] = hdr;
    const unsigned nd = (cnt + 31u) & ~31u, pj = p.rank * PK_JOBS + job;
#pragma unroll
    for (int c = 0; c < 5; ++c) {
      const int nl = (int)(nd >> 2) - c * 64;
      if (nl > 0) {
        const v4u d = *(const LAS v4u*)(pk + c * 256 + lane * 4);
        const unsigned vc = PK_OFF + (pj >> 5) * PK_BLK + (c < 3 ? (pj & 31u) * 3072u : 95232u + (pj & 31u) * 2048u) + c * 1024u + lane * 16u;
        const unsigned long long em = nl >= 64 ? ~0ull : (1ull << nl) - 1ull;
        asm volatile("s_mov_b64 exec, %9\n\t" ST7("%0", "%1", PKM) "s_mov_b64 exec, -1\n\ts_nop 1"
                     :: "v"(vc), "v"(d), "s"(wsd[0]), "s"(wsd[1]), "s"(wsd[2]), "s"(wsd[3]), "s"(wsd[4]), "s"(wsd[5]), "s"(wsd[6]), "s"(em) : "memory");
      }
    }
    asm volatile("s_waitcnt vmcnt(0)" : "+v"(nx) :: "memory");
    job = __builtin_amdgcn_readfirstlane(nx) + NPA * 8u;
    const bool more = job < PK_JOBS;
    {
      unsigned vo = lane * 16u; const unsigned char* src = p.out[p.rank] + pk_off(p.rank, more ? job : 0u);
      asm volatile("s_cmp_ge_u32 %6, %8\n\ts_cbranch_scc1 OE_%=\n\tglobal_load_dwordx4 %0, %4, %5\n\tv_add_u32 %4, %7, %4\n\tglobal_load_dwordx4 %1, %4, %5\n\tv_add_u32 %4, %7, %4\n\t"
                   "global_load_dwordx4 %2, %4, %5\n\tv_add_u32 %4, %7, %4\n\tglobal_load_dwordx4 %3, %4, %5\nOE_%=:"
                   : "=&v"(X[0]), "=&v"(X[1]), "=&v"(X[2]), "=&v"(X[3]), "+v"(vo) : "s"(src), "s"(job), "n"(N_ * 4), "n"(PK_JOBS) : "memory", "scc");
    }
    store_flag7(fa + OFLAG_OFF + pj * 4u, call);
    if (!more) break;
    asm volatile("s_waitcnt vmcnt(1)" : TIE4(X) :: "memory");
  }
  TS(1); TS(9);
}
#define NG 118u
#define XCNT 768u
__device__ __forceinline__ unsigned blk_job(int rank, unsigned q) { const unsigned bl = q / 7u; return ((rank + 1 + (q - bl * 7u)) & 7u) * PK_JOBS + bl * 32u; }
__device__ __forceinline__ void expand(const Params& p, LAS unsigned char* lds, unsigned call, int wid, unsigned q, unsigned mode) {
  unsigned char* const myws = p.ws[p.rank];
  LAS unsigned* lctl = (LAS unsigned*)(lds + LCTL);
  const int lane = fresh_lane();
  const unsigned lbase = (unsigned)(size_t)lds;
  for (unsigned r = 0;; ++r) {
    if (mode == 0u) {
      if (threadIdx.x == 0) lctl[6 + (r & 1u)] = __hip_atomic_fetch_add((unsigned*)(myws + WCT_OFF + (call & 1u) * 128u), 1u, __ATOMIC_RELAXED, __HIP_MEMORY_SCOPE_AGENT);
      __syncthreads();
      q = __builtin_amdgcn_readfirstlane(lctl[6 + (r & 1u)]) + NG;
    }
    if (q >= NBLK) break;
    const unsigned pj = blk_job(p.rank, q), snd = pj / PK_JOBS, vb = PK_OFF + (pj >> 5) * PK_BLK + lane * 16u;
    bool ok = true;
    if (mode != 2u) {
      ok = wait_flags(myws, OFLAG_OFF + (pj + wid + (lane & 24)) * 4u, call);
#pragma unroll
      for (int h = 0; h < 4; ++h) {
        const unsigned la = lbase + (wid + h * 8) * 3072u + (h >> 1) * 16384u, vo = vb + (wid + h * 8) * 3072u;
        asm volatile("s_mov_b32 m0, %0\n\ts_nop 0\n\tglobal_load_lds_dwordx4 %1, %2 sc0 sc1\n\tglobal_load_lds_dwordx4 %1, %2 offset:1024 sc0 sc1\n\t"
                     "global_load_lds_dwordx4 %1, %2 offset:2048 sc0 sc1" :: "s"(la), "v"(vo), "s"(myws) : "memory");
      }
      WAIT_V(0);
    }
#pragma unroll 2
    for (int h = 0; h < 4; ++h) {
      const unsigned j = wid + h * 8, oo = pk_off(snd, pj - snd * PK_JOBS + j) + lane * 16u, so = j * 3072u + (h >> 1) * 16384u;
      const LAS unsigned* pk = (const LAS unsigned*)(lds + so);
      const unsigned hdr = pk[lane & 31];
      unsigned pos[16], cnt = 32u; unsigned long long m[16];
#pragma unroll
      for (int i = 0; i < 16; ++i) {
        const unsigned lo = __builtin_amdgcn_readlane(hdr, 2 * i), hi = __builtin_amdgcn_readlane(hdr, 2 * i + 1);
        m[i] = (unsigned long long)hi << 32 | lo;
        pos[i] = __builtin_amdgcn_mbcnt_hi(hi, __builtin_amdgcn_mbcnt_lo(lo, cnt));
        cnt += (unsigned)__builtin_popcountll(m[i]);
      }
      if (cnt > XCNT) {
        const unsigned s2 = 49152u + wid * 2048u, la = lbase + s2, vo = vb + 98304u + j * 2048u, d2 = (s2 - so - 3072u) >> 2;
        asm volatile("s_waitcnt lgkmcnt(0)\n\ts_mov_b32 m0, %0\n\ts_nop 0\n\tglobal_load_lds_dwordx4 %1, %2 sc0 sc1\n\tglobal_load_lds_dwordx4 %1, %2 offset:1024 sc0 sc1\n\ts_waitcnt vmcnt(0)"
                     :: "s"(la), "v"(vo), "s"(myws) : "memory");
#pragma unroll
        for (int i = 0; i < 16; ++i) pos[i] = (pos[i] + (pos[i] >= 768u ? d2 : 0u)) & 0x3fffffffu;
      }
#pragma unroll
      for (int rr = 0; rr < 4; ++rr) {
        v4u v;
#pragma unroll
        for (int c = 0; c < 4; ++c) {
          const unsigned x = *(const LAS unsigned*)(lds + (so + pos[4 * rr + c] * 4u));
          v[c] = !ok ? 0x7fc00000u : (m[4 * rr + c] >> lane & 1ull) ? x : 0u;
        }
        const unsigned vo_ = oo + rr * (N_ * 4u);
        asm volatile("global_store_dwordx4 %0, %1, %2" XSM "\n\ts_nop 1" :: "v"(vo_), "v"(v), "s"(p.out[p.rank]) : "memory");
      }
    }
    if (mode) break;
  }
}

__global__ __launch_bounds__(512, 2) void fused_kernel(const Params p) {
  extern __shared__ __attribute__((aligned(16))) unsigned char smem[];
  LAS unsigned char* lds = (LAS unsigned char*)smem;
  const int tid = threadIdx.x, wid = __builtin_amdgcn_readfirstlane(tid >> 6), wc = wid & 3;
  unsigned char* myws = p.ws[p.rank];
  const int b = blockIdx.x, rank = p.rank, g = b - NPUSH;
  const bool push = b < NPUSH;
  LAS unsigned* lctl = (LAS unsigned*)(lds + LCTL);
  const unsigned lbase = (unsigned)(size_t)lds;
  const int xc = b & 7, L = (xc >> 2) * 30 + (g >> 3), cn = L >> 2;
  const int pm = (xc & 3) * 4 + (L & 3), si = (L & 3) * 4 + (xc & 3);
  const bool skip = cn == 0 && si >= NSH;
  const unsigned nvis = skip ? 96u : 128u;
  auto tile_of = [&](int u, int& tm, int& tn) __attribute__((always_inline)) {
    const int v = u + (skip && u > 0), idx = NSH * rank + si;
    const bool sh = v == 1 && cn == 0;
    tm = sh ? idx & 15 : pm; tn = sh ? SHC + (idx >> 4) : v == 0 ? cn : 15 * v - 1 + cn;
  };
  unsigned call, imgo, flo;
  {
    const int lane = tid & 63;
    unsigned R[16], R2[16]; v4u X[4], X2[4];
    int t1, k1, j1 = wid;
    if (push) { j1 = (b % NTURN) * WPT + wid % WPT; stream_slab(wid / WPT * SPT + b / NTURN, rank, t1, k1); } else { t1 = g >> 2; k1 = g & 3; }
    load_job(p, t1, k1, j1, lane, R, X);
    const bool two = !push && g < 80;
    if (two) load_job(p, t1 + 60, k1, wid, lane, R2, X2);
    if (tid == 0) {
      const unsigned t = __hip_atomic_fetch_add((unsigned*)(myws + CTR_OFF), 1u, __ATOMIC_RELAXED, __HIP_MEMORY_SCOPE_AGENT);
      lctl[0] = t / (unsigned)GRID + 1u; lctl[1] = INITK; lctl[2] = 0u; lctl[3] = 0u; lctl[5] = 0u;
      if (b == 0) {
        __hip_atomic_store((unsigned*)(myws + WCT_OFF + (t / (unsigned)GRID & 1u) * 128u), 0u, __ATOMIC_RELAXED, __HIP_MEMORY_SCOPE_AGENT);
        __hip_atomic_store((unsigned*)(myws + WCT_OFF + 64u + (t / (unsigned)GRID & 1u) * 128u), 0u, __ATOMIC_RELAXED, __HIP_MEMORY_SCOPE_AGENT);
        __hip_atomic_store((unsigned*)(myws + WCT_OFF + 256u + (t / (unsigned)GRID & 1u) * 128u), 0u, __ATOMIC_RELAXED, __HIP_MEMORY_SCOPE_AGENT);
      }
    }
    if (wid >= 4 && !push) {
      const int u = wid - 4, l = lane & 31, src = l >> 2, ks = l & 3;
      int tm, tn; tile_of(u, tm, tn);
      const int ax = stream_idx(tm, ks, src), aw = stream_idx(16 + tn, ks, src);
#define CORD 9
      const int key = src == rank ? ks : u >= CORD ? 32 + ks * 8 + ((src - rank) & 7) : ((ax > aw ? ax : aw) + 1) * 32 + l;
      int rk = 0;
#pragma unroll
      for (int s = 0; s < 32; ++s) rk += __builtin_amdgcn_readlane(key, s) < key;
      const unsigned ent = (unsigned)((16 + tn) * 32 + l) << 15 | (unsigned)(tm * 32 + l);
      if (u * 32u < nvis) *(LAS unsigned*)(lds + LTAB + (u * 32 + rk) * 4) = ent;
      if (u == 0 && rk < 4) *(LAS unsigned*)(lds + LTAB + (nvis + rk) * 4) = ent;
    }
    asm volatile("s_waitcnt vmcnt(0)" : TIE16(R), TIE4(X) :: "memory");
    asm volatile("" : TIE16(R2) :: "memory");
    __syncthreads();
    call = __builtin_amdgcn_readfirstlane(lctl[0]);
    imgo = IMG_OFF + (call & 1u) * IMG_BYTES;
    flo = FLAG_OFF + (call & 1u) * FLAG_BYTES;
    TS(0);
    if (push) {
      if (b < NPA) pusher(p, lds, imgo, flo, call, b, wid, lane, R, X);
      WAIT_V(0);
      expand(p, lds, call, wid, 0u, 0u);
      TS(14);
      return;
    }
    v4u D[4];
    const unsigned s1 = t1 * 32 + rank * 4 + k1;
    retile(t1 < 16, R, X, D);
    store_job(myws, imgo + s1 * SLAB + job_off(wid, lane) + 4096, D);
    if (two) {
      retile(false, R2, X2, D);
      store_job(myws, imgo + (s1 + 60 * 32) * SLAB + job_off(wid, lane) + 4096, D);
    }
    WAIT_V(0);
    store_flag1(myws, flo + s1 * 64 + wid * 4, call);
    if (two) store_flag1(myws, flo + (s1 + 60 * 32) * 64 + wid * 4, call);
    TS(1);
  }
  const float scale = __builtin_bit_cast(float, __builtin_amdgcn_readfirstlane(__builtin_bit_cast(int, p.sx[0] * p.sw[0])));
  WAIT_V(0);

  auto run = [&](auto WRC) __attribute__((always_inline)) {
    constexpr int wr = decltype(WRC)::value;
    unsigned ppend = 0, pk = INITK, pw = 4u;
    auto peval = [&]() __attribute__((always_inline)) {
      const v4u f = *(const LAS v4u*)(lds + LCP + (unsigned)fresh_lane() * 16);
      unsigned long long bm = __ballot(f[0] == call && f[1] == call && f[2] == call && f[3] == call);
      bm &= bm >> 1; bm &= bm >> 2;
      bm = ~bm & 0x1111111111111111ull;
      unsigned c = bm ? (unsigned)__builtin_ctzll(bm) >> 2 : 16u;
      c = c < pw ? c : pw;
      pw = c == pw ? 16u : PWIN;
      const unsigned n = pk + c < nvis ? pk + c : nvis;
      if (n > pk) {
        pk = n; lctl[1] = n;
      }
      ppend = 0;
    };
    auto pissue = [&]() __attribute__((always_inline)) {
      const unsigned ln = (unsigned)fresh_lane();
      unsigned e = pk + ((ln >> 2) < pw ? ln >> 2 : 0u); e = e < nvis ? e : nvis - 1;
      const unsigned kk = *(const LAS unsigned*)(lds + LTAB + e * 4);
      const unsigned la = lbase + LCP, fo = flo + ((ln & 2) ? kk >> 15 : kk & 0x7fffu) * 64 + (ln & 1) * 16; unsigned char* const w_ = myws;
      asm volatile("s_mov_b32 m0, %0\n\ts_nop 0\n\tglobal_load_lds_dwordx4 %1, %2 sc0 sc1" :: "s"(la), "v"(fo), "s"(w_) : "memory");
      ppend = 1;
    };
    auto poll = [&](bool blocking, unsigned need) __attribute__((always_inline)) {
      if (!blocking) {
        if (ppend) {
          unsigned st; asm volatile("s_getreg_b32 %0, hwreg(HW_REG_IB_STS)" : "=s"(st) :: "memory");
          st = __builtin_amdgcn_readfirstlane(st);
          if ((st & 15u) | ((st >> 18) & 48u)) return;
          WAIT_V(0);
          peval();
        } else { pw = 16u; pissue(); }
        return;
      }
      WAIT_V(0);
      if (ppend) { peval(); if (pk < need) __builtin_amdgcn_s_sleep(PSLEEP); }
      if (pk < nvis) pissue();
      WAIT_L(0);
    };
    auto slow = [&](unsigned need) __attribute__((always_inline)) {
      const unsigned long long t0 = __builtin_amdgcn_s_memrealtime();
      for (;;) {
        if (wr == 1 && wc == 0) poll(true, need); else __builtin_amdgcn_s_sleep(1);
        const unsigned long long el = __builtin_amdgcn_s_memrealtime() - t0;
        if (wr == 1 && wc == 0 && el > 100000ull) {
          unsigned* tmo = (unsigned*)(myws + TMO_OFF);
          if (__builtin_amdgcn_readfirstlane(__hip_atomic_load(tmo, __ATOMIC_RELAXED, __HIP_MEMORY_SCOPE_AGENT)) == call || el > DEADLINE_TICKS) {
            __hip_atomic_store(tmo, call, __ATOMIC_RELAXED, __HIP_MEMORY_SCOPE_AGENT);
            lctl[2] = 1u; lctl[1] = DEADV; pk = nvis; ppend = 0;
          }
          WAIT_V(0);
        }
        if (__builtin_amdgcn_readfirstlane(__hip_atomic_load(lctl + 1, __ATOMIC_RELAXED, __HIP_MEMORY_SCOPE_WORKGROUP)) >= need || el > 2 * DEADLINE_TICKS) break;
      }
      asm volatile("" ::: "memory");
    };

#define GT0 26
    auto grab = [&](int t) __attribute__((always_inline)) {
      const unsigned ln = (unsigned)fresh_lane() & 31u, q = __builtin_amdgcn_readfirstlane(lctl[4]), pj = blk_job(rank, q);
      if (q >= NBLK) return;
      if (t == GT0) {
        const unsigned la = lbase + LCTL + 64u, fo = OFLAG_OFF + (pj + ln) * 4u; unsigned char* const w_ = myws;
        asm volatile("s_mov_b32 m0, %0\n\ts_mov_b32 exec_hi, 0\n\tglobal_load_lds_dword %1, %2 sc0 sc1\n\ts_mov_b32 exec_hi, -1" :: "s"(la), "v"(fo), "s"(w_) : "memory");
      } else {
        WAIT_V(0);
        lctl[5] = __ballot(lctl[16 + ln] != call) == 0ull ? PK_OFF + (pj >> 5) * PK_BLK - imgo : 0u;
      }
    };
    const unsigned char* XI = myws + imgo;
    const unsigned voff = (unsigned)(wc * 4096 + (tid & 63) * 16);
    const unsigned ldsw = (unsigned)wc * 4096u;
    const int aoff = wr * 8192 + (tid & 63) * 16, boff = wc * 4096 + (tid & 63) * 16;
#define SA(b_, h) (((b_) * 2 + (h)) * HTB)
#define SB(b_, h) ((4 + (b_) * 2 + (h)) * HTB)
#define STG1(bufoff, va, o) __builtin_amdgcn_global_load_lds((const unsigned*)(XI + (va)), (LAS unsigned*)(lds + (bufoff) + ldsw), 16, o, 0)
#define STAGE(bufoff, goff) do { if (wr == 0) { const unsigned va_ = voff + (goff); STG1(bufoff, va_, 0); STG1(bufoff, va_, 1024); STG1(bufoff, va_, 2048); STG1(bufoff, va_, 3072); } } while (0)
#define LDA(dst, b_, h) do { _Pragma("unroll") for (int m = 0; m < 4; ++m) { \
    v4i lo = *(const LAS v4i*)(lds + SA(b_, h) + aoff + m * 2048); v4i hi = *(const LAS v4i*)(lds + SA(b_, h) + aoff + m * 2048 + 1024); dst[m] = cat8(lo, hi); } } while (0)
#define LDB(dst, b_, h) do { _Pragma("unroll") for (int n = 0; n < 2; ++n) { \
    v4i lo = *(const LAS v4i*)(lds + SB(b_, h) + boff + n * 2048); v4i hi = *(const LAS v4i*)(lds + SB(b_, h) + boff + n * 2048 + 1024); dst[n] = cat8(lo, hi); } } while (0)
#define MMA(ai, bj, At, Bt) do { __builtin_amdgcn_s_setprio(1); _Pragma("unroll") for (int m = 0; m < 4; ++m) _Pragma("unroll") for (int n = 0; n < 2; ++n) \
    acc[ai][bj][m][n] = __builtin_amdgcn_mfma_scale_f32_16x16x128_f8f6f4(Bt[n], At[m], acc[ai][bj][m][n], 0, 1, 0, 0, 0, 0); __builtin_amdgcn_s_setprio(0); } while (0)
#define MF1(ai, bj, At, Bt, q) acc[ai][bj][(q) >> 1][(q) & 1] = __builtin_amdgcn_mfma_scale_f32_16x16x128_f8f6f4(Bt[(q) & 1], At[(q) >> 1], acc[ai][bj][(q) >> 1][(q) & 1], 0, 1, 0, 0, 0, 0)
#define WV12 WAIT_V(12)
#define MEMSTEP do { if (wr == 0) WV12; else if (wc == 0 && (ppend || (pk < nvis && pk < need + PTH))) poll(false, 0u); } while (0)
#define KOFF(i) ((unsigned)__builtin_amdgcn_readlane(kv, (i)))
    f32x4 acc[2][2][4][2];
#pragma unroll
    for (int a = 0; a < 2; ++a)
#pragma unroll
      for (int c = 0; c < 2; ++c)
#pragma unroll
        for (int m = 0; m < 4; ++m)
#pragma unroll
          for (int n = 0; n < 2; ++n) acc[a][c][m][n] = (f32x4){0.f, 0.f, 0.f, 0.f};
    v8i At[4], B0[2], B1[2];
    constexpr unsigned hstep = HTB;
#define OFA(e_) ((e_) & ~0x7fffu)
#define OFB(e_) (((e_) & 0x7fffu) << 15)
    slow(4u);
    unsigned known = 4u, need = 4u;
    const unsigned e0 = __builtin_amdgcn_readfirstlane(*(const LAS unsigned*)(lds + LTAB));
    {
      const unsigned cA = OFA(e0), cB = OFB(e0);
      STAGE(SB(0, 0), cB); STAGE(SA(0, 0), cA); STAGE(SB(0, 1), cB + hstep); STAGE(SA(0, 1), cA + hstep);
      if (wr == 1) BAR;
      if (wr == 0) WAIT_V(8);
      BAR;
      STAGE(SB(1, 0), cB + SLAB); STAGE(SA(1, 0), cA + SLAB); STAGE(SB(1, 1), cB + SLAB + hstep);
      if (wr == 0) WAIT_V(12);
      BAR;
    }
    unsigned a1 = OFA(e0) + SLAB, a2 = a1 + SLAB, a3 = a2 + SLAB, b2 = OFB(e0) + 2 * SLAB, b3 = b2 + SLAB;
    for (int ui = 0;; ++ui) {
      const bool has_next = (unsigned)(ui * 32 + 32) < nvis;
      int kv = 0;
      if (wr == 0) { const unsigned ln = ui * 32u + (unsigned)fresh_lane(); kv = *(const LAS int*)(lds + LTAB + (ln < 131u ? ln : 131u) * 4); }
      for (int t = 0; t < NKS; t += 2) {
        if (known < need) { if (wr == 0) BAR; slow(need); if (wr == 1) BAR; known = need; }
        LDB(B0, 0, 0); SCHED; LDA(At, 0, 0); SCHED; STAGE(SA(1, 1), a1 + hstep);
        WAIT_L(8); BAR; WAIT_L(0); MMA(0, 0, At, B0); BAR; SCHED;
        LDB(B1, 0, 1); SCHED; STAGE(SB(0, 0), b2);
        BAR; WAIT_L(0); MMA(0, 1, At, B1); BAR;
        LDA(At, 0, 1); SCHED; STAGE(SA(0, 0), a2);
        BAR; WAIT_L(0); MMA(1, 0, At, B0); BAR; SCHED;
        STAGE(SB(0, 1), b2 + hstep); MEMSTEP;
        if (wr == 1 && wc == 1 && ui == 3 && (t == GT0 || t == NKS - 4)) grab(t);
        BAR; MMA(1, 1, At, B1); BAR;
        LDB(B0, 1, 0); SCHED; LDA(At, 1, 0); SCHED; STAGE(SA(0, 1), a2 + hstep);
        WAIT_L(8); BAR; WAIT_L(0); MMA(0, 0, At, B0); BAR; SCHED;
        LDB(B1, 1, 1); const unsigned kn = lctl[1]; SCHED; STAGE(SB(1, 0), b3);
        BAR; WAIT_L(0); MMA(0, 1, At, B1); BAR;
        LDA(At, 1, 1); SCHED; STAGE(SA(1, 0), a3);
        BAR; WAIT_L(0); MMA(1, 0, At, B0); BAR; SCHED;
        STAGE(SB(1, 1), b3 + hstep); if (wr == 1 || has_next || t < NKS - 2) MEMSTEP;
        BAR; SCHED;
        __builtin_amdgcn_s_setprio(1);
        MF1(1, 1, At, B1, 0); SCHED;
        unsigned o2 = KOFF(t + 4), o3 = KOFF(t + 5); known = __builtin_amdgcn_readfirstlane(kn);
        asm volatile("" : "+s"(o2), "+s"(o3), "+s"(known)); SCHED;
        MF1(1, 1, At, B1, 1); SCHED;
        a1 = a3; a2 = OFA(o2); b2 = OFB(o2);
        asm volatile("" : "+s"(a1), "+s"(a2), "+s"(b2)); SCHED;
        MF1(1, 1, At, B1, 2); SCHED;
        a3 = OFA(o3); b3 = OFB(o3);
        if (wr == 0 && !has_next && t == NKS - 4) {
          const unsigned ob_ = __builtin_amdgcn_readfirstlane(lctl[5]);
          a2 = ob_ ? ob_ : a1; a3 = ob_ ? ob_ + 2 * hstep : a1; b2 = ob_ ? ob_ + 3 * hstep : a1; b3 = ob_ ? ob_ + 5 * hstep : a1;
        }
        asm volatile("" : "+s"(a2), "+s"(b2), "+s"(a3), "+s"(b3)); SCHED;
        MF1(1, 1, At, B1, 3); SCHED;
        need = need + 2u < nvis ? need + 2u : nvis;
        asm volatile("" : "+s"(need)); SCHED;
        MF1(1, 1, At, B1, 4); MF1(1, 1, At, B1, 5); MF1(1, 1, At, B1, 6); MF1(1, 1, At, B1, 7);
        __builtin_amdgcn_s_setprio(0);
        BAR;
      }
      int tm, tn; tile_of(ui, tm, tn);
      const bool dead = __builtin_amdgcn_readfirstlane(lctl[2]) != 0u, sht = tn >= SHC;
      TS(2 + 2 * ui);
      unsigned gv = 1u;
      const int ng = __builtin_amdgcn_readfirstlane((int)!(wc == 1 && ui == 2 && has_next));
      {
        const int ln = fresh_lane();
        const unsigned vo = (unsigned)((tm * 256 + wc * 32 + 4 * (ln >> 4)) * N_ + tn * 256 + wr * 64 + 4 * (ln & 15)) * 4u;
        const float qnan = __builtin_nanf("");
        const float sc_ = dead ? qnan : scale, lo_ = dead ? qnan : 0.f;
        const int shs = __builtin_amdgcn_readfirstlane(sht ? 1 : ui < PLN ? 2 : 0);
        unsigned char* const ob = p.out[rank];
        if (wr == 1) {
          const unsigned co = WCT_OFF + 256u + (call & 1u) * 128u; unsigned char* const w_ = myws;
          asm volatile("s_cmp_lg_u32 %3, 0\n\ts_cbranch_scc1 GE_%=\n\ts_mov_b64 exec, 1\n\tglobal_atomic_add %0, %1, %0, %2 sc0\n\ts_mov_b64 exec, -1\nGE_%=:"
                       : "+v"(gv) : "v"(co), "s"(w_), "s"(ng) : "memory", "scc");
        }
#pragma unroll
        for (int bj = 0; bj < 2; ++bj)
#pragma unroll
          for (int n = 0; n < 2; ++n)
#pragma unroll
            for (int r4 = 0; r4 < 4; ++r4) {
              unsigned char* rowp = ob + (size_t)(bj * 128 + n * 16 + r4) * N_ * 4;
#pragma unroll
              for (int ai = 0; ai < 2; ++ai) {
                f32x4 v;
#pragma unroll
                for (int c = 0; c < 4; ++c) {
                  const float x = acc[ai][bj][c][n][r4] * sc_;
                  asm("v_max_f32 %0, %1, %2" : "=v"(v[c]) : "s"(lo_), "v"(x));
                }
                asm volatile("s_cmp_lg_u32 %4, 0\n\ts_cbranch_scc1 SW_%=\n\tglobal_store_dwordx4 %0, %1, %2 offset:%3" OSM "\n\ts_branch SE_%=\n"
                             "SW_%=:\n\tglobal_store_dwordx4 %0, %1, %2 offset:%3 sc0 sc1" SHM "\nSE_%=:\n\ts_nop 1"
                             :: "v"(vo), "v"(v), "s"(rowp), "n"(ai * 512), "s"(shs) : "memory", "scc");
              }
            }
      }
      if (!has_next && wr == 0) WAIT_V(32);
      if (wr == 1) {
        asm volatile("s_waitcnt vmcnt(32)" : "+v"(gv) :: "memory");
        if (!ng) { const unsigned q_ = __builtin_amdgcn_readfirstlane(gv); lctl[4] = q_ < NG ? q_ : NBLK; }
      }
      if (sht) {
        WAIT_V(0);
        unsigned o_ = 0;
        if (fresh_lane() == 0) o_ = __hip_atomic_fetch_add(lctl + 3, 1u, __ATOMIC_RELAXED, __HIP_MEMORY_SCOPE_WORKGROUP);
        if (__builtin_amdgcn_readfirstlane(o_) == 7u) store_flag1(myws, DONE_OFF + si * 64, call);
        if (wr == 0) WAIT_V(0);
      }
      TS(3 + 2 * ui);
      if (!has_next) break;
#pragma unroll
      for (int a = 0; a < 2; ++a)
#pragma unroll
        for (int c = 0; c < 2; ++c)
#pragma unroll
          for (int m = 0; m < 4; ++m)
#pragma unroll
            for (int n = 0; n < 2; ++n) acc[a][c][m][n] = (f32x4){0.f, 0.f, 0.f, 0.f};
    }
    if (wr == 0) BAR;
  };
  if (wid < 4) run(std::integral_constant<int, 0>{}); else run(std::integral_constant<int, 1>{});
  expand(p, lds, call, wid, __builtin_amdgcn_readfirstlane(lctl[4]), skip ? 0u : __builtin_amdgcn_readfirstlane(lctl[5]) ? 2u : 1u);
  TS(14);
}
extern "C" void kernel_launch(int rank, int nranks, ncclComm_t comm,
                              hipStream_t stream,
                              void* const* d_in, const int* in_sizes, int n_in,
                              void* d_out, int out_size,
                              const int* in_dtypes, int out_dtype,
                              void* const* d_out_all,
                              void* const* d_ws_all, size_t ws_bytes) {
  if (nranks != NR || (unsigned)rank >= (unsigned)NR || ws_bytes < WS_NEED || n_in < 4) {
    fprintf(stderr, "kernel_launch: unexpected configuration\n"); return;
  }
  static bool attr_done[64] = {};
  int dev = 0; HIP_CHECK(hipGetDevice(&dev));
  if (!attr_done[dev & 63]) {
    HIP_CHECK(hipFuncSetAttribute((const void*)fused_kernel, hipFuncAttributeMaxDynamicSharedMemorySize, LDS_BYTES));
    attr_done[dev & 63] = true;
  }
  Params p;
  p.x = (const unsigned char*)d_in[0]; p.w = (const unsigned char*)d_in[1];
  p.sx = (const float*)d_in[2]; p.sw = (const float*)d_in[3];
  for (int i = 0; i < NR; ++i) { p.ws[i] = (unsigned char*)d_ws_all[i]; p.out[i] = (unsigned char*)(i == rank ? d_out : d_out_all[i]); }
  p.rank = rank;
  fused_kernel<<<GRID, 512, LDS_BYTES, stream>>>(p);
}
```

```cpp
#include <hip/hip_runtime.h>
#include <rccl/rccl.h>
#include <cstdio>
#include <type_traits>

#define HIP_CHECK(expr) do { hipError_t _e = (expr); if (_e != hipSuccess) { \
    fprintf(stderr, "HIP error %s at %s:%d\n", hipGetErrorString(_e), __FILE__, __LINE__); \
    return; } } while (0)

typedef int v8i __attribute__((ext_vector_type(8)));
typedef int v4i __attribute__((ext_vector_type(4)));
typedef unsigned v4u __attribute__((ext_vector_type(4)));
typedef float f32x4 __attribute__((ext_vector_type(4)));
#define LAS __attribute__((address_space(3)))

constexpr int NR = 8;
constexpr int M_ = 4096, N_ = 16384, KL = 512;
constexpr int NKS = 32;
constexpr int HTB = 16384;
constexpr int STAGE_BYTES = 8 * HTB;
constexpr unsigned LCTL = STAGE_BYTES, LTAB = STAGE_BYTES + 256, LCP = STAGE_BYTES + 1024;
constexpr int LDS_BYTES = STAGE_BYTES + 2048;
constexpr unsigned SLAB = 32768;
constexpr unsigned IMG_BYTES = 2560u * SLAB;
constexpr unsigned IMG_OFF = 1u << 20;
constexpr unsigned PK_OFF = IMG_OFF + 2 * IMG_BYTES, PK_SLOT = 5120, PK_JOBS = 1280;
constexpr unsigned PK_BLK = 32 * PK_SLOT, NBLK = 7 * PK_JOBS / 32;
constexpr unsigned CW_OFF = PK_OFF + NR * (PK_JOBS / 32) * PK_BLK - 512u * SLAB, CW_BYTES = 2048u * SLAB;
constexpr size_t WS_NEED = (size_t)CW_OFF + 512u * SLAB + (size_t)CW_BYTES;
constexpr unsigned CTR_OFF = 0, TMO_OFF = 128, WCT_OFF = 256, DONE_OFF = 4096, OFLAG_OFF = 8192, FLAG_OFF = 65536, FLAG_BYTES = 2560u * 64u, DBG_OFF = 524288;
constexpr int GRID = 256;
constexpr int NPUSH = 16, NCOM = 32;
constexpr int NSH = 20, SHC = 54;
constexpr unsigned DEADV = 0x7fffffffu;
constexpr unsigned long long DEADLINE_TICKS = 90000000ull;
#define WPT 2

struct Params {
  const unsigned char* x;
  const unsigned char* w;
  const float* sx; const float* sw;
  unsigned char* ws[NR];
  unsigned char* out[NR];
  int rank;
};

#define TS(slot) do {} while (0)
#define TV(slot, v) do {} while (0)
__device__ __forceinline__ v8i cat8(v4i a, v4i b) { return __builtin_shufflevector(a, b, 0, 1, 2, 3, 4, 5, 6, 7); }
#define WAIT_V(n) asm volatile("s_waitcnt vmcnt(" #n ")" ::: "memory")
#define WAIT_L(n) asm volatile("s_waitcnt lgkmcnt(" #n ")" ::: "memory")
#define BAR __builtin_amdgcn_s_barrier()
#define SCHED __builtin_amdgcn_sched_barrier(0)
#define TIE16(R) "+v"(R[0]), "+v"(R[1]), "+v"(R[2]), "+v"(R[3]), "+v"(R[4]), "+v"(R[5]), "+v"(R[6]), "+v"(R[7]), \
                 "+v"(R[8]), "+v"(R[9]), "+v"(R[10]), "+v"(R[11]), "+v"(R[12]), "+v"(R[13]), "+v"(R[14]), "+v"(R[15])
#define TIE4(X) "+v"(X[0]), "+v"(X[1]), "+v"(X[2]), "+v"(X[3])

#define PKM ""
#define IMM ""
#define LDM ""
#define SHM ""
__device__ __forceinline__ int fresh_lane() { int l; asm volatile("v_mbcnt_lo_u32_b32 %0, -1, 0\n\tv_mbcnt_hi_u32_b32 %0, -1, %0" : "=v"(l)); return l; }

__device__ __forceinline__ void load_job(const Params& p, int tile, int ks, int jw, int lane, unsigned (&R)[16], v4u (&X)[4]) {
  const int i = lane & 15, g = lane >> 4;
  const bool isx = tile < 16, iso = tile >= 80;
  unsigned vo = isx ? i * KL + g * 32 : iso ? lane * 16 : g * 32 * N_ + 4 * i;
  const unsigned char* src = isx ? p.x + (size_t)(tile * 256 + (jw >> 1) * 64) * KL + ks * 128 + (jw & 1) * 16
                           : iso ? p.out[p.rank] + (unsigned)ks
                                 : p.w + (size_t)(ks * 128 + (jw & 1) * 16) * N_ + (tile - 16) * 256 + (jw >> 1) * 64;
  const unsigned st = isx ? 16 * KL : iso ? 4 * N_ : N_;
#define LD1(r) "global_load_dword %" #r ", %20, %21" LDM "\n\tv_add_u32 %20, %22, %20\n\t"
#define LD4(r) "global_load_dwordx4 %" #r ", %20, %21" LDM "\n\tv_add_u32 %20, %22, %20\n\t"
  asm volatile("s_cmp_eq_u32 %22, %23\n\ts_cbranch_scc1 LW_%=\n\t" LD4(16) LD4(17) LD4(18) LD4(19) "s_branch LE_%=\n"
               "LW_%=:\n\t" LD1(0) LD1(1) LD1(2) LD1(3) LD1(4) LD1(5) LD1(6) LD1(7) LD1(8) LD1(9) LD1(10) LD1(11) LD1(12) LD1(13) LD1(14) LD1(15) "\n"
               "LE_%=:"
               : "=&v"(R[0]), "=&v"(R[1]), "=&v"(R[2]), "=&v"(R[3]), "=&v"(R[4]), "=&v"(R[5]), "=&v"(R[6]), "=&v"(R[7]),
                 "=&v"(R[8]), "=&v"(R[9]), "=&v"(R[10]), "=&v"(R[11]), "=&v"(R[12]), "=&v"(R[13]), "=&v"(R[14]), "=&v"(R[15]),
                 "=&v"(X[0]), "=&v"(X[1]), "=&v"(X[2]), "=&v"(X[3]), "+v"(vo)
               : "s"(src), "s"(st), "n"(N_) : "memory", "scc");
}
__device__ __forceinline__ void retile(bool plain, const unsigned (&R)[16], const v4u (&X)[4], v4u (&D)[4]) {
  if (plain) {
#pragma unroll
    for (int e = 0; e < 4; ++e) D[e] = X[e];
  } else {
#pragma unroll
    for (int d = 0; d < 4; ++d) {
      const unsigned a = R[4 * d], bb = R[4 * d + 1], c = R[4 * d + 2], f = R[4 * d + 3];
      const unsigned t0_ = __builtin_amdgcn_perm(bb, a, 0x05010400u), t1_ = __builtin_amdgcn_perm(bb, a, 0x07030602u);
      const unsigned t2_ = __builtin_amdgcn_perm(f, c, 0x05010400u), t3_ = __builtin_amdgcn_perm(f, c, 0x07030602u);
      D[0][d] = __builtin_amdgcn_perm(t2_, t0_, 0x05040100u); D[1][d] = __builtin_amdgcn_perm(t2_, t0_, 0x07060302u);
      D[2][d] = __builtin_amdgcn_perm(t3_, t1_, 0x05040100u); D[3][d] = __builtin_amdgcn_perm(t3_, t1_, 0x07060302u);
    }
  }
}
__device__ __forceinline__ void store_job(unsigned char* ws, unsigned vo, const v4u (&D)[4]) {
  asm volatile("global_store_dwordx4 %0, %1, %5 offset:-4096 sc0 sc1\n\tglobal_store_dwordx4 %0, %2, %5 offset:-2048 sc0 sc1\n\t"
               "global_store_dwordx4 %0, %3, %5 sc0 sc1\n\tglobal_store_dwordx4 %0, %4, %5 offset:2048 sc0 sc1\n\ts_nop 1"
               :: "v"(vo), "v"(D[0]), "v"(D[1]), "v"(D[2]), "v"(D[3]), "s"(ws) : "memory");
}
__device__ __forceinline__ void store_flag1(unsigned char* ws, unsigned fo, unsigned v) {
  asm volatile("s_mov_b64 exec, 1\n\tglobal_store_dword %0, %1, %2 sc0 sc1\n\ts_mov_b64 exec, -1" :: "v"(fo), "v"(v), "s"(ws) : "memory");
}
__device__ __forceinline__ void store_flag7(unsigned long long fa, unsigned v) {
  asm volatile("s_mov_b64 exec, 0x7f\n\tglobal_store_dword %0, %1, off sc0 sc1\n\ts_mov_b64 exec, -1" :: "v"(fa), "v"(v) : "memory");
}
__device__ __forceinline__ unsigned job_off(int jw, int lane) { return (jw >> 1) * 8192 + (jw & 1) * 1024 + lane * 16; }

__device__ __forceinline__ void stream_slab(int j, int snd, int& tile, int& ks) {
  if (j < 120) { ks = j / 30; int q = j - ks * 30; if (q < 14) { q += 2 * snd; if (q >= 14) q -= 14; tile = 16 + q; } else tile = (q - 14 + 2 * snd) & 15; }
  else if (j < 168) { const int u = j - 120; ks = u / 12; int q = u - ks * 12 + snd; if (q >= 12) q -= 12; tile = 30 + q; }
  else { const int u = j - 168, gq = u >= 56, v = u - gq * 56; ks = v / 14; int q = v - ks * 14 + 2 * snd; if (q >= 14) q -= 14; tile = 42 + 14 * gq + q; }
}
__device__ __forceinline__ int stream_idx(int tile, int ks, int snd) {
  if (tile < 16) return ks * 30 + 14 + ((tile - 2 * snd) & 15);
  if (tile < 30) { int q = tile - 16 - 2 * snd; if (q < 0) q += 14; return ks * 30 + q; }
  if (tile < 42 || tile >= 70) { int q = (tile < 42 ? tile - 30 : tile - 70) - snd; if (q < 0) q += 12; return 120 + ks * 12 + q; }
  const int gq = tile >= 56; int q = tile - 42 - 14 * gq - 2 * snd; if (q < 0) q += 14; return 168 + gq * 56 + ks * 14 + q;
}
__device__ __forceinline__ unsigned shared_off(int r, int i) { const int idx = NSH * r + i; return (unsigned)((idx & 15) * 256 * N_ + (SHC + (idx >> 4)) * 256) * 4u; }

#define INITK 0u
#define LAG2 120
#define LAG 1
#define PWIN 4u
#define PTH 8u
#define INV 0
#define LASTWT 0
#define OST 1
#define OSM " nt"
#define PLN 0
#define XST OST
#define XSM " nt"
#define PSLEEP 8
__device__ __forceinline__ bool wait_flags(unsigned char* ws, unsigned fo, unsigned call) {
  const unsigned long long t0 = __builtin_amdgcn_s_memrealtime();
  unsigned* tmo = (unsigned*)(ws + TMO_OFF);
  for (;;) {
    unsigned v, tm; const unsigned to = TMO_OFF;
    asm volatile("global_load_dword %0, %2, %4 sc0 sc1\n\tglobal_load_dword %1, %3, %4 sc0 sc1\n\ts_waitcnt vmcnt(0)" : "=&v"(v), "=&v"(tm) : "v"(fo), "v"(to), "s"(ws) : "memory");
    if (__ballot(v != call) == 0ull) return true;
    if (__builtin_amdgcn_readfirstlane(tm) == call || __builtin_amdgcn_s_memrealtime() - t0 > DEADLINE_TICKS) {
      __hip_atomic_store(tmo, call, __ATOMIC_RELAXED, __HIP_MEMORY_SCOPE_AGENT);
      return false;
    }
    __builtin_amdgcn_s_sleep(16);
  }
}
constexpr int NTURN = 8 / WPT;
#define NPA 16
constexpr int SPT = NPA * WPT / 8, NT = 280 / SPT;
__device__ __forceinline__ unsigned pk_off(int r, unsigned jj) { return shared_off(r, jj >> 6) + ((jj & 63u) * 4u) * (N_ * 4u); }
#define ST1(vc, d, k, md) "global_store_dwordx4 " vc ", " d ", %" #k " sc0 sc1" md "\n\t"
#define ST7(vc, d, md) ST1(vc, d, 2, md) ST1(vc, d, 3, md) ST1(vc, d, 4, md) ST1(vc, d, 5, md) ST1(vc, d, 6, md) ST1(vc, d, 7, md) ST1(vc, d, 8, md)
__device__ __forceinline__ unsigned bfi(unsigned m, unsigned a, unsigned b) { return (a & m) | (b & ~m); }
__device__ __forceinline__ unsigned encode(v4u (&S)[4], LAS unsigned char* el, int lane, unsigned long long& em) {
  unsigned sm = 0;
#pragma unroll
  for (int d = 0; d < 4; ++d) sm = __builtin_amdgcn_udot4(S[0][d] & 0x7c7c7c7cu, 0x01010101u, sm, false);
  sm += __builtin_amdgcn_update_dpp(0, (int)sm, 0x111, 0xf, 0xf, true); sm += __builtin_amdgcn_update_dpp(0, (int)sm, 0x112, 0xf, 0xf, true);
  sm += __builtin_amdgcn_update_dpp(0, (int)sm, 0x114, 0xf, 0xf, true); sm += __builtin_amdgcn_update_dpp(0, (int)sm, 0x118, 0xf, 0xf, true);
  int e0 = (__builtin_amdgcn_readlane(sm, 15) + __builtin_amdgcn_readlane(sm, 31) + __builtin_amdgcn_readlane(sm, 47) + __builtin_amdgcn_readlane(sm, 63) - 17121) >> 12;
  e0 = e0 < 0 ? 0 : e0 > 24 ? 24 : e0;
  const unsigned c1 = (unsigned)(128 - 4 * e0) * 0x01010101u, ix = (unsigned)((lane & 7) * 4 | (lane >> 3) << 13);
  unsigned cnt = 0, hdr = 0; v4u U[4];
#pragma unroll
  for (int c = 0; c < 4; ++c) {
#pragma unroll
    for (int d = 0; d < 4; ++d) {
      const unsigned v = S[c][d], u = (v & 0x7f7f7f7fu) + c1;
      const bool esc = (u & 0xe0e0e0e0u) != 0x80808080u;
      const unsigned long long m = __ballot(esc);
      const unsigned pos = __builtin_amdgcn_mbcnt_hi((unsigned)(m >> 32), __builtin_amdgcn_mbcnt_lo((unsigned)m, cnt));
      *(LAS unsigned short*)(el + 4 + 2 * (esc ? pos : 1100u + (unsigned)lane)) = (unsigned short)(__builtin_amdgcn_udot4(v & 0x60606060u, 0x40100401u, ix, false) + d);
      cnt += (unsigned)__builtin_popcountll(m);
      U[c][d] = bfi(0x80808080u, v, u);
    }
    hdr |= (cnt & 127u) << (8 * c);
  }
  em = ~0ull;
  if (cnt > 126u) return 32u;
  if (lane == 0) *(LAS unsigned*)el = hdr;
#pragma unroll
  for (int d = 0; d < 4; ++d) {
    const unsigned z = U[3][d];
    S[0][d] = bfi(0x60606060u, z << 4, U[0][d]); S[1][d] = bfi(0x60606060u, z << 2, U[1][d]); S[2][d] = bfi(0x60606060u, __builtin_amdgcn_alignbit(z, z, 2), U[2][d]);
  }
  S[3] = *(const LAS v4u*)(el + (lane & 15) * 16);
  em = (1ull << ((cnt + 33u) >> 5) * 4u) - 1ull;
  return (unsigned)e0;
}
__device__ __forceinline__ void pusher(const Params& p, LAS unsigned char* lds, unsigned imgo, unsigned flo, unsigned call, int b, int wid, int lane, unsigned (&R)[16], v4u (&X)[4]) {
  unsigned char* const myws = p.ws[p.rank];
  const int tau = wid / WPT, e = tau * SPT + b / NTURN, jw = (b % NTURN) * WPT + wid % WPT;
  unsigned long long fa = 0;
  unsigned char* wsd[7]; int pr[7];
#pragma unroll
  for (int k = 0; k < 7; ++k) {
    pr[k] = (p.rank + 1 + (wid + b + k) % 7) & 7; wsd[k] = p.ws[pr[k]];
    if (lane == k) fa = (unsigned long long)wsd[k];
  }
  const unsigned jo = job_off(jw, lane), dno = DONE_OFF + (lane < NSH ? lane : 0) * 64u, cwo = CW_OFF;
  LAS unsigned* const pk = (LAS unsigned*)(lds + wid * PK_SLOT);
  int tile, ks; stream_slab(e, p.rank, tile, ks);
  unsigned job = e * 8 + jw, cd = 32u;
  unsigned long long em = ~0ull;
  v4u D[4];
  retile(tile < 16, R, X, D);
  if (tile >= 16) cd = encode(D, (LAS unsigned char*)pk, lane, em);
  v4u D2[4]; unsigned cd2 = 0u; unsigned long long em2 = 0ull;
  for (int i = 0; i < tau; ++i) BAR;
  for (int T = tau; T < NT; T += NTURN) {
    const unsigned sid = tile * 32 + p.rank * 4 + ks, vb = imgo + jo + sid * SLAB;
    const unsigned fo = flo + sid * 64u + jw * 4u + (tile >= 16 ? 32u : 0u), fv = tile >= 16 ? call << 6 | cd : call;
    const bool nar = tile >= 30 && tile < 40;
    unsigned long long fn = 0;
    if (nar) {
      const int ra = (tile - 30) * 16 / NSH, rb = ((tile - 30) * 16 + 15) / NSH;
      unsigned char* const da = ra != p.rank ? p.ws[ra & 7] : myws + (cwo - imgo);
      unsigned char* const db = rb != p.rank && rb != ra ? p.ws[rb & 7] : myws + (cwo - imgo);
      fn = (unsigned long long)(lane ? p.ws[ra & 7] : p.ws[rb & 7]) + fo + 40u * 32u * 64u;
#pragma unroll
      for (int c = 0; c < 4; ++c) {
        const unsigned vc = vb + 40u * 32u * SLAB + c * 2048u;
        const unsigned long long em_ = c == 3 ? em2 : ~0ull;
        asm volatile("s_mov_b64 exec, %4\n\t" ST1("%0", "%1", 2, "") ST1("%0", "%1", 3, "") "s_mov_b64 exec, -1\n\ts_nop 1" :: "v"(vc), "v"(D2[c]), "s"(da), "s"(db), "s"(em_) : "memory");
      }
    }
    stream_slab((T + NTURN < NT ? T + NTURN : T) * SPT + b / NTURN, p.rank, tile, ks);
    load_job(p, tile, ks, jw, lane, R, X);
#pragma unroll
    for (int c = 0; c < 4; ++c) {
      const unsigned vc = vb + c * 2048u;
      const unsigned long long em_ = c == 3 ? em : ~0ull;
      asm volatile("s_mov_b64 exec, %9\n\t" ST7("%0", "%1", IMM) "s_mov_b64 exec, -1\n\ts_nop 1"
                   :: "v"(vc), "v"(D[c]), "s"(wsd[0]), "s"(wsd[1]), "s"(wsd[2]), "s"(wsd[3]), "s"(wsd[4]), "s"(wsd[5]), "s"(wsd[6]), "s"(em_) : "memory");
    }
    const int nb = NT - 1 - T < NTURN ? NT - 1 - T : NTURN;
    const int lag2 = __builtin_amdgcn_readfirstlane((int)(T * SPT >= LAG2));
    asm volatile("s_cmp_eq_u32 %20, 0\n\ts_cbranch_scc1 PB_%=\n\ts_barrier\nPB_%=:\n\ts_waitcnt vmcnt(28)\n\ts_cmp_lg_u32 %21, 0\n\ts_cbranch_scc1 PC_%=\n\ts_waitcnt vmcnt(0)\nPC_%=:"
                 : TIE16(R), TIE4(X) : "s"(nb), "s"(lag2) : "memory", "scc");
    if (nar) asm volatile("s_mov_b64 exec, 3\n\tglobal_store_dword %0, %1, off sc0 sc1\n\ts_mov_b64 exec, -1" :: "v"(fn), "v"(call << 6 | cd2) : "memory");
    if (!lag2) { store_flag7(fa + fo, fv); if (nb > 1 && NTURN > 2) BAR; }
    retile(tile < 16, R, X, D);
    cd = 32u; em = ~0ull;
    if (tile >= 16) cd = encode(D, (LAS unsigned char*)pk, lane, em);
    if (lag2) { if (nb > 1 && NTURN > 2) BAR; WAIT_V(0); store_flag7(fa + fo, fv); }
    if (tile >= 30 && tile < 40) {
      load_job(p, tile + 40, ks, jw, lane, R, X);
      asm volatile("s_waitcnt vmcnt(0)" : TIE16(R), TIE4(X) :: "memory");
      retile(false, R, X, D2);
      cd2 = encode(D2, (LAS unsigned char*)pk, lane, em2);
    }
    for (int i = NTURN > 2 ? 2 : 1; i < nb; ++i) BAR;
  }
  const bool ok = wait_flags(myws, dno, call);
  load_job(p, 80, (int)pk_off(p.rank, job), jw, lane, R, X);
  asm volatile("s_waitcnt vmcnt(0)" : TIE16(R), TIE4(X) :: "memory");
  for (;;) {
    unsigned nx = 1u; const unsigned co = WCT_OFF + 64u + (call & 1u) * 128u;
    asm volatile("s_mov_b64 exec, 1\n\tglobal_atomic_add %0, %1, %0, %2 sc0\n\ts_mov_b64 exec, -1" : "+v"(nx) : "v"(co), "s"(myws) : "memory");
    unsigned cnt = 32u, hdr = 0u;
#pragma unroll
    for (int j = 0; j < 16; ++j) {
      const unsigned v = ok ? X[j >> 2][j & 3] : 0x7fc00000u;
      const unsigned long long m = __ballot(v != 0u);
      const unsigned lo = (unsigned)m, hi = (unsigned)(m >> 32);
      pk[v != 0u ? __builtin_amdgcn_mbcnt_hi(hi, __builtin_amdgcn_mbcnt_lo(lo, cnt)) : 1100u + (unsigned)lane] = v;
      hdr = lane == 2 * j ? lo : lane == 2 * j + 1 ? hi : hdr;
      cnt += (unsigned)__builtin_popcountll(m);
    }
    pk[lane < 32 ? (unsigned)lane : 1100u + (unsigned)lane] = hdr;
    const unsigned nd = (cnt + 31u) & ~31u, pj = p.rank * PK_JOBS + job;
#pragma unroll
    for (int c = 0; c < 5; ++c) {
      const int nl = (int)(nd >> 2) - c * 64;
      if (nl > 0) {
        const v4u d = *(const LAS v4u*)(pk + c * 256 + lane * 4);
        const unsigned vc = PK_OFF + (pj >> 5) * PK_BLK + (c < 3 ? (pj & 31u) * 3072u : 95232u + (pj & 31u) * 2048u) + c * 1024u + lane * 16u;
        const unsigned long long em = nl >= 64 ? ~0ull : (1ull << nl) - 1ull;
        asm volatile("s_mov_b64 exec, %9\n\t" ST7("%0", "%1", PKM) "s_mov_b64 exec, -1\n\ts_nop 1"
                     :: "v"(vc), "v"(d), "s"(wsd[0]), "s"(wsd[1]), "s"(wsd[2]), "s"(wsd[3]), "s"(wsd[4]), "s"(wsd[5]), "s"(wsd[6]), "s"(em) : "memory");
      }
    }
    asm volatile("s_waitcnt vmcnt(0)" : "+v"(nx) :: "memory");
    job = __builtin_amdgcn_readfirstlane(nx) + NPA * 8u;
    const bool more = job < PK_JOBS;
    {
      unsigned vo = lane * 16u; const unsigned char* src = p.out[p.rank] + pk_off(p.rank, more ? job : 0u);
      asm volatile("s_cmp_ge_u32 %6, %8\n\ts_cbranch_scc1 OE_%=\n\tglobal_load_dwordx4 %0, %4, %5\n\tv_add_u32 %4, %7, %4\n\tglobal_load_dwordx4 %1, %4, %5\n\tv_add_u32 %4, %7, %4\n\t"
                   "global_load_dwordx4 %2, %4, %5\n\tv_add_u32 %4, %7, %4\n\tglobal_load_dwordx4 %3, %4, %5\nOE_%=:"
                   : "=&v"(X[0]), "=&v"(X[1]), "=&v"(X[2]), "=&v"(X[3]), "+v"(vo) : "s"(src), "s"(job), "n"(N_ * 4), "n"(PK_JOBS) : "memory", "scc");
    }
    store_flag7(fa + OFLAG_OFF + pj * 4u, call);
    if (!more) break;
    asm volatile("s_waitcnt vmcnt(1)" : TIE4(X) :: "memory");
  }
  TS(1); TS(9);
}
constexpr unsigned NUNIT = 196, UTAB = 120 * 1024;
#define UDLY 12
#define UWIN 24u
__device__ __forceinline__ void unpacker(const Params& p, LAS unsigned char* lds, unsigned imgo, unsigned flo, unsigned call, int b, int wid, int lane) {
  unsigned char* const myws = p.ws[p.rank];
  LAS unsigned* lctl = (LAS unsigned*)(lds + LCTL);
  const unsigned long long t0 = __builtin_amdgcn_s_memrealtime();
  if (wid == 0) {
    unsigned i = lane, done = 0;
    for (;;) {
      const unsigned q = (i < NUNIT ? i : 0u) * 16u + (b - NPUSH), wp = q / 14u, r = q - wp * 14u;
      const int snd = (p.rank + 1 + (r >> 1)) & 7;
      const int k14 = wp / 14u, gp = k14 >> 2, ks = k14 & 3, q14 = wp - k14 * 14;
      int qq = q14 + (gp == 1 ? snd : 2 * snd); qq -= qq >= (gp == 1 ? 12 : 14) ? (gp == 1 ? 12 : 14) : 0;
      const int tile = gp == 1 && q14 >= 12 ? 58 + ((NSH * p.rank) >> 4) + q14 : (gp == 0 ? 16 : 14 * gp + (gp == 1 ? 16 : 14)) + qq;
      const unsigned hs = (tile * 32 + snd * 4 + ks) * 2 + (r & 1u), fo = flo + (hs >> 1) * 64u + 32u + (hs & 1u) * 16u, to = TMO_OFF;
      const unsigned long long pm = __ballot(i < NUNIT && i < done + UWIN);
      v4u v = {0u, 0u, 0u, 0u}; unsigned tm;
      asm volatile("s_mov_b64 exec, %5\n\tglobal_load_dwordx4 %0, %2, %4 sc0 sc1\n\ts_mov_b64 exec, -1\n\tglobal_load_dword %1, %3, %4 sc0 sc1\n\ts_waitcnt vmcnt(0)"
                   : "+v"(v), "=&v"(tm) : "v"(fo), "v"(to), "s"(myws), "s"(pm) : "memory");
      const unsigned tag = call & 0x3ffffffu;
      const bool ok = (pm >> lane & 1ull) && v[0] >> 6 == tag && v[1] >> 6 == tag && v[2] >> 6 == tag && v[3] >> 6 == tag;
      if (ok) {
        *(LAS unsigned long long*)(lds + UTAB + i * 8u) = (unsigned long long)((v[0] & 63u) | (v[1] & 63u) << 8 | (v[2] & 63u) << 16 | (v[3] & 63u) << 24) << 32 | hs;
        i += 64u;
      }
      const unsigned nk = (unsigned)__builtin_popcountll(__ballot(ok));
      done += nk;
      if (done >= NUNIT) break;
      if (__builtin_amdgcn_readfirstlane(tm) == call || __builtin_amdgcn_s_memrealtime() - t0 > DEADLINE_TICKS) {
        __hip_atomic_store((unsigned*)(myws + TMO_OFF), call, __ATOMIC_RELAXED, __HIP_MEMORY_SCOPE_AGENT);
        lctl[2] = 1u;
        break;
      }
      if (!nk) __builtin_amdgcn_s_sleep(4);
    }
    TS(3);
    return;
  }
  const unsigned cwo = CW_OFF, lb = (unsigned)(size_t)lds + (wid - 1) * 16640u, msk = 0x60606060u;
  LAS unsigned char* const buf = lds + (wid - 1) * 16640u;
  unsigned pfo = DONE_OFF + 4032u, pend = 0;
  const v4u cv = {call, call, call, call};
#define UFLAG asm volatile("s_mov_b64 exec, 1\n\tglobal_store_dwordx4 %0, %1, %2 sc0 sc1\n\ts_mov_b64 exec, -1\n\ts_nop 1" :: "v"(pfo), "v"(cv), "s"(myws) : "memory")
  for (unsigned i = wid - 1; i < NUNIT; i += 7u) {
    unsigned hs, cds;
    for (;;) {
      const unsigned long long e = __hip_atomic_load((LAS unsigned long long*)(lds + UTAB + i * 8u), __ATOMIC_RELAXED, __HIP_MEMORY_SCOPE_WORKGROUP);
      hs = __builtin_amdgcn_readfirstlane((unsigned)e); cds = __builtin_amdgcn_readfirstlane((unsigned)(e >> 32));
      if (hs) break;
      if (pend) { WAIT_V(0); UFLAG; pend = 0; continue; }
      if (__builtin_amdgcn_readfirstlane(__hip_atomic_load(lctl + 2, __ATOMIC_RELAXED, __HIP_MEMORY_SCOPE_WORKGROUP)) || __builtin_amdgcn_s_memrealtime() - t0 > 2 * DEADLINE_TICKS) return;
      __builtin_amdgcn_s_sleep(2);
    }
    const unsigned so = (hs >> 1) * SLAB + (hs & 1u) * 16384u;
    v4u C[4][3]; unsigned E[4];
    {
      unsigned va = imgo + so + lane * 16u + 4096u, ve = imgo + so + lane * 4u + 4096u;
#define UL(j, o) "global_load_dwordx4 %" #j ", %16, %18 offset:" #o " sc0 sc1\n\t"
      asm volatile(UL(0, -4096) UL(1, -2048) UL(2, 0) "global_load_dword %12, %17, %18 offset:2048 sc0 sc1\n\t"
                   UL(3, -3072) UL(4, -1024) UL(5, 1024) "global_load_dword %13, %17, %18 offset:3072 sc0 sc1\n\t"
                   "v_add_u32 %16, 0x2000, %16\n\tv_add_u32 %17, 0x2000, %17\n\t"
                   UL(6, -4096) UL(7, -2048) UL(8, 0) "global_load_dword %14, %17, %18 offset:2048 sc0 sc1\n\t"
                   UL(9, -3072) UL(10, -1024) UL(11, 1024) "global_load_dword %15, %17, %18 offset:3072 sc0 sc1\n\t"
                   "s_waitcnt vmcnt(16)"
                   : "=&v"(C[0][0]), "=&v"(C[0][1]), "=&v"(C[0][2]), "=&v"(C[1][0]), "=&v"(C[1][1]), "=&v"(C[1][2]),
                     "=&v"(C[2][0]), "=&v"(C[2][1]), "=&v"(C[2][2]), "=&v"(C[3][0]), "=&v"(C[3][1]), "=&v"(C[3][2]),
                     "=&v"(E[0]), "=&v"(E[1]), "=&v"(E[2]), "=&v"(E[3]), "+v"(va), "+v"(ve) : "s"(myws) : "memory");
    }
    UFLAG;
#pragma unroll
    for (int j = 0; j < 4; ++j) {
      asm volatile("s_waitcnt vmcnt(13)" : "+v"(C[j][0]), "+v"(C[j][1]), "+v"(C[j][2]), "+v"(E[j]) :: "memory");
      const unsigned cd = cds >> (8 * j) & 63u, c2 = (cd & 31u) * 0x04040404u;
      v4u D[4];
#pragma unroll
      for (int d = 0; d < 4; ++d) {
        const unsigned t0_ = C[j][0][d] & msk, t1_ = C[j][1][d] & msk, t2_ = C[j][2][d] & msk;
        asm("v_xad_u32 %0, %1, %2, %3" : "=v"(D[0][d]) : "v"(C[j][0][d]), "v"(t0_), "s"(c2));
        asm("v_xad_u32 %0, %1, %2, %3" : "=v"(D[1][d]) : "v"(C[j][1][d]), "v"(t1_), "s"(c2));
        asm("v_xad_u32 %0, %1, %2, %3" : "=v"(D[2][d]) : "v"(C[j][2][d]), "v"(t2_), "s"(c2));
        D[3][d] = __builtin_amdgcn_alignbit(t2_, t2_, 30) + (t1_ >> 2) + (t0_ >> 4) + c2;
      }
#pragma unroll
      for (int c = 0; c < 4; ++c) *(LAS v4u*)(buf + j * 4096 + c * 1024 + lane * 16) = D[c];
      const unsigned h = __builtin_amdgcn_readfirstlane(E[j]);
#pragma unroll
      for (int k = 0; k < 2; ++k) {
        const unsigned en = k ? E[j] >> 16 : E[j], ps = 2u * lane - 2u + k, y = en & 0x1fe0u, t = y | y << 6;
        const unsigned ch = (ps >= (h & 127u)) + (ps >= (h >> 8 & 127u)) + (ps >= (h >> 16 & 127u));
        const unsigned ad = lb + (ps < (h >> 24 & 127u) ? j * 4096u + ch * 1024u + (en & 31u) * 4u + (en >> 13 & 7u) * 128u : 16384u + lane * 4u), mv = (t | t << 12) & msk;
        asm volatile("ds_mskor_b32 %0, %1, %2" :: "v"(ad), "v"(msk), "v"(mv) : "memory");
      }
#pragma unroll
      for (int c = 0; c < 4; ++c) D[c] = *(const LAS v4u*)(buf + j * 4096 + c * 1024 + lane * 16);
      store_job(myws, (cd < 32u ? imgo : cwo) + so + (j >> 1) * 8192u + (j & 1) * 1024u + lane * 16u + 4096u, D);
    }
    pfo = flo + (hs >> 1) * 64u + (hs & 1u) * 16u; pend = 1;
  }
  WAIT_V(0); UFLAG;
}
#define NG 120u
#define N1 100u
#define XCNT 768u
__device__ __forceinline__ unsigned blk_job(int rank, unsigned q) { const unsigned bl = q / 7u; return ((rank + 1 + (q - bl * 7u)) & 7u) * PK_JOBS + bl * 32u; }
__device__ __forceinline__ void expand(const Params& p, LAS unsigned char* lds, unsigned call, int wid, unsigned q, unsigned mode) {
  unsigned char* const myws = p.ws[p.rank];
  LAS unsigned* lctl = (LAS unsigned*)(lds + LCTL);
  const int lane = fresh_lane();
  const unsigned lbase = (unsigned)(size_t)lds;
  for (unsigned r = 0;; ++r) {
    if (mode == 0u) {
      if (threadIdx.x == 0) lctl[6 + (r & 1u)] = __hip_atomic_fetch_add((unsigned*)(myws + WCT_OFF + (call & 1u) * 128u), 1u, __ATOMIC_RELAXED, __HIP_MEMORY_SCOPE_AGENT);
      __syncthreads();
      q = __builtin_amdgcn_readfirstlane(lctl[6 + (r & 1u)]); q += q < N1 ? 0u : NG;
    }
    if (q >= NBLK) break;
    const unsigned pj = blk_job(p.rank, q), snd = pj / PK_JOBS, vb = PK_OFF + (pj >> 5) * PK_BLK + lane * 16u;
    bool ok = true;
    if (mode != 2u) {
      ok = wait_flags(myws, OFLAG_OFF + (pj + wid + (lane & 24)) * 4u, call);
#pragma unroll
      for (int h = 0; h < 4; ++h) {
        const unsigned la = lbase + (wid + h * 8) * 3072u + (h >> 1) * 16384u, vo = vb + (wid + h * 8) * 3072u;
        asm volatile("s_mov_b32 m0, %0\n\ts_nop 0\n\tglobal_load_lds_dwordx4 %1, %2 sc0 sc1\n\tglobal_load_lds_dwordx4 %1, %2 offset:1024 sc0 sc1\n\t"
                     "global_load_lds_dwordx4 %1, %2 offset:2048 sc0 sc1" :: "s"(la), "v"(vo), "s"(myws) : "memory");
      }
      WAIT_V(0);
    }
#pragma unroll 2
    for (int h = 0; h < 4; ++h) {
      const unsigned j = wid + h * 8, oo = pk_off(snd, pj - snd * PK_JOBS + j) + lane * 16u, so = j * 3072u + (h >> 1) * 16384u;
      const LAS unsigned* pk = (const LAS unsigned*)(lds + so);
      const unsigned hdr = pk[lane & 31];
      unsigned pos[16], cnt = 32u; unsigned long long m[16];
#pragma unroll
      for (int i = 0; i < 16; ++i) {
        const unsigned lo = __builtin_amdgcn_readlane(hdr, 2 * i), hi = __builtin_amdgcn_readlane(hdr, 2 * i + 1);
        m[i] = (unsigned long long)hi << 32 | lo;
        pos[i] = __builtin_amdgcn_mbcnt_hi(hi, __builtin_amdgcn_mbcnt_lo(lo, cnt));
        cnt += (unsigned)__builtin_popcountll(m[i]);
      }
      if (cnt > XCNT) {
        const unsigned s2 = 49152u + wid * 2048u, la = lbase + s2, vo = vb + 98304u + j * 2048u, d2 = (s2 - so - 3072u) >> 2;
        asm volatile("s_waitcnt lgkmcnt(0)\n\ts_mov_b32 m0, %0\n\ts_nop 0\n\tglobal_load_lds_dwordx4 %1, %2 sc0 sc1\n\tglobal_load_lds_dwordx4 %1, %2 offset:1024 sc0 sc1\n\ts_waitcnt vmcnt(0)"
                     :: "s"(la), "v"(vo), "s"(myws) : "memory");
#pragma unroll
        for (int i = 0; i < 16; ++i) pos[i] = (pos[i] + (pos[i] >= 768u ? d2 : 0u)) & 0x3fffffffu;
      }
#pragma unroll
      for (int rr = 0; rr < 4; ++rr) {
        v4u v;
#pragma unroll
        for (int c = 0; c < 4; ++c) {
          const unsigned x = *(const LAS unsigned*)(lds + (so + pos[4 * rr + c] * 4u));
          v[c] = !ok ? 0x7fc00000u : (m[4 * rr + c] >> lane & 1ull) ? x : 0u;
        }
        const unsigned vo_ = oo + rr * (N_ * 4u);
        asm volatile("global_store_dwordx4 %0, %1, %2" XSM "\n\ts_nop 1" :: "v"(vo_), "v"(v), "s"(p.out[p.rank]) : "memory");
      }
    }
    if (mode) break;
  }
}

__global__ __launch_bounds__(512, 2) void fused_kernel(const Params p) {
  extern __shared__ __attribute__((aligned(16))) unsigned char smem[];
  LAS unsigned char* lds = (LAS unsigned char*)smem;
  const int tid = threadIdx.x, wid = __builtin_amdgcn_readfirstlane(tid >> 6), wc = wid & 3;
  unsigned char* myws = p.ws[p.rank];
  const int b = blockIdx.x, rank = p.rank, g = b - NCOM, gl = b - NPUSH;
  const bool push = b < NPUSH;
  LAS unsigned* lctl = (LAS unsigned*)(lds + LCTL);
  const unsigned lbase = (unsigned)(size_t)lds;
  const int xc = b & 7, L = (xc >> 2) * 28 + (g >> 3), cn = L >> 2;
  const int pm = (xc & 3) * 4 + (L & 3), si = L * 4 + (xc & 3);
  const bool skip = cn < 2 && si >= NSH;
  const unsigned nvis = skip ? 96u : 128u;
  auto tile_of = [&](int u, int& tm, int& tn) __attribute__((always_inline)) {
    const int v = u + (skip && u > 0), idx = NSH * rank + si;
    const bool sh = v == 1 && cn < 2;
    tm = sh ? idx & 15 : pm; tn = sh ? SHC + (idx >> 4) : v == 0 ? cn : 14 * v - 2 + cn;
  };
  unsigned call, imgo, flo;
  {
    const int lane = tid & 63;
    unsigned R[16], R2[16]; v4u X[4], X2[4];
    int t1, k1, j1 = wid;
    if (push) { j1 = (b % NTURN) * WPT + wid % WPT; stream_slab(wid / WPT * SPT + b / NTURN, rank, t1, k1); } else { t1 = gl >> 2; k1 = gl & 3; }
    load_job(p, t1, k1, j1, lane, R, X);
    const bool two = !push && gl < 80;
    if (two) load_job(p, t1 + 60, k1, wid, lane, R2, X2);
    if (tid == 0) {
      const unsigned t = __hip_atomic_fetch_add((unsigned*)(myws + CTR_OFF), 1u, __ATOMIC_RELAXED, __HIP_MEMORY_SCOPE_AGENT);
      lctl[0] = t / (unsigned)GRID + 1u; lctl[1] = INITK; lctl[2] = 0u; lctl[3] = 0u; lctl[5] = 0u;
      if (b == 0) {
        __hip_atomic_store((unsigned*)(myws + WCT_OFF + (t / (unsigned)GRID & 1u) * 128u), 0u, __ATOMIC_RELAXED, __HIP_MEMORY_SCOPE_AGENT);
        __hip_atomic_store((unsigned*)(myws + WCT_OFF + 64u + (t / (unsigned)GRID & 1u) * 128u), 0u, __ATOMIC_RELAXED, __HIP_MEMORY_SCOPE_AGENT);
        __hip_atomic_store((unsigned*)(myws + WCT_OFF + 256u + (t / (unsigned)GRID & 1u) * 128u), 0u, __ATOMIC_RELAXED, __HIP_MEMORY_SCOPE_AGENT);
      }
    }
    if (!push && g < 0 && tid < (int)NUNIT) *(LAS unsigned long long*)(lds + UTAB + tid * 8) = 0ull;
    if (wid >= 4 && g >= 0) {
      const int u = wid - 4, l = lane & 31, src = l >> 2, ks = l & 3;
      int tm, tn; tile_of(u, tm, tn);
      const int ax = stream_idx(tm, ks, src), aw = stream_idx(16 + tn, ks, src) + UDLY;
#define CORD 9
      const int key = src == rank ? ks : u >= CORD ? 32 + ks * 8 + ((src - rank) & 7) : ((ax > aw ? ax : aw) + 1) * 32 + l;
      int rk = 0;
#pragma unroll
      for (int s = 0; s < 32; ++s) rk += __builtin_amdgcn_readlane(key, s) < key;
      const unsigned ent = (unsigned)((16 + tn) * 32 + l) << 15 | (unsigned)(tm * 32 + l);
      if (u * 32u < nvis) *(LAS unsigned*)(lds + LTAB + (u * 32 + rk) * 4) = ent;
      if (u == 0 && rk < 4) *(LAS unsigned*)(lds + LTAB + (nvis + rk) * 4) = ent;
    }
    asm volatile("s_waitcnt vmcnt(0)" : TIE16(R), TIE4(X) :: "memory");
    asm volatile("" : TIE16(R2) :: "memory");
    __syncthreads();
    call = __builtin_amdgcn_readfirstlane(lctl[0]);
    imgo = IMG_OFF + (call & 1u) * IMG_BYTES;
    flo = FLAG_OFF + (call & 1u) * FLAG_BYTES;
    TS(0);
    if (push) {
      if (b < NPA) pusher(p, lds, imgo, flo, call, b, wid, lane, R, X);
      WAIT_V(0);
      expand(p, lds, call, wid, 0u, 0u);
      TS(14);
      return;
    }
    v4u D[4];
    const unsigned s1 = t1 * 32 + rank * 4 + k1;
    retile(t1 < 16, R, X, D);
    store_job(myws, imgo + s1 * SLAB + job_off(wid, lane) + 4096, D);
    if (two) {
      retile(false, R2, X2, D);
      store_job(myws, imgo + (s1 + 60 * 32) * SLAB + job_off(wid, lane) + 4096, D);
    }
    WAIT_V(0);
    store_flag1(myws, flo + s1 * 64 + wid * 4, call);
    if (two) store_flag1(myws, flo + (s1 + 60 * 32) * 64 + wid * 4, call);
    TS(1);
    if (g < 0) {
      unpacker(p, lds, imgo, flo, call, b, wid, lane);
      __syncthreads();
      TS(4);
      expand(p, lds, call, wid, 0u, 0u);
      TS(14);
      return;
    }
  }
  const float scale = __builtin_bit_cast(float, __builtin_amdgcn_readfirstlane(__builtin_bit_cast(int, p.sx[0] * p.sw[0])));
  WAIT_V(0);

  auto run = [&](auto WRC) __attribute__((always_inline)) {
    constexpr int wr = decltype(WRC)::value;
    unsigned ppend = 0, pk = INITK, pw = 4u;
    auto peval = [&]() __attribute__((always_inline)) {
      const v4u f = *(const LAS v4u*)(lds + LCP + (unsigned)fresh_lane() * 16);
      unsigned long long bm = __ballot(f[0] == call && f[1] == call && f[2] == call && f[3] == call);
      bm &= bm >> 1; bm &= bm >> 2;
      bm = ~bm & 0x1111111111111111ull;
      unsigned c = bm ? (unsigned)__builtin_ctzll(bm) >> 2 : 16u;
      c = c < pw ? c : pw;
      pw = c == pw ? 16u : PWIN;
      const unsigned n = pk + c < nvis ? pk + c : nvis;
      if (n > pk) {
        pk = n; lctl[1] = n;
      }
      ppend = 0;
    };
    auto pissue = [&]() __attribute__((always_inline)) {
      const unsigned ln = (unsigned)fresh_lane();
      unsigned e = pk + ((ln >> 2) < pw ? ln >> 2 : 0u); e = e < nvis ? e : nvis - 1;
      const unsigned kk = *(const LAS unsigned*)(lds + LTAB + e * 4);
      const unsigned la = lbase + LCP, fo = flo + ((ln & 2) ? kk >> 15 : kk & 0x7fffu) * 64 + (ln & 1) * 16; unsigned char* const w_ = myws;
      asm volatile("s_mov_b32 m0, %0\n\ts_nop 0\n\tglobal_load_lds_dwordx4 %1, %2 sc0 sc1" :: "s"(la), "v"(fo), "s"(w_) : "memory");
      ppend = 1;
    };
    auto poll = [&](bool blocking, unsigned need) __attribute__((always_inline)) {
      if (!blocking) {
        if (ppend) {
          unsigned st; asm volatile("s_getreg_b32 %0, hwreg(HW_REG_IB_STS)" : "=s"(st) :: "memory");
          st = __builtin_amdgcn_readfirstlane(st);
          if ((st & 15u) | ((st >> 18) & 48u)) return;
          WAIT_V(0);
          peval();
        } else { pw = 16u; pissue(); }
        return;
      }
      WAIT_V(0);
      if (ppend) { peval(); if (pk < need) __builtin_amdgcn_s_sleep(PSLEEP); }
      if (pk < nvis) pissue();
      WAIT_L(0);
    };
    auto slow = [&](unsigned need) __attribute__((always_inline)) {
      const unsigned long long t0 = __builtin_amdgcn_s_memrealtime();
      for (;;) {
        if (wr == 1 && wc == 0) poll(true, need); else __builtin_amdgcn_s_sleep(1);
        const unsigned long long el = __builtin_amdgcn_s_memrealtime() - t0;
        if (wr == 1 && wc == 0 && el > 100000ull) {
          unsigned* tmo = (unsigned*)(myws + TMO_OFF);
          if (__builtin_amdgcn_readfirstlane(__hip_atomic_load(tmo, __ATOMIC_RELAXED, __HIP_MEMORY_SCOPE_AGENT)) == call || el > DEADLINE_TICKS) {
            __hip_atomic_store(tmo, call, __ATOMIC_RELAXED, __HIP_MEMORY_SCOPE_AGENT);
            lctl[2] = 1u; lctl[1] = DEADV; pk = nvis; ppend = 0;
          }
          WAIT_V(0);
        }
        if (__builtin_amdgcn_readfirstlane(__hip_atomic_load(lctl + 1, __ATOMIC_RELAXED, __HIP_MEMORY_SCOPE_WORKGROUP)) >= need || el > 2 * DEADLINE_TICKS) break;
      }
      asm volatile("" ::: "memory");
    };

#define GT0 26
    auto grab = [&](int t) __attribute__((always_inline)) {
      const unsigned ln = (unsigned)fresh_lane() & 31u, q = __builtin_amdgcn_readfirstlane(lctl[4]), pj = blk_job(rank, q);
      if (q >= NBLK) return;
      if (t == GT0) {
        const unsigned la = lbase + LCTL + 64u, fo = OFLAG_OFF + (pj + ln) * 4u; unsigned char* const w_ = myws;
        asm volatile("s_mov_b32 m0, %0\n\ts_mov_b32 exec_hi, 0\n\tglobal_load_lds_dword %1, %2 sc0 sc1\n\ts_mov_b32 exec_hi, -1" :: "s"(la), "v"(fo), "s"(w_) : "memory");
      } else {
        WAIT_V(0);
        lctl[5] = __ballot(lctl[16 + ln] != call) == 0ull ? PK_OFF + (pj >> 5) * PK_BLK - imgo : 0u;
      }
    };
    const unsigned char* XI = myws + imgo;
    const unsigned voff = (unsigned)(wc * 4096 + (tid & 63) * 16);
    const unsigned ldsw = (unsigned)wc * 4096u;
    const int aoff = wr * 8192 + (tid & 63) * 16, boff = wc * 4096 + (tid & 63) * 16;
#define SA(b_, h) (((b_) * 2 + (h)) * HTB)
#define SB(b_, h) ((4 + (b_) * 2 + (h)) * HTB)
#define STG1(bufoff, va, o) __builtin_amdgcn_global_load_lds((const unsigned*)(XI + (va)), (LAS unsigned*)(lds + (bufoff) + ldsw), 16, o, 0)
#define STAGE(bufoff, goff) do { if (wr == 0) { const unsigned va_ = voff + (goff); STG1(bufoff, va_, 0); STG1(bufoff, va_, 1024); STG1(bufoff, va_, 2048); STG1(bufoff, va_, 3072); } } while (0)
#define LDA(dst, b_, h) do { _Pragma("unroll") for (int m = 0; m < 4; ++m) { \
    v4i lo = *(const LAS v4i*)(lds + SA(b_, h) + aoff + m * 2048); v4i hi = *(const LAS v4i*)(lds + SA(b_, h) + aoff + m * 2048 + 1024); dst[m] = cat8(lo, hi); } } while (0)
#define LDB(dst, b_, h) do { _Pragma("unroll") for (int n = 0; n < 2; ++n) { \
    v4i lo = *(const LAS v4i*)(lds + SB(b_, h) + boff + n * 2048); v4i hi = *(const LAS v4i*)(lds + SB(b_, h) + boff + n * 2048 + 1024); dst[n] = cat8(lo, hi); } } while (0)
#define MMA(ai, bj, At, Bt) do { __builtin_amdgcn_s_setprio(1); _Pragma("unroll") for (int m = 0; m < 4; ++m) _Pragma("unroll") for (int n = 0; n < 2; ++n) \
    acc[ai][bj][m][n] = __builtin_amdgcn_mfma_scale_f32_16x16x128_f8f6f4(Bt[n], At[m], acc[ai][bj][m][n], 0, 1, 0, 0, 0, 0); __builtin_amdgcn_s_setprio(0); } while (0)
#define MF1(ai, bj, At, Bt, q) acc[ai][bj][(q) >> 1][(q) & 1] = __builtin_amdgcn_mfma_scale_f32_16x16x128_f8f6f4(Bt[(q) & 1], At[(q) >> 1], acc[ai][bj][(q) >> 1][(q) & 1], 0, 1, 0, 0, 0, 0)
#define WV12 WAIT_V(12)
#define MEMSTEP do { if (wr == 0) WV12; else if (wc == 0 && (ppend || (pk < nvis && pk < need + PTH))) poll(false, 0u); } while (0)
#define KOFF(i) ((unsigned)__builtin_amdgcn_readlane(kv, (i)))
    f32x4 acc[2][2][4][2];
#pragma unroll
    for (int a = 0; a < 2; ++a)
#pragma unroll
      for (int c = 0; c < 2; ++c)
#pragma unroll
        for (int m = 0; m < 4; ++m)
#pragma unroll
          for (int n = 0; n < 2; ++n) acc[a][c][m][n] = (f32x4){0.f, 0.f, 0.f, 0.f};
    v8i At[4], B0[2], B1[2];
    constexpr unsigned hstep = HTB;
#define OFA(e_) ((e_) & ~0x7fffu)
#define OFB(e_) (((e_) & 0x7fffu) << 15)
    slow(4u);
    unsigned known = 4u, need = 4u;
    const unsigned e0 = __builtin_amdgcn_readfirstlane(*(const LAS unsigned*)(lds + LTAB));
    {
      const unsigned cA = OFA(e0), cB = OFB(e0);
      STAGE(SB(0, 0), cB); STAGE(SA(0, 0), cA); STAGE(SB(0, 1), cB + hstep); STAGE(SA(0, 1), cA + hstep);
      if (wr == 1) BAR;
      if (wr == 0) WAIT_V(8);
      BAR;
      STAGE(SB(1, 0), cB + SLAB); STAGE(SA(1, 0), cA + SLAB); STAGE(SB(1, 1), cB + SLAB + hstep);
      if (wr == 0) WAIT_V(12);
      BAR;
    }
    unsigned a1 = OFA(e0) + SLAB, a2 = a1 + SLAB, a3 = a2 + SLAB, b2 = OFB(e0) + 2 * SLAB, b3 = b2 + SLAB;
    for (int ui = 0;; ++ui) {
      const bool has_next = (unsigned)(ui * 32 + 32) < nvis;
      int kv = 0;
      if (wr == 0) { const unsigned ln = ui * 32u + (unsigned)fresh_lane(); kv = *(const LAS int*)(lds + LTAB + (ln < 131u ? ln : 131u) * 4); }
      for (int t = 0; t < NKS; t += 2) {
        if (known < need) { if (wr == 0) BAR; slow(need); if (wr == 1) BAR; known = need; }
        LDB(B0, 0, 0); SCHED; LDA(At, 0, 0); SCHED; STAGE(SA(1, 1), a1 + hstep);
        WAIT_L(8); BAR; WAIT_L(0); MMA(0, 0, At, B0); BAR; SCHED;
        LDB(B1, 0, 1); SCHED; STAGE(SB(0, 0), b2);
        BAR; WAIT_L(0); MMA(0, 1, At, B1); BAR;
        LDA(At, 0, 1); SCHED; STAGE(SA(0, 0), a2);
        BAR; WAIT_L(0); MMA(1, 0, At, B0); BAR; SCHED;
        STAGE(SB(0, 1), b2 + hstep); MEMSTEP;
        if (wr == 1 && wc == 1 && ui == 3 && (t == GT0 || t == NKS - 4)) grab(t);
        BAR; MMA(1, 1, At, B1); BAR;
        LDB(B0, 1, 0); SCHED; LDA(At, 1, 0); SCHED; STAGE(SA(0, 1), a2 + hstep);
        WAIT_L(8); BAR; WAIT_L(0); MMA(0, 0, At, B0); BAR; SCHED;
        LDB(B1, 1, 1); const unsigned kn = lctl[1]; SCHED; STAGE(SB(1, 0), b3);
        BAR; WAIT_L(0); MMA(0, 1, At, B1); BAR;
        LDA(At, 1, 1); SCHED; STAGE(SA(1, 0), a3);
        BAR; WAIT_L(0); MMA(1, 0, At, B0); BAR; SCHED;
        STAGE(SB(1, 1), b3 + hstep); if (wr == 1 || has_next || t < NKS - 2) MEMSTEP;
        BAR; SCHED;
        __builtin_amdgcn_s_setprio(1);
        MF1(1, 1, At, B1, 0); SCHED;
        unsigned o2 = KOFF(t + 4), o3 = KOFF(t + 5); known = __builtin_amdgcn_readfirstlane(kn);
        asm volatile("" : "+s"(o2), "+s"(o3), "+s"(known)); SCHED;
        MF1(1, 1, At, B1, 1); SCHED;
        a1 = a3; a2 = OFA(o2); b2 = OFB(o2);
        asm volatile("" : "+s"(a1), "+s"(a2), "+s"(b2)); SCHED;
        MF1(1, 1, At, B1, 2); SCHED;
        a3 = OFA(o3); b3 = OFB(o3);
        if (wr == 0 && !has_next && t == NKS - 4) {
          const unsigned ob_ = __builtin_amdgcn_readfirstlane(lctl[5]);
          a2 = ob_ ? ob_ : a1; a3 = ob_ ? ob_ + 2 * hstep : a1; b2 = ob_ ? ob_ + 3 * hstep : a1; b3 = ob_ ? ob_ + 5 * hstep : a1;
        }
        asm volatile("" : "+s"(a2), "+s"(b2), "+s"(a3), "+s"(b3)); SCHED;
        MF1(1, 1, At, B1, 3); SCHED;
        need = need + 2u < nvis ? need + 2u : nvis;
        asm volatile("" : "+s"(need)); SCHED;
        MF1(1, 1, At, B1, 4); MF1(1, 1, At, B1, 5); MF1(1, 1, At, B1, 6); MF1(1, 1, At, B1, 7);
        __builtin_amdgcn_s_setprio(0);
        BAR;
      }
      int tm, tn; tile_of(ui, tm, tn);
      const bool dead = __builtin_amdgcn_readfirstlane(lctl[2]) != 0u, sht = tn >= SHC;
      TS(2 + 2 * ui);
      unsigned gv = 1u;
      const int ng = __builtin_amdgcn_readfirstlane((int)!(wc == 1 && ui == 2 && has_next));
      {
        const int ln = fresh_lane();
        const unsigned vo = (unsigned)((tm * 256 + wc * 32 + 4 * (ln >> 4)) * N_ + tn * 256 + wr * 64 + 4 * (ln & 15)) * 4u;
        const float qnan = __builtin_nanf("");
        const float sc_ = dead ? qnan : scale, lo_ = dead ? qnan : 0.f;
        const int shs = __builtin_amdgcn_readfirstlane(sht ? 1 : ui < PLN ? 2 : 0);
        unsigned char* const ob = p.out[rank];
        if (wr == 1) {
          const unsigned co = WCT_OFF + 256u + (call & 1u) * 128u; unsigned char* const w_ = myws;
          asm volatile("s_cmp_lg_u32 %3, 0\n\ts_cbranch_scc1 GE_%=\n\ts_mov_b64 exec, 1\n\tglobal_atomic_add %0, %1, %0, %2 sc0\n\ts_mov_b64 exec, -1\nGE_%=:"
                       : "+v"(gv) : "v"(co), "s"(w_), "s"(ng) : "memory", "scc");
        }
#pragma unroll
        for (int bj = 0; bj < 2; ++bj)
#pragma unroll
          for (int n = 0; n < 2; ++n)
#pragma unroll
            for (int r4 = 0; r4 < 4; ++r4) {
              unsigned char* rowp = ob + (size_t)(bj * 128 + n * 16 + r4) * N_ * 4;
#pragma unroll
              for (int ai = 0; ai < 2; ++ai) {
                f32x4 v;
#pragma unroll
                for (int c = 0; c < 4; ++c) {
                  const float x = acc[ai][bj][c][n][r4] * sc_;
                  asm("v_max_f32 %0, %1, %2" : "=v"(v[c]) : "s"(lo_), "v"(x));
                }
                asm volatile("s_cmp_lg_u32 %4, 0\n\ts_cbranch_scc1 SW_%=\n\tglobal_store_dwordx4 %0, %1, %2 offset:%3" OSM "\n\ts_branch SE_%=\n"
                             "SW_%=:\n\tglobal_store_dwordx4 %0, %1, %2 offset:%3 sc0 sc1" SHM "\nSE_%=:\n\ts_nop 1"
                             :: "v"(vo), "v"(v), "s"(rowp), "n"(ai * 512), "s"(shs) : "memory", "scc");
              }
            }
      }
      if (!has_next && wr == 0) WAIT_V(32);
      if (wr == 1) {
        asm volatile("s_waitcnt vmcnt(32)" : "+v"(gv) :: "memory");
        if (!ng) { const unsigned q_ = __builtin_amdgcn_readfirstlane(gv); lctl[4] = q_ < NG ? q_ + N1 : NBLK; }
      }
      if (sht) {
        WAIT_V(0);
        unsigned o_ = 0;
        if (fresh_lane() == 0) o_ = __hip_atomic_fetch_add(lctl + 3, 1u, __ATOMIC_RELAXED, __HIP_MEMORY_SCOPE_WORKGROUP);
        if (__builtin_amdgcn_readfirstlane(o_) == 7u) store_flag1(myws, DONE_OFF + si * 64, call);
        if (wr == 0) WAIT_V(0);
      }
      TS(3 + 2 * ui);
      if (!has_next) break;
#pragma unroll
      for (int a = 0; a < 2; ++a)
#pragma unroll
        for (int c = 0; c < 2; ++c)
#pragma unroll
          for (int m = 0; m < 4; ++m)
#pragma unroll
            for (int n = 0; n < 2; ++n) acc[a][c][m][n] = (f32x4){0.f, 0.f, 0.f, 0.f};
    }
    if (wr == 0) BAR;
  };
  if (wid < 4) run(std::integral_constant<int, 0>{}); else run(std::integral_constant<int, 1>{});
  expand(p, lds, call, wid, __builtin_amdgcn_readfirstlane(lctl[4]), skip ? 0u : __builtin_amdgcn_readfirstlane(lctl[5]) ? 2u : 1u);
  TS(14);
}
extern "C" void kernel_launch(int rank, int nranks, ncclComm_t comm,
                              hipStream_t stream,
                              void* const* d_in, const int* in_sizes, int n_in,
                              void* d_out, int out_size,
                              const int* in_dtypes, int out_dtype,
                              void* const* d_out_all,
                              void* const* d_ws_all, size_t ws_bytes) {
  if (nranks != NR || (unsigned)rank >= (unsigned)NR || ws_bytes < WS_NEED || n_in < 4) {
    fprintf(stderr, "kernel_launch: unexpected configuration\n"); return;
  }
  static bool attr_done[64] = {};
  int dev = 0; HIP_CHECK(hipGetDevice(&dev));
  if (!attr_done[dev & 63]) {
    HIP_CHECK(hipFuncSetAttribute((const void*)fused_kernel, hipFuncAttributeMaxDynamicSharedMemorySize, LDS_BYTES));
    attr_done[dev & 63] = true;
  }
  Params p;
  p.x = (const unsigned char*)d_in[0]; p.w = (const unsigned char*)d_in[1];
  p.sx = (const float*)d_in[2]; p.sw = (const float*)d_in[3];
  for (int i = 0; i < NR; ++i) { p.ws[i] = (unsigned char*)d_ws_all[i]; p.out[i] = (unsigned char*)(i == rank ? d_out : d_out_all[i]); }
  p.rank = rank;
  fused_kernel<<<GRID, 512, LDS_BYTES, stream>>>(p);
}
```

```cpp
#include <hip/hip_runtime.h>
#include <rccl/rccl.h>
#include <cstdio>
#include <type_traits>

#define HIP_CHECK(expr) do { hipError_t _e = (expr); if (_e != hipSuccess) { \
    fprintf(stderr, "HIP error %s at %s:%d\n", hipGetErrorString(_e), __FILE__, __LINE__); \
    return; } } while (0)

typedef int v8i __attribute__((ext_vector_type(8)));
typedef int v4i __attribute__((ext_vector_type(4)));
typedef unsigned v4u __attribute__((ext_vector_type(4)));
typedef float f32x4 __attribute__((ext_vector_type(4)));
#define LAS __attribute__((address_space(3)))

constexpr int NR = 8;
constexpr int M_ = 4096, N_ = 16384, KL = 512;
constexpr int NKS = 32;
constexpr int HTB = 16384;
constexpr int STAGE_BYTES = 8 * HTB;
constexpr unsigned LCTL = STAGE_BYTES, LTAB = STAGE_BYTES + 256, LCP = STAGE_BYTES + 1024;
constexpr unsigned LTCH = STAGE_BYTES + 2048;
constexpr int LDS_BYTES = STAGE_BYTES + 2048 + 256;
constexpr unsigned SLAB = 32768;
constexpr unsigned IMG_BYTES = 2560u * SLAB;
constexpr unsigned IMG_OFF = 1u << 20;
constexpr unsigned PK_OFF = IMG_OFF + 2 * IMG_BYTES, PK_SLOT = 5120, PK_JOBS = 1280;
constexpr unsigned PK_BLK = 32 * PK_SLOT, NBLK = 7 * PK_JOBS / 32;
constexpr unsigned CW_OFF = PK_OFF + NR * (PK_JOBS / 32) * PK_BLK - 512u * SLAB, CW_BYTES = 2048u * SLAB;
constexpr size_t WS_NEED = (size_t)CW_OFF + 512u * SLAB + (size_t)CW_BYTES;
constexpr unsigned CTR_OFF = 0, TMO_OFF = 128, WCT_OFF = 256, DONE_OFF = 4096, OFLAG_OFF = 8192, FLAG_OFF = 65536, FLAG_BYTES = 2560u * 64u, DBG_OFF = 524288;
constexpr int GRID = 256;
constexpr int NPUSH = 16, NCOM = 32;
constexpr int NSH = 20, SHC = 54;
constexpr unsigned DEADV = 0x7fffffffu;
constexpr unsigned long long DEADLINE_TICKS = 90000000ull;
#define WPT 2

struct Params {
  const unsigned char* x;
  const unsigned char* w;
  const float* sx; const float* sw;
  unsigned char* ws[NR];
  unsigned char* out[NR];
  int rank;
};

#define TS(slot) do {} while (0)
#define TV(slot, v) do {} while (0)
__device__ __forceinline__ v8i cat8(v4i a, v4i b) { return __builtin_shufflevector(a, b, 0, 1, 2, 3, 4, 5, 6, 7); }
#define WAIT_V(n) asm volatile("s_waitcnt vmcnt(" #n ")" ::: "memory")
#define WAIT_L(n) asm volatile("s_waitcnt lgkmcnt(" #n ")" ::: "memory")
#define BAR __builtin_amdgcn_s_barrier()
#define SCHED __builtin_amdgcn_sched_barrier(0)
#define TIE16(R) "+v"(R[0]), "+v"(R[1]), "+v"(R[2]), "+v"(R[3]), "+v"(R[4]), "+v"(R[5]), "+v"(R[6]), "+v"(R[7]), \
                 "+v"(R[8]), "+v"(R[9]), "+v"(R[10]), "+v"(R[11]), "+v"(R[12]), "+v"(R[13]), "+v"(R[14]), "+v"(R[15])
#define TIE4(X) "+v"(X[0]), "+v"(X[1]), "+v"(X[2]), "+v"(X[3])

#define PKM ""
#define IMM ""
#define LDM ""
#define SHM ""
__device__ __forceinline__ int fresh_lane() { int l; asm volatile("v_mbcnt_lo_u32_b32 %0, -1, 0\n\tv_mbcnt_hi_u32_b32 %0, -1, %0" : "=v"(l)); return l; }

__device__ __forceinline__ void load_job(const Params& p, int tile, int ks, int jw, int lane, unsigned (&R)[16], v4u (&X)[4]) {
  const int i = lane & 15, g = lane >> 4;
  const bool isx = tile < 16, iso = tile >= 80;
  unsigned vo = isx ? i * KL + g * 32 : iso ? lane * 16 : g * 32 * N_ + 4 * i;
  const unsigned char* src = isx ? p.x + (size_t)(tile * 256 + (jw >> 1) * 64) * KL + ks * 128 + (jw & 1) * 16
                           : iso ? p.out[p.rank] + (unsigned)ks
                                 : p.w + (size_t)(ks * 128 + (jw & 1) * 16) * N_ + (tile - 16) * 256 + (jw >> 1) * 64;
  const unsigned st = isx ? 16 * KL : iso ? 4 * N_ : N_;
#define LD1(r) "global_load_dword %" #r ", %20, %21" LDM "\n\tv_add_u32 %20, %22, %20\n\t"
#define LD4(r) "global_load_dwordx4 %" #r ", %20, %21" LDM "\n\tv_add_u32 %20, %22, %20\n\t"
  asm volatile("s_cmp_eq_u32 %22, %23\n\ts_cbranch_scc1 LW_%=\n\t" LD4(16) LD4(17) LD4(18) LD4(19) "s_branch LE_%=\n"
               "LW_%=:\n\t" LD1(0) LD1(1) LD1(2) LD1(3) LD1(4) LD1(5) LD1(6) LD1(7) LD1(8) LD1(9) LD1(10) LD1(11) LD1(12) LD1(13) LD1(14) LD1(15) "\n"
               "LE_%=:"
               : "=&v"(R[0]), "=&v"(R[1]), "=&v"(R[2]), "=&v"(R[3]), "=&v"(R[4]), "=&v"(R[5]), "=&v"(R[6]), "=&v"(R[7]),
                 "=&v"(R[8]), "=&v"(R[9]), "=&v"(R[10]), "=&v"(R[11]), "=&v"(R[12]), "=&v"(R[13]), "=&v"(R[14]), "=&v"(R[15]),
                 "=&v"(X[0]), "=&v"(X[1]), "=&v"(X[2]), "=&v"(X[3]), "+v"(vo)
               : "s"(src), "s"(st), "n"(N_) : "memory", "scc");
}
__device__ __forceinline__ void retile(bool plain, const unsigned (&R)[16], const v4u (&X)[4], v4u (&D)[4]) {
  if (plain) {
#pragma unroll
    for (int e = 0; e < 4; ++e) D[e] = X[e];
  } else {
#pragma unroll
    for (int d = 0; d < 4; ++d) {
      const unsigned a = R[4 * d], bb = R[4 * d + 1], c = R[4 * d + 2], f = R[4 * d + 3];
      const unsigned t0_ = __builtin_amdgcn_perm(bb, a, 0x05010400u), t1_ = __builtin_amdgcn_perm(bb, a, 0x07030602u);
      const unsigned t2_ = __builtin_amdgcn_perm(f, c, 0x05010400u), t3_ = __builtin_amdgcn_perm(f, c, 0x07030602u);
      D[0][d] = __builtin_amdgcn_perm(t2_, t0_, 0x05040100u); D[1][d] = __builtin_amdgcn_perm(t2_, t0_, 0x07060302u);
      D[2][d] = __builtin_amdgcn_perm(t3_, t1_, 0x05040100u); D[3][d] = __builtin_amdgcn_perm(t3_, t1_, 0x07060302u);
    }
  }
}
__device__ __forceinline__ void store_job(unsigned char* ws, unsigned vo, const v4u (&D)[4]) {
  asm volatile("global_store_dwordx4 %0, %1, %5 offset:-4096 sc0 sc1\n\tglobal_store_dwordx4 %0, %2, %5 offset:-2048 sc0 sc1\n\t"
               "global_store_dwordx4 %0, %3, %5 sc0 sc1\n\tglobal_store_dwordx4 %0, %4, %5 offset:2048 sc0 sc1\n\ts_nop 1"
               :: "v"(vo), "v"(D[0]), "v"(D[1]), "v"(D[2]), "v"(D[3]), "s"(ws) : "memory");
}
__device__ __forceinline__ void store_flag1(unsigned char* ws, unsigned fo, unsigned v) {
  asm volatile("s_mov_b64 exec, 1\n\tglobal_store_dword %0, %1, %2 sc0 sc1\n\ts_mov_b64 exec, -1" :: "v"(fo), "v"(v), "s"(ws) : "memory");
}
__device__ __forceinline__ void store_flag7(unsigned long long fa, unsigned v) {
  asm volatile("s_mov_b64 exec, 0x7f\n\tglobal_store_dword %0, %1, off sc0 sc1\n\ts_mov_b64 exec, -1" :: "v"(fa), "v"(v) : "memory");
}
__device__ __forceinline__ unsigned job_off(int jw, int lane) { return (jw >> 1) * 8192 + (jw & 1) * 1024 + lane * 16; }

__device__ __forceinline__ void stream_slab(int j, int snd, int& tile, int& ks) {
  if (j < 120) { ks = j / 30; int q = j - ks * 30; if (q < 14) { q += 2 * snd; if (q >= 14) q -= 14; tile = 16 + q; } else tile = (q - 14 + 2 * snd) & 15; }
  else if (j < 168) { const int u = j - 120; ks = u / 12; int q = u - ks * 12 + snd; if (q >= 12) q -= 12; tile = 30 + q; }
  else { const int u = j - 168, gq = u >= 56, v = u - gq * 56; ks = v / 14; int q = v - ks * 14 + 2 * snd; if (q >= 14) q -= 14; tile = 42 + 14 * gq + q; }
}
__device__ __forceinline__ int stream_idx(int tile, int ks, int snd) {
  if (tile < 16) return ks * 30 + 14 + ((tile - 2 * snd) & 15);
  if (tile < 30) { int q = tile - 16 - 2 * snd; if (q < 0) q += 14; return ks * 30 + q; }
  if (tile < 42 || tile >= 70) { int q = (tile < 42 ? tile - 30 : tile - 70) - snd; if (q < 0) q += 12; return 120 + ks * 12 + q; }
  const int gq = tile >= 56; int q = tile - 42 - 14 * gq - 2 * snd; if (q < 0) q += 14; return 168 + gq * 56 + ks * 14 + q;
}
__device__ __forceinline__ unsigned shared_off(int r, int i) { const int idx = NSH * r + i; return (unsigned)((idx & 15) * 256 * N_ + (SHC + (idx >> 4)) * 256) * 4u; }

#define INITK 0u
#define LAG2 120
#define LAG 1
#define PWIN 4u
#define PTH 8u
#define INV 0
#define LASTWT 0
#define OST 1
#define OSM " nt"
#define PLN 0
#define XST OST
#define XSM " nt"
#define PSLEEP 8
__device__ __forceinline__ bool wait_flags(unsigned char* ws, unsigned fo, unsigned call) {
  const unsigned long long t0 = __builtin_amdgcn_s_memrealtime();
  unsigned* tmo = (unsigned*)(ws + TMO_OFF);
  for (;;) {
    unsigned v, tm; const unsigned to = TMO_OFF;
    asm volatile("global_load_dword %0, %2, %4 sc0 sc1\n\tglobal_load_dword %1, %3, %4 sc0 sc1\n\ts_waitcnt vmcnt(0)" : "=&v"(v), "=&v"(tm) : "v"(fo), "v"(to), "s"(ws) : "memory");
    if (__ballot(v != call) == 0ull) return true;
    if (__builtin_amdgcn_readfirstlane(tm) == call || __builtin_amdgcn_s_memrealtime() - t0 > DEADLINE_TICKS) {
      __hip_atomic_store(tmo, call, __ATOMIC_RELAXED, __HIP_MEMORY_SCOPE_AGENT);
      return false;
    }
    __builtin_amdgcn_s_sleep(16);
  }
}
constexpr int NTURN = 8 / WPT;
#define NPA 16
constexpr int SPT = NPA * WPT / 8, NT = 280 / SPT;
__device__ __forceinline__ unsigned pk_off(int r, unsigned jj) { return shared_off(r, jj >> 6) + ((jj & 63u) * 4u) * (N_ * 4u); }
#define ST1(vc, d, k, md) "global_store_dwordx4 " vc ", " d ", %" #k " sc0 sc1" md "\n\t"
#define ST7(vc, d, md) ST1(vc, d, 2, md) ST1(vc, d, 3, md) ST1(vc, d, 4, md) ST1(vc, d, 5, md) ST1(vc, d, 6, md) ST1(vc, d, 7, md) ST1(vc, d, 8, md)
__device__ __forceinline__ unsigned bfi(unsigned m, unsigned a, unsigned b) { return (a & m) | (b & ~m); }
__device__ __forceinline__ unsigned encode(v4u (&S)[4], LAS unsigned char* el, int lane, unsigned long long& em) {
  unsigned sm = 0;
#pragma unroll
  for (int d = 0; d < 4; ++d) sm = __builtin_amdgcn_udot4(S[0][d] & 0x7c7c7c7cu, 0x01010101u, sm, false);
  sm += __builtin_amdgcn_update_dpp(0, (int)sm, 0x111, 0xf, 0xf, true); sm += __builtin_amdgcn_update_dpp(0, (int)sm, 0x112, 0xf, 0xf, true);
  sm += __builtin_amdgcn_update_dpp(0, (int)sm, 0x114, 0xf, 0xf, true); sm += __builtin_amdgcn_update_dpp(0, (int)sm, 0x118, 0xf, 0xf, true);
  int e0 = (__builtin_amdgcn_readlane(sm, 15) + __builtin_amdgcn_readlane(sm, 31) + __builtin_amdgcn_readlane(sm, 47) + __builtin_amdgcn_readlane(sm, 63) - 17121) >> 12;
  e0 = e0 < 0 ? 0 : e0 > 24 ? 24 : e0;
  const unsigned c1 = (unsigned)(128 - 4 * e0) * 0x01010101u, ix = (unsigned)((lane & 7) * 4 | (lane >> 3) << 13);
  unsigned cnt = 0, hdr = 0; v4u U[4];
#pragma unroll
  for (int c = 0; c < 4; ++c) {
#pragma unroll
    for (int d = 0; d < 4; ++d) {
      const unsigned v = S[c][d], u = (v & 0x7f7f7f7fu) + c1;
      const bool esc = (u & 0xe0e0e0e0u) != 0x80808080u;
      const unsigned long long m = __ballot(esc);
      const unsigned pos = __builtin_amdgcn_mbcnt_hi((unsigned)(m >> 32), __builtin_amdgcn_mbcnt_lo((unsigned)m, cnt));
      *(LAS unsigned short*)(el + 4 + 2 * (esc ? pos : 1100u + (unsigned)lane)) = (unsigned short)(__builtin_amdgcn_udot4(v & 0x60606060u, 0x40100401u, ix, false) + d);
      cnt += (unsigned)__builtin_popcountll(m);
      U[c][d] = bfi(0x80808080u, v, u);
    }
    hdr |= (cnt & 127u) << (8 * c);
  }
  em = ~0ull;
  if (cnt > 126u) return 32u;
  if (lane == 0) *(LAS unsigned*)el = hdr;
#pragma unroll
  for (int d = 0; d < 4; ++d) {
    const unsigned z = U[3][d];
    S[0][d] = bfi(0x60606060u, z << 4, U[0][d]); S[1][d] = bfi(0x60606060u, z << 2, U[1][d]); S[2][d] = bfi(0x60606060u, __builtin_amdgcn_alignbit(z, z, 2), U[2][d]);
  }
  S[3] = *(const LAS v4u*)(el + (lane & 15) * 16);
  em = (1ull << ((cnt + 33u) >> 5) * 4u) - 1ull;
  return (unsigned)e0;
}
__device__ __forceinline__ void pusher(const Params& p, LAS unsigned char* lds, unsigned imgo, unsigned flo, unsigned call, int b, int wid, int lane, unsigned (&R)[16], v4u (&X)[4]) {
  unsigned char* const myws = p.ws[p.rank];
  const int tau = wid / WPT, e = tau * SPT + b / NTURN, jw = (b % NTURN) * WPT + wid % WPT;
  unsigned long long fa = 0;
  unsigned char* wsd[7]; int pr[7];
#pragma unroll
  for (int k = 0; k < 7; ++k) {
    pr[k] = (p.rank + 1 + (wid + b + k) % 7) & 7; wsd[k] = p.ws[pr[k]];
    if (lane == k) fa = (unsigned long long)wsd[k];
  }
  const unsigned jo = job_off(jw, lane), dno = DONE_OFF + (lane < NSH ? lane : 0) * 64u, cwo = CW_OFF;
  LAS unsigned* const pk = (LAS unsigned*)(lds + wid * PK_SLOT);
  int tile, ks; stream_slab(e, p.rank, tile, ks);
  unsigned job = e * 8 + jw, cd = 32u;
  unsigned long long em = ~0ull;
  v4u D[4];
  retile(tile < 16, R, X, D);
  if (tile >= 16) cd = encode(D, (LAS unsigned char*)pk, lane, em);
  v4u D2[4]; unsigned cd2 = 0u; unsigned long long em2 = 0ull;
  for (int i = 0; i < tau; ++i) BAR;
  for (int T = tau; T < NT; T += NTURN) {
    const unsigned sid = tile * 32 + p.rank * 4 + ks, vb = imgo + jo + sid * SLAB;
    const unsigned fo = flo + sid * 64u + jw * 4u + (tile >= 16 ? 32u : 0u), fv = tile >= 16 ? call << 6 | cd : call;
    const bool nar = tile >= 30 && tile < 40;
    unsigned long long fn = 0;
    if (nar) {
      const int ra = (tile - 30) * 16 / NSH, rb = ((tile - 30) * 16 + 15) / NSH;
      unsigned char* const da = ra != p.rank ? p.ws[ra & 7] : myws + (cwo - imgo);
      unsigned char* const db = rb != p.rank && rb != ra ? p.ws[rb & 7] : myws + (cwo - imgo);
      fn = (unsigned long long)(lane ? p.ws[ra & 7] : p.ws[rb & 7]) + fo + 40u * 32u * 64u;
#pragma unroll
      for (int c = 0; c < 4; ++c) {
        const unsigned vc = vb + 40u * 32u * SLAB + c * 2048u;
        const unsigned long long em_ = c == 3 ? em2 : ~0ull;
        asm volatile("s_mov_b64 exec, %4\n\t" ST1("%0", "%1", 2, "") ST1("%0", "%1", 3, "") "s_mov_b64 exec, -1\n\ts_nop 1" :: "v"(vc), "v"(D2[c]), "s"(da), "s"(db), "s"(em_) : "memory");
      }
    }
    stream_slab((T + NTURN < NT ? T + NTURN : T) * SPT + b / NTURN, p.rank, tile, ks);
    load_job(p, tile, ks, jw, lane, R, X);
#pragma unroll
    for (int c = 0; c < 4; ++c) {
      const unsigned vc = vb + c * 2048u;
      const unsigned long long em_ = c == 3 ? em : ~0ull;
      asm volatile("s_mov_b64 exec, %9\n\t" ST7("%0", "%1", IMM) "s_mov_b64 exec, -1\n\ts_nop 1"
                   :: "v"(vc), "v"(D[c]), "s"(wsd[0]), "s"(wsd[1]), "s"(wsd[2]), "s"(wsd[3]), "s"(wsd[4]), "s"(wsd[5]), "s"(wsd[6]), "s"(em_) : "memory");
    }
    const int nb = NT - 1 - T < NTURN ? NT - 1 - T : NTURN;
    const int lag2 = __builtin_amdgcn_readfirstlane((int)(T * SPT >= LAG2));
    asm volatile("s_cmp_eq_u32 %20, 0\n\ts_cbranch_scc1 PB_%=\n\ts_barrier\nPB_%=:\n\ts_waitcnt vmcnt(28)\n\ts_cmp_lg_u32 %21, 0\n\ts_cbranch_scc1 PC_%=\n\ts_waitcnt vmcnt(0)\nPC_%=:"
                 : TIE16(R), TIE4(X) : "s"(nb), "s"(lag2) : "memory", "scc");
    if (nar) asm volatile("s_mov_b64 exec, 3\n\tglobal_store_dword %0, %1, off sc0 sc1\n\ts_mov_b64 exec, -1" :: "v"(fn), "v"(call << 6 | cd2) : "memory");
    if (!lag2) { store_flag7(fa + fo, fv); if (nb > 1 && NTURN > 2) BAR; }
    retile(tile < 16, R, X, D);
    cd = 32u; em = ~0ull;
    if (tile >= 16) cd = encode(D, (LAS unsigned char*)pk, lane, em);
    if (lag2) { if (nb > 1 && NTURN > 2) BAR; WAIT_V(0); store_flag7(fa + fo, fv); }
    if (tile >= 30 && tile < 40) {
      load_job(p, tile + 40, ks, jw, lane, R, X);
      asm volatile("s_waitcnt vmcnt(0)" : TIE16(R), TIE4(X) :: "memory");
      retile(false, R, X, D2);
      cd2 = encode(D2, (LAS unsigned char*)pk, lane, em2);
    }
    for (int i = NTURN > 2 ? 2 : 1; i < nb; ++i) BAR;
  }
  const bool ok = wait_flags(myws, dno, call);
  load_job(p, 80, (int)pk_off(p.rank, job), jw, lane, R, X);
  asm volatile("s_waitcnt vmcnt(0)" : TIE16(R), TIE4(X) :: "memory");
  for (;;) {
    unsigned nx = 1u; const unsigned co = WCT_OFF + 64u + (call & 1u) * 128u;
    asm volatile("s_nop 1\n\ts_mov_b64 exec, 1\n\tglobal_atomic_add %0, %1, %0, %2 sc0\n\ts_mov_b64 exec, -1" : "+v"(nx) : "v"(co), "s"(myws) : "memory");
    unsigned cnt = 32u, hdr = 0u;
#pragma unroll
    for (int j = 0; j < 16; ++j) {
      const unsigned v = ok ? X[j >> 2][j & 3] : 0x7fc00000u;
      const unsigned long long m = __ballot(v != 0u);
      const unsigned lo = (unsigned)m, hi = (unsigned)(m >> 32);
      pk[v != 0u ? __builtin_amdgcn_mbcnt_hi(hi, __builtin_amdgcn_mbcnt_lo(lo, cnt)) : 1100u + (unsigned)lane] = v;
      hdr = lane == 2 * j ? lo : lane == 2 * j + 1 ? hi : hdr;
      cnt += (unsigned)__builtin_popcountll(m);
    }
    pk[lane < 32 ? (unsigned)lane : 1100u + (unsigned)lane] = hdr;
    const unsigned nd = (cnt + 31u) & ~31u, pj = p.rank * PK_JOBS + job;
#pragma unroll
    for (int c = 0; c < 5; ++c) {
      const int nl = (int)(nd >> 2) - c * 64;
      if (nl > 0) {
        const v4u d = *(const LAS v4u*)(pk + c * 256 + lane * 4);
        const unsigned vc = PK_OFF + (pj >> 5) * PK_BLK + (c < 3 ? (pj & 31u) * 3072u : 95232u + (pj & 31u) * 2048u) + c * 1024u + lane * 16u;
        const unsigned long long em = nl >= 64 ? ~0ull : (1ull << nl) - 1ull;
        asm volatile("s_mov_b64 exec, %9\n\t" ST7("%0", "%1", PKM) "s_mov_b64 exec, -1\n\ts_nop 1"
                     :: "v"(vc), "v"(d), "s"(wsd[0]), "s"(wsd[1]), "s"(wsd[2]), "s"(wsd[3]), "s"(wsd[4]), "s"(wsd[5]), "s"(wsd[6]), "s"(em) : "memory");
      }
    }
    asm volatile("s_waitcnt vmcnt(0)" : "+v"(nx) :: "memory");
    job = __builtin_amdgcn_readfirstlane(nx) + NPA * 8u;
    const bool more = job < PK_JOBS;
    {
      unsigned vo = lane * 16u; const unsigned char* src = p.out[p.rank] + pk_off(p.rank, more ? job : 0u);
      asm volatile("s_cmp_ge_u32 %6, %8\n\ts_cbranch_scc1 OE_%=\n\tglobal_load_dwordx4 %0, %4, %5\n\tv_add_u32 %4, %7, %4\n\tglobal_load_dwordx4 %1, %4, %5\n\tv_add_u32 %4, %7, %4\n\t"
                   "global_load_dwordx4 %2, %4, %5\n\tv_add_u32 %4, %7, %4\n\tglobal_load_dwordx4 %3, %4, %5\nOE_%=:"
                   : "=&v"(X[0]), "=&v"(X[1]), "=&v"(X[2]), "=&v"(X[3]), "+v"(vo) : "s"(src), "s"(job), "n"(N_ * 4), "n"(PK_JOBS) : "memory", "scc");
    }
    store_flag7(fa + OFLAG_OFF + pj * 4u, call);
    if (!more) break;
    asm volatile("s_waitcnt vmcnt(1)" : TIE4(X) :: "memory");
  }
  TS(1); TS(9);
}
constexpr unsigned NUNIT = 196, UTAB = 120 * 1024;
#define UDLY 12
#define UWIN 24u
__device__ __forceinline__ void unpacker(const Params& p, LAS unsigned char* lds, unsigned imgo, unsigned flo, unsigned call, int b, int wid, int lane) {
  unsigned char* const myws = p.ws[p.rank];
  LAS unsigned* lctl = (LAS unsigned*)(lds + LCTL);
  const unsigned long long t0 = __builtin_amdgcn_s_memrealtime();
  if (wid == 0) {
    unsigned i = lane, done = 0;
    for (;;) {
      const unsigned q = (i < NUNIT ? i : 0u) * 16u + (b - NPUSH), wp = q / 14u, r = q - wp * 14u;
      const int snd = (p.rank + 1 + (r >> 1)) & 7;
      const int k14 = wp / 14u, gp = k14 >> 2, ks = k14 & 3, q14 = wp - k14 * 14;
      int qq = q14 + (gp == 1 ? snd : 2 * snd); qq -= qq >= (gp == 1 ? 12 : 14) ? (gp == 1 ? 12 : 14) : 0;
      const int tile = gp == 1 && q14 >= 12 ? 58 + ((NSH * p.rank) >> 4) + q14 : (gp == 0 ? 16 : 14 * gp + (gp == 1 ? 16 : 14)) + qq;
      const unsigned hs = (tile * 32 + snd * 4 + ks) * 2 + (r & 1u), fo = flo + (hs >> 1) * 64u + 32u + (hs & 1u) * 16u, to = TMO_OFF;
      const unsigned long long pm = __ballot(i < NUNIT && i < done + UWIN);
      v4u v = {0u, 0u, 0u, 0u}; unsigned tm;
      asm volatile("s_mov_b64 exec, %5\n\tglobal_load_dwordx4 %0, %2, %4 sc0 sc1\n\ts_mov_b64 exec, -1\n\tglobal_load_dword %1, %3, %4 sc0 sc1\n\ts_waitcnt vmcnt(0)"
                   : "+v"(v), "=&v"(tm) : "v"(fo), "v"(to), "s"(myws), "s"(pm) : "memory");
      const unsigned tag = call & 0x3ffffffu;
      const bool ok = (pm >> lane & 1ull) && v[0] >> 6 == tag && v[1] >> 6 == tag && v[2] >> 6 == tag && v[3] >> 6 == tag;
      if (ok) {
        *(LAS unsigned long long*)(lds + UTAB + i * 8u) = (unsigned long long)((v[0] & 63u) | (v[1] & 63u) << 8 | (v[2] & 63u) << 16 | (v[3] & 63u) << 24) << 32 | hs;
        i += 64u;
      }
      const unsigned nk = (unsigned)__builtin_popcountll(__ballot(ok));
      done += nk;
      if (done >= NUNIT) break;
      if (__builtin_amdgcn_readfirstlane(tm) == call || __builtin_amdgcn_s_memrealtime() - t0 > DEADLINE_TICKS) {
        __hip_atomic_store((unsigned*)(myws + TMO_OFF), call, __ATOMIC_RELAXED, __HIP_MEMORY_SCOPE_AGENT);
        lctl[2] = 1u;
        break;
      }
      if (!nk) __builtin_amdgcn_s_sleep(4);
    }
    TS(3);
    return;
  }
  const unsigned cwo = CW_OFF, lb = (unsigned)(size_t)lds + (wid - 1) * 16640u, msk = 0x60606060u;
  LAS unsigned char* const buf = lds + (wid - 1) * 16640u;
  unsigned pfo = DONE_OFF + 4032u, pend = 0;
  const v4u cv = {call, call, call, call};
#define UFLAG asm volatile("s_mov_b64 exec, 1\n\tglobal_store_dwordx4 %0, %1, %2 sc0 sc1\n\ts_mov_b64 exec, -1\n\ts_nop 1" :: "v"(pfo), "v"(cv), "s"(myws) : "memory")
  for (unsigned i = wid - 1; i < NUNIT; i += 7u) {
    unsigned hs, cds;
    for (;;) {
      const unsigned long long e = __hip_atomic_load((LAS unsigned long long*)(lds + UTAB + i * 8u), __ATOMIC_RELAXED, __HIP_MEMORY_SCOPE_WORKGROUP);
      hs = __builtin_amdgcn_readfirstlane((unsigned)e); cds = __builtin_amdgcn_readfirstlane((unsigned)(e >> 32));
      if (hs) break;
      if (pend) { WAIT_V(0); UFLAG; pend = 0; continue; }
      if (__builtin_amdgcn_readfirstlane(__hip_atomic_load(lctl + 2, __ATOMIC_RELAXED, __HIP_MEMORY_SCOPE_WORKGROUP)) || __builtin_amdgcn_s_memrealtime() - t0 > 2 * DEADLINE_TICKS) return;
      __builtin_amdgcn_s_sleep(2);
    }
    const unsigned so = (hs >> 1) * SLAB + (hs & 1u) * 16384u;
    v4u C[4][3]; unsigned E[4];
    {
      unsigned va = imgo + so + lane * 16u + 4096u, ve = imgo + so + lane * 4u + 4096u;
#define UL(j, o) "global_load_dwordx4 %" #j ", %16, %18 offset:" #o " sc0 sc1\n\t"
      asm volatile(UL(0, -4096) UL(1, -2048) UL(2, 0) "global_load_dword %12, %17, %18 offset:2048 sc0 sc1\n\t"
                   UL(3, -3072) UL(4, -1024) UL(5, 1024) "global_load_dword %13, %17, %18 offset:3072 sc0 sc1\n\t"
                   "v_add_u32 %16, 0x2000, %16\n\tv_add_u32 %17, 0x2000, %17\n\t"
                   UL(6, -4096) UL(7, -2048) UL(8, 0) "global_load_dword %14, %17, %18 offset:2048 sc0 sc1\n\t"
                   UL(9, -3072) UL(10, -1024) UL(11, 1024) "global_load_dword %15, %17, %18 offset:3072 sc0 sc1\n\t"
                   "s_waitcnt vmcnt(16)"
                   : "=&v"(C[0][0]), "=&v"(C[0][1]), "=&v"(C[0][2]), "=&v"(C[1][0]), "=&v"(C[1][1]), "=&v"(C[1][2]),
                     "=&v"(C[2][0]), "=&v"(C[2][1]), "=&v"(C[2][2]), "=&v"(C[3][0]), "=&v"(C[3][1]), "=&v"(C[3][2]),
                     "=&v"(E[0]), "=&v"(E[1]), "=&v"(E[2]), "=&v"(E[3]), "+v"(va), "+v"(ve) : "s"(myws) : "memory");
    }
    UFLAG;
#pragma unroll
    for (int j = 0; j < 4; ++j) {
      asm volatile("s_waitcnt vmcnt(13)" : "+v"(C[j][0]), "+v"(C[j][1]), "+v"(C[j][2]), "+v"(E[j]) :: "memory");
      const unsigned cd = cds >> (8 * j) & 63u, c2 = (cd & 31u) * 0x04040404u;
      v4u D[4];
#pragma unroll
      for (int d = 0; d < 4; ++d) {
        const unsigned t0_ = C[j][0][d] & msk, t1_ = C[j][1][d] & msk, t2_ = C[j][2][d] & msk;
        asm("v_xad_u32 %0, %1, %2, %3" : "=v"(D[0][d]) : "v"(C[j][0][d]), "v"(t0_), "s"(c2));
        asm("v_xad_u32 %0, %1, %2, %3" : "=v"(D[1][d]) : "v"(C[j][1][d]), "v"(t1_), "s"(c2));
        asm("v_xad_u32 %0, %1, %2, %3" : "=v"(D[2][d]) : "v"(C[j][2][d]), "v"(t2_), "s"(c2));
        D[3][d] = __builtin_amdgcn_alignbit(t2_, t2_, 30) + (t1_ >> 2) + (t0_ >> 4) + c2;
      }
#pragma unroll
      for (int c = 0; c < 4; ++c) *(LAS v4u*)(buf + j * 4096 + c * 1024 + lane * 16) = D[c];
      const unsigned h = __builtin_amdgcn_readfirstlane(E[j]);
#pragma unroll
      for (int k = 0; k < 2; ++k) {
        const unsigned en = k ? E[j] >> 16 : E[j], ps = 2u * lane - 2u + k, y = en & 0x1fe0u, t = y | y << 6;
        const unsigned ch = (ps >= (h & 127u)) + (ps >= (h >> 8 & 127u)) + (ps >= (h >> 16 & 127u));
        const unsigned ad = lb + (ps < (h >> 24 & 127u) ? j * 4096u + ch * 1024u + (en & 31u) * 4u + (en >> 13 & 7u) * 128u : 16384u + lane * 4u), mv = (t | t << 12) & msk;
        asm volatile("ds_mskor_b32 %0, %1, %2" :: "v"(ad), "v"(msk), "v"(mv) : "memory");
      }
#pragma unroll
      for (int c = 0; c < 4; ++c) D[c] = *(const LAS v4u*)(buf + j * 4096 + c * 1024 + lane * 16);
      store_job(myws, (cd < 32u ? imgo : cwo) + so + (j >> 1) * 8192u + (j & 1) * 1024u + lane * 16u + 4096u, D);
    }
    pfo = flo + (hs >> 1) * 64u + (hs & 1u) * 16u; pend = 1;
  }
  WAIT_V(0); UFLAG;
}
#define NG 120u
#define N1 100u
#define XCNT 768u
__device__ __forceinline__ unsigned blk_job(int rank, unsigned q) { const unsigned bl = q / 7u; return ((rank + 1 + (q - bl * 7u)) & 7u) * PK_JOBS + bl * 32u; }
__device__ __forceinline__ void expand(const Params& p, LAS unsigned char* lds, unsigned call, int wid, unsigned q, unsigned mode) {
  unsigned char* const myws = p.ws[p.rank];
  LAS unsigned* lctl = (LAS unsigned*)(lds + LCTL);
  const int lane = fresh_lane();
  const unsigned lbase = (unsigned)(size_t)lds;
  for (unsigned r = 0;; ++r) {
    if (mode == 0u) {
      if (threadIdx.x == 0) lctl[6 + (r & 1u)] = __hip_atomic_fetch_add((unsigned*)(myws + WCT_OFF + (call & 1u) * 128u), 1u, __ATOMIC_RELAXED, __HIP_MEMORY_SCOPE_AGENT);
      __syncthreads();
      q = __builtin_amdgcn_readfirstlane(lctl[6 + (r & 1u)]); q += q < N1 ? 0u : NG;
    }
    if (q >= NBLK) break;
    const unsigned pj = blk_job(p.rank, q), snd = pj / PK_JOBS, vb = PK_OFF + (pj >> 5) * PK_BLK + lane * 16u;
    bool ok = true;
    if (mode != 2u) {
      ok = wait_flags(myws, OFLAG_OFF + (pj + wid + (lane & 24)) * 4u, call);
#pragma unroll
      for (int h = 0; h < 4; ++h) {
        const unsigned la = lbase + (wid + h * 8) * 3072u + (h >> 1) * 16384u, vo = vb + (wid + h * 8) * 3072u;
        asm volatile("s_mov_b32 m0, %0\n\ts_nop 0\n\tglobal_load_lds_dwordx4 %1, %2 sc0 sc1\n\tglobal_load_lds_dwordx4 %1, %2 offset:1024 sc0 sc1\n\t"
                     "global_load_lds_dwordx4 %1, %2 offset:2048 sc0 sc1" :: "s"(la), "v"(vo), "s"(myws) : "memory");
      }
      WAIT_V(0);
    }
#pragma unroll 2
    for (int h = 0; h < 4; ++h) {
      const unsigned j = wid + h * 8, oo = pk_off(snd, pj - snd * PK_JOBS + j) + lane * 16u, so = j * 3072u + (h >> 1) * 16384u;
      const LAS unsigned* pk = (const LAS unsigned*)(lds + so);
      const unsigned hdr = pk[lane & 31];
      unsigned pos[16], cnt = 32u; unsigned long long m[16];
#pragma unroll
      for (int i = 0; i < 16; ++i) {
        const unsigned lo = __builtin_amdgcn_readlane(hdr, 2 * i), hi = __builtin_amdgcn_readlane(hdr, 2 * i + 1);
        m[i] = (unsigned long long)hi << 32 | lo;
        pos[i] = __builtin_amdgcn_mbcnt_hi(hi, __builtin_amdgcn_mbcnt_lo(lo, cnt));
        cnt += (unsigned)__builtin_popcountll(m[i]);
      }
      if (cnt > XCNT) {
        const unsigned s2 = 49152u + wid * 2048u, la = lbase + s2, vo = vb + 98304u + j * 2048u, d2 = (s2 - so - 3072u) >> 2;
        asm volatile("s_waitcnt lgkmcnt(0)\n\ts_mov_b32 m0, %0\n\ts_nop 0\n\tglobal_load_lds_dwordx4 %1, %2 sc0 sc1\n\tglobal_load_lds_dwordx4 %1, %2 offset:1024 sc0 sc1\n\ts_waitcnt vmcnt(0)"
                     :: "s"(la), "v"(vo), "s"(myws) : "memory");
#pragma unroll
        for (int i = 0; i < 16; ++i) pos[i] = (pos[i] + (pos[i] >= 768u ? d2 : 0u)) & 0x3fffffffu;
      }
#pragma unroll
      for (int rr = 0; rr < 4; ++rr) {
        v4u v;
#pragma unroll
        for (int c = 0; c < 4; ++c) {
          const unsigned x = *(const LAS unsigned*)(lds + (so + pos[4 * rr + c] * 4u));
          v[c] = !ok ? 0x7fc00000u : (m[4 * rr + c] >> lane & 1ull) ? x : 0u;
        }
        const unsigned vo_ = oo + rr * (N_ * 4u);
        asm volatile("global_store_dwordx4 %0, %1, %2" XSM "\n\ts_nop 1" :: "v"(vo_), "v"(v), "s"(p.out[p.rank]) : "memory");
      }
    }
    if (mode) break;
  }
}

__global__ __launch_bounds__(512, 2) void fused_kernel(const Params p) {
  extern __shared__ __attribute__((aligned(16))) unsigned char smem[];
  LAS unsigned char* lds = (LAS unsigned char*)smem;
  const int tid = threadIdx.x, wid = __builtin_amdgcn_readfirstlane(tid >> 6), wc = wid & 3;
  unsigned char* myws = p.ws[p.rank];
  const int b = blockIdx.x, rank = p.rank, g = b - NCOM, gl = b - NPUSH;
  const bool push = b < NPUSH;
  LAS unsigned* lctl = (LAS unsigned*)(lds + LCTL);
  const unsigned lbase = (unsigned)(size_t)lds;
  const int xc = b & 7, L = (xc >> 2) * 28 + (g >> 3), cn = L >> 2;
  const int pm = (xc & 3) * 4 + (L & 3), si = L * 4 + (xc & 3);
  const bool skip = cn < 2 && si >= NSH;
  const unsigned nvis = skip ? 96u : 128u;
  auto tile_of = [&](int u, int& tm, int& tn) __attribute__((always_inline)) {
    const int v = u + (skip && u > 0), idx = NSH * rank + si;
    const bool sh = v == 1 && cn < 2;
    tm = sh ? idx & 15 : pm; tn = sh ? SHC + (idx >> 4) : v == 0 ? cn : 14 * v - 2 + cn;
  };
  unsigned call, imgo, flo;
  {
    const int lane = tid & 63;
    unsigned R[16], R2[16]; v4u X[4], X2[4];
    int t1, k1, j1 = wid;
    if (push) { j1 = (b % NTURN) * WPT + wid % WPT; stream_slab(wid / WPT * SPT + b / NTURN, rank, t1, k1); } else { t1 = gl >> 2; k1 = gl & 3; }
    load_job(p, t1, k1, j1, lane, R, X);
    const bool two = !push && gl < 80;
    if (two) load_job(p, t1 + 60, k1, wid, lane, R2, X2);
    if (tid == 0) {
      const unsigned t = __hip_atomic_fetch_add((unsigned*)(myws + CTR_OFF), 1u, __ATOMIC_RELAXED, __HIP_MEMORY_SCOPE_AGENT);
      lctl[0] = t / (unsigned)GRID + 1u; lctl[1] = INITK; lctl[2] = 0u; lctl[3] = 0u; lctl[5] = 0u;
      if (b == 0) {
        __hip_atomic_store((unsigned*)(myws + WCT_OFF + (t / (unsigned)GRID & 1u) * 128u), 0u, __ATOMIC_RELAXED, __HIP_MEMORY_SCOPE_AGENT);
        __hip_atomic_store((unsigned*)(myws + WCT_OFF + 64u + (t / (unsigned)GRID & 1u) * 128u), 0u, __ATOMIC_RELAXED, __HIP_MEMORY_SCOPE_AGENT);
        __hip_atomic_store((unsigned*)(myws + WCT_OFF + 256u + (t / (unsigned)GRID & 1u) * 128u), 0u, __ATOMIC_RELAXED, __HIP_MEMORY_SCOPE_AGENT);
      }
    }
    if (!push && g < 0 && tid < (int)NUNIT) *(LAS unsigned long long*)(lds + UTAB + tid * 8) = 0ull;
    if (wid >= 4 && g >= 0) {
      const int u = wid - 4, l = lane & 31, src = l >> 2, ks = l & 3;
      int tm, tn; tile_of(u, tm, tn);
      const int ax = stream_idx(tm, ks, src), aw = stream_idx(16 + tn, ks, src) + UDLY;
#define CORD 9
      const int key = src == rank ? ks : u >= CORD ? 32 + ks * 8 + ((src - rank) & 7) : ((ax > aw ? ax : aw) + 1) * 32 + l;
      int rk = 0;
#pragma unroll
      for (int s = 0; s < 32; ++s) rk += __builtin_amdgcn_readlane(key, s) < key;
      const unsigned ent = (unsigned)((16 + tn) * 32 + l) << 15 | (unsigned)(tm * 32 + l);
      if (u * 32u < nvis) *(LAS unsigned*)(lds + LTAB + (u * 32 + rk) * 4) = ent;
      if (u == 0 && rk < 4) *(LAS unsigned*)(lds + LTAB + (nvis + rk) * 4) = ent;
    }
    asm volatile("s_waitcnt vmcnt(0)" : TIE16(R), TIE4(X) :: "memory");
    asm volatile("" : TIE16(R2) :: "memory");
    __syncthreads();
    call = __builtin_amdgcn_readfirstlane(lctl[0]);
    imgo = IMG_OFF + (call & 1u) * IMG_BYTES;
    flo = FLAG_OFF + (call & 1u) * FLAG_BYTES;
    TS(0);
    if (push) {
      if (b < NPA) pusher(p, lds, imgo, flo, call, b, wid, lane, R, X);
      WAIT_V(0);
      expand(p, lds, call, wid, 0u, 0u);
      TS(14);
      return;
    }
    v4u D[4];
    const unsigned s1 = t1 * 32 + rank * 4 + k1;
    retile(t1 < 16, R, X, D);
    store_job(myws, imgo + s1 * SLAB + job_off(wid, lane) + 4096, D);
    if (two) {
      retile(false, R2, X2, D);
      store_job(myws, imgo + (s1 + 60 * 32) * SLAB + job_off(wid, lane) + 4096, D);
    }
    WAIT_V(0);
    store_flag1(myws, flo + s1 * 64 + wid * 4, call);
    if (two) store_flag1(myws, flo + (s1 + 60 * 32) * 64 + wid * 4, call);
    TS(1);
    if (g < 0) {
      unpacker(p, lds, imgo, flo, call, b, wid, lane);
      __syncthreads();
      TS(4);
      expand(p, lds, call, wid, 0u, 0u);
      TS(14);
      return;
    }
  }
  const float scale = __builtin_bit_cast(float, __builtin_amdgcn_readfirstlane(__builtin_bit_cast(int, p.sx[0] * p.sw[0])));
  WAIT_V(0);

  auto run = [&](auto WRC) __attribute__((always_inline)) {
    constexpr int wr = decltype(WRC)::value;
    unsigned ppend = 0, pk = INITK, pw = 4u;
    auto peval = [&]() __attribute__((always_inline)) {
      const v4u f = *(const LAS v4u*)(lds + LCP + (unsigned)fresh_lane() * 16);
      unsigned long long bm = __ballot(f[0] == call && f[1] == call && f[2] == call && f[3] == call);
      bm &= bm >> 1; bm &= bm >> 2;
      bm = ~bm & 0x1111111111111111ull;
      unsigned c = bm ? (unsigned)__builtin_ctzll(bm) >> 2 : 16u;
      c = c < pw ? c : pw;
      pw = c == pw ? 16u : PWIN;
      const unsigned n = pk + c < nvis ? pk + c : nvis;
      if (n > pk) {
        pk = n; lctl[1] = n;
      }
      ppend = 0;
    };
    auto pissue = [&]() __attribute__((always_inline)) {
      const unsigned ln = (unsigned)fresh_lane();
      unsigned e = pk + ((ln >> 2) < pw ? ln >> 2 : 0u); e = e < nvis ? e : nvis - 1;
      const unsigned kk = *(const LAS unsigned*)(lds + LTAB + e * 4);
      const unsigned la = lbase + LCP, fo = flo + ((ln & 2) ? kk >> 15 : kk & 0x7fffu) * 64 + (ln & 1) * 16; unsigned char* const w_ = myws;
      asm volatile("s_mov_b32 m0, %0\n\ts_nop 0\n\tglobal_load_lds_dwordx4 %1, %2 sc0 sc1" :: "s"(la), "v"(fo), "s"(w_) : "memory");
      ppend = 1;
    };
    auto poll = [&](bool blocking, unsigned need) __attribute__((always_inline)) {
      if (!blocking) {
        if (ppend) {
          unsigned st; asm volatile("s_getreg_b32 %0, hwreg(HW_REG_IB_STS)" : "=s"(st) :: "memory");
          st = __builtin_amdgcn_readfirstlane(st);
          if ((st & 15u) | ((st >> 18) & 48u)) return;
          WAIT_V(0);
          peval();
        } else { pw = 16u; pissue(); }
        return;
      }
      WAIT_V(0);
      if (ppend) { peval(); if (pk < need) __builtin_amdgcn_s_sleep(PSLEEP); }
      if (pk < nvis) pissue();
      WAIT_L(0);
    };
    auto slow = [&](unsigned need) __attribute__((always_inline)) {
      const unsigned long long t0 = __builtin_amdgcn_s_memrealtime();
      for (;;) {
        if (wr == 1 && wc == 0) poll(true, need); else __builtin_amdgcn_s_sleep(1);
        const unsigned long long el = __builtin_amdgcn_s_memrealtime() - t0;
        if (wr == 1 && wc == 0 && el > 100000ull) {
          unsigned* tmo = (unsigned*)(myws + TMO_OFF);
          if (__builtin_amdgcn_readfirstlane(__hip_atomic_load(tmo, __ATOMIC_RELAXED, __HIP_MEMORY_SCOPE_AGENT)) == call || el > DEADLINE_TICKS) {
            __hip_atomic_store(tmo, call, __ATOMIC_RELAXED, __HIP_MEMORY_SCOPE_AGENT);
            lctl[2] = 1u; lctl[1] = DEADV; pk = nvis; ppend = 0;
          }
          WAIT_V(0);
        }
        if (__builtin_amdgcn_readfirstlane(__hip_atomic_load(lctl + 1, __ATOMIC_RELAXED, __HIP_MEMORY_SCOPE_WORKGROUP)) >= need || el > 2 * DEADLINE_TICKS) break;
      }
      asm volatile("" ::: "memory");
    };

#define GT0 26
    auto grab = [&](int t) __attribute__((always_inline)) {
      const unsigned ln = (unsigned)fresh_lane() & 31u, q = __builtin_amdgcn_readfirstlane(lctl[4]), pj = blk_job(rank, q);
      if (q >= NBLK) return;
      if (t == GT0) {
        const unsigned la = lbase + LCTL + 64u, fo = OFLAG_OFF + (pj + ln) * 4u; unsigned char* const w_ = myws;
        asm volatile("s_mov_b32 m0, %0\n\ts_mov_b32 exec_hi, 0\n\tglobal_load_lds_dword %1, %2 sc0 sc1\n\ts_mov_b32 exec_hi, -1" :: "s"(la), "v"(fo), "s"(w_) : "memory");
      } else {
        WAIT_V(0);
        lctl[5] = __ballot(lctl[16 + ln] != call) == 0ull ? PK_OFF + (pj >> 5) * PK_BLK - imgo : 0u;
      }
    };
#define TBURST 3u
#define TEND 12604u
    unsigned ti = (unsigned)(g * 3 + wc - 1);
    auto touch = [&]() __attribute__((always_inline)) {
      const unsigned k = (ti >= 960u) + (ti >= 1920u) + (ti >= 2880u), r = ti - k * 960u, i2 = ti - 9244u, bq = i2 / 12u;
      const unsigned o = ti < 3840u ? imgo + ((r >> 2) * 4u + k) * SLAB + (r & 3u) * 8192u
                       : ti < 3848u ? (ti - 3840u) * 8192u
                       : ti < 3868u ? flo + (ti - 3848u) * 8192u
                       : ti < 4124u ? imgo + (70u + ((NSH * rank) >> 4)) * 32u * SLAB + (ti - 3868u) * 8192u
                       : ti < 9244u ? imgo + 30u * 32u * SLAB + (ti - 4124u) * 8192u
                                    : PK_OFF + (blk_job(rank, bq) >> 5) * PK_BLK + (i2 - bq * 12u) * 8192u;
      const unsigned la = lbase + LTCH, fo = o + (unsigned)fresh_lane() * 128u; unsigned char* const w_ = myws;
      asm volatile("s_mov_b32 m0, %0\n\ts_nop 0\n\tglobal_load_lds_dword %1, %2 sc0 sc1" :: "s"(la), "v"(fo), "s"(w_) : "memory");
      ti += 672u;
    };
    const unsigned char* XI = myws + imgo;
    const unsigned voff = (unsigned)(wc * 4096 + (tid & 63) * 16);
    const unsigned ldsw = (unsigned)wc * 4096u;
    const int aoff = wr * 8192 + (tid & 63) * 16, boff = wc * 4096 + (tid & 63) * 16;
#define SA(b_, h) (((b_) * 2 + (h)) * HTB)
#define SB(b_, h) ((4 + (b_) * 2 + (h)) * HTB)
#define STG1(bufoff, va, o) __builtin_amdgcn_global_load_lds((const unsigned*)(XI + (va)), (LAS unsigned*)(lds + (bufoff) + ldsw), 16, o, 0)
#define STAGE(bufoff, goff) do { if (wr == 0) { const unsigned va_ = voff + (goff); STG1(bufoff, va_, 0); STG1(bufoff, va_, 1024); STG1(bufoff, va_, 2048); STG1(bufoff, va_, 3072); } } while (0)
#define LDA(dst, b_, h) do { _Pragma("unroll") for (int m = 0; m < 4; ++m) { \
    v4i lo = *(const LAS v4i*)(lds + SA(b_, h) + aoff + m * 2048); v4i hi = *(const LAS v4i*)(lds + SA(b_, h) + aoff + m * 2048 + 1024); dst[m] = cat8(lo, hi); } } while (0)
#define LDB(dst, b_, h) do { _Pragma("unroll") for (int n = 0; n < 2; ++n) { \
    v4i lo = *(const LAS v4i*)(lds + SB(b_, h) + boff + n * 2048); v4i hi = *(const LAS v4i*)(lds + SB(b_, h) + boff + n * 2048 + 1024); dst[n] = cat8(lo, hi); } } while (0)
#define MMA(ai, bj, At, Bt) do { __builtin_amdgcn_s_setprio(1); _Pragma("unroll") for (int m = 0; m < 4; ++m) _Pragma("unroll") for (int n = 0; n < 2; ++n) \
    acc[ai][bj][m][n] = __builtin_amdgcn_mfma_scale_f32_16x16x128_f8f6f4(Bt[n], At[m], acc[ai][bj][m][n], 0, 1, 0, 0, 0, 0); __builtin_amdgcn_s_setprio(0); } while (0)
#define MF1(ai, bj, At, Bt, q) acc[ai][bj][(q) >> 1][(q) & 1] = __builtin_amdgcn_mfma_scale_f32_16x16x128_f8f6f4(Bt[(q) & 1], At[(q) >> 1], acc[ai][bj][(q) >> 1][(q) & 1], 0, 1, 0, 0, 0, 0)
#define WV12 WAIT_V(12)
#define MEMSTEP do { if (wr == 0) WV12; else if (wc == 0 && (ppend || (pk < nvis && pk < need + PTH))) poll(false, 0u); } while (0)
#define KOFF(i) ((unsigned)__builtin_amdgcn_readlane(kv, (i)))
    f32x4 acc[2][2][4][2];
#pragma unroll
    for (int a = 0; a < 2; ++a)
#pragma unroll
      for (int c = 0; c < 2; ++c)
#pragma unroll
        for (int m = 0; m < 4; ++m)
#pragma unroll
          for (int n = 0; n < 2; ++n) acc[a][c][m][n] = (f32x4){0.f, 0.f, 0.f, 0.f};
    v8i At[4], B0[2], B1[2];
    constexpr unsigned hstep = HTB;
#define OFA(e_) ((e_) & ~0x7fffu)
#define OFB(e_) (((e_) & 0x7fffu) << 15)
    if (wr == 1 && wc > 0) for (unsigned i = 0; i < TBURST; ++i) touch();
    slow(4u);
    unsigned known = 4u, need = 4u;
    const unsigned e0 = __builtin_amdgcn_readfirstlane(*(const LAS unsigned*)(lds + LTAB));
    {
      const unsigned cA = OFA(e0), cB = OFB(e0);
      STAGE(SB(0, 0), cB); STAGE(SA(0, 0), cA); STAGE(SB(0, 1), cB + hstep); STAGE(SA(0, 1), cA + hstep);
      if (wr == 1) BAR;
      if (wr == 0) WAIT_V(8);
      BAR;
      STAGE(SB(1, 0), cB + SLAB); STAGE(SA(1, 0), cA + SLAB); STAGE(SB(1, 1), cB + SLAB + hstep);
      if (wr == 0) WAIT_V(12);
      BAR;
    }
    unsigned a1 = OFA(e0) + SLAB, a2 = a1 + SLAB, a3 = a2 + SLAB, b2 = OFB(e0) + 2 * SLAB, b3 = b2 + SLAB;
    for (int ui = 0;; ++ui) {
      const bool has_next = (unsigned)(ui * 32 + 32) < nvis;
      int kv = 0;
      if (wr == 0) { const unsigned ln = ui * 32u + (unsigned)fresh_lane(); kv = *(const LAS int*)(lds + LTAB + (ln < 131u ? ln : 131u) * 4); }
      for (int t = 0; t < NKS; t += 2) {
        if (known < need) { if (wr == 0) BAR; slow(need); if (wr == 1) BAR; known = need; }
        LDB(B0, 0, 0); SCHED; LDA(At, 0, 0); SCHED; STAGE(SA(1, 1), a1 + hstep);
        WAIT_L(8); BAR; WAIT_L(0); MMA(0, 0, At, B0); BAR; SCHED;
        LDB(B1, 0, 1); SCHED; STAGE(SB(0, 0), b2);
        BAR; WAIT_L(0); MMA(0, 1, At, B1); BAR;
        LDA(At, 0, 1); SCHED; STAGE(SA(0, 0), a2);
        BAR; WAIT_L(0); MMA(1, 0, At, B0); BAR; SCHED;
        STAGE(SB(0, 1), b2 + hstep); MEMSTEP;
        if (wr == 1 && wc == 1 && ui == 3 && (t == GT0 || t == NKS - 4)) grab(t);
        if (wr == 1 && wc > 0 && ti < TEND) touch();
        BAR; MMA(1, 1, At, B1); BAR;
        LDB(B0, 1, 0); SCHED; LDA(At, 1, 0); SCHED; STAGE(SA(0, 1), a2 + hstep);
        WAIT_L(8); BAR; WAIT_L(0); MMA(0, 0, At, B0); BAR; SCHED;
        LDB(B1, 1, 1); const unsigned kn = lctl[1]; SCHED; STAGE(SB(1, 0), b3);
        BAR; WAIT_L(0); MMA(0, 1, At, B1); BAR;
        LDA(At, 1, 1); SCHED; STAGE(SA(1, 0), a3);
        BAR; WAIT_L(0); MMA(1, 0, At, B0); BAR; SCHED;
        STAGE(SB(1, 1), b3 + hstep); if (wr == 1 || has_next || t < NKS - 2) MEMSTEP;
        BAR; SCHED;
        __builtin_amdgcn_s_setprio(1);
        MF1(1, 1, At, B1, 0); SCHED;
        unsigned o2 = KOFF(t + 4), o3 = KOFF(t + 5); known = __builtin_amdgcn_readfirstlane(kn);
        asm volatile("" : "+s"(o2), "+s"(o3), "+s"(known)); SCHED;
        MF1(1, 1, At, B1, 1); SCHED;
        a1 = a3; a2 = OFA(o2); b2 = OFB(o2);
        asm volatile("" : "+s"(a1), "+s"(a2), "+s"(b2)); SCHED;
        MF1(1, 1, At, B1, 2); SCHED;
        a3 = OFA(o3); b3 = OFB(o3);
        if (wr == 0 && !has_next && t == NKS - 4) {
          const unsigned ob_ = __builtin_amdgcn_readfirstlane(lctl[5]);
          a2 = ob_ ? ob_ : a1; a3 = ob_ ? ob_ + 2 * hstep : a1; b2 = ob_ ? ob_ + 3 * hstep : a1; b3 = ob_ ? ob_ + 5 * hstep : a1;
        }
        asm volatile("" : "+s"(a2), "+s"(b2), "+s"(a3), "+s"(b3)); SCHED;
        MF1(1, 1, At, B1, 3); SCHED;
        need = need + 2u < nvis ? need + 2u : nvis;
        asm volatile("" : "+s"(need)); SCHED;
        MF1(1, 1, At, B1, 4); MF1(1, 1, At, B1, 5); MF1(1, 1, At, B1, 6); MF1(1, 1, At, B1, 7);
        __builtin_amdgcn_s_setprio(0);
        BAR;
      }
      int tm, tn; tile_of(ui, tm, tn);
      const bool dead = __builtin_amdgcn_readfirstlane(lctl[2]) != 0u, sht = tn >= SHC;
      TS(2 + 2 * ui);
      unsigned gv = 1u;
      const int ng = __builtin_amdgcn_readfirstlane((int)!(wc == 1 && ui == 2 && has_next));
      {
        const int ln = fresh_lane();
        const unsigned vo = (unsigned)((tm * 256 + wc * 32 + 4 * (ln >> 4)) * N_ + tn * 256 + wr * 64 + 4 * (ln & 15)) * 4u;
        const float qnan = __builtin_nanf("");
        const float sc_ = dead ? qnan : scale, lo_ = dead ? qnan : 0.f;
        const int shs = __builtin_amdgcn_readfirstlane(sht ? 1 : ui < PLN ? 2 : 0);
        unsigned char* const ob = p.out[rank];
        if (wr == 1) {
          const unsigned co = WCT_OFF + 256u + (call & 1u) * 128u; unsigned char* const w_ = myws;
          asm volatile("s_cmp_lg_u32 %3, 0\n\ts_cbranch_scc1 GE_%=\n\ts_mov_b64 exec, 1\n\tglobal_atomic_add %0, %1, %0, %2 sc0\n\ts_mov_b64 exec, -1\nGE_%=:"
                       : "+v"(gv) : "v"(co), "s"(w_), "s"(ng) : "memory", "scc");
        }
#pragma unroll
        for (int bj = 0; bj < 2; ++bj)
#pragma unroll
          for (int n = 0; n < 2; ++n)
#pragma unroll
            for (int r4 = 0; r4 < 4; ++r4) {
              unsigned char* rowp = ob + (size_t)(bj * 128 + n * 16 + r4) * N_ * 4;
#pragma unroll
              for (int ai = 0; ai < 2; ++ai) {
                f32x4 v;
#pragma unroll
                for (int c = 0; c < 4; ++c) {
                  const float x = acc[ai][bj][c][n][r4] * sc_;
                  asm("v_max_f32 %0, %1, %2" : "=v"(v[c]) : "s"(lo_), "v"(x));
                }
                asm volatile("s_cmp_lg_u32 %4, 0\n\ts_cbranch_scc1 SW_%=\n\tglobal_store_dwordx4 %0, %1, %2 offset:%3" OSM "\n\ts_branch SE_%=\n"
                             "SW_%=:\n\tglobal_store_dwordx4 %0, %1, %2 offset:%3 sc0 sc1" SHM "\nSE_%=:\n\ts_nop 1"
                             :: "v"(vo), "v"(v), "s"(rowp), "n"(ai * 512), "s"(shs) : "memory", "scc");
              }
            }
      }
      if (!has_next && wr == 0) WAIT_V(32);
      if (wr == 1) {
        asm volatile("s_waitcnt vmcnt(32)" : "+v"(gv) :: "memory");
        if (!ng) { const unsigned q_ = __builtin_amdgcn_readfirstlane(gv); lctl[4] = q_ < NG ? q_ + N1 : NBLK; }
      }
      if (sht) {
        WAIT_V(0);
        unsigned o_ = 0;
        if (fresh_lane() == 0) o_ = __hip_atomic_fetch_add(lctl + 3, 1u, __ATOMIC_RELAXED, __HIP_MEMORY_SCOPE_WORKGROUP);
        if (__builtin_amdgcn_readfirstlane(o_) == 7u) store_flag1(myws, DONE_OFF + si * 64, call);
        if (wr == 0) WAIT_V(0);
      }
      TS(3 + 2 * ui);
      if (!has_next) break;
#pragma unroll
      for (int a = 0; a < 2; ++a)
#pragma unroll
        for (int c = 0; c < 2; ++c)
#pragma unroll
          for (int m = 0; m < 4; ++m)
#pragma unroll
            for (int n = 0; n < 2; ++n) acc[a][c][m][n] = (f32x4){0.f, 0.f, 0.f, 0.f};
    }
    if (wr == 0) BAR;
  };
  if (wid < 4) run(std::integral_constant<int, 0>{}); else run(std::integral_constant<int, 1>{});
  expand(p, lds, call, wid, __builtin_amdgcn_readfirstlane(lctl[4]), skip ? 0u : __builtin_amdgcn_readfirstlane(lctl[5]) ? 2u : 1u);
  TS(14);
}
extern "C" void kernel_launch(int rank, int nranks, ncclComm_t comm,
                              hipStream_t stream,
                              void* const* d_in, const int* in_sizes, int n_in,
                              void* d_out, int out_size,
                              const int* in_dtypes, int out_dtype,
                              void* const* d_out_all,
                              void* const* d_ws_all, size_t ws_bytes) {
  if (nranks != NR || (unsigned)rank >= (unsigned)NR || ws_bytes < WS_NEED || n_in < 4) {
    fprintf(stderr, "kernel_launch: unexpected configuration\n"); return;
  }
  static bool attr_done[64] = {};
  int dev = 0; HIP_CHECK(hipGetDevice(&dev));
  if (!attr_done[dev & 63]) {
    HIP_CHECK(hipFuncSetAttribute((const void*)fused_kernel, hipFuncAttributeMaxDynamicSharedMemorySize, LDS_BYTES));
    attr_done[dev & 63] = true;
  }
  Params p;
  p.x = (const unsigned char*)d_in[0]; p.w = (const unsigned char*)d_in[1];
  p.sx = (const float*)d_in[2]; p.sw = (const float*)d_in[3];
  for (int i = 0; i < NR; ++i) { p.ws[i] = (unsigned char*)d_ws_all[i]; p.out[i] = (unsigned char*)(i == rank ? d_out : d_out_all[i]); }
  p.rank = rank;
  fused_kernel<<<GRID, 512, LDS_BYTES, stream>>>(p);
}
```

```cpp
#include <hip/hip_runtime.h>
#include <rccl/rccl.h>
#include <cstdio>
#include <type_traits>

#define HIP_CHECK(expr) do { hipError_t _e = (expr); if (_e != hipSuccess) { \
    fprintf(stderr, "HIP error %s at %s:%d\n", hipGetErrorString(_e), __FILE__, __LINE__); \
    return; } } while (0)

typedef int v8i __attribute__((ext_vector_type(8)));
typedef int v4i __attribute__((ext_vector_type(4)));
typedef unsigned v4u __attribute__((ext_vector_type(4)));
typedef float f32x4 __attribute__((ext_vector_type(4)));
#define LAS __attribute__((address_space(3)))

constexpr int NR = 8;
constexpr int M_ = 4096, N_ = 16384, KL = 512;
constexpr int NKS = 32;
constexpr int HTB = 16384;
constexpr int STAGE_BYTES = 8 * HTB;
constexpr unsigned LCTL = STAGE_BYTES, LTAB = STAGE_BYTES + 256, LCP = STAGE_BYTES + 1024;
constexpr unsigned LTCH = STAGE_BYTES + 2048;
constexpr int LDS_BYTES = STAGE_BYTES + 2048 + 256;
constexpr unsigned SLAB = 32768;
constexpr unsigned IMG_BYTES = 2560u * SLAB;
constexpr unsigned IMG_OFF = 1u << 20;
constexpr unsigned PK_OFF = IMG_OFF + 2 * IMG_BYTES, PK_SLOT = 5120, PK_JOBS = 1280;
constexpr unsigned PK_BLK = 32 * PK_SLOT, NBLK = 7 * PK_JOBS / 32;
constexpr unsigned CW_OFF = PK_OFF + NR * (PK_JOBS / 32) * PK_BLK - 512u * SLAB, CW_BYTES = 2048u * SLAB;
constexpr size_t WS_NEED = (size_t)CW_OFF + 512u * SLAB + (size_t)CW_BYTES;
constexpr unsigned CTR_OFF = 0, TMO_OFF = 128, WCT_OFF = 256, DONE_OFF = 4096, OFLAG_OFF = 8192, FLAG_OFF = 65536, FLAG_BYTES = 2560u * 64u, DBG_OFF = 524288;
constexpr int GRID = 256;
#define CTS 4
constexpr unsigned CTB_OFF = 49152;
constexpr int NPUSH = 16, NCOM = 32;
constexpr int NSH = 20, SHC = 54;
constexpr unsigned DEADV = 0x7fffffffu;
constexpr unsigned long long DEADLINE_TICKS = 90000000ull;
#define WPT 2

struct Params {
  const unsigned char* x;
  const unsigned char* w;
  const float* sx; const float* sw;
  unsigned char* ws[NR];
  unsigned char* out[NR];
  int rank;
};

#define TS(slot) do {} while (0)
#define TV(slot, v) do {} while (0)
__device__ __forceinline__ v8i cat8(v4i a, v4i b) { return __builtin_shufflevector(a, b, 0, 1, 2, 3, 4, 5, 6, 7); }
#define WAIT_V(n) asm volatile("s_waitcnt vmcnt(" #n ")" ::: "memory")
#define WAIT_L(n) asm volatile("s_waitcnt lgkmcnt(" #n ")" ::: "memory")
#define BAR __builtin_amdgcn_s_barrier()
#define SCHED __builtin_amdgcn_sched_barrier(0)
#define TIE16(R) "+v"(R[0]), "+v"(R[1]), "+v"(R[2]), "+v"(R[3]), "+v"(R[4]), "+v"(R[5]), "+v"(R[6]), "+v"(R[7]), \
                 "+v"(R[8]), "+v"(R[9]), "+v"(R[10]), "+v"(R[11]), "+v"(R[12]), "+v"(R[13]), "+v"(R[14]), "+v"(R[15])
#define TIE4(X) "+v"(X[0]), "+v"(X[1]), "+v"(X[2]), "+v"(X[3])

#define PKM ""
#define IMM ""
#define LDM ""
#define SHM ""
__device__ __forceinline__ int fresh_lane() { int l; asm volatile("v_mbcnt_lo_u32_b32 %0, -1, 0\n\tv_mbcnt_hi_u32_b32 %0, -1, %0" : "=v"(l)); return l; }

__device__ __forceinline__ void load_job(const Params& p, int tile, int ks, int jw, int lane, unsigned (&R)[16], v4u (&X)[4]) {
  const int i = lane & 15, g = lane >> 4;
  const bool isx = tile < 16, iso = tile >= 80;
  unsigned vo = isx ? i * KL + g * 32 : iso ? lane * 16 : g * 32 * N_ + 4 * i;
  const unsigned char* src = isx ? p.x + (size_t)(tile * 256 + (jw >> 1) * 64) * KL + ks * 128 + (jw & 1) * 16
                           : iso ? p.out[p.rank] + (unsigned)ks
                                 : p.w + (size_t)(ks * 128 + (jw & 1) * 16) * N_ + (tile - 16) * 256 + (jw >> 1) * 64;
  const unsigned st = isx ? 16 * KL : iso ? 4 * N_ : N_;
#define LD1(r) "global_load_dword %" #r ", %20, %21" LDM "\n\tv_add_u32 %20, %22, %20\n\t"
#define LD4(r) "global_load_dwordx4 %" #r ", %20, %21" LDM "\n\tv_add_u32 %20, %22, %20\n\t"
  asm volatile("s_cmp_eq_u32 %22, %23\n\ts_cbranch_scc1 LW_%=\n\t" LD4(16) LD4(17) LD4(18) LD4(19) "s_branch LE_%=\n"
               "LW_%=:\n\t" LD1(0) LD1(1) LD1(2) LD1(3) LD1(4) LD1(5) LD1(6) LD1(7) LD1(8) LD1(9) LD1(10) LD1(11) LD1(12) LD1(13) LD1(14) LD1(15) "\n"
               "LE_%=:"
               : "=&v"(R[0]), "=&v"(R[1]), "=&v"(R[2]), "=&v"(R[3]), "=&v"(R[4]), "=&v"(R[5]), "=&v"(R[6]), "=&v"(R[7]),
                 "=&v"(R[8]), "=&v"(R[9]), "=&v"(R[10]), "=&v"(R[11]), "=&v"(R[12]), "=&v"(R[13]), "=&v"(R[14]), "=&v"(R[15]),
                 "=&v"(X[0]), "=&v"(X[1]), "=&v"(X[2]), "=&v"(X[3]), "+v"(vo)
               : "s"(src), "s"(st), "n"(N_) : "memory", "scc");
}
__device__ __forceinline__ void retile(bool plain, const unsigned (&R)[16], const v4u (&X)[4], v4u (&D)[4]) {
  if (plain) {
#pragma unroll
    for (int e = 0; e < 4; ++e) D[e] = X[e];
  } else {
#pragma unroll
    for (int d = 0; d < 4; ++d) {
      const unsigned a = R[4 * d], bb = R[4 * d + 1], c = R[4 * d + 2], f = R[4 * d + 3];
      const unsigned t0_ = __builtin_amdgcn_perm(bb, a, 0x05010400u), t1_ = __builtin_amdgcn_perm(bb, a, 0x07030602u);
      const unsigned t2_ = __builtin_amdgcn_perm(f, c, 0x05010400u), t3_ = __builtin_amdgcn_perm(f, c, 0x07030602u);
      D[0][d] = __builtin_amdgcn_perm(t2_, t0_, 0x05040100u); D[1][d] = __builtin_amdgcn_perm(t2_, t0_, 0x07060302u);
      D[2][d] = __builtin_amdgcn_perm(t3_, t1_, 0x05040100u); D[3][d] = __builtin_amdgcn_perm(t3_, t1_, 0x07060302u);
    }
  }
}
__device__ __forceinline__ void store_job(unsigned char* ws, unsigned vo, const v4u (&D)[4]) {
  asm volatile("global_store_dwordx4 %0, %1, %5 offset:-4096 sc0 sc1\n\tglobal_store_dwordx4 %0, %2, %5 offset:-2048 sc0 sc1\n\t"
               "global_store_dwordx4 %0, %3, %5 sc0 sc1\n\tglobal_store_dwordx4 %0, %4, %5 offset:2048 sc0 sc1\n\ts_nop 1"
               :: "v"(vo), "v"(D[0]), "v"(D[1]), "v"(D[2]), "v"(D[3]), "s"(ws) : "memory");
}
__device__ __forceinline__ void store_flag1(unsigned char* ws, unsigned fo, unsigned v) {
  asm volatile("s_mov_b64 exec, 1\n\tglobal_store_dword %0, %1, %2 sc0 sc1\n\ts_mov_b64 exec, -1" :: "v"(fo), "v"(v), "s"(ws) : "memory");
}
__device__ __forceinline__ void store_flag7(unsigned long long fa, unsigned v) {
  asm volatile("s_mov_b64 exec, 0x7f\n\tglobal_store_dword %0, %1, off sc0 sc1\n\ts_mov_b64 exec, -1" :: "v"(fa), "v"(v) : "memory");
}
__device__ __forceinline__ unsigned job_off(int jw, int lane) { return (jw >> 1) * 8192 + (jw & 1) * 1024 + lane * 16; }

#define NMIX 4
__device__ __forceinline__ void stream_slab(int j, int snd, int& tile, int& ks) {
  if (j < 120) { ks = j / 30; int q = j - ks * 30; if (q < 14) { q += 2 * snd; if (q >= 14) q -= 14; tile = 16 + q; } else tile = (q - 14 + 2 * snd) & 15; }
  else if (j < 168) { const int u = j - 120; ks = u / 12; int q = u - ks * 12 + snd; if (q >= 12) q -= 12; tile = 30 + q; }
  else { const int u = j - 168, gq = u >= 56, v = u - gq * 56; ks = v / 14; int q = v - ks * 14 + 2 * snd; if (q >= 14) q -= 14; tile = 42 + 14 * gq + q; }
}
__device__ __forceinline__ int stream_idx(int tile, int ks, int snd) {
  if (tile < 16) return ks * 30 + 14 + ((tile - 2 * snd) & 15);
  if (tile < 30) { int q = tile - 16 - 2 * snd; if (q < 0) q += 14; return ks * 30 + q; }
  if (NMIX && tile >= 70) return 131 + ks * 12;
  if (tile < 42 || tile >= 70) { int q = (tile < 42 ? tile - 30 : tile - 70) - snd; if (q < 0) q += 12; return 120 + ks * 12 + q; }
  const int gq = tile >= 56; int q = tile - 42 - 14 * gq - 2 * snd; if (q < 0) q += 14; return 168 + gq * 56 + ks * 14 + q;
}
__device__ __forceinline__ unsigned shared_off(int r, int i) { const int idx = NSH * r + i; return (unsigned)((idx & 15) * 256 * N_ + (SHC + (idx >> 4)) * 256) * 4u; }

#define INITK 0u
#define LAG2 120
#define LAG 1
#define PWIN 4u
#define PTH 8u
#define INV 0
#define LASTWT 0
#define OST 1
#define OSM " nt"
#define PLN 0
#define XST OST
#define XSM " nt"
#define PSLEEP 8
__device__ __forceinline__ bool wait_flags(unsigned char* ws, unsigned fo, unsigned call) {
  const unsigned long long t0 = __builtin_amdgcn_s_memrealtime();
  unsigned* tmo = (unsigned*)(ws + TMO_OFF);
  for (;;) {
    unsigned v, tm; const unsigned to = TMO_OFF;
    asm volatile("global_load_dword %0, %2, %4 sc0 sc1\n\tglobal_load_dword %1, %3, %4 sc0 sc1\n\ts_waitcnt vmcnt(0)" : "=&v"(v), "=&v"(tm) : "v"(fo), "v"(to), "s"(ws) : "memory");
    if (__ballot(v != call) == 0ull) return true;
    if (__builtin_amdgcn_readfirstlane(tm) == call || __builtin_amdgcn_s_memrealtime() - t0 > DEADLINE_TICKS) {
      __hip_atomic_store(tmo, call, __ATOMIC_RELAXED, __HIP_MEMORY_SCOPE_AGENT);
      return false;
    }
    __builtin_amdgcn_s_sleep(16);
  }
}
constexpr int NTURN = 8 / WPT;
#define NPA 16
constexpr int SPT = NPA * WPT / 8, NT = 280 / SPT;
__device__ __forceinline__ unsigned pk_off(int r, unsigned jj) { return shared_off(r, jj >> 6) + ((jj & 63u) * 4u) * (N_ * 4u); }
#define ST1(vc, d, k, md) "global_store_dwordx4 " vc ", " d ", %" #k " sc0 sc1" md "\n\t"
#define ST7(vc, d, md) ST1(vc, d, 2, md) ST1(vc, d, 3, md) ST1(vc, d, 4, md) ST1(vc, d, 5, md) ST1(vc, d, 6, md) ST1(vc, d, 7, md) ST1(vc, d, 8, md)
__device__ __forceinline__ unsigned bfi(unsigned m, unsigned a, unsigned b) { return (a & m) | (b & ~m); }
__device__ __forceinline__ unsigned encode(v4u (&S)[4], LAS unsigned char* el, int lane, unsigned long long& em) {
  unsigned sm = 0;
#pragma unroll
  for (int d = 0; d < 4; ++d) sm = __builtin_amdgcn_udot4(S[0][d] & 0x7c7c7c7cu, 0x01010101u, sm, false);
  sm += __builtin_amdgcn_update_dpp(0, (int)sm, 0x111, 0xf, 0xf, true); sm += __builtin_amdgcn_update_dpp(0, (int)sm, 0x112, 0xf, 0xf, true);
  sm += __builtin_amdgcn_update_dpp(0, (int)sm, 0x114, 0xf, 0xf, true); sm += __builtin_amdgcn_update_dpp(0, (int)sm, 0x118, 0xf, 0xf, true);
  int e0 = (__builtin_amdgcn_readlane(sm, 15) + __builtin_amdgcn_readlane(sm, 31) + __builtin_amdgcn_readlane(sm, 47) + __builtin_amdgcn_readlane(sm, 63) - 17121) >> 12;
  e0 = e0 < 0 ? 0 : e0 > 24 ? 24 : e0;
  const unsigned c1 = (unsigned)(128 - 4 * e0) * 0x01010101u, ix = (unsigned)((lane & 7) * 4 | (lane >> 3) << 13);
  unsigned cnt = 0, hdr = 0; v4u U[4];
#pragma unroll
  for (int c = 0; c < 4; ++c) {
#pragma unroll
    for (int d = 0; d < 4; ++d) {
      const unsigned v = S[c][d], u = (v & 0x7f7f7f7fu) + c1;
      const bool esc = (u & 0xe0e0e0e0u) != 0x80808080u;
      const unsigned long long m = __ballot(esc);
      const unsigned pos = __builtin_amdgcn_mbcnt_hi((unsigned)(m >> 32), __builtin_amdgcn_mbcnt_lo((unsigned)m, cnt));
      *(LAS unsigned short*)(el + 4 + 2 * (esc ? pos : 1100u + (unsigned)lane)) = (unsigned short)(__builtin_amdgcn_udot4(v & 0x60606060u, 0x40100401u, ix, false) + d);
      cnt += (unsigned)__builtin_popcountll(m);
      U[c][d] = bfi(0x80808080u, v, u);
    }
    hdr |= (cnt & 127u) << (8 * c);
  }
  em = ~0ull;
  if (cnt > 126u) return 32u;
  if (lane == 0) *(LAS unsigned*)el = hdr;
#pragma unroll
  for (int d = 0; d < 4; ++d) {
    const unsigned z = U[3][d];
    S[0][d] = bfi(0x60606060u, z << 4, U[0][d]); S[1][d] = bfi(0x60606060u, z << 2, U[1][d]); S[2][d] = bfi(0x60606060u, __builtin_amdgcn_alignbit(z, z, 2), U[2][d]);
  }
  S[3] = *(const LAS v4u*)(el + (lane & 15) * 16);
  em = (1ull << ((cnt + 33u) >> 5) * 4u) - 1ull;
  return (unsigned)e0;
}
__device__ __forceinline__ void pusher(const Params& p, LAS unsigned char* lds, unsigned imgo, unsigned flo, unsigned call, int b, int wid, int lane, unsigned (&R)[16], v4u (&X)[4]) {
  unsigned char* const myws = p.ws[p.rank];
  const int tau = wid / WPT, e = tau * SPT + b / NTURN, jw = (b % NTURN) * WPT + wid % WPT;
  unsigned long long fa = 0;
  unsigned char* wsd[7]; int pr[7];
#pragma unroll
  for (int k = 0; k < 7; ++k) {
    pr[k] = (p.rank + 1 + (wid + b + k) % 7) & 7; wsd[k] = p.ws[pr[k]];
    if (lane == k) fa = (unsigned long long)wsd[k];
  }
  const unsigned jo = job_off(jw, lane), dno = DONE_OFF + (lane < NSH ? lane : 0) * 64u, cwo = CW_OFF;
  LAS unsigned* const pk = (LAS unsigned*)(lds + wid * PK_SLOT);
  int tile, ks; stream_slab(e, p.rank, tile, ks);
  unsigned job = e * 8 + jw, cd = 32u;
  unsigned long long em = ~0ull;
  v4u D[4];
  retile(tile < 16, R, X, D);
  if (tile >= 16) cd = encode(D, (LAS unsigned char*)pk, lane, em);
  static_assert(NTURN == 4, "turn structure");
  v4u D2[4]; unsigned R2[16], cd2 = 0u; unsigned long long em2 = 0ull;
  int tn, kn;
  auto next = [&](int Tn) __attribute__((always_inline)) {
    stream_slab(Tn * SPT + b / NTURN, p.rank, tn, kn);
    load_job(p, tn, kn, jw, lane, R, X);
    unsigned vo = (lane >> 4) * 32 * N_ + 4 * (lane & 15);
    const unsigned char* src = p.w + (size_t)(kn * 128 + (jw & 1) * 16) * N_ + ((unsigned)(tn - 30) < 10u ? SHC + (tn - 30 + NMIX * jw) % 10 : 0) * 256 + (jw >> 1) * 64;
#define LN1(r) "global_load_dword %" #r ", %16, %17\n\tv_add_u32 %16, %19, %16\n\t"
    asm volatile("s_cmp_ge_u32 %18, 10\n\ts_cbranch_scc1 LN_%=\n\t" LN1(0) LN1(1) LN1(2) LN1(3) LN1(4) LN1(5) LN1(6) LN1(7) LN1(8) LN1(9) LN1(10) LN1(11) LN1(12) LN1(13) LN1(14) LN1(15) "\nLN_%=:"
                 : "=&v"(R2[0]), "=&v"(R2[1]), "=&v"(R2[2]), "=&v"(R2[3]), "=&v"(R2[4]), "=&v"(R2[5]), "=&v"(R2[6]), "=&v"(R2[7]),
                   "=&v"(R2[8]), "=&v"(R2[9]), "=&v"(R2[10]), "=&v"(R2[11]), "=&v"(R2[12]), "=&v"(R2[13]), "=&v"(R2[14]), "=&v"(R2[15]), "+v"(vo)
                 : "s"(src), "s"((unsigned)(tn - 30)), "n"(N_) : "memory", "scc");
  };
  next(tau + NTURN);
  for (int i = 0; i < tau; ++i) BAR;
  for (int T = tau; T < NT; T += NTURN) {
    const unsigned sid = tile * 32 + p.rank * 4 + ks, vb = imgo + jo + sid * SLAB;
    const unsigned fo = flo + sid * 64u + jw * 4u + (tile >= 16 ? 32u : 0u), fv = tile >= 16 ? call << 6 | cd : call;
    const bool nar = tile >= 30 && tile < 40;
    unsigned long long fn = 0;
    if (nar) {
      const int nq = (tile - 30 + NMIX * jw) % 10, ra = nq * 16 / NSH, rb = (nq * 16 + 15) / NSH;
      const unsigned ds = (unsigned)(70 + nq - tile) * 32u;
      unsigned char* const da = ra != p.rank ? p.ws[ra & 7] : myws + (cwo - imgo);
      unsigned char* const db = rb != p.rank && rb != ra ? p.ws[rb & 7] : myws + (cwo - imgo);
      fn = (unsigned long long)(lane ? p.ws[ra & 7] : p.ws[rb & 7]) + fo + ds * 64u;
#pragma unroll
      for (int c = 0; c < 4; ++c) {
        const unsigned vc = vb + ds * SLAB + c * 2048u;
        const unsigned long long em_ = c == 3 ? em2 : ~0ull;
        asm volatile("s_mov_b64 exec, %4\n\t" ST1("%0", "%1", 2, "") ST1("%0", "%1", 3, "") "s_mov_b64 exec, -1\n\ts_nop 1" :: "v"(vc), "v"(D2[c]), "s"(da), "s"(db), "s"(em_) : "memory");
      }
    }
#pragma unroll
    for (int c = 0; c < 4; ++c) {
      const unsigned vc = vb + c * 2048u;
      const unsigned long long em_ = c == 3 ? em : ~0ull;
      asm volatile("s_mov_b64 exec, %9\n\t" ST7("%0", "%1", IMM) "s_mov_b64 exec, -1\n\ts_nop 1"
                   :: "v"(vc), "v"(D[c]), "s"(wsd[0]), "s"(wsd[1]), "s"(wsd[2]), "s"(wsd[3]), "s"(wsd[4]), "s"(wsd[5]), "s"(wsd[6]), "s"(em_) : "memory");
    }
    const int nb = NT - 1 - T < NTURN ? NT - 1 - T : NTURN;
    const int lag2 = __builtin_amdgcn_readfirstlane((int)(T * SPT >= LAG2));
    asm volatile("s_cmp_eq_u32 %20, 0\n\ts_cbranch_scc1 PB_%=\n\ts_barrier\nPB_%=:\n\ts_waitcnt vmcnt(36)\n\ts_cmp_lt_u32 %22, 10\n\ts_cbranch_scc1 PD_%=\n\ts_waitcnt vmcnt(28)\n"
                 "PD_%=:\n\ts_cmp_lg_u32 %21, 0\n\ts_cbranch_scc1 PC_%=\n\ts_waitcnt vmcnt(0)\nPC_%=:"
                 : TIE16(R), TIE4(X) : "s"(nb), "s"(lag2), "s"((unsigned)(tile - 30)) : "memory", "scc");
    asm volatile("" : TIE16(R2) :: "memory");
    const unsigned fv2 = call << 6 | cd2;
#define NFLAG if (nar) asm volatile("s_mov_b64 exec, 3\n\tglobal_store_dword %0, %1, off sc0 sc1\n\ts_mov_b64 exec, -1" :: "v"(fn), "v"(fv2) : "memory")
    if (!lag2) { store_flag7(fa + fo, fv); NFLAG; if (nb > 1 && NTURN > 2) BAR; }
    tile = tn; ks = kn;
    const bool nn = (unsigned)(tile - 30) < 10u;
    retile(tile < 16, R, X, D);
    cd = 32u; em = ~0ull;
    if (tile >= 16) cd = encode(D, (LAS unsigned char*)pk, lane, em);
    if (lag2) { if (nb > 1 && NTURN > 2) BAR; WAIT_V(0); store_flag7(fa + fo, fv); NFLAG; }
#define LDI 3
    if (nn) {
      retile(false, R2, X, D2);
      cd2 = encode(D2, (LAS unsigned char*)pk, lane, em2);
    }
    if (LDI == 3 && nb > 2) BAR;
    next(T + 2 * NTURN < NT ? T + 2 * NTURN : T);
    for (int i = LDI; i < nb; ++i) BAR;
  }
  const bool ok = wait_flags(myws, dno, call);
  load_job(p, 80, (int)pk_off(p.rank, job), jw, lane, R, X);
  asm volatile("s_waitcnt vmcnt(0)" : TIE16(R), TIE4(X) :: "memory");
  for (;;) {
    unsigned nx = 1u; const unsigned co = WCT_OFF + 64u + (call & 1u) * 128u;
    asm volatile("s_nop 1\n\ts_mov_b64 exec, 1\n\tglobal_atomic_add %0, %1, %0, %2 sc0\n\ts_mov_b64 exec, -1" : "+v"(nx) : "v"(co), "s"(myws) : "memory");
    unsigned cnt = 32u, hdr = 0u;
#pragma unroll
    for (int j = 0; j < 16; ++j) {
      const unsigned v = ok ? X[j >> 2][j & 3] : 0x7fc00000u;
      const unsigned long long m = __ballot(v != 0u);
      const unsigned lo = (unsigned)m, hi = (unsigned)(m >> 32);
      pk[v != 0u ? __builtin_amdgcn_mbcnt_hi(hi, __builtin_amdgcn_mbcnt_lo(lo, cnt)) : 1100u + (unsigned)lane] = v;
      hdr = lane == 2 * j ? lo : lane == 2 * j + 1 ? hi : hdr;
      cnt += (unsigned)__builtin_popcountll(m);
    }
    pk[lane < 32 ? (unsigned)lane : 1100u + (unsigned)lane] = hdr;
    const unsigned nd = (cnt + 31u) & ~31u, pj = p.rank * PK_JOBS + job;
#pragma unroll
    for (int c = 0; c < 5; ++c) {
      const int nl = (int)(nd >> 2) - c * 64;
      if (nl > 0) {
        const v4u d = *(const LAS v4u*)(pk + c * 256 + lane * 4);
        const unsigned vc = PK_OFF + (pj >> 5) * PK_BLK + (c < 3 ? (pj & 31u) * 3072u : 95232u + (pj & 31u) * 2048u) + c * 1024u + lane * 16u;
        const unsigned long long em = nl >= 64 ? ~0ull : (1ull << nl) - 1ull;
        asm volatile("s_mov_b64 exec, %9\n\t" ST7("%0", "%1", PKM) "s_mov_b64 exec, -1\n\ts_nop 1"
                     :: "v"(vc), "v"(d), "s"(wsd[0]), "s"(wsd[1]), "s"(wsd[2]), "s"(wsd[3]), "s"(wsd[4]), "s"(wsd[5]), "s"(wsd[6]), "s"(em) : "memory");
      }
    }
    asm volatile("s_waitcnt vmcnt(0)" : "+v"(nx) :: "memory");
    job = __builtin_amdgcn_readfirstlane(nx) + NPA * 8u;
    const bool more = job < PK_JOBS;
    {
      unsigned vo = lane * 16u; const unsigned char* src = p.out[p.rank] + pk_off(p.rank, more ? job : 0u);
      asm volatile("s_cmp_ge_u32 %6, %8\n\ts_cbranch_scc1 OE_%=\n\tglobal_load_dwordx4 %0, %4, %5\n\tv_add_u32 %4, %7, %4\n\tglobal_load_dwordx4 %1, %4, %5\n\tv_add_u32 %4, %7, %4\n\t"
                   "global_load_dwordx4 %2, %4, %5\n\tv_add_u32 %4, %7, %4\n\tglobal_load_dwordx4 %3, %4, %5\nOE_%=:"
                   : "=&v"(X[0]), "=&v"(X[1]), "=&v"(X[2]), "=&v"(X[3]), "+v"(vo) : "s"(src), "s"(job), "n"(N_ * 4), "n"(PK_JOBS) : "memory", "scc");
    }
    store_flag7(fa + OFLAG_OFF + pj * 4u, call);
    if (!more) break;
    asm volatile("s_waitcnt vmcnt(1)" : TIE4(X) :: "memory");
  }
  TS(1); TS(9);
}
constexpr unsigned NUNIT = 196, UTAB = 120 * 1024;
#define UDLY 12
#define UWIN 24u
__device__ __forceinline__ void unpacker(const Params& p, LAS unsigned char* lds, unsigned imgo, unsigned flo, unsigned call, int b, int wid, int lane) {
  unsigned char* const myws = p.ws[p.rank];
  LAS unsigned* lctl = (LAS unsigned*)(lds + LCTL);
  const unsigned long long t0 = __builtin_amdgcn_s_memrealtime();
  if (wid == 0) {
    unsigned i = lane, done = 0;
    for (;;) {
      const unsigned q = (i < NUNIT ? i : 0u) * 16u + (b - NPUSH), wp = q / 14u, r = q - wp * 14u;
      const int snd = (p.rank + 1 + (r >> 1)) & 7;
      const int k14 = wp / 14u, gp = k14 >> 2, ks = k14 & 3, q14 = wp - k14 * 14;
      int qq = q14 + (gp == 1 ? snd : 2 * snd); qq -= qq >= (gp == 1 ? 12 : 14) ? (gp == 1 ? 12 : 14) : 0;
      const int tile = gp == 1 && q14 >= 12 ? 58 + ((NSH * p.rank) >> 4) + q14 : (gp == 0 ? 16 : 14 * gp + (gp == 1 ? 16 : 14)) + qq;
      const unsigned hs = (tile * 32 + snd * 4 + ks) * 2 + (r & 1u), fo = flo + (hs >> 1) * 64u + 32u + (hs & 1u) * 16u, to = TMO_OFF;
      const unsigned long long pm = __ballot(i < NUNIT && i < done + UWIN);
      v4u v = {0u, 0u, 0u, 0u}; unsigned tm;
      asm volatile("s_mov_b64 exec, %5\n\tglobal_load_dwordx4 %0, %2, %4 sc0 sc1\n\ts_mov_b64 exec, -1\n\tglobal_load_dword %1, %3, %4 sc0 sc1\n\ts_waitcnt vmcnt(0)"
                   : "+v"(v), "=&v"(tm) : "v"(fo), "v"(to), "s"(myws), "s"(pm) : "memory");
      const unsigned tag = call & 0x3ffffffu;
      const bool ok = (pm >> lane & 1ull) && v[0] >> 6 == tag && v[1] >> 6 == tag && v[2] >> 6 == tag && v[3] >> 6 == tag;
      if (ok) {
        *(LAS unsigned long long*)(lds + UTAB + i * 8u) = (unsigned long long)((v[0] & 63u) | (v[1] & 63u) << 8 | (v[2] & 63u) << 16 | (v[3] & 63u) << 24) << 32 | hs;
        i += 64u;
      }
      const unsigned nk = (unsigned)__builtin_popcountll(__ballot(ok));
      done += nk;
      if (done >= NUNIT) break;
      if (__builtin_amdgcn_readfirstlane(tm) == call || __builtin_amdgcn_s_memrealtime() - t0 > DEADLINE_TICKS) {
        __hip_atomic_store((unsigned*)(myws + TMO_OFF), call, __ATOMIC_RELAXED, __HIP_MEMORY_SCOPE_AGENT);
        lctl[2] = 1u;
        break;
      }
      if (!nk) __builtin_amdgcn_s_sleep(4);
    }
    TS(3);
    return;
  }
  const unsigned cwo = CW_OFF, lb = (unsigned)(size_t)lds + (wid - 1) * 16640u, msk = 0x60606060u;
  LAS unsigned char* const buf = lds + (wid - 1) * 16640u;
  unsigned pfo = DONE_OFF + 4032u, pend = 0;
  const v4u cv = {call, call, call, call};
#define UFLAG asm volatile("s_mov_b64 exec, 1\n\tglobal_store_dwordx4 %0, %1, %2 sc0 sc1\n\ts_mov_b64 exec, -1\n\ts_nop 1" :: "v"(pfo), "v"(cv), "s"(myws) : "memory")
  for (unsigned i = wid - 1; i < NUNIT; i += 7u) {
    unsigned hs, cds;
    for (;;) {
      const unsigned long long e = __hip_atomic_load((LAS unsigned long long*)(lds + UTAB + i * 8u), __ATOMIC_RELAXED, __HIP_MEMORY_SCOPE_WORKGROUP);
      hs = __builtin_amdgcn_readfirstlane((unsigned)e); cds = __builtin_amdgcn_readfirstlane((unsigned)(e >> 32));
      if (hs) break;
      if (pend) { WAIT_V(0); UFLAG; pend = 0; continue; }
      if (__builtin_amdgcn_readfirstlane(__hip_atomic_load(lctl + 2, __ATOMIC_RELAXED, __HIP_MEMORY_SCOPE_WORKGROUP)) || __builtin_amdgcn_s_memrealtime() - t0 > 2 * DEADLINE_TICKS) return;
      __builtin_amdgcn_s_sleep(2);
    }
    const unsigned so = (hs >> 1) * SLAB + (hs & 1u) * 16384u;
    v4u C[4][3]; unsigned E[4];
    {
      unsigned va = imgo + so + lane * 16u + 4096u, ve = imgo + so + lane * 4u + 4096u;
#define UL(j, o) "global_load_dwordx4 %" #j ", %16, %18 offset:" #o " sc0 sc1\n\t"
      asm volatile(UL(0, -4096) UL(1, -2048) UL(2, 0) "global_load_dword %12, %17, %18 offset:2048 sc0 sc1\n\t"
                   UL(3, -3072) UL(4, -1024) UL(5, 1024) "global_load_dword %13, %17, %18 offset:3072 sc0 sc1\n\t"
                   "v_add_u32 %16, 0x2000, %16\n\tv_add_u32 %17, 0x2000, %17\n\t"
                   UL(6, -4096) UL(7, -2048) UL(8, 0) "global_load_dword %14, %17, %18 offset:2048 sc0 sc1\n\t"
                   UL(9, -3072) UL(10, -1024) UL(11, 1024) "global_load_dword %15, %17, %18 offset:3072 sc0 sc1\n\t"
                   "s_waitcnt vmcnt(16)"
                   : "=&v"(C[0][0]), "=&v"(C[0][1]), "=&v"(C[0][2]), "=&v"(C[1][0]), "=&v"(C[1][1]), "=&v"(C[1][2]),
                     "=&v"(C[2][0]), "=&v"(C[2][1]), "=&v"(C[2][2]), "=&v"(C[3][0]), "=&v"(C[3][1]), "=&v"(C[3][2]),
                     "=&v"(E[0]), "=&v"(E[1]), "=&v"(E[2]), "=&v"(E[3]), "+v"(va), "+v"(ve) : "s"(myws) : "memory");
    }
    UFLAG;
#pragma unroll
    for (int j = 0; j < 4; ++j) {
      asm volatile("s_waitcnt vmcnt(13)" : "+v"(C[j][0]), "+v"(C[j][1]), "+v"(C[j][2]), "+v"(E[j]) :: "memory");
      const unsigned cd = cds >> (8 * j) & 63u, c2 = (cd & 31u) * 0x04040404u;
      v4u D[4];
#pragma unroll
      for (int d = 0; d < 4; ++d) {
        const unsigned t0_ = C[j][0][d] & msk, t1_ = C[j][1][d] & msk, t2_ = C[j][2][d] & msk;
        asm("v_xad_u32 %0, %1, %2, %3" : "=v"(D[0][d]) : "v"(C[j][0][d]), "v"(t0_), "s"(c2));
        asm("v_xad_u32 %0, %1, %2, %3" : "=v"(D[1][d]) : "v"(C[j][1][d]), "v"(t1_), "s"(c2));
        asm("v_xad_u32 %0, %1, %2, %3" : "=v"(D[2][d]) : "v"(C[j][2][d]), "v"(t2_), "s"(c2));
        D[3][d] = __builtin_amdgcn_alignbit(t2_, t2_, 30) + (t1_ >> 2) + (t0_ >> 4) + c2;
      }
#pragma unroll
      for (int c = 0; c < 4; ++c) *(LAS v4u*)(buf + j * 4096 + c * 1024 + lane * 16) = D[c];
      const unsigned h = __builtin_amdgcn_readfirstlane(E[j]);
#pragma unroll
      for (int k = 0; k < 2; ++k) {
        const unsigned en = k ? E[j] >> 16 : E[j], ps = 2u * lane - 2u + k, y = en & 0x1fe0u, t = y | y << 6;
        const unsigned ch = (ps >= (h & 127u)) + (ps >= (h >> 8 & 127u)) + (ps >= (h >> 16 & 127u));
        const unsigned ad = lb + (ps < (h >> 24 & 127u) ? j * 4096u + ch * 1024u + (en & 31u) * 4u + (en >> 13 & 7u) * 128u : 16384u + lane * 4u), mv = (t | t << 12) & msk;
        asm volatile("ds_mskor_b32 %0, %1, %2" :: "v"(ad), "v"(msk), "v"(mv) : "memory");
      }
#pragma unroll
      for (int c = 0; c < 4; ++c) D[c] = *(const LAS v4u*)(buf + j * 4096 + c * 1024 + lane * 16);
      store_job(myws, (cd < 32u ? imgo : cwo) + so + (j >> 1) * 8192u + (j & 1) * 1024u + lane * 16u + 4096u, D);
    }
    pfo = flo + (hs >> 1) * 64u + (hs & 1u) * 16u; pend = 1;
  }
  WAIT_V(0); UFLAG;
}
#define NG 120u
#define N1 100u
#define XCNT 768u
__device__ __forceinline__ unsigned blk_job(int rank, unsigned q) { const unsigned bl = q / 7u; return ((rank + 1 + (q - bl * 7u)) & 7u) * PK_JOBS + bl * 32u; }
__device__ __forceinline__ void expand(const Params& p, LAS unsigned char* lds, unsigned call, int wid, unsigned q, unsigned mode) {
  unsigned char* const myws = p.ws[p.rank];
  LAS unsigned* lctl = (LAS unsigned*)(lds + LCTL);
  const int lane = fresh_lane();
  const unsigned lbase = (unsigned)(size_t)lds;
  for (unsigned r = 0;; ++r) {
    if (mode == 0u) {
      if (threadIdx.x == 0) lctl[6 + (r & 1u)] = __hip_atomic_fetch_add((unsigned*)(myws + WCT_OFF + (call & 1u) * 128u), 1u, __ATOMIC_RELAXED, __HIP_MEMORY_SCOPE_AGENT);
      __syncthreads();
      q = __builtin_amdgcn_readfirstlane(lctl[6 + (r & 1u)]); q += q < N1 ? 0u : NG;
    }
    if (q >= NBLK) break;
    const unsigned pj = blk_job(p.rank, q), snd = pj / PK_JOBS, vb = PK_OFF + (pj >> 5) * PK_BLK + lane * 16u;
    bool ok = true;
    if (mode != 2u) {
      ok = wait_flags(myws, OFLAG_OFF + (pj + wid + (lane & 24)) * 4u, call);
#pragma unroll
      for (int h = 0; h < 4; ++h) {
        const unsigned la = lbase + (wid + h * 8) * 3072u + (h >> 1) * 16384u, vo = vb + (wid + h * 8) * 3072u;
        asm volatile("s_mov_b32 m0, %0\n\ts_nop 0\n\tglobal_load_lds_dwordx4 %1, %2 sc0 sc1\n\tglobal_load_lds_dwordx4 %1, %2 offset:1024 sc0 sc1\n\t"
                     "global_load_lds_dwordx4 %1, %2 offset:2048 sc0 sc1" :: "s"(la), "v"(vo), "s"(myws) : "memory");
      }
      WAIT_V(0);
    }
#pragma unroll 2
    for (int h = 0; h < 4; ++h) {
      const unsigned j = wid + h * 8, oo = pk_off(snd, pj - snd * PK_JOBS + j) + lane * 16u, so = j * 3072u + (h >> 1) * 16384u;
      const LAS unsigned* pk = (const LAS unsigned*)(lds + so);
      const unsigned hdr = pk[lane & 31];
      unsigned pos[16], cnt = 32u; unsigned long long m[16];
#pragma unroll
      for (int i = 0; i < 16; ++i) {
        const unsigned lo = __builtin_amdgcn_readlane(hdr, 2 * i), hi = __builtin_amdgcn_readlane(hdr, 2 * i + 1);
        m[i] = (unsigned long long)hi << 32 | lo;
        pos[i] = __builtin_amdgcn_mbcnt_hi(hi, __builtin_amdgcn_mbcnt_lo(lo, cnt));
        cnt += (unsigned)__builtin_popcountll(m[i]);
      }
      if (cnt > XCNT) {
        const unsigned s2 = 49152u + wid * 2048u, la = lbase + s2, vo = vb + 98304u + j * 2048u, d2 = (s2 - so - 3072u) >> 2;
        asm volatile("s_waitcnt lgkmcnt(0)\n\ts_mov_b32 m0, %0\n\ts_nop 0\n\tglobal_load_lds_dwordx4 %1, %2 sc0 sc1\n\tglobal_load_lds_dwordx4 %1, %2 offset:1024 sc0 sc1\n\ts_waitcnt vmcnt(0)"
                     :: "s"(la), "v"(vo), "s"(myws) : "memory");
#pragma unroll
        for (int i = 0; i < 16; ++i) pos[i] = (pos[i] + (pos[i] >= 768u ? d2 : 0u)) & 0x3fffffffu;
      }
#pragma unroll
      for (int rr = 0; rr < 4; ++rr) {
        v4u v;
#pragma unroll
        for (int c = 0; c < 4; ++c) {
          const unsigned x = *(const LAS unsigned*)(lds + (so + pos[4 * rr + c] * 4u));
          v[c] = !ok ? 0x7fc00000u : (m[4 * rr + c] >> lane & 1ull) ? x : 0u;
        }
        const unsigned vo_ = oo + rr * (N_ * 4u);
        asm volatile("global_store_dwordx4 %0, %1, %2" XSM "\n\ts_nop 1" :: "v"(vo_), "v"(v), "s"(p.out[p.rank]) : "memory");
      }
    }
    if (mode) break;
  }
}

__global__ __launch_bounds__(512, 2) void fused_kernel(const Params p) {
  extern __shared__ __attribute__((aligned(16))) unsigned char smem[];
  LAS unsigned char* lds = (LAS unsigned char*)smem;
  const int tid = threadIdx.x, wid = __builtin_amdgcn_readfirstlane(tid >> 6), wc = wid & 3;
  unsigned char* myws = p.ws[p.rank];
  const int b = blockIdx.x, rank = p.rank, g = b - NCOM, gl = b - NPUSH;
  const bool push = b < NPUSH;
  LAS unsigned* lctl = (LAS unsigned*)(lds + LCTL);
  const unsigned lbase = (unsigned)(size_t)lds;
  const int xc = b & 7, L = (xc >> 2) * 28 + (g >> 3), cn = L >> 2;
  const int pm = (xc & 3) * 4 + (L & 3), si = L * 4 + (xc & 3);
  const bool skip = cn < 2 && si >= NSH;
  const unsigned nvis = skip ? 96u : 128u;
  auto tile_of = [&](int u, int& tm, int& tn) __attribute__((always_inline)) {
    const int v = u + (skip && u > 0), idx = NSH * rank + si;
    const bool sh = v == 1 && cn < 2;
    tm = sh ? idx & 15 : pm; tn = sh ? SHC + (idx >> 4) : v == 0 ? cn : 14 * v - 2 + cn;
  };
  unsigned call, imgo, flo;
  {
    const int lane = tid & 63;
    unsigned R[16], R2[16]; v4u X[4], X2[4];
    int t1, k1, j1 = wid;
    if (push) { j1 = (b % NTURN) * WPT + wid % WPT; stream_slab(wid / WPT * SPT + b / NTURN, rank, t1, k1); } else { t1 = gl >> 2; k1 = gl & 3; }
    load_job(p, t1, k1, j1, lane, R, X);
    const bool two = !push && gl < 80;
    if (two) load_job(p, t1 + 60, k1, wid, lane, R2, X2);
    if (tid == 0) {
      unsigned* const cp = (unsigned*)(myws + CTB_OFF + b * CTS);
      const unsigned t = __hip_atomic_load(cp, __ATOMIC_RELAXED, __HIP_MEMORY_SCOPE_AGENT) * (unsigned)GRID;
      __hip_atomic_store(cp, t / (unsigned)GRID + 1u, __ATOMIC_RELAXED, __HIP_MEMORY_SCOPE_AGENT);
      lctl[0] = t / (unsigned)GRID + 1u; lctl[1] = INITK; lctl[2] = 0u; lctl[3] = 0u; lctl[5] = 0u;
      if (b == 0) {
        __hip_atomic_store((unsigned*)(myws + WCT_OFF + (t / (unsigned)GRID & 1u) * 128u), 0u, __ATOMIC_RELAXED, __HIP_MEMORY_SCOPE_AGENT);
        __hip_atomic_store((unsigned*)(myws + WCT_OFF + 64u + (t / (unsigned)GRID & 1u) * 128u), 0u, __ATOMIC_RELAXED, __HIP_MEMORY_SCOPE_AGENT);
        __hip_atomic_store((unsigned*)(myws + WCT_OFF + 256u + (t / (unsigned)GRID & 1u) * 128u), 0u, __ATOMIC_RELAXED, __HIP_MEMORY_SCOPE_AGENT);
      }
    }
    if (!push && g < 0 && tid < (int)NUNIT) *(LAS unsigned long long*)(lds + UTAB + tid * 8) = 0ull;
    if (wid >= 4 && g >= 0) {
      const int u = wid - 4, l = lane & 31, src = l >> 2, ks = l & 3;
      int tm, tn; tile_of(u, tm, tn);
      const int ax = stream_idx(tm, ks, src), aw = stream_idx(16 + tn, ks, src) + UDLY;
#define CORD 9
      const int key = src == rank ? ks : u >= CORD ? 32 + ks * 8 + ((src - rank) & 7) : ((ax > aw ? ax : aw) + 1) * 32 + l;
      int rk = 0;
#pragma unroll
      for (int s = 0; s < 32; ++s) rk += __builtin_amdgcn_readlane(key, s) < key;
      const unsigned ent = (unsigned)((16 + tn) * 32 + l) << 15 | (unsigned)(tm * 32 + l);
      if (u * 32u < nvis) *(LAS unsigned*)(lds + LTAB + (u * 32 + rk) * 4) = ent;
      if (u == 0 && rk < 4) *(LAS unsigned*)(lds + LTAB + (nvis + rk) * 4) = ent;
    }
    asm volatile("s_waitcnt vmcnt(0)" : TIE16(R), TIE4(X) :: "memory");
    asm volatile("" : TIE16(R2) :: "memory");
    __syncthreads();
    call = __builtin_amdgcn_readfirstlane(lctl[0]);
    imgo = IMG_OFF + (call & 1u) * IMG_BYTES;
    flo = FLAG_OFF + (call & 1u) * FLAG_BYTES;
    TS(0);
    if (push) {
      if (b < NPA) pusher(p, lds, imgo, flo, call, b, wid, lane, R, X);
      WAIT_V(0);
      expand(p, lds, call, wid, 0u, 0u);
      TS(14);
      return;
    }
    v4u D[4];
    const unsigned s1 = t1 * 32 + rank * 4 + k1;
    retile(t1 < 16, R, X, D);
    store_job(myws, imgo + s1 * SLAB + job_off(wid, lane) + 4096, D);
    if (two) {
      retile(false, R2, X2, D);
      store_job(myws, imgo + (s1 + 60 * 32) * SLAB + job_off(wid, lane) + 4096, D);
    }
    WAIT_V(0);
    store_flag1(myws, flo + s1 * 64 + wid * 4, call);
    if (two) store_flag1(myws, flo + (s1 + 60 * 32) * 64 + wid * 4, call);
    TS(1);
    if (g < 0) {
      unpacker(p, lds, imgo, flo, call, b, wid, lane);
      __syncthreads();
      TS(4);
      expand(p, lds, call, wid, 0u, 0u);
      TS(14);
      return;
    }
  }
  const float scale = __builtin_bit_cast(float, __builtin_amdgcn_readfirstlane(__builtin_bit_cast(int, p.sx[0] * p.sw[0])));
  WAIT_V(0);

  auto run = [&](auto WRC) __attribute__((always_inline)) {
    constexpr int wr = decltype(WRC)::value;
    unsigned ppend = 0, pk = INITK, pw = 4u;
    auto peval = [&]() __attribute__((always_inline)) {
      const v4u f = *(const LAS v4u*)(lds + LCP + (unsigned)fresh_lane() * 16);
      unsigned long long bm = __ballot(f[0] == call && f[1] == call && f[2] == call && f[3] == call);
      bm &= bm >> 1; bm &= bm >> 2;
      bm = ~bm & 0x1111111111111111ull;
      unsigned c = bm ? (unsigned)__builtin_ctzll(bm) >> 2 : 16u;
      c = c < pw ? c : pw;
      pw = c == pw ? 16u : PWIN;
      const unsigned n = pk + c < nvis ? pk + c : nvis;
      if (n > pk) {
        pk = n; lctl[1] = n;
      }
      ppend = 0;
    };
    auto pissue = [&]() __attribute__((always_inline)) {
      const unsigned ln = (unsigned)fresh_lane();
      unsigned e = pk + ((ln >> 2) < pw ? ln >> 2 : 0u); e = e < nvis ? e : nvis - 1;
      const unsigned kk = *(const LAS unsigned*)(lds + LTAB + e * 4);
      const unsigned la = lbase + LCP, fo = flo + ((ln & 2) ? kk >> 15 : kk & 0x7fffu) * 64 + (ln & 1) * 16; unsigned char* const w_ = myws;
      asm volatile("s_mov_b32 m0, %0\n\ts_nop 0\n\tglobal_load_lds_dwordx4 %1, %2 sc0 sc1" :: "s"(la), "v"(fo), "s"(w_) : "memory");
      ppend = 1;
    };
    auto poll = [&](bool blocking, unsigned need) __attribute__((always_inline)) {
      if (!blocking) {
        if (ppend) {
          unsigned st; asm volatile("s_getreg_b32 %0, hwreg(HW_REG_IB_STS)" : "=s"(st) :: "memory");
          st = __builtin_amdgcn_readfirstlane(st);
          if ((st & 15u) | ((st >> 18) & 48u)) return;
          WAIT_V(0);
          peval();
        } else { pw = 16u; pissue(); }
        return;
      }
      WAIT_V(0);
      if (ppend) { peval(); if (pk < need) __builtin_amdgcn_s_sleep(PSLEEP); }
      if (pk < nvis) pissue();
      WAIT_L(0);
    };
    auto slow = [&](unsigned need) __attribute__((always_inline)) {
      const unsigned long long t0 = __builtin_amdgcn_s_memrealtime();
      for (;;) {
        if (wr == 1 && wc == 0) poll(true, need); else __builtin_amdgcn_s_sleep(1);
        const unsigned long long el = __builtin_amdgcn_s_memrealtime() - t0;
        if (wr == 1 && wc == 0 && el > 100000ull) {
          unsigned* tmo = (unsigned*)(myws + TMO_OFF);
          if (__builtin_amdgcn_readfirstlane(__hip_atomic_load(tmo, __ATOMIC_RELAXED, __HIP_MEMORY_SCOPE_AGENT)) == call || el > DEADLINE_TICKS) {
            __hip_atomic_store(tmo, call, __ATOMIC_RELAXED, __HIP_MEMORY_SCOPE_AGENT);
            lctl[2] = 1u; lctl[1] = DEADV; pk = nvis; ppend = 0;
          }
          WAIT_V(0);
        }
        if (__builtin_amdgcn_readfirstlane(__hip_atomic_load(lctl + 1, __ATOMIC_RELAXED, __HIP_MEMORY_SCOPE_WORKGROUP)) >= need || el > 2 * DEADLINE_TICKS) break;
      }
      asm volatile("" ::: "memory");
    };

#define GT0 26
    auto grab = [&](int t) __attribute__((always_inline)) {
      const unsigned ln = (unsigned)fresh_lane() & 31u, q = __builtin_amdgcn_readfirstlane(lctl[4]), pj = blk_job(rank, q);
      if (q >= NBLK) return;
      if (t == GT0) {
        const unsigned la = lbase + LCTL + 64u, fo = OFLAG_OFF + (pj + ln) * 4u; unsigned char* const w_ = myws;
        asm volatile("s_mov_b32 m0, %0\n\ts_mov_b32 exec_hi, 0\n\tglobal_load_lds_dword %1, %2 sc0 sc1\n\ts_mov_b32 exec_hi, -1" :: "s"(la), "v"(fo), "s"(w_) : "memory");
      } else {
        WAIT_V(0);
        lctl[5] = __ballot(lctl[16 + ln] != call) == 0ull ? PK_OFF + (pj >> 5) * PK_BLK - imgo : 0u;
      }
    };
#define TBURST 3u
#define TEND 12604u
    unsigned ti = (unsigned)(g * 3 + wc - 1);
    auto touch = [&]() __attribute__((always_inline)) {
      const unsigned k = (ti >= 960u) + (ti >= 1920u) + (ti >= 2880u), r = ti - k * 960u, i2 = ti - 9244u, bq = i2 / 12u;
      const unsigned o = ti < 3840u ? imgo + ((r >> 2) * 4u + k) * SLAB + (r & 3u) * 8192u
                       : ti < 3848u ? (ti - 3840u) * 8192u
                       : ti < 3868u ? flo + (ti - 3848u) * 8192u
                       : ti < 4124u ? imgo + (70u + ((NSH * rank) >> 4)) * 32u * SLAB + (ti - 3868u) * 8192u
                       : ti < 9244u ? imgo + 30u * 32u * SLAB + (ti - 4124u) * 8192u
                                    : PK_OFF + (blk_job(rank, bq) >> 5) * PK_BLK + (i2 - bq * 12u) * 8192u;
      const unsigned la = lbase + LTCH, fo = o + (unsigned)fresh_lane() * 128u; unsigned char* const w_ = myws;
      asm volatile("s_mov_b32 m0, %0\n\ts_nop 0\n\tglobal_load_lds_dword %1, %2 sc0 sc1" :: "s"(la), "v"(fo), "s"(w_) : "memory");
      ti += 672u;
    };
    const unsigned char* XI = myws + imgo;
    const unsigned voff = (unsigned)(wc * 4096 + (tid & 63) * 16);
    const unsigned ldsw = (unsigned)wc * 4096u;
    const int aoff = wr * 8192 + (tid & 63) * 16, boff = wc * 4096 + (tid & 63) * 16;
#define SA(b_, h) (((b_) * 2 + (h)) * HTB)
#define SB(b_, h) ((4 + (b_) * 2 + (h)) * HTB)
#define STG1(bufoff, va, o) __builtin_amdgcn_global_load_lds((const unsigned*)(XI + (va)), (LAS unsigned*)(lds + (bufoff) + ldsw), 16, o, 0)
#define STAGE(bufoff, goff) do { if (wr == 0) { const unsigned va_ = voff + (goff); STG1(bufoff, va_, 0); STG1(bufoff, va_, 1024); STG1(bufoff, va_, 2048); STG1(bufoff, va_, 3072); } } while (0)
#define LDA(dst, b_, h) do { _Pragma("unroll") for (int m = 0; m < 4; ++m) { \
    v4i lo = *(const LAS v4i*)(lds + SA(b_, h) + aoff + m * 2048); v4i hi = *(const LAS v4i*)(lds + SA(b_, h) + aoff + m * 2048 + 1024); dst[m] = cat8(lo, hi); } } while (0)
#define LDB(dst, b_, h) do { _Pragma("unroll") for (int n = 0; n < 2; ++n) { \
    v4i lo = *(const LAS v4i*)(lds + SB(b_, h) + boff + n * 2048); v4i hi = *(const LAS v4i*)(lds + SB(b_, h) + boff + n * 2048 + 1024); dst[n] = cat8(lo, hi); } } while (0)
#define MMA(ai, bj, At, Bt) do { __builtin_amdgcn_s_setprio(1); _Pragma("unroll") for (int m = 0; m < 4; ++m) _Pragma("unroll") for (int n = 0; n < 2; ++n) \
    acc[ai][bj][m][n] = __builtin_amdgcn_mfma_scale_f32_16x16x128_f8f6f4(Bt[n], At[m], acc[ai][bj][m][n], 0, 1, 0, 0, 0, 0); __builtin_amdgcn_s_setprio(0); } while (0)
#define MF1(ai, bj, At, Bt, q) acc[ai][bj][(q) >> 1][(q) & 1] = __builtin_amdgcn_mfma_scale_f32_16x16x128_f8f6f4(Bt[(q) & 1], At[(q) >> 1], acc[ai][bj][(q) >> 1][(q) & 1], 0, 1, 0, 0, 0, 0)
#define WV12 WAIT_V(12)
#define MEMSTEP do { if (wr == 0) WV12; else if (wc == 0 && (ppend || (pk < nvis && pk < need + PTH))) poll(false, 0u); } while (0)
#define KOFF(i) ((unsigned)__builtin_amdgcn_readlane(kv, (i)))
    f32x4 acc[2][2][4][2];
#pragma unroll
    for (int a = 0; a < 2; ++a)
#pragma unroll
      for (int c = 0; c < 2; ++c)
#pragma unroll
        for (int m = 0; m < 4; ++m)
#pragma unroll
          for (int n = 0; n < 2; ++n) acc[a][c][m][n] = (f32x4){0.f, 0.f, 0.f, 0.f};
    v8i At[4], B0[2], B1[2];
    constexpr unsigned hstep = HTB;
#define OFA(e_) ((e_) & ~0x7fffu)
#define OFB(e_) (((e_) & 0x7fffu) << 15)
    if (wr == 1 && wc > 0) for (unsigned i = 0; i < TBURST; ++i) touch();
    slow(4u);
    unsigned known = 4u, need = 4u;
    const unsigned e0 = __builtin_amdgcn_readfirstlane(*(const LAS unsigned*)(lds + LTAB));
    {
      const unsigned cA = OFA(e0), cB = OFB(e0);
      STAGE(SB(0, 0), cB); STAGE(SA(0, 0), cA); STAGE(SB(0, 1), cB + hstep); STAGE(SA(0, 1), cA + hstep);
      if (wr == 1) BAR;
      if (wr == 0) WAIT_V(8);
      BAR;
      STAGE(SB(1, 0), cB + SLAB); STAGE(SA(1, 0), cA + SLAB); STAGE(SB(1, 1), cB + SLAB + hstep);
      if (wr == 0) WAIT_V(12);
      BAR;
    }
    unsigned a1 = OFA(e0) + SLAB, a2 = a1 + SLAB, a3 = a2 + SLAB, b2 = OFB(e0) + 2 * SLAB, b3 = b2 + SLAB;
    for (int ui = 0;; ++ui) {
      const bool has_next = (unsigned)(ui * 32 + 32) < nvis;
      int kv = 0;
      if (wr == 0) { const unsigned ln = ui * 32u + (unsigned)fresh_lane(); kv = *(const LAS int*)(lds + LTAB + (ln < 131u ? ln : 131u) * 4); }
      for (int t = 0; t < NKS; t += 2) {
        if (known < need) { if (wr == 0) BAR; slow(need); if (wr == 1) BAR; known = need; }
        LDB(B0, 0, 0); SCHED; LDA(At, 0, 0); SCHED; STAGE(SA(1, 1), a1 + hstep);
        WAIT_L(8); BAR; WAIT_L(0); MMA(0, 0, At, B0); BAR; SCHED;
        LDB(B1, 0, 1); SCHED; STAGE(SB(0, 0), b2);
        BAR; WAIT_L(0); MMA(0, 1, At, B1); BAR;
        LDA(At, 0, 1); SCHED; STAGE(SA(0, 0), a2);
        BAR; WAIT_L(0); MMA(1, 0, At, B0); BAR; SCHED;
        STAGE(SB(0, 1), b2 + hstep); MEMSTEP;
        if (wr == 1 && wc == 1 && ui == 3 && (t == GT0 || t == NKS - 4)) grab(t);
        if (wr == 1 && wc > 0 && ti < TEND) touch();
        BAR; MMA(1, 1, At, B1); BAR;
        LDB(B0, 1, 0); SCHED; LDA(At, 1, 0); SCHED; STAGE(SA(0, 1), a2 + hstep);
        WAIT_L(8); BAR; WAIT_L(0); MMA(0, 0, At, B0); BAR; SCHED;
        LDB(B1, 1, 1); const unsigned kn = lctl[1]; SCHED; STAGE(SB(1, 0), b3);
        BAR; WAIT_L(0); MMA(0, 1, At, B1); BAR;
        LDA(At, 1, 1); SCHED; STAGE(SA(1, 0), a3);
        BAR; WAIT_L(0); MMA(1, 0, At, B0); BAR; SCHED;
        STAGE(SB(1, 1), b3 + hstep); if (wr == 1 || has_next || t < NKS - 2) MEMSTEP;
        BAR; SCHED;
        __builtin_amdgcn_s_setprio(1);
        MF1(1, 1, At, B1, 0); SCHED;
        unsigned o2 = KOFF(t + 4), o3 = KOFF(t + 5); known = __builtin_amdgcn_readfirstlane(kn);
        asm volatile("" : "+s"(o2), "+s"(o3), "+s"(known)); SCHED;
        MF1(1, 1, At, B1, 1); SCHED;
        a1 = a3; a2 = OFA(o2); b2 = OFB(o2);
        asm volatile("" : "+s"(a1), "+s"(a2), "+s"(b2)); SCHED;
        MF1(1, 1, At, B1, 2); SCHED;
        a3 = OFA(o3); b3 = OFB(o3);
        if (wr == 0 && !has_next && t == NKS - 4) {
          const unsigned ob_ = __builtin_amdgcn_readfirstlane(lctl[5]);
          a2 = ob_ ? ob_ : a1; a3 = ob_ ? ob_ + 2 * hstep : a1; b2 = ob_ ? ob_ + 3 * hstep : a1; b3 = ob_ ? ob_ + 5 * hstep : a1;
        }
        asm volatile("" : "+s"(a2), "+s"(b2), "+s"(a3), "+s"(b3)); SCHED;
        MF1(1, 1, At, B1, 3); SCHED;
        need = need + 2u < nvis ? need + 2u : nvis;
        asm volatile("" : "+s"(need)); SCHED;
        MF1(1, 1, At, B1, 4); MF1(1, 1, At, B1, 5); MF1(1, 1, At, B1, 6); MF1(1, 1, At, B1, 7);
        __builtin_amdgcn_s_setprio(0);
        BAR;
      }
      int tm, tn; tile_of(ui, tm, tn);
      const bool dead = __builtin_amdgcn_readfirstlane(lctl[2]) != 0u, sht = tn >= SHC;
      TS(2 + 2 * ui);
      unsigned gv = 1u;
      const int ng = __builtin_amdgcn_readfirstlane((int)!(wc == 1 && ui == 2 && has_next));
      {
        const int ln = fresh_lane();
        const unsigned vo = (unsigned)((tm * 256 + wc * 32 + 4 * (ln >> 4)) * N_ + tn * 256 + wr * 64 + 4 * (ln & 15)) * 4u;
        const float qnan = __builtin_nanf("");
        const float sc_ = dead ? qnan : scale, lo_ = dead ? qnan : 0.f;
        const int shs = __builtin_amdgcn_readfirstlane(sht ? 1 : ui < PLN ? 2 : 0);
        unsigned char* const ob = p.out[rank];
        if (wr == 1) {
          const unsigned co = WCT_OFF + 256u + (call & 1u) * 128u; unsigned char* const w_ = myws;
          asm volatile("s_cmp_lg_u32 %3, 0\n\ts_cbranch_scc1 GE_%=\n\ts_mov_b64 exec, 1\n\tglobal_atomic_add %0, %1, %0, %2 sc0\n\ts_mov_b64 exec, -1\nGE_%=:"
                       : "+v"(gv) : "v"(co), "s"(w_), "s"(ng) : "memory", "scc");
        }
#pragma unroll
        for (int bj = 0; bj < 2; ++bj)
#pragma unroll
          for (int n = 0; n < 2; ++n)
#pragma unroll
            for (int r4 = 0; r4 < 4; ++r4) {
              unsigned char* rowp = ob + (size_t)(bj * 128 + n * 16 + r4) * N_ * 4;
#pragma unroll
              for (int ai = 0; ai < 2; ++ai) {
                f32x4 v;
#pragma unroll
                for (int c = 0; c < 4; ++c) {
                  const float x = acc[ai][bj][c][n][r4] * sc_;
                  asm("v_max_f32 %0, %1, %2" : "=v"(v[c]) : "s"(lo_), "v"(x));
                }
                asm volatile("s_cmp_lg_u32 %4, 0\n\ts_cbranch_scc1 SW_%=\n\tglobal_store_dwordx4 %0, %1, %2 offset:%3" OSM "\n\ts_branch SE_%=\n"
                             "SW_%=:\n\tglobal_store_dwordx4 %0, %1, %2 offset:%3 sc0 sc1" SHM "\nSE_%=:\n\ts_nop 1"
                             :: "v"(vo), "v"(v), "s"(rowp), "n"(ai * 512), "s"(shs) : "memory", "scc");
              }
            }
      }
      if (!has_next && wr == 0) WAIT_V(32);
      if (wr == 1) {
        asm volatile("s_waitcnt vmcnt(32)" : "+v"(gv) :: "memory");
        if (!ng) { const unsigned q_ = __builtin_amdgcn_readfirstlane(gv); lctl[4] = q_ < NG ? q_ + N1 : NBLK; }
      }
      if (sht) {
        WAIT_V(0);
        unsigned o_ = 0;
        if (fresh_lane() == 0) o_ = __hip_atomic_fetch_add(lctl + 3, 1u, __ATOMIC_RELAXED, __HIP_MEMORY_SCOPE_WORKGROUP);
        if (__builtin_amdgcn_readfirstlane(o_) == 7u) store_flag1(myws, DONE_OFF + si * 64, call);
        if (wr == 0) WAIT_V(0);
      }
      TS(3 + 2 * ui);
      if (!has_next) break;
#pragma unroll
      for (int a = 0; a < 2; ++a)
#pragma unroll
        for (int c = 0; c < 2; ++c)
#pragma unroll
          for (int m = 0; m < 4; ++m)
#pragma unroll
            for (int n = 0; n < 2; ++n) acc[a][c][m][n] = (f32x4){0.f, 0.f, 0.f, 0.f};
    }
    if (wr == 0) BAR;
  };
  if (wid < 4) run(std::integral_constant<int, 0>{}); else run(std::integral_constant<int, 1>{});
  expand(p, lds, call, wid, __builtin_amdgcn_readfirstlane(lctl[4]), skip ? 0u : __builtin_amdgcn_readfirstlane(lctl[5]) ? 2u : 1u);
  TS(14);
}
extern "C" void kernel_launch(int rank, int nranks, ncclComm_t comm,
                              hipStream_t stream,
                              void* const* d_in, const int* in_sizes, int n_in,
                              void* d_out, int out_size,
                              const int* in_dtypes, int out_dtype,
                              void* const* d_out_all,
                              void* const* d_ws_all, size_t ws_bytes) {
  if (nranks != NR || (unsigned)rank >= (unsigned)NR || ws_bytes < WS_NEED || n_in < 4) {
    fprintf(stderr, "kernel_launch: unexpected configuration\n"); return;
  }
  static bool attr_done[64] = {};
  int dev = 0; HIP_CHECK(hipGetDevice(&dev));
  if (!attr_done[dev & 63]) {
    HIP_CHECK(hipFuncSetAttribute((const void*)fused_kernel, hipFuncAttributeMaxDynamicSharedMemorySize, LDS_BYTES));
    attr_done[dev & 63] = true;
  }
  Params p;
  p.x = (const unsigned char*)d_in[0]; p.w = (const unsigned char*)d_in[1];
  p.sx = (const float*)d_in[2]; p.sw = (const float*)d_in[3];
  for (int i = 0; i < NR; ++i) { p.ws[i] = (unsigned char*)d_ws_all[i]; p.out[i] = (unsigned char*)(i == rank ? d_out : d_out_all[i]); }
  p.rank = rank;
  fused_kernel<<<GRID, 512, LDS_BYTES, stream>>>(p);
}
```

```cpp
#include <hip/hip_runtime.h>
#include <rccl/rccl.h>
#include <cstdio>
#include <type_traits>

#define HIP_CHECK(expr) do { hipError_t _e = (expr); if (_e != hipSuccess) { \
    fprintf(stderr, "HIP error %s at %s:%d\n", hipGetErrorString(_e), __FILE__, __LINE__); \
    return; } } while (0)

typedef int v8i __attribute__((ext_vector_type(8)));
typedef int v4i __attribute__((ext_vector_type(4)));
typedef unsigned v4u __attribute__((ext_vector_type(4)));
typedef unsigned v2u __attribute__((ext_vector_type(2)));
typedef float f32x4 __attribute__((ext_vector_type(4)));
#define LAS __attribute__((address_space(3)))

constexpr int NR = 8;
constexpr int M_ = 4096, N_ = 16384, KL = 512;
constexpr int NKS = 32;
constexpr int HTB = 16384;
constexpr int STAGE_BYTES = 8 * HTB;
constexpr unsigned LCTL = STAGE_BYTES, LTAB = STAGE_BYTES + 256, LCP = STAGE_BYTES + 1024;
constexpr unsigned LTCH = STAGE_BYTES + 2048;
constexpr int LDS_BYTES = STAGE_BYTES + 2048 + 256;
constexpr unsigned SLAB = 32768;
constexpr unsigned IMG_BYTES = 2560u * SLAB;
constexpr unsigned IMG_OFF = 1u << 20;
constexpr unsigned PK_OFF = IMG_OFF + 2 * IMG_BYTES, PK_SLOT = 5120, PK_JOBS = 1280;
constexpr unsigned PK_BLK = 32 * PK_SLOT, NBLK = 7 * PK_JOBS / 32;
constexpr unsigned CW_OFF = PK_OFF + NR * (PK_JOBS / 32) * PK_BLK - 512u * SLAB, CW_BYTES = 2048u * SLAB;
constexpr size_t WS_NEED = (size_t)CW_OFF + 512u * SLAB + (size_t)CW_BYTES;
constexpr unsigned CTR_OFF = 0, TMO_OFF = 128, WCT_OFF = 256, DONE_OFF = 4096, OFLAG_OFF = 8192, FLAG_OFF = 65536, FLAG_BYTES = 2560u * 64u, DBG_OFF = 524288;
constexpr int GRID = 256;
#define CTS 4
constexpr unsigned CTB_OFF = 49152;
constexpr int NPUSH = 16, NCOM = 32;
constexpr int NSH = 20, SHC = 54;
constexpr unsigned DEADV = 0x7fffffffu;
constexpr unsigned long long DEADLINE_TICKS = 90000000ull;
#define WPT 2

struct Params {
  const unsigned char* x;
  const unsigned char* w;
  const float* sx; const float* sw;
  unsigned char* ws[NR];
  unsigned char* out[NR];
  int rank;
};

#define TS(slot) do {} while (0)
#define TV(slot, v) do {} while (0)
__device__ __forceinline__ v8i cat8(v4i a, v4i b) { return __builtin_shufflevector(a, b, 0, 1, 2, 3, 4, 5, 6, 7); }
#define WAIT_V(n) asm volatile("s_waitcnt vmcnt(" #n ")" ::: "memory")
#define WAIT_L(n) asm volatile("s_waitcnt lgkmcnt(" #n ")" ::: "memory")
#define BAR __builtin_amdgcn_s_barrier()
#define SCHED __builtin_amdgcn_sched_barrier(0)
#define TIE16(R) "+v"(R[0]), "+v"(R[1]), "+v"(R[2]), "+v"(R[3]), "+v"(R[4]), "+v"(R[5]), "+v"(R[6]), "+v"(R[7]), \
                 "+v"(R[8]), "+v"(R[9]), "+v"(R[10]), "+v"(R[11]), "+v"(R[12]), "+v"(R[13]), "+v"(R[14]), "+v"(R[15])
#define TIE4(X) "+v"(X[0]), "+v"(X[1]), "+v"(X[2]), "+v"(X[3])

#define PKM ""
#define IMM ""
#define LDM ""
#define SHM ""
__device__ __forceinline__ int fresh_lane() { int l; asm volatile("v_mbcnt_lo_u32_b32 %0, -1, 0\n\tv_mbcnt_hi_u32_b32 %0, -1, %0" : "=v"(l)); return l; }

__device__ __forceinline__ void load_job(const Params& p, int tile, int ks, int jw, int lane, unsigned (&R)[16], v4u (&X)[4]) {
  const int i = lane & 15, g = lane >> 4;
  const bool isx = tile < 16, iso = tile >= 80;
  unsigned vo = isx ? i * KL + g * 32 : iso ? lane * 16 : g * 32 * N_ + 4 * i;
  const unsigned char* src = isx ? p.x + (size_t)(tile * 256 + (jw >> 1) * 64) * KL + ks * 128 + (jw & 1) * 16
                           : iso ? p.out[p.rank] + (unsigned)ks
                                 : p.w + (size_t)(ks * 128 + (jw & 1) * 16) * N_ + (tile - 16) * 256 + (jw >> 1) * 64;
  const unsigned st = isx ? 16 * KL : iso ? 4 * N_ : N_;
#define LD1(r) "global_load_dword %" #r ", %20, %21" LDM "\n\tv_add_u32 %20, %22, %20\n\t"
#define LD4(r) "global_load_dwordx4 %" #r ", %20, %21" LDM "\n\tv_add_u32 %20, %22, %20\n\t"
  asm volatile("s_cmp_eq_u32 %22, %23\n\ts_cbranch_scc1 LW_%=\n\t" LD4(16) LD4(17) LD4(18) LD4(19) "s_branch LE_%=\n"
               "LW_%=:\n\t" LD1(0) LD1(1) LD1(2) LD1(3) LD1(4) LD1(5) LD1(6) LD1(7) LD1(8) LD1(9) LD1(10) LD1(11) LD1(12) LD1(13) LD1(14) LD1(15) "\n"
               "LE_%=:"
               : "=&v"(R[0]), "=&v"(R[1]), "=&v"(R[2]), "=&v"(R[3]), "=&v"(R[4]), "=&v"(R[5]), "=&v"(R[6]), "=&v"(R[7]),
                 "=&v"(R[8]), "=&v"(R[9]), "=&v"(R[10]), "=&v"(R[11]), "=&v"(R[12]), "=&v"(R[13]), "=&v"(R[14]), "=&v"(R[15]),
                 "=&v"(X[0]), "=&v"(X[1]), "=&v"(X[2]), "=&v"(X[3]), "+v"(vo)
               : "s"(src), "s"(st), "n"(N_) : "memory", "scc");
}
__device__ __forceinline__ void retile(bool plain, const unsigned (&R)[16], const v4u (&X)[4], v4u (&D)[4]) {
  if (plain) {
#pragma unroll
    for (int e = 0; e < 4; ++e) D[e] = X[e];
  } else {
#pragma unroll
    for (int d = 0; d < 4; ++d) {
      const unsigned a = R[4 * d], bb = R[4 * d + 1], c = R[4 * d + 2], f = R[4 * d + 3];
      const unsigned t0_ = __builtin_amdgcn_perm(bb, a, 0x05010400u), t1_ = __builtin_amdgcn_perm(bb, a, 0x07030602u);
      const unsigned t2_ = __builtin_amdgcn_perm(f, c, 0x05010400u), t3_ = __builtin_amdgcn_perm(f, c, 0x07030602u);
      D[0][d] = __builtin_amdgcn_perm(t2_, t0_, 0x05040100u); D[1][d] = __builtin_amdgcn_perm(t2_, t0_, 0x07060302u);
      D[2][d] = __builtin_amdgcn_perm(t3_, t1_, 0x05040100u); D[3][d] = __builtin_amdgcn_perm(t3_, t1_, 0x07060302u);
    }
  }
}
__device__ __forceinline__ void store_job(unsigned char* ws, unsigned vo, const v4u (&D)[4]) {
  asm volatile("global_store_dwordx4 %0, %1, %5 offset:-4096 sc0 sc1\n\tglobal_store_dwordx4 %0, %2, %5 offset:-2048 sc0 sc1\n\t"
               "global_store_dwordx4 %0, %3, %5 sc0 sc1\n\tglobal_store_dwordx4 %0, %4, %5 offset:2048 sc0 sc1\n\ts_nop 1"
               :: "v"(vo), "v"(D[0]), "v"(D[1]), "v"(D[2]), "v"(D[3]), "s"(ws) : "memory");
}
__device__ __forceinline__ void store_flag1(unsigned char* ws, unsigned fo, unsigned v) {
  asm volatile("s_mov_b64 exec, 1\n\tglobal_store_dword %0, %1, %2 sc0 sc1\n\ts_mov_b64 exec, -1" :: "v"(fo), "v"(v), "s"(ws) : "memory");
}
__device__ __forceinline__ void store_flag7(unsigned long long fa, unsigned v) {
  asm volatile("s_mov_b64 exec, 0x7f\n\tglobal_store_dword %0, %1, off sc0 sc1\n\ts_mov_b64 exec, -1" :: "v"(fa), "v"(v) : "memory");
}
__device__ __forceinline__ unsigned job_off(int jw, int lane) { return (jw >> 1) * 8192 + (jw & 1) * 1024 + lane * 16; }

#define NMIX 4
__device__ __forceinline__ void stream_slab(int j, int snd, int& tile, int& ks) {
  if (j < 120) { ks = j / 30; int q = j - ks * 30; if (q < 14) { q += 2 * snd; if (q >= 14) q -= 14; tile = 16 + q; } else tile = (q - 14 + 2 * snd) & 15; }
  else if (j < 168) { const int u = j - 120; ks = u / 12; int q = u - ks * 12 + snd; if (q >= 12) q -= 12; tile = 30 + q; }
  else { const int u = j - 168, gq = u >= 56, v = u - gq * 56; ks = v / 14; int q = v - ks * 14 + 2 * snd; if (q >= 14) q -= 14; tile = 42 + 14 * gq + q; }
}
__device__ __forceinline__ int stream_idx(int tile, int ks, int snd) {
  if (tile < 16) return ks * 30 + 14 + ((tile - 2 * snd) & 15);
  if (tile < 30) { int q = tile - 16 - 2 * snd; if (q < 0) q += 14; return ks * 30 + q; }
  if (NMIX && tile >= 70) return 131 + ks * 12;
  if (tile < 42 || tile >= 70) { int q = (tile < 42 ? tile - 30 : tile - 70) - snd; if (q < 0) q += 12; return 120 + ks * 12 + q; }
  const int gq = tile >= 56; int q = tile - 42 - 14 * gq - 2 * snd; if (q < 0) q += 14; return 168 + gq * 56 + ks * 14 + q;
}
__device__ __forceinline__ unsigned shared_off(int r, int i) { const int idx = NSH * r + i; return (unsigned)((idx & 15) * 256 * N_ + (SHC + (idx >> 4)) * 256) * 4u; }

#define INITK 0u
#define LAG2 120
#define LAG 1
#define PWIN 4u
#define PTH 8u
#define INV 0
#define LASTWT 0
#define OST 1
#define OSM " nt"
#define PLN 0
#define XST OST
#define XSM " nt"
#define PSLEEP 8
__device__ __forceinline__ bool wait_flags(unsigned char* ws, unsigned fo, unsigned call) {
  const unsigned long long t0 = __builtin_amdgcn_s_memrealtime();
  unsigned* tmo = (unsigned*)(ws + TMO_OFF);
  for (;;) {
    unsigned v, tm; const unsigned to = TMO_OFF;
    asm volatile("global_load_dword %0, %2, %4 sc0 sc1\n\tglobal_load_dword %1, %3, %4 sc0 sc1\n\ts_waitcnt vmcnt(0)" : "=&v"(v), "=&v"(tm) : "v"(fo), "v"(to), "s"(ws) : "memory");
    if (__ballot(v != call) == 0ull) return true;
    if (__builtin_amdgcn_readfirstlane(tm) == call || __builtin_amdgcn_s_memrealtime() - t0 > DEADLINE_TICKS) {
      __hip_atomic_store(tmo, call, __ATOMIC_RELAXED, __HIP_MEMORY_SCOPE_AGENT);
      return false;
    }
    __builtin_amdgcn_s_sleep(16);
  }
}
constexpr int NTURN = 8 / WPT;
#define NPA 16
constexpr int SPT = NPA * WPT / 8, NT = 280 / SPT;
__device__ __forceinline__ unsigned pk_off(int r, unsigned jj) { return shared_off(r, jj >> 6) + ((jj & 63u) * 4u) * (N_ * 4u); }
#define ST1(vc, d, k, md) "global_store_dwordx4 " vc ", " d ", %" #k " sc0 sc1" md "\n\t"
#define ST7(vc, d, md) ST1(vc, d, 2, md) ST1(vc, d, 3, md) ST1(vc, d, 4, md) ST1(vc, d, 5, md) ST1(vc, d, 6, md) ST1(vc, d, 7, md) ST1(vc, d, 8, md)
__device__ __forceinline__ unsigned bfi(unsigned m, unsigned a, unsigned b) { return (a & m) | (b & ~m); }
#define XLIM 104
__device__ __forceinline__ unsigned encode(v4u (&S)[4], LAS unsigned char* el, int lane, unsigned long long& em, bool isx) {
  unsigned sm = 0;
  const unsigned me = isx ? 0x78787878u : 0x7c7c7c7cu, mt = isx ? 0xc0c0c0c0u : 0xe0e0e0e0u;
#pragma unroll
  for (int d = 0; d < 4; ++d) sm = __builtin_amdgcn_udot4(S[0][d] & me, 0x01010101u, sm, false);
  sm += __builtin_amdgcn_update_dpp(0, (int)sm, 0x111, 0xf, 0xf, true); sm += __builtin_amdgcn_update_dpp(0, (int)sm, 0x112, 0xf, 0xf, true);
  sm += __builtin_amdgcn_update_dpp(0, (int)sm, 0x114, 0xf, 0xf, true); sm += __builtin_amdgcn_update_dpp(0, (int)sm, 0x118, 0xf, 0xf, true);
  int e0 = (__builtin_amdgcn_readlane(sm, 15) + __builtin_amdgcn_readlane(sm, 31) + __builtin_amdgcn_readlane(sm, 47) + __builtin_amdgcn_readlane(sm, 63) - (isx ? 34242 : 17121)) >> (isx ? 13 : 12);
  e0 = e0 < 0 ? 0 : e0 > (isx ? 8 : 24) ? (isx ? 8 : 24) : e0;
  const unsigned c1 = (unsigned)(128 - (isx ? 8 : 4) * e0) * 0x01010101u, ix = (unsigned)((lane & 7) * 4 | (lane >> 3) << 13);
  unsigned cnt = 0, hdr = 0; v4u U[4];
#pragma unroll
  for (int c = 0; c < 4; ++c) {
#pragma unroll
    for (int d = 0; d < 4; ++d) {
      const unsigned v = S[c][d], u = (v & 0x7f7f7f7fu) + c1;
      const bool esc = (u & mt) != 0x80808080u;
      const unsigned long long m = __ballot(esc);
      const unsigned pos = __builtin_amdgcn_mbcnt_hi((unsigned)(m >> 32), __builtin_amdgcn_mbcnt_lo((unsigned)m, cnt));
      *(LAS unsigned short*)(el + 4 + 2 * (esc ? pos : 1100u + (unsigned)lane)) = (unsigned short)(__builtin_amdgcn_udot4(v & 0x60606060u, 0x40100401u, ix, false) + d);
      cnt += (unsigned)__builtin_popcountll(m);
      U[c][d] = bfi(0x80808080u, v, u);
    }
    hdr |= (cnt & 127u) << (8 * c);
  }
  em = ~0ull;
  if (cnt > 126u) return 32u;
  if (lane == 0) *(LAS unsigned*)el = hdr;
  if (isx) {
#pragma unroll
    for (int k = 0; k < 14; ++k) {
      const unsigned z = U[3][2 + k / 7];
      S[k >> 2][k & 3] = bfi(0x40404040u, k % 7 == 6 ? z >> 1 : z << (6 - k % 7), U[k >> 2][k & 3]);
    }
    *(LAS unsigned long long*)(el + 2560 + lane * 8) = (unsigned long long)S[3][1] << 32 | S[3][0];
    S[3] = *(const LAS v4u*)(el + (lane < 32 ? 2560 + lane * 16 : (lane & 15) * 16));
    em = (1ull << (32u + ((cnt + 33u) >> 5) * 4u)) - 1ull;
    return (unsigned)e0;
  }
#pragma unroll
  for (int d = 0; d < 4; ++d) {
    const unsigned z = U[3][d];
    S[0][d] = bfi(0x60606060u, z << 4, U[0][d]); S[1][d] = bfi(0x60606060u, z << 2, U[1][d]); S[2][d] = bfi(0x60606060u, __builtin_amdgcn_alignbit(z, z, 2), U[2][d]);
  }
  S[3] = *(const LAS v4u*)(el + (lane & 15) * 16);
  em = (1ull << ((cnt + 33u) >> 5) * 4u) - 1ull;
  return (unsigned)e0;
}
__device__ __forceinline__ void pusher(const Params& p, LAS unsigned char* lds, unsigned imgo, unsigned flo, unsigned call, int b, int wid, int lane, unsigned (&R)[16], v4u (&X)[4]) {
  unsigned char* const myws = p.ws[p.rank];
  const int tau = wid / WPT, e = tau * SPT + b / NTURN, jw = (b % NTURN) * WPT + wid % WPT;
  unsigned long long fa = 0;
  unsigned char* wsd[7]; int pr[7];
#pragma unroll
  for (int k = 0; k < 7; ++k) {
    pr[k] = (p.rank + 1 + (wid + b + k) % 7) & 7; wsd[k] = p.ws[pr[k]];
    if (lane == k) fa = (unsigned long long)wsd[k];
  }
  const unsigned jo = job_off(jw, lane), dno = DONE_OFF + (lane < NSH ? lane : 0) * 64u, cwo = CW_OFF;
  LAS unsigned* const pk = (LAS unsigned*)(lds + wid * PK_SLOT);
  int tile, ks; stream_slab(e, p.rank, tile, ks);
  unsigned job = e * 8 + jw, cd = 32u;
  unsigned long long em = ~0ull;
  v4u D[4];
  retile(tile < 16, R, X, D);
  bool cod = true;
  cd = encode(D, (LAS unsigned char*)pk, lane, em, tile < 16);
  static_assert(NTURN == 4, "turn structure");
  v4u D2[4]; unsigned R2[16], cd2 = 0u; unsigned long long em2 = 0ull;
  int tn, kn;
  auto next = [&](int Tn) __attribute__((always_inline)) {
    stream_slab(Tn * SPT + b / NTURN, p.rank, tn, kn);
    load_job(p, tn, kn, jw, lane, R, X);
    unsigned vo = (lane >> 4) * 32 * N_ + 4 * (lane & 15);
    const unsigned char* src = p.w + (size_t)(kn * 128 + (jw & 1) * 16) * N_ + ((unsigned)(tn - 30) < 10u ? SHC + (tn - 30 + NMIX * jw) % 10 : 0) * 256 + (jw >> 1) * 64;
#define LN1(r) "global_load_dword %" #r ", %16, %17\n\tv_add_u32 %16, %19, %16\n\t"
    asm volatile("s_cmp_ge_u32 %18, 10\n\ts_cbranch_scc1 LN_%=\n\t" LN1(0) LN1(1) LN1(2) LN1(3) LN1(4) LN1(5) LN1(6) LN1(7) LN1(8) LN1(9) LN1(10) LN1(11) LN1(12) LN1(13) LN1(14) LN1(15) "\nLN_%=:"
                 : "=&v"(R2[0]), "=&v"(R2[1]), "=&v"(R2[2]), "=&v"(R2[3]), "=&v"(R2[4]), "=&v"(R2[5]), "=&v"(R2[6]), "=&v"(R2[7]),
                   "=&v"(R2[8]), "=&v"(R2[9]), "=&v"(R2[10]), "=&v"(R2[11]), "=&v"(R2[12]), "=&v"(R2[13]), "=&v"(R2[14]), "=&v"(R2[15]), "+v"(vo)
                 : "s"(src), "s"((unsigned)(tn - 30)), "n"(N_) : "memory", "scc");
  };
  next(tau + NTURN);
  for (int i = 0; i < tau; ++i) BAR;
  for (int T = tau; T < NT; T += NTURN) {
    const unsigned sid = tile * 32 + p.rank * 4 + ks, vb = imgo + jo + sid * SLAB;
    const unsigned fo = flo + sid * 64u + jw * 4u + (cod ? 32u : 0u), fv = cod ? call << 6 | cd : call;
    const bool nar = tile >= 30 && tile < 40;
    unsigned long long fn = 0;
    if (nar) {
      const int nq = (tile - 30 + NMIX * jw) % 10, ra = nq * 16 / NSH, rb = (nq * 16 + 15) / NSH;
      const unsigned ds = (unsigned)(70 + nq - tile) * 32u;
      unsigned char* const da = ra != p.rank ? p.ws[ra & 7] : myws + (cwo - imgo);
      unsigned char* const db = rb != p.rank && rb != ra ? p.ws[rb & 7] : myws + (cwo - imgo);
      fn = (unsigned long long)(lane ? p.ws[ra & 7] : p.ws[rb & 7]) + fo + ds * 64u;
#pragma unroll
      for (int c = 0; c < 4; ++c) {
        const unsigned vc = vb + ds * SLAB + c * 2048u;
        const unsigned long long em_ = c == 3 ? em2 : ~0ull;
        asm volatile("s_mov_b64 exec, %4\n\t" ST1("%0", "%1", 2, "") ST1("%0", "%1", 3, "") "s_mov_b64 exec, -1\n\ts_nop 1" :: "v"(vc), "v"(D2[c]), "s"(da), "s"(db), "s"(em_) : "memory");
      }
    }
#pragma unroll
    for (int c = 0; c < 4; ++c) {
      const unsigned vc = vb + c * 2048u;
      const unsigned long long em_ = c == 3 ? em : ~0ull;
      asm volatile("s_mov_b64 exec, %9\n\t" ST7("%0", "%1", IMM) "s_mov_b64 exec, -1\n\ts_nop 1"
                   :: "v"(vc), "v"(D[c]), "s"(wsd[0]), "s"(wsd[1]), "s"(wsd[2]), "s"(wsd[3]), "s"(wsd[4]), "s"(wsd[5]), "s"(wsd[6]), "s"(em_) : "memory");
    }
    const int nb = NT - 1 - T < NTURN ? NT - 1 - T : NTURN;
    const int lag2 = __builtin_amdgcn_readfirstlane((int)(T * SPT >= LAG2));
    asm volatile("s_cmp_eq_u32 %20, 0\n\ts_cbranch_scc1 PB_%=\n\ts_barrier\nPB_%=:\n\ts_waitcnt vmcnt(36)\n\ts_cmp_lt_u32 %22, 10\n\ts_cbranch_scc1 PD_%=\n\ts_waitcnt vmcnt(28)\n"
                 "PD_%=:\n\ts_cmp_lg_u32 %21, 0\n\ts_cbranch_scc1 PC_%=\n\ts_waitcnt vmcnt(0)\nPC_%=:"
                 : TIE16(R), TIE4(X) : "s"(nb), "s"(lag2), "s"((unsigned)(tile - 30)) : "memory", "scc");
    asm volatile("" : TIE16(R2) :: "memory");
    const unsigned fv2 = call << 6 | cd2;
#define NFLAG if (nar) asm volatile("s_mov_b64 exec, 3\n\tglobal_store_dword %0, %1, off sc0 sc1\n\ts_mov_b64 exec, -1" :: "v"(fn), "v"(fv2) : "memory")
    if (!lag2) { store_flag7(fa + fo, fv); NFLAG; if (nb > 1 && NTURN > 2) BAR; }
    tile = tn; ks = kn;
    const bool nn = (unsigned)(tile - 30) < 10u;
    retile(tile < 16, R, X, D);
    cd = 32u; em = ~0ull;
    cod = tile >= 16 || (T + NTURN) * SPT + b / NTURN < XLIM;
    if (cod) cd = encode(D, (LAS unsigned char*)pk, lane, em, tile < 16);
    if (lag2) { if (nb > 1 && NTURN > 2) BAR; WAIT_V(0); store_flag7(fa + fo, fv); NFLAG; }
#define LDI 3
    if (nn) {
      retile(false, R2, X, D2);
      cd2 = encode(D2, (LAS unsigned char*)pk, lane, em2, false);
    }
    if (LDI == 3 && nb > 2) BAR;
    next(T + 2 * NTURN < NT ? T + 2 * NTURN : T);
    for (int i = LDI; i < nb; ++i) BAR;
  }
  const bool ok = wait_flags(myws, dno, call);
  load_job(p, 80, (int)pk_off(p.rank, job), jw, lane, R, X);
  asm volatile("s_waitcnt vmcnt(0)" : TIE16(R), TIE4(X) :: "memory");
  for (;;) {
    unsigned nx = 1u; const unsigned co = WCT_OFF + 64u + (call & 1u) * 128u;
    asm volatile("s_nop 1\n\ts_mov_b64 exec, 1\n\tglobal_atomic_add %0, %1, %0, %2 sc0\n\ts_mov_b64 exec, -1" : "+v"(nx) : "v"(co), "s"(myws) : "memory");
    unsigned cnt = 32u, hdr = 0u;
#pragma unroll
    for (int j = 0; j < 16; ++j) {
      const unsigned v = ok ? X[j >> 2][j & 3] : 0x7fc00000u;
      const unsigned long long m = __ballot(v != 0u);
      const unsigned lo = (unsigned)m, hi = (unsigned)(m >> 32);
      pk[v != 0u ? __builtin_amdgcn_mbcnt_hi(hi, __builtin_amdgcn_mbcnt_lo(lo, cnt)) : 1100u + (unsigned)lane] = v;
      hdr = lane == 2 * j ? lo : lane == 2 * j + 1 ? hi : hdr;
      cnt += (unsigned)__builtin_popcountll(m);
    }
    pk[lane < 32 ? (unsigned)lane : 1100u + (unsigned)lane] = hdr;
    const unsigned nd = (cnt + 31u) & ~31u, pj = p.rank * PK_JOBS + job;
#pragma unroll
    for (int c = 0; c < 5; ++c) {
      const int nl = (int)(nd >> 2) - c * 64;
      if (nl > 0) {
        const v4u d = *(const LAS v4u*)(pk + c * 256 + lane * 4);
        const unsigned vc = PK_OFF + (pj >> 5) * PK_BLK + (c < 3 ? (pj & 31u) * 3072u : 95232u + (pj & 31u) * 2048u) + c * 1024u + lane * 16u;
        const unsigned long long em = nl >= 64 ? ~0ull : (1ull << nl) - 1ull;
        asm volatile("s_mov_b64 exec, %9\n\t" ST7("%0", "%1", PKM) "s_mov_b64 exec, -1\n\ts_nop 1"
                     :: "v"(vc), "v"(d), "s"(wsd[0]), "s"(wsd[1]), "s"(wsd[2]), "s"(wsd[3]), "s"(wsd[4]), "s"(wsd[5]), "s"(wsd[6]), "s"(em) : "memory");
      }
    }
    asm volatile("s_waitcnt vmcnt(0)" : "+v"(nx) :: "memory");
    job = __builtin_amdgcn_readfirstlane(nx) + NPA * 8u;
    const bool more = job < PK_JOBS;
    {
      unsigned vo = lane * 16u; const unsigned char* src = p.out[p.rank] + pk_off(p.rank, more ? job : 0u);
      asm volatile("s_cmp_ge_u32 %6, %8\n\ts_cbranch_scc1 OE_%=\n\tglobal_load_dwordx4 %0, %4, %5\n\tv_add_u32 %4, %7, %4\n\tglobal_load_dwordx4 %1, %4, %5\n\tv_add_u32 %4, %7, %4\n\t"
                   "global_load_dwordx4 %2, %4, %5\n\tv_add_u32 %4, %7, %4\n\tglobal_load_dwordx4 %3, %4, %5\nOE_%=:"
                   : "=&v"(X[0]), "=&v"(X[1]), "=&v"(X[2]), "=&v"(X[3]), "+v"(vo) : "s"(src), "s"(job), "n"(N_ * 4), "n"(PK_JOBS) : "memory", "scc");
    }
    store_flag7(fa + OFLAG_OFF + pj * 4u, call);
    if (!more) break;
    asm volatile("s_waitcnt vmcnt(1)" : TIE4(X) :: "memory");
  }
  TS(1); TS(9);
}
static_assert(XLIM % 8 == 0 && XLIM >= 104 && XLIM <= 120, "whole units");
constexpr unsigned NUNIT = (XLIM + 168) * 14 / 16, UTAB = 120 * 1024;
#define UDLY 12
#define UDLX UDLY
#define UWIN 24u
__device__ __forceinline__ void unpacker(const Params& p, LAS unsigned char* lds, unsigned imgo, unsigned flo, unsigned call, int b, int wid, int lane) {
  unsigned char* const myws = p.ws[p.rank];
  LAS unsigned* lctl = (LAS unsigned*)(lds + LCTL);
  const unsigned long long t0 = __builtin_amdgcn_s_memrealtime();
  if (wid == 0) {
    unsigned i = lane, done = 0;
    for (;;) {
      const unsigned q = (i < NUNIT ? i : 0u) * 16u + (b - NPUSH), sp = q / 14u, r = q - sp * 14u, wp = sp < XLIM ? 0u : sp - XLIM + 56u;
      const int snd = (p.rank + 1 + (r >> 1)) & 7;
      const int k14 = wp / 14u, gp = k14 >> 2, q14 = wp - k14 * 14;
      int ks = k14 & 3;
      int qq = q14 + (gp == 1 ? snd : 2 * snd); qq -= qq >= (gp == 1 ? 12 : 14) ? (gp == 1 ? 12 : 14) : 0;
      int tile = gp == 1 && q14 >= 12 ? 58 + ((NSH * p.rank) >> 4) + q14 : (gp == 0 ? 16 : 14 * gp + (gp == 1 ? 16 : 14)) + qq;
      if (sp < XLIM) stream_slab((int)sp, snd, tile, ks);
      const unsigned hs = (tile * 32 + snd * 4 + ks) * 2 + (r & 1u), fo = flo + (hs >> 1) * 64u + 32u + (hs & 1u) * 16u, to = TMO_OFF;
      const unsigned long long pm = __ballot(i < NUNIT && i < done + UWIN);
      v4u v = {0u, 0u, 0u, 0u}; unsigned tm;
      asm volatile("s_mov_b64 exec, %5\n\tglobal_load_dwordx4 %0, %2, %4 sc0 sc1\n\ts_mov_b64 exec, -1\n\tglobal_load_dword %1, %3, %4 sc0 sc1\n\ts_waitcnt vmcnt(0)"
                   : "+v"(v), "=&v"(tm) : "v"(fo), "v"(to), "s"(myws), "s"(pm) : "memory");
      const unsigned tag = call & 0x3ffffffu;
      const bool ok = (pm >> lane & 1ull) && v[0] >> 6 == tag && v[1] >> 6 == tag && v[2] >> 6 == tag && v[3] >> 6 == tag;
      if (ok) {
        *(LAS unsigned long long*)(lds + UTAB + i * 8u) = (unsigned long long)((v[0] & 63u) | (v[1] & 63u) << 8 | (v[2] & 63u) << 16 | (v[3] & 63u) << 24) << 32 | hs | 0x80000000u;
        i += 64u;
      }
      const unsigned nk = (unsigned)__builtin_popcountll(__ballot(ok));
      done += nk;
      if (done >= NUNIT) break;
      if (__builtin_amdgcn_readfirstlane(tm) == call || __builtin_amdgcn_s_memrealtime() - t0 > DEADLINE_TICKS) {
        __hip_atomic_store((unsigned*)(myws + TMO_OFF), call, __ATOMIC_RELAXED, __HIP_MEMORY_SCOPE_AGENT);
        lctl[2] = 1u;
        break;
      }
      if (!nk) __builtin_amdgcn_s_sleep(4);
    }
    TS(3);
    return;
  }
  const unsigned cwo = CW_OFF, lb = (unsigned)(size_t)lds + (wid - 1) * 16640u, msk = 0x60606060u;
  LAS unsigned char* const buf = lds + (wid - 1) * 16640u;
  unsigned pfo = DONE_OFF + 4032u, pend = 0;
  const v4u cv = {call, call, call, call};
#define UFLAG asm volatile("s_mov_b64 exec, 1\n\tglobal_store_dwordx4 %0, %1, %2 sc0 sc1\n\ts_mov_b64 exec, -1\n\ts_nop 1" :: "v"(pfo), "v"(cv), "s"(myws) : "memory")
  for (unsigned i = wid - 1; i < NUNIT; i += 7u) {
    unsigned hs, cds;
    for (;;) {
      const unsigned long long e = __hip_atomic_load((LAS unsigned long long*)(lds + UTAB + i * 8u), __ATOMIC_RELAXED, __HIP_MEMORY_SCOPE_WORKGROUP);
      hs = __builtin_amdgcn_readfirstlane((unsigned)e); cds = __builtin_amdgcn_readfirstlane((unsigned)(e >> 32));
      if (hs) break;
      if (pend) { WAIT_V(0); UFLAG; pend = 0; continue; }
      if (__builtin_amdgcn_readfirstlane(__hip_atomic_load(lctl + 2, __ATOMIC_RELAXED, __HIP_MEMORY_SCOPE_WORKGROUP)) || __builtin_amdgcn_s_memrealtime() - t0 > 2 * DEADLINE_TICKS) return;
      __builtin_amdgcn_s_sleep(2);
    }
    hs &= 0x7fffffffu;
    const unsigned so = (hs >> 1) * SLAB + (hs & 1u) * 16384u;
    auto unit = [&](auto isx_) __attribute__((always_inline)) {
      constexpr bool isx = decltype(isx_)::value;
      v4u C[4][3]; unsigned E[4]; v2u P[4];
      unsigned va = imgo + so + lane * 16u + 4096u, ve = imgo + so + lane * 4u + 4096u, vp = imgo + so + lane * 8u + 4096u;
#define UL(j, o) "global_load_dwordx4 %" #j ", %16, %18 offset:" #o " sc0 sc1\n\t"
#define UE(j, o) "global_load_dword %" #j ", %17, %18 offset:" #o " sc0 sc1\n\t"
#define UOUT "=&v"(C[0][0]), "=&v"(C[0][1]), "=&v"(C[0][2]), "=&v"(C[1][0]), "=&v"(C[1][1]), "=&v"(C[1][2]), \
             "=&v"(C[2][0]), "=&v"(C[2][1]), "=&v"(C[2][2]), "=&v"(C[3][0]), "=&v"(C[3][1]), "=&v"(C[3][2]), \
             "=&v"(E[0]), "=&v"(E[1]), "=&v"(E[2]), "=&v"(E[3]), "+v"(va), "+v"(ve)
      if constexpr (isx) {
#define XL(j, o) "global_load_dwordx4 %" #j ", %16, %23 offset:" #o " sc0 sc1\n\t"
#define XE(j, o) "global_load_dword %" #j ", %17, %23 offset:" #o " sc0 sc1\n\t"
#define XP(j, o) "global_load_dwordx2 %" #j ", %22, %23 offset:" #o " sc0 sc1\n\t"
        asm volatile(XL(0, -4096) XL(1, -2048) XL(2, 0) XP(18, 2048) XE(12, 2560)
                     XL(3, -3072) XL(4, -1024) XL(5, 1024) XP(19, 3072) XE(13, 3584)
                     "v_add_u32 %16, 0x2000, %16\n\tv_add_u32 %17, 0x2000, %17\n\tv_add_u32 %22, 0x2000, %22\n\t"
                     XL(6, -4096) XL(7, -2048) XL(8, 0) XP(20, 2048) XE(14, 2560)
                     XL(9, -3072) XL(10, -1024) XL(11, 1024) XP(21, 3072) XE(15, 3584)
                     "s_waitcnt vmcnt(20)"
                     : UOUT, "=&v"(P[0]), "=&v"(P[1]), "=&v"(P[2]), "=&v"(P[3]), "+v"(vp) : "s"(myws) : "memory");
      } else {
        asm volatile(UL(0, -4096) UL(1, -2048) UL(2, 0) UE(12, 2048) UL(3, -3072) UL(4, -1024) UL(5, 1024) UE(13, 3072)
                     "v_add_u32 %16, 0x2000, %16\n\tv_add_u32 %17, 0x2000, %17\n\t"
                     UL(6, -4096) UL(7, -2048) UL(8, 0) UE(14, 2048) UL(9, -3072) UL(10, -1024) UL(11, 1024) UE(15, 3072)
                     "s_waitcnt vmcnt(16)"
                     : UOUT : "s"(myws) : "memory");
      }
      UFLAG;
#pragma unroll
      for (int j = 0; j < 4; ++j) {
        const unsigned cd = cds >> (8 * j) & 63u, c2 = (cd & 31u) * (isx ? 0x08080808u : 0x04040404u);
        v4u D[4];
        if constexpr (isx) {
          asm volatile("s_waitcnt vmcnt(%5)" : "+v"(C[j][0]), "+v"(C[j][1]), "+v"(C[j][2]), "+v"(E[j]), "+v"(P[j]) : "n"(16 - j) : "memory");
          unsigned z[2] = {c2, c2};
#pragma unroll
          for (int k = 0; k < 14; ++k) {
            const unsigned c = k < 12 ? C[j][k >> 2][k & 3] : P[j][k & 1], t = c & 0x40404040u;
            asm("v_xad_u32 %0, %1, %2, %3" : "=v"(D[k >> 2][k & 3]) : "v"(c), "v"(t), "s"(c2));
            z[k / 7] += k % 7 == 6 ? t << 1 : t >> (6 - k % 7);
          }
          D[3][2] = z[0]; D[3][3] = z[1];
        } else {
          asm volatile("s_waitcnt vmcnt(13)" : "+v"(C[j][0]), "+v"(C[j][1]), "+v"(C[j][2]), "+v"(E[j]) :: "memory");
#pragma unroll
          for (int d = 0; d < 4; ++d) {
            const unsigned t0_ = C[j][0][d] & msk, t1_ = C[j][1][d] & msk, t2_ = C[j][2][d] & msk;
            asm("v_xad_u32 %0, %1, %2, %3" : "=v"(D[0][d]) : "v"(C[j][0][d]), "v"(t0_), "s"(c2));
            asm("v_xad_u32 %0, %1, %2, %3" : "=v"(D[1][d]) : "v"(C[j][1][d]), "v"(t1_), "s"(c2));
            asm("v_xad_u32 %0, %1, %2, %3" : "=v"(D[2][d]) : "v"(C[j][2][d]), "v"(t2_), "s"(c2));
            D[3][d] = __builtin_amdgcn_alignbit(t2_, t2_, 30) + (t1_ >> 2) + (t0_ >> 4) + c2;
          }
        }
#pragma unroll
        for (int c = 0; c < 4; ++c) *(LAS v4u*)(buf + j * 4096 + c * 1024 + lane * 16) = D[c];
        const unsigned h = __builtin_amdgcn_readfirstlane(E[j]);
#pragma unroll
        for (int k = 0; k < 2; ++k) {
          const unsigned en = k ? E[j] >> 16 : E[j], ps = 2u * lane - 2u + k, y = en & 0x1fe0u, t = y | y << 6;
          const unsigned ch = (ps >= (h & 127u)) + (ps >= (h >> 8 & 127u)) + (ps >= (h >> 16 & 127u));
          const unsigned ad = lb + (ps < (h >> 24 & 127u) ? j * 4096u + ch * 1024u + (en & 31u) * 4u + (en >> 13 & 7u) * 128u : 16384u + lane * 4u), mv = (t | t << 12) & msk;
          asm volatile("ds_mskor_b32 %0, %1, %2" :: "v"(ad), "v"(msk), "v"(mv) : "memory");
        }
#pragma unroll
        for (int c = 0; c < 4; ++c) D[c] = *(const LAS v4u*)(buf + j * 4096 + c * 1024 + lane * 16);
        store_job(myws, (cd < 32u ? imgo : cwo + (isx ? 512u * SLAB : 0u)) + so + (j >> 1) * 8192u + (j & 1) * 1024u + lane * 16u + 4096u, D);
      }
    };
    if (hs < 1024u) unit(std::true_type{}); else unit(std::false_type{});
    pfo = flo + (hs >> 1) * 64u + (hs & 1u) * 16u; pend = 1;
  }
  WAIT_V(0); UFLAG;
}
#define NG 120u
#define N1 100u
#define XCNT 768u
__device__ __forceinline__ unsigned blk_job(int rank, unsigned q) { const unsigned bl = q / 7u; return ((rank + 1 + (q - bl * 7u)) & 7u) * PK_JOBS + bl * 32u; }
__device__ __forceinline__ void expand(const Params& p, LAS unsigned char* lds, unsigned call, int wid, unsigned q, unsigned mode) {
  unsigned char* const myws = p.ws[p.rank];
  LAS unsigned* lctl = (LAS unsigned*)(lds + LCTL);
  const int lane = fresh_lane();
  const unsigned lbase = (unsigned)(size_t)lds;
  for (unsigned r = 0;; ++r) {
    if (mode == 0u) {
      if (threadIdx.x == 0) lctl[6 + (r & 1u)] = __hip_atomic_fetch_add((unsigned*)(myws + WCT_OFF + (call & 1u) * 128u), 1u, __ATOMIC_RELAXED, __HIP_MEMORY_SCOPE_AGENT);
      __syncthreads();
      q = __builtin_amdgcn_readfirstlane(lctl[6 + (r & 1u)]); q += q < N1 ? 0u : NG;
    }
    if (q >= NBLK) break;
    const unsigned pj = blk_job(p.rank, q), snd = pj / PK_JOBS, vb = PK_OFF + (pj >> 5) * PK_BLK + lane * 16u;
    bool ok = true;
    if (mode != 2u) {
      ok = wait_flags(myws, OFLAG_OFF + (pj + wid + (lane & 24)) * 4u, call);
#pragma unroll
      for (int h = 0; h < 4; ++h) {
        const unsigned la = lbase + (wid + h * 8) * 3072u + (h >> 1) * 16384u, vo = vb + (wid + h * 8) * 3072u;
        asm volatile("s_mov_b32 m0, %0\n\ts_nop 0\n\tglobal_load_lds_dwordx4 %1, %2 sc0 sc1\n\tglobal_load_lds_dwordx4 %1, %2 offset:1024 sc0 sc1\n\t"
                     "global_load_lds_dwordx4 %1, %2 offset:2048 sc0 sc1" :: "s"(la), "v"(vo), "s"(myws) : "memory");
      }
      WAIT_V(0);
    }
#pragma unroll 2
    for (int h = 0; h < 4; ++h) {
      const unsigned j = wid + h * 8, oo = pk_off(snd, pj - snd * PK_JOBS + j) + lane * 16u, so = j * 3072u + (h >> 1) * 16384u;
      const LAS unsigned* pk = (const LAS unsigned*)(lds + so);
      const unsigned hdr = pk[lane & 31];
      unsigned pos[16], cnt = 32u; unsigned long long m[16];
#pragma unroll
      for (int i = 0; i < 16; ++i) {
        const unsigned lo = __builtin_amdgcn_readlane(hdr, 2 * i), hi = __builtin_amdgcn_readlane(hdr, 2 * i + 1);
        m[i] = (unsigned long long)hi << 32 | lo;
        pos[i] = __builtin_amdgcn_mbcnt_hi(hi, __builtin_amdgcn_mbcnt_lo(lo, cnt));
        cnt += (unsigned)__builtin_popcountll(m[i]);
      }
      if (cnt > XCNT) {
        const unsigned s2 = 49152u + wid * 2048u, la = lbase + s2, vo = vb + 98304u + j * 2048u, d2 = (s2 - so - 3072u) >> 2;
        asm volatile("s_waitcnt lgkmcnt(0)\n\ts_mov_b32 m0, %0\n\ts_nop 0\n\tglobal_load_lds_dwordx4 %1, %2 sc0 sc1\n\tglobal_load_lds_dwordx4 %1, %2 offset:1024 sc0 sc1\n\ts_waitcnt vmcnt(0)"
                     :: "s"(la), "v"(vo), "s"(myws) : "memory");
#pragma unroll
        for (int i = 0; i < 16; ++i) pos[i] = (pos[i] + (pos[i] >= 768u ? d2 : 0u)) & 0x3fffffffu;
      }
#pragma unroll
      for (int rr = 0; rr < 4; ++rr) {
        v4u v;
#pragma unroll
        for (int c = 0; c < 4; ++c) {
          const unsigned x = *(const LAS unsigned*)(lds + (so + pos[4 * rr + c] * 4u));
          v[c] = !ok ? 0x7fc00000u : (m[4 * rr + c] >> lane & 1ull) ? x : 0u;
        }
        const unsigned vo_ = oo + rr * (N_ * 4u);
        asm volatile("global_store_dwordx4 %0, %1, %2" XSM "\n\ts_nop 1" :: "v"(vo_), "v"(v), "s"(p.out[p.rank]) : "memory");
      }
    }
    if (mode) break;
  }
}

__global__ __launch_bounds__(512, 2) void fused_kernel(const Params p) {
  extern __shared__ __attribute__((aligned(16))) unsigned char smem[];
  LAS unsigned char* lds = (LAS unsigned char*)smem;
  const int tid = threadIdx.x, wid = __builtin_amdgcn_readfirstlane(tid >> 6), wc = wid & 3;
  unsigned char* myws = p.ws[p.rank];
  const int b = blockIdx.x, rank = p.rank, g = b - NCOM, gl = b - NPUSH;
  const bool push = b < NPUSH;
  LAS unsigned* lctl = (LAS unsigned*)(lds + LCTL);
  const unsigned lbase = (unsigned)(size_t)lds;
  const int xc = b & 7, L = (xc >> 2) * 28 + (g >> 3), cn = L >> 2;
  const int pm = (xc & 3) * 4 + (L & 3), si = L * 4 + (xc & 3);
  const bool skip = cn < 2 && si >= NSH;
  const unsigned nvis = skip ? 96u : 128u;
  auto tile_of = [&](int u, int& tm, int& tn) __attribute__((always_inline)) {
    const int v = u + (skip && u > 0), idx = NSH * rank + si;
    const bool sh = v == 1 && cn < 2;
    tm = sh ? idx & 15 : pm; tn = sh ? SHC + (idx >> 4) : v == 0 ? cn : 14 * v - 2 + cn;
  };
  unsigned call, imgo, flo;
  {
    const int lane = tid & 63;
    unsigned R[16], R2[16]; v4u X[4], X2[4];
    int t1, k1, j1 = wid;
    if (push) { j1 = (b % NTURN) * WPT + wid % WPT; stream_slab(wid / WPT * SPT + b / NTURN, rank, t1, k1); } else { t1 = gl >> 2; k1 = gl & 3; }
    load_job(p, t1, k1, j1, lane, R, X);
    const bool two = !push && gl < 80;
    if (two) load_job(p, t1 + 60, k1, wid, lane, R2, X2);
    if (tid == 0) {
      unsigned* const cp = (unsigned*)(myws + CTB_OFF + b * CTS);
      const unsigned t = __hip_atomic_load(cp, __ATOMIC_RELAXED, __HIP_MEMORY_SCOPE_AGENT) * (unsigned)GRID;
      __hip_atomic_store(cp, t / (unsigned)GRID + 1u, __ATOMIC_RELAXED, __HIP_MEMORY_SCOPE_AGENT);
      lctl[0] = t / (unsigned)GRID + 1u; lctl[1] = INITK; lctl[2] = 0u; lctl[3] = 0u; lctl[5] = 0u;
      if (b == 0) {
        __hip_atomic_store((unsigned*)(myws + WCT_OFF + (t / (unsigned)GRID & 1u) * 128u), 0u, __ATOMIC_RELAXED, __HIP_MEMORY_SCOPE_AGENT);
        __hip_atomic_store((unsigned*)(myws + WCT_OFF + 64u + (t / (unsigned)GRID & 1u) * 128u), 0u, __ATOMIC_RELAXED, __HIP_MEMORY_SCOPE_AGENT);
        __hip_atomic_store((unsigned*)(myws + WCT_OFF + 256u + (t / (unsigned)GRID & 1u) * 128u), 0u, __ATOMIC_RELAXED, __HIP_MEMORY_SCOPE_AGENT);
      }
    }
    if (!push && g < 0 && tid < (int)NUNIT) *(LAS unsigned long long*)(lds + UTAB + tid * 8) = 0ull;
    if (wid >= 4 && g >= 0) {
      const int u = wid - 4, l = lane & 31, src = l >> 2, ks = l & 3;
      int tm, tn; tile_of(u, tm, tn);
      const int a0 = stream_idx(tm, ks, src), ax = a0 + (a0 < XLIM ? UDLX : 0), aw = stream_idx(16 + tn, ks, src) + UDLY;
#define CORD 9
      const int key = src == rank ? ks : u >= CORD ? 32 + ks * 8 + ((src - rank) & 7) : ((ax > aw ? ax : aw) + 1) * 32 + l;
      int rk = 0;
#pragma unroll
      for (int s = 0; s < 32; ++s) rk += __builtin_amdgcn_readlane(key, s) < key;
      const unsigned ent = (unsigned)((16 + tn) * 32 + l) << 15 | (unsigned)(tm * 32 + l);
      if (u * 32u < nvis) *(LAS unsigned*)(lds + LTAB + (u * 32 + rk) * 4) = ent;
      if (u == 0 && rk < 4) *(LAS unsigned*)(lds + LTAB + (nvis + rk) * 4) = ent;
    }
    asm volatile("s_waitcnt vmcnt(0)" : TIE16(R), TIE4(X) :: "memory");
    asm volatile("" : TIE16(R2) :: "memory");
    __syncthreads();
    call = __builtin_amdgcn_readfirstlane(lctl[0]);
    imgo = IMG_OFF + (call & 1u) * IMG_BYTES;
    flo = FLAG_OFF + (call & 1u) * FLAG_BYTES;
    TS(0);
    if (push) {
      if (b < NPA) pusher(p, lds, imgo, flo, call, b, wid, lane, R, X);
      WAIT_V(0);
      expand(p, lds, call, wid, 0u, 0u);
      TS(14);
      return;
    }
    v4u D[4];
    const unsigned s1 = t1 * 32 + rank * 4 + k1;
    retile(t1 < 16, R, X, D);
    store_job(myws, imgo + s1 * SLAB + job_off(wid, lane) + 4096, D);
    if (two) {
      retile(false, R2, X2, D);
      store_job(myws, imgo + (s1 + 60 * 32) * SLAB + job_off(wid, lane) + 4096, D);
    }
    WAIT_V(0);
    store_flag1(myws, flo + s1 * 64 + wid * 4, call);
    if (two) store_flag1(myws, flo + (s1 + 60 * 32) * 64 + wid * 4, call);
    TS(1);
    if (g < 0) {
      unpacker(p, lds, imgo, flo, call, b, wid, lane);
      __syncthreads();
      TS(4);
      expand(p, lds, call, wid, 0u, 0u);
      TS(14);
      return;
    }
  }
  const float scale = __builtin_bit_cast(float, __builtin_amdgcn_readfirstlane(__builtin_bit_cast(int, p.sx[0] * p.sw[0])));
  WAIT_V(0);

  auto run = [&](auto WRC) __attribute__((always_inline)) {
    constexpr int wr = decltype(WRC)::value;
    unsigned ppend = 0, pk = INITK, pw = 4u;
    auto peval = [&]() __attribute__((always_inline)) {
      const v4u f = *(const LAS v4u*)(lds + LCP + (unsigned)fresh_lane() * 16);
      unsigned long long bm = __ballot(f[0] == call && f[1] == call && f[2] == call && f[3] == call);
      bm &= bm >> 1; bm &= bm >> 2;
      bm = ~bm & 0x1111111111111111ull;
      unsigned c = bm ? (unsigned)__builtin_ctzll(bm) >> 2 : 16u;
      c = c < pw ? c : pw;
      pw = c == pw ? 16u : PWIN;
      const unsigned n = pk + c < nvis ? pk + c : nvis;
      if (n > pk) {
        pk = n; lctl[1] = n;
      }
      ppend = 0;
    };
    auto pissue = [&]() __attribute__((always_inline)) {
      const unsigned ln = (unsigned)fresh_lane();
      unsigned e = pk + ((ln >> 2) < pw ? ln >> 2 : 0u); e = e < nvis ? e : nvis - 1;
      const unsigned kk = *(const LAS unsigned*)(lds + LTAB + e * 4);
      const unsigned la = lbase + LCP, fo = flo + ((ln & 2) ? kk >> 15 : kk & 0x7fffu) * 64 + (ln & 1) * 16; unsigned char* const w_ = myws;
      asm volatile("s_mov_b32 m0, %0\n\ts_nop 0\n\tglobal_load_lds_dwordx4 %1, %2 sc0 sc1" :: "s"(la), "v"(fo), "s"(w_) : "memory");
      ppend = 1;
    };
    auto poll = [&](bool blocking, unsigned need) __attribute__((always_inline)) {
      if (!blocking) {
        if (ppend) {
          unsigned st; asm volatile("s_getreg_b32 %0, hwreg(HW_REG_IB_STS)" : "=s"(st) :: "memory");
          st = __builtin_amdgcn_readfirstlane(st);
          if ((st & 15u) | ((st >> 18) & 48u)) return;
          WAIT_V(0);
          peval();
        } else { pw = 16u; pissue(); }
        return;
      }
      WAIT_V(0);
      if (ppend) { peval(); if (pk < need) __builtin_amdgcn_s_sleep(PSLEEP); }
      if (pk < nvis) pissue();
      WAIT_L(0);
    };
    auto slow = [&](unsigned need) __attribute__((always_inline)) {
      const unsigned long long t0 = __builtin_amdgcn_s_memrealtime();
      for (;;) {
        if (wr == 1 && wc == 0) poll(true, need); else __builtin_amdgcn_s_sleep(1);
        const unsigned long long el = __builtin_amdgcn_s_memrealtime() - t0;
        if (wr == 1 && wc == 0 && el > 100000ull) {
          unsigned* tmo = (unsigned*)(myws + TMO_OFF);
          if (__builtin_amdgcn_readfirstlane(__hip_atomic_load(tmo, __ATOMIC_RELAXED, __HIP_MEMORY_SCOPE_AGENT)) == call || el > DEADLINE_TICKS) {
            __hip_atomic_store(tmo, call, __ATOMIC_RELAXED, __HIP_MEMORY_SCOPE_AGENT);
            lctl[2] = 1u; lctl[1] = DEADV; pk = nvis; ppend = 0;
          }
          WAIT_V(0);
        }
        if (__builtin_amdgcn_readfirstlane(__hip_atomic_load(lctl + 1, __ATOMIC_RELAXED, __HIP_MEMORY_SCOPE_WORKGROUP)) >= need || el > 2 * DEADLINE_TICKS) break;
      }
      asm volatile("" ::: "memory");
    };

#define GT0 26
    auto grab = [&](int t) __attribute__((always_inline)) {
      const unsigned ln = (unsigned)fresh_lane() & 31u, q = __builtin_amdgcn_readfirstlane(lctl[4]), pj = blk_job(rank, q);
      if (q >= NBLK) return;
      if (t == GT0) {
        const unsigned la = lbase + LCTL + 64u, fo = OFLAG_OFF + (pj + ln) * 4u; unsigned char* const w_ = myws;
        asm volatile("s_mov_b32 m0, %0\n\ts_mov_b32 exec_hi, 0\n\tglobal_load_lds_dword %1, %2 sc0 sc1\n\ts_mov_b32 exec_hi, -1" :: "s"(la), "v"(fo), "s"(w_) : "memory");
      } else {
        WAIT_V(0);
        lctl[5] = __ballot(lctl[16 + ln] != call) == 0ull ? PK_OFF + (pj >> 5) * PK_BLK - imgo : 0u;
      }
    };
#define TBURST 3u
#define TEND 12604u
    unsigned ti = (unsigned)(g * 3 + wc - 1);
    auto touch = [&]() __attribute__((always_inline)) {
      const unsigned k = (ti >= 960u) + (ti >= 1920u) + (ti >= 2880u), r = ti - k * 960u, i2 = ti - 9244u, bq = i2 / 12u;
      const unsigned o = ti < 3840u ? imgo + ((r >> 2) * 4u + k) * SLAB + (r & 3u) * 8192u
                       : ti < 3848u ? (ti - 3840u) * 8192u
                       : ti < 3868u ? flo + (ti - 3848u) * 8192u
                       : ti < 4124u ? imgo + (70u + ((NSH * rank) >> 4)) * 32u * SLAB + (ti - 3868u) * 8192u
                       : ti < 9244u ? imgo + 30u * 32u * SLAB + (ti - 4124u) * 8192u
                                    : PK_OFF + (blk_job(rank, bq) >> 5) * PK_BLK + (i2 - bq * 12u) * 8192u;
      const unsigned la = lbase + LTCH, fo = o + (unsigned)fresh_lane() * 128u; unsigned char* const w_ = myws;
      asm volatile("s_mov_b32 m0, %0\n\ts_nop 0\n\tglobal_load_lds_dword %1, %2 sc0 sc1" :: "s"(la), "v"(fo), "s"(w_) : "memory");
      ti += 672u;
    };
    const unsigned char* XI = myws + imgo;
    const unsigned voff = (unsigned)(wc * 4096 + (tid & 63) * 16);
    const unsigned ldsw = (unsigned)wc * 4096u;
    const int aoff = wr * 8192 + (tid & 63) * 16, boff = wc * 4096 + (tid & 63) * 16;
#define SA(b_, h) (((b_) * 2 + (h)) * HTB)
#define SB(b_, h) ((4 + (b_) * 2 + (h)) * HTB)
#define STG1(bufoff, va, o) __builtin_amdgcn_global_load_lds((const unsigned*)(XI + (va)), (LAS unsigned*)(lds + (bufoff) + ldsw), 16, o, 0)
#define STAGE(bufoff, goff) do { if (wr == 0) { const unsigned va_ = voff + (goff); STG1(bufoff, va_, 0); STG1(bufoff, va_, 1024); STG1(bufoff, va_, 2048); STG1(bufoff, va_, 3072); } } while (0)
#define LDA(dst, b_, h) do { _Pragma("unroll") for (int m = 0; m < 4; ++m) { \
    v4i lo = *(const LAS v4i*)(lds + SA(b_, h) + aoff + m * 2048); v4i hi = *(const LAS v4i*)(lds + SA(b_, h) + aoff + m * 2048 + 1024); dst[m] = cat8(lo, hi); } } while (0)
#define LDB(dst, b_, h) do { _Pragma("unroll") for (int n = 0; n < 2; ++n) { \
    v4i lo = *(const LAS v4i*)(lds + SB(b_, h) + boff + n * 2048); v4i hi = *(const LAS v4i*)(lds + SB(b_, h) + boff + n * 2048 + 1024); dst[n] = cat8(lo, hi); } } while (0)
#define MMA(ai, bj, At, Bt) do { __builtin_amdgcn_s_setprio(1); _Pragma("unroll") for (int m = 0; m < 4; ++m) _Pragma("unroll") for (int n = 0; n < 2; ++n) \
    acc[ai][bj][m][n] = __builtin_amdgcn_mfma_scale_f32_16x16x128_f8f6f4(Bt[n], At[m], acc[ai][bj][m][n], 0, 1, 0, 0, 0, 0); __builtin_amdgcn_s_setprio(0); } while (0)
#define MF1(ai, bj, At, Bt, q) acc[ai][bj][(q) >> 1][(q) & 1] = __builtin_amdgcn_mfma_scale_f32_16x16x128_f8f6f4(Bt[(q) & 1], At[(q) >> 1], acc[ai][bj][(q) >> 1][(q) & 1], 0, 1, 0, 0, 0, 0)
#define WV12 WAIT_V(12)
#define MEMSTEP do { if (wr == 0) WV12; else if (wc == 0 && (ppend || (pk < nvis && pk < need + PTH))) poll(false, 0u); } while (0)
#define KOFF(i) ((unsigned)__builtin_amdgcn_readlane(kv, (i)))
    f32x4 acc[2][2][4][2];
#pragma unroll
    for (int a = 0; a < 2; ++a)
#pragma unroll
      for (int c = 0; c < 2; ++c)
#pragma unroll
        for (int m = 0; m < 4; ++m)
#pragma unroll
          for (int n = 0; n < 2; ++n) acc[a][c][m][n] = (f32x4){0.f, 0.f, 0.f, 0.f};
    v8i At[4], B0[2], B1[2];
    constexpr unsigned hstep = HTB;
#define OFA(e_) ((e_) & ~0x7fffu)
#define OFB(e_) (((e_) & 0x7fffu) << 15)
    if (wr == 1 && wc > 0) for (unsigned i = 0; i < TBURST; ++i) touch();
    slow(4u);
    unsigned known = 4u, need = 4u;
    const unsigned e0 = __builtin_amdgcn_readfirstlane(*(const LAS unsigned*)(lds + LTAB));
    {
      const unsigned cA = OFA(e0), cB = OFB(e0);
      STAGE(SB(0, 0), cB); STAGE(SA(0, 0), cA); STAGE(SB(0, 1), cB + hstep); STAGE(SA(0, 1), cA + hstep);
      if (wr == 1) BAR;
      if (wr == 0) WAIT_V(8);
      BAR;
      STAGE(SB(1, 0), cB + SLAB); STAGE(SA(1, 0), cA + SLAB); STAGE(SB(1, 1), cB + SLAB + hstep);
      if (wr == 0) WAIT_V(12);
      BAR;
    }
    unsigned a1 = OFA(e0) + SLAB, a2 = a1 + SLAB, a3 = a2 + SLAB, b2 = OFB(e0) + 2 * SLAB, b3 = b2 + SLAB;
    for (int ui = 0;; ++ui) {
      const bool has_next = (unsigned)(ui * 32 + 32) < nvis;
      int kv = 0;
      if (wr == 0) { const unsigned ln = ui * 32u + (unsigned)fresh_lane(); kv = *(const LAS int*)(lds + LTAB + (ln < 131u ? ln : 131u) * 4); }
      for (int t = 0; t < NKS; t += 2) {
        if (known < need) { if (wr == 0) BAR; slow(need); if (wr == 1) BAR; known = need; }
        LDB(B0, 0, 0); SCHED; LDA(At, 0, 0); SCHED; STAGE(SA(1, 1), a1 + hstep);
        WAIT_L(8); BAR; WAIT_L(0); MMA(0, 0, At, B0); BAR; SCHED;
        LDB(B1, 0, 1); SCHED; STAGE(SB(0, 0), b2);
        BAR; WAIT_L(0); MMA(0, 1, At, B1); BAR;
        LDA(At, 0, 1); SCHED; STAGE(SA(0, 0), a2);
        BAR; WAIT_L(0); MMA(1, 0, At, B0); BAR; SCHED;
        STAGE(SB(0, 1), b2 + hstep); MEMSTEP;
        if (wr == 1 && wc == 1 && ui == 3 && (t == GT0 || t == NKS - 4)) grab(t);
        if (wr == 1 && wc > 0 && ti < TEND) touch();
        BAR; MMA(1, 1, At, B1); BAR;
        LDB(B0, 1, 0); SCHED; LDA(At, 1, 0); SCHED; STAGE(SA(0, 1), a2 + hstep);
        WAIT_L(8); BAR; WAIT_L(0); MMA(0, 0, At, B0); BAR; SCHED;
        LDB(B1, 1, 1); const unsigned kn = lctl[1]; SCHED; STAGE(SB(1, 0), b3);
        BAR; WAIT_L(0); MMA(0, 1, At, B1); BAR;
        LDA(At, 1, 1); SCHED; STAGE(SA(1, 0), a3);
        BAR; WAIT_L(0); MMA(1, 0, At, B0); BAR; SCHED;
        STAGE(SB(1, 1), b3 + hstep); if (wr == 1 || has_next || t < NKS - 2) MEMSTEP;
        BAR; SCHED;
        __builtin_amdgcn_s_setprio(1);
        MF1(1, 1, At, B1, 0); SCHED;
        unsigned o2 = KOFF(t + 4), o3 = KOFF(t + 5); known = __builtin_amdgcn_readfirstlane(kn);
        asm volatile("" : "+s"(o2), "+s"(o3), "+s"(known)); SCHED;
        MF1(1, 1, At, B1, 1); SCHED;
        a1 = a3; a2 = OFA(o2); b2 = OFB(o2);
        asm volatile("" : "+s"(a1), "+s"(a2), "+s"(b2)); SCHED;
        MF1(1, 1, At, B1, 2); SCHED;
        a3 = OFA(o3); b3 = OFB(o3);
        if (wr == 0 && !has_next && t == NKS - 4) {
          const unsigned ob_ = __builtin_amdgcn_readfirstlane(lctl[5]);
          a2 = ob_ ? ob_ : a1; a3 = ob_ ? ob_ + 2 * hstep : a1; b2 = ob_ ? ob_ + 3 * hstep : a1; b3 = ob_ ? ob_ + 5 * hstep : a1;
        }
        asm volatile("" : "+s"(a2), "+s"(b2), "+s"(a3), "+s"(b3)); SCHED;
        MF1(1, 1, At, B1, 3); SCHED;
        need = need + 2u < nvis ? need + 2u : nvis;
        asm volatile("" : "+s"(need)); SCHED;
        MF1(1, 1, At, B1, 4); MF1(1, 1, At, B1, 5); MF1(1, 1, At, B1, 6); MF1(1, 1, At, B1, 7);
        __builtin_amdgcn_s_setprio(0);
        BAR;
      }
      int tm, tn; tile_of(ui, tm, tn);
      const bool dead = __builtin_amdgcn_readfirstlane(lctl[2]) != 0u, sht = tn >= SHC;
      TS(2 + 2 * ui);
      unsigned gv = 1u;
      const int ng = __builtin_amdgcn_readfirstlane((int)!(wc == 1 && ui == 2 && has_next));
      {
        const int ln = fresh_lane();
        const unsigned vo = (unsigned)((tm * 256 + wc * 32 + 4 * (ln >> 4)) * N_ + tn * 256 + wr * 64 + 4 * (ln & 15)) * 4u;
        const float qnan = __builtin_nanf("");
        const float sc_ = dead ? qnan : scale, lo_ = dead ? qnan : 0.f;
        const int shs = __builtin_amdgcn_readfirstlane(sht ? 1 : ui < PLN ? 2 : 0);
        unsigned char* const ob = p.out[rank];
        if (wr == 1) {
          const unsigned co = WCT_OFF + 256u + (call & 1u) * 128u; unsigned char* const w_ = myws;
          asm volatile("s_cmp_lg_u32 %3, 0\n\ts_cbranch_scc1 GE_%=\n\ts_mov_b64 exec, 1\n\tglobal_atomic_add %0, %1, %0, %2 sc0\n\ts_mov_b64 exec, -1\nGE_%=:"
                       : "+v"(gv) : "v"(co), "s"(w_), "s"(ng) : "memory", "scc");
        }
#pragma unroll
        for (int bj = 0; bj < 2; ++bj)
#pragma unroll
          for (int n = 0; n < 2; ++n)
#pragma unroll
            for (int r4 = 0; r4 < 4; ++r4) {
              unsigned char* rowp = ob + (size_t)(bj * 128 + n * 16 + r4) * N_ * 4;
#pragma unroll
              for (int ai = 0; ai < 2; ++ai) {
                f32x4 v;
#pragma unroll
                for (int c = 0; c < 4; ++c) {
                  const float x = acc[ai][bj][c][n][r4] * sc_;
                  asm("v_max_f32 %0, %1, %2" : "=v"(v[c]) : "s"(lo_), "v"(x));
                }
                asm volatile("s_cmp_lg_u32 %4, 0\n\ts_cbranch_scc1 SW_%=\n\tglobal_store_dwordx4 %0, %1, %2 offset:%3" OSM "\n\ts_branch SE_%=\n"
                             "SW_%=:\n\tglobal_store_dwordx4 %0, %1, %2 offset:%3 sc0 sc1" SHM "\nSE_%=:\n\ts_nop 1"
                             :: "v"(vo), "v"(v), "s"(rowp), "n"(ai * 512), "s"(shs) : "memory", "scc");
              }
            }
      }
      if (!has_next && wr == 0) WAIT_V(32);
      if (wr == 1) {
        asm volatile("s_waitcnt vmcnt(32)" : "+v"(gv) :: "memory");
        if (!ng) { const unsigned q_ = __builtin_amdgcn_readfirstlane(gv); lctl[4] = q_ < NG ? q_ + N1 : NBLK; }
      }
      if (sht) {
        WAIT_V(0);
        unsigned o_ = 0;
        if (fresh_lane() == 0) o_ = __hip_atomic_fetch_add(lctl + 3, 1u, __ATOMIC_RELAXED, __HIP_MEMORY_SCOPE_WORKGROUP);
        if (__builtin_amdgcn_readfirstlane(o_) == 7u) store_flag1(myws, DONE_OFF + si * 64, call);
        if (wr == 0) WAIT_V(0);
      }
      TS(3 + 2 * ui);
      if (!has_next) break;
#pragma unroll
      for (int a = 0; a < 2; ++a)
#pragma unroll
        for (int c = 0; c < 2; ++c)
#pragma unroll
          for (int m = 0; m < 4; ++m)
#pragma unroll
            for (int n = 0; n < 2; ++n) acc[a][c][m][n] = (f32x4){0.f, 0.f, 0.f, 0.f};
    }
    if (wr == 0) BAR;
  };
  if (wid < 4) run(std::integral_constant<int, 0>{}); else run(std::integral_constant<int, 1>{});
  expand(p, lds, call, wid, __builtin_amdgcn_readfirstlane(lctl[4]), skip ? 0u : __builtin_amdgcn_readfirstlane(lctl[5]) ? 2u : 1u);
  TS(14);
}
extern "C" void kernel_launch(int rank, int nranks, ncclComm_t comm,
                              hipStream_t stream,
                              void* const* d_in, const int* in_sizes, int n_in,
                              void* d_out, int out_size,
                              const int* in_dtypes, int out_dtype,
                              void* const* d_out_all,
                              void* const* d_ws_all, size_t ws_bytes) {
  if (nranks != NR || (unsigned)rank >= (unsigned)NR || ws_bytes < WS_NEED || n_in < 4) {
    fprintf(stderr, "kernel_launch: unexpected configuration\n"); return;
  }
  static bool attr_done[64] = {};
  int dev = 0; HIP_CHECK(hipGetDevice(&dev));
  if (!attr_done[dev & 63]) {
    HIP_CHECK(hipFuncSetAttribute((const void*)fused_kernel, hipFuncAttributeMaxDynamicSharedMemorySize, LDS_BYTES));
    attr_done[dev & 63] = true;
  }
  Params p;
  p.x = (const unsigned char*)d_in[0]; p.w = (const unsigned char*)d_in[1];
  p.sx = (const float*)d_in[2]; p.sw = (const float*)d_in[3];
  for (int i = 0; i < NR; ++i) { p.ws[i] = (unsigned char*)d_ws_all[i]; p.out[i] = (unsigned char*)(i == rank ? d_out : d_out_all[i]); }
  p.rank = rank;
  fused_kernel<<<GRID, 512, LDS_BYTES, stream>>>(p);
}
```

```cpp
#include <hip/hip_runtime.h>
#include <rccl/rccl.h>
#include <cstdio>
#include <type_traits>

#define HIP_CHECK(expr) do { hipError_t _e = (expr); if (_e != hipSuccess) { \
    fprintf(stderr, "HIP error %s at %s:%d\n", hipGetErrorString(_e), __FILE__, __LINE__); \
    return; } } while (0)

typedef int v8i __attribute__((ext_vector_type(8)));
typedef int v4i __attribute__((ext_vector_type(4)));
typedef unsigned v4u __attribute__((ext_vector_type(4)));
typedef unsigned v2u __attribute__((ext_vector_type(2)));
typedef float f32x4 __attribute__((ext_vector_type(4)));
#define LAS __attribute__((address_space(3)))

constexpr int NR = 8;
constexpr int M_ = 4096, N_ = 16384, KL = 512;
constexpr int NKS = 32;
constexpr int HTB = 16384;
constexpr int STAGE_BYTES = 8 * HTB;
constexpr unsigned LCTL = STAGE_BYTES, LTAB = STAGE_BYTES + 256, LCP = STAGE_BYTES + 1024;
constexpr unsigned LTCH = STAGE_BYTES + 2048;
constexpr int LDS_BYTES = STAGE_BYTES + 2048 + 256;
constexpr unsigned SLAB = 32768;
constexpr unsigned IMG_BYTES = 2560u * SLAB;
constexpr unsigned IMG_OFF = 1u << 20;
constexpr unsigned PK_OFF = IMG_OFF + 2 * IMG_BYTES, PK_SLOT = 5120, PK_JOBS = 1280;
constexpr unsigned PK_BLK = 32 * PK_SLOT, NBLK = 7 * PK_JOBS / 32;
constexpr unsigned CW_OFF = PK_OFF + NR * (PK_JOBS / 32) * PK_BLK - 512u * SLAB, CW_BYTES = 2048u * SLAB;
constexpr size_t WS_NEED = (size_t)CW_OFF + 512u * SLAB + (size_t)CW_BYTES;
constexpr unsigned CTR_OFF = 0, TMO_OFF = 128, WCT_OFF = 256, DONE_OFF = 4096, OFLAG_OFF = 8192, FLAG_OFF = 65536, FLAG_BYTES = 2560u * 64u, DBG_OFF = 524288;
constexpr int GRID = 256;
#define CTS 4
constexpr unsigned CTB_OFF = 49152;
constexpr int NPUSH = 16, NCOM = 32;
constexpr int NSH = 20, SHC = 54;
constexpr unsigned DEADV = 0x7fffffffu;
constexpr unsigned long long DEADLINE_TICKS = 90000000ull;
#define WPT 2

struct Params {
  const unsigned char* x;
  const unsigned char* w;
  const float* sx; const float* sw;
  unsigned char* ws[NR];
  unsigned char* out[NR];
  int rank;
};

#define TS(slot) do {} while (0)
#define TV(slot, v) do {} while (0)
__device__ __forceinline__ v8i cat8(v4i a, v4i b) { return __builtin_shufflevector(a, b, 0, 1, 2, 3, 4, 5, 6, 7); }
#define WAIT_V(n) asm volatile("s_waitcnt vmcnt(" #n ")" ::: "memory")
#define WAIT_L(n) asm volatile("s_waitcnt lgkmcnt(" #n ")" ::: "memory")
#define BAR __builtin_amdgcn_s_barrier()
#define SCHED __builtin_amdgcn_sched_barrier(0)
#define TIE16(R) "+v"(R[0]), "+v"(R[1]), "+v"(R[2]), "+v"(R[3]), "+v"(R[4]), "+v"(R[5]), "+v"(R[6]), "+v"(R[7]), \
                 "+v"(R[8]), "+v"(R[9]), "+v"(R[10]), "+v"(R[11]), "+v"(R[12]), "+v"(R[13]), "+v"(R[14]), "+v"(R[15])
#define TIE4(X) "+v"(X[0]), "+v"(X[1]), "+v"(X[2]), "+v"(X[3])

#define PKM ""
#define IMM ""
#define LDM ""
#define SHM ""
__device__ __forceinline__ int fresh_lane() { int l; asm volatile("v_mbcnt_lo_u32_b32 %0, -1, 0\n\tv_mbcnt_hi_u32_b32 %0, -1, %0" : "=v"(l)); return l; }

__device__ __forceinline__ void load_job(const Params& p, int tile, int ks, int jw, int lane, unsigned (&R)[16], v4u (&X)[4]) {
  const int i = lane & 15, g = lane >> 4;
  const bool isx = tile < 16, iso = tile >= 80;
  unsigned vo = isx ? i * KL + g * 32 : iso ? lane * 16 : g * 32 * N_ + 4 * i;
  const unsigned char* src = isx ? p.x + (size_t)(tile * 256 + (jw >> 1) * 64) * KL + ks * 128 + (jw & 1) * 16
                           : iso ? p.out[p.rank] + (unsigned)ks
                                 : p.w + (size_t)(ks * 128 + (jw & 1) * 16) * N_ + (tile - 16) * 256 + (jw >> 1) * 64;
  const unsigned st = isx ? 16 * KL : iso ? 4 * N_ : N_;
#define LD1(r) "global_load_dword %" #r ", %20, %21" LDM "\n\tv_add_u32 %20, %22, %20\n\t"
#define LD4(r) "global_load_dwordx4 %" #r ", %20, %21" LDM "\n\tv_add_u32 %20, %22, %20\n\t"
  asm volatile("s_cmp_eq_u32 %22, %23\n\ts_cbranch_scc1 LW_%=\n\t" LD4(16) LD4(17) LD4(18) LD4(19) "s_branch LE_%=\n"
               "LW_%=:\n\t" LD1(0) LD1(1) LD1(2) LD1(3) LD1(4) LD1(5) LD1(6) LD1(7) LD1(8) LD1(9) LD1(10) LD1(11) LD1(12) LD1(13) LD1(14) LD1(15) "\n"
               "LE_%=:"
               : "=&v"(R[0]), "=&v"(R[1]), "=&v"(R[2]), "=&v"(R[3]), "=&v"(R[4]), "=&v"(R[5]), "=&v"(R[6]), "=&v"(R[7]),
                 "=&v"(R[8]), "=&v"(R[9]), "=&v"(R[10]), "=&v"(R[11]), "=&v"(R[12]), "=&v"(R[13]), "=&v"(R[14]), "=&v"(R[15]),
                 "=&v"(X[0]), "=&v"(X[1]), "=&v"(X[2]), "=&v"(X[3]), "+v"(vo)
               : "s"(src), "s"(st), "n"(N_) : "memory", "scc");
}
__device__ __forceinline__ void retile(bool plain, const unsigned (&R)[16], const v4u (&X)[4], v4u (&D)[4]) {
  if (plain) {
#pragma unroll
    for (int e = 0; e < 4; ++e) D[e] = X[e];
  } else {
#pragma unroll
    for (int d = 0; d < 4; ++d) {
      const unsigned a = R[4 * d], bb = R[4 * d + 1], c = R[4 * d + 2], f = R[4 * d + 3];
      const unsigned t0_ = __builtin_amdgcn_perm(bb, a, 0x05010400u), t1_ = __builtin_amdgcn_perm(bb, a, 0x07030602u);
      const unsigned t2_ = __builtin_amdgcn_perm(f, c, 0x05010400u), t3_ = __builtin_amdgcn_perm(f, c, 0x07030602u);
      D[0][d] = __builtin_amdgcn_perm(t2_, t0_, 0x05040100u); D[1][d] = __builtin_amdgcn_perm(t2_, t0_, 0x07060302u);
      D[2][d] = __builtin_amdgcn_perm(t3_, t1_, 0x05040100u); D[3][d] = __builtin_amdgcn_perm(t3_, t1_, 0x07060302u);
    }
  }
}
__device__ __forceinline__ void store_job(unsigned char* ws, unsigned vo, const v4u (&D)[4]) {
  asm volatile("global_store_dwordx4 %0, %1, %5 offset:-4096 sc0 sc1\n\tglobal_store_dwordx4 %0, %2, %5 offset:-2048 sc0 sc1\n\t"
               "global_store_dwordx4 %0, %3, %5 sc0 sc1\n\tglobal_store_dwordx4 %0, %4, %5 offset:2048 sc0 sc1\n\ts_nop 1"
               :: "v"(vo), "v"(D[0]), "v"(D[1]), "v"(D[2]), "v"(D[3]), "s"(ws) : "memory");
}
__device__ __forceinline__ void store_flag1(unsigned char* ws, unsigned fo, unsigned v) {
  asm volatile("s_mov_b64 exec, 1\n\tglobal_store_dword %0, %1, %2 sc0 sc1\n\ts_mov_b64 exec, -1" :: "v"(fo), "v"(v), "s"(ws) : "memory");
}
__device__ __forceinline__ void store_flag7(unsigned long long fa, unsigned v) {
  asm volatile("s_mov_b64 exec, 0x7f\n\tglobal_store_dword %0, %1, off sc0 sc1\n\ts_mov_b64 exec, -1" :: "v"(fa), "v"(v) : "memory");
}
__device__ __forceinline__ unsigned job_off(int jw, int lane) { return (jw >> 1) * 8192 + (jw & 1) * 1024 + lane * 16; }

#define NMIX 4
#define XORD 2
#define XFST 0
__device__ __forceinline__ void stream_slab(int j, int snd, int& tile, int& ks) {
  if (j < 120) { ks = j / 30; int q = j - ks * 30; if (q < 14) { q += 2 * snd; if (q >= 14) q -= 14; tile = 16 + q; }
    else if (ks >= XORD) { const int r = (q - 14 + snd) & 7; tile = (r & 3) + (r >> 2) * 8 + (q < 22 ? XFST : 4 - XFST); }
    else tile = (q - 14 + 2 * snd) & 15; }
  else if (j < 168) { const int u = j - 120; ks = u / 12; int q = u - ks * 12 + snd; if (q >= 12) q -= 12; tile = 30 + q; }
  else { const int u = j - 168, gq = u >= 56, v = u - gq * 56; ks = v / 14; int q = v - ks * 14 + 2 * snd; if (q >= 14) q -= 14; tile = 42 + 14 * gq + q; }
}
__device__ __forceinline__ int stream_idx(int tile, int ks, int snd) {
  if (tile < 16 && ks >= XORD) return ks * 30 + 14 + (((tile & 3) + (tile >> 3) * 4 - snd) & 7) + ((tile & 4) == XFST ? 0 : 8);
  if (tile < 16) return ks * 30 + 14 + ((tile - 2 * snd) & 15);
  if (tile < 30) { int q = tile - 16 - 2 * snd; if (q < 0) q += 14; return ks * 30 + q; }
  if (NMIX && tile >= 70) return 131 + ks * 12;
  if (tile < 42 || tile >= 70) { int q = (tile < 42 ? tile - 30 : tile - 70) - snd; if (q < 0) q += 12; return 120 + ks * 12 + q; }
  const int gq = tile >= 56; int q = tile - 42 - 14 * gq - 2 * snd; if (q < 0) q += 14; return 168 + gq * 56 + ks * 14 + q;
}
__device__ __forceinline__ unsigned shared_off(int r, int i) { const int idx = NSH * r + i; return (unsigned)((idx & 15) * 256 * N_ + (SHC + (idx >> 4)) * 256) * 4u; }

#define INITK 0u
#define LAG2 120
#define LAG 1
#define PWIN 4u
#define PTH 8u
#define INV 0
#define LASTWT 0
#define OST 1
#define OSM " nt"
#define PLN 0
#define XST OST
#define XSM " nt"
#define PSLEEP 8
__device__ __forceinline__ bool wait_flags(unsigned char* ws, unsigned fo, unsigned call) {
  const unsigned long long t0 = __builtin_amdgcn_s_memrealtime();
  unsigned* tmo = (unsigned*)(ws + TMO_OFF);
  for (;;) {
    unsigned v, tm; const unsigned to = TMO_OFF;
    asm volatile("global_load_dword %0, %2, %4 sc0 sc1\n\tglobal_load_dword %1, %3, %4 sc0 sc1\n\ts_waitcnt vmcnt(0)" : "=&v"(v), "=&v"(tm) : "v"(fo), "v"(to), "s"(ws) : "memory");
    if (__ballot(v != call) == 0ull) return true;
    if (__builtin_amdgcn_readfirstlane(tm) == call || __builtin_amdgcn_s_memrealtime() - t0 > DEADLINE_TICKS) {
      __hip_atomic_store(tmo, call, __ATOMIC_RELAXED, __HIP_MEMORY_SCOPE_AGENT);
      return false;
    }
    __builtin_amdgcn_s_sleep(16);
  }
}
constexpr int NTURN = 8 / WPT;
#define NPA 16
constexpr int SPT = NPA * WPT / 8, NT = 280 / SPT;
__device__ __forceinline__ unsigned pk_off(int r, unsigned jj) { return shared_off(r, jj >> 6) + ((jj & 63u) * 4u) * (N_ * 4u); }
#define ST1(vc, d, k, md) "global_store_dwordx4 " vc ", " d ", %" #k " sc0 sc1" md "\n\t"
#define ST7(vc, d, md) ST1(vc, d, 2, md) ST1(vc, d, 3, md) ST1(vc, d, 4, md) ST1(vc, d, 5, md) ST1(vc, d, 6, md) ST1(vc, d, 7, md) ST1(vc, d, 8, md)
__device__ __forceinline__ unsigned bfi(unsigned m, unsigned a, unsigned b) { return (a & m) | (b & ~m); }
#define XLIM 104
__device__ __forceinline__ unsigned encode(v4u (&S)[4], LAS unsigned char* el, int lane, unsigned long long& em, bool isx) {
  unsigned sm = 0;
  const unsigned me = isx ? 0x78787878u : 0x7c7c7c7cu, mt = isx ? 0xc0c0c0c0u : 0xe0e0e0e0u;
#pragma unroll
  for (int d = 0; d < 4; ++d) sm = __builtin_amdgcn_udot4(S[0][d] & me, 0x01010101u, sm, false);
  sm += __builtin_amdgcn_update_dpp(0, (int)sm, 0x111, 0xf, 0xf, true); sm += __builtin_amdgcn_update_dpp(0, (int)sm, 0x112, 0xf, 0xf, true);
  sm += __builtin_amdgcn_update_dpp(0, (int)sm, 0x114, 0xf, 0xf, true); sm += __builtin_amdgcn_update_dpp(0, (int)sm, 0x118, 0xf, 0xf, true);
  int e0 = (__builtin_amdgcn_readlane(sm, 15) + __builtin_amdgcn_readlane(sm, 31) + __builtin_amdgcn_readlane(sm, 47) + __builtin_amdgcn_readlane(sm, 63) - (isx ? 34242 : 17121)) >> (isx ? 13 : 12);
  e0 = e0 < 0 ? 0 : e0 > (isx ? 8 : 24) ? (isx ? 8 : 24) : e0;
  const unsigned c1 = (unsigned)(128 - (isx ? 8 : 4) * e0) * 0x01010101u, ix = (unsigned)((lane & 7) * 4 | (lane >> 3) << 13);
  unsigned cnt = 0, hdr = 0; v4u U[4];
#pragma unroll
  for (int c = 0; c < 4; ++c) {
#pragma unroll
    for (int d = 0; d < 4; ++d) {
      const unsigned v = S[c][d], u = (v & 0x7f7f7f7fu) + c1;
      const bool esc = (u & mt) != 0x80808080u;
      const unsigned long long m = __ballot(esc);
      const unsigned pos = __builtin_amdgcn_mbcnt_hi((unsigned)(m >> 32), __builtin_amdgcn_mbcnt_lo((unsigned)m, cnt));
      *(LAS unsigned short*)(el + 4 + 2 * (esc ? pos : 1100u + (unsigned)lane)) = (unsigned short)(__builtin_amdgcn_udot4(v & 0x60606060u, 0x40100401u, ix, false) + d);
      cnt += (unsigned)__builtin_popcountll(m);
      U[c][d] = bfi(0x80808080u, v, u);
    }
    hdr |= (cnt & 127u) << (8 * c);
  }
  em = ~0ull;
  if (cnt > 126u) return 32u;
  if (lane == 0) *(LAS unsigned*)el = hdr;
  if (isx) {
#pragma unroll
    for (int k = 0; k < 14; ++k) {
      const unsigned z = U[3][2 + k / 7];
      S[k >> 2][k & 3] = bfi(0x40404040u, k % 7 == 6 ? z >> 1 : z << (6 - k % 7), U[k >> 2][k & 3]);
    }
    *(LAS unsigned long long*)(el + 2560 + lane * 8) = (unsigned long long)S[3][1] << 32 | S[3][0];
    S[3] = *(const LAS v4u*)(el + (lane < 32 ? 2560 + lane * 16 : (lane & 15) * 16));
    em = (1ull << (32u + ((cnt + 33u) >> 5) * 4u)) - 1ull;
    return (unsigned)e0;
  }
#pragma unroll
  for (int d = 0; d < 4; ++d) {
    const unsigned z = U[3][d];
    S[0][d] = bfi(0x60606060u, z << 4, U[0][d]); S[1][d] = bfi(0x60606060u, z << 2, U[1][d]); S[2][d] = bfi(0x60606060u, __builtin_amdgcn_alignbit(z, z, 2), U[2][d]);
  }
  S[3] = *(const LAS v4u*)(el + (lane & 15) * 16);
  em = (1ull << ((cnt + 33u) >> 5) * 4u) - 1ull;
  return (unsigned)e0;
}
__device__ __forceinline__ void pusher(const Params& p, LAS unsigned char* lds, unsigned imgo, unsigned flo, unsigned call, int b, int wid, int lane, unsigned (&R)[16], v4u (&X)[4]) {
  unsigned char* const myws = p.ws[p.rank];
  const int tau = wid / WPT, e = tau * SPT + b / NTURN, jw = (b % NTURN) * WPT + wid % WPT;
  unsigned long long fa = 0;
  unsigned char* wsd[7]; int pr[7];
#pragma unroll
  for (int k = 0; k < 7; ++k) {
    pr[k] = (p.rank + 1 + (wid + b + k) % 7) & 7; wsd[k] = p.ws[pr[k]];
    if (lane == k) fa = (unsigned long long)wsd[k];
  }
  const unsigned jo = job_off(jw, lane), dno = DONE_OFF + (lane < NSH ? lane : 0) * 64u, cwo = CW_OFF;
  LAS unsigned* const pk = (LAS unsigned*)(lds + wid * PK_SLOT);
  int tile, ks; stream_slab(e, p.rank, tile, ks);
  unsigned job = e * 8 + jw, cd = 32u;
  unsigned long long em = ~0ull;
  v4u D[4];
  retile(tile < 16, R, X, D);
  bool cod = true;
  cd = encode(D, (LAS unsigned char*)pk, lane, em, tile < 16);
  static_assert(NTURN == 4, "turn structure");
  v4u D2[4]; unsigned R2[16], cd2 = 0u; unsigned long long em2 = 0ull;
  int tn, kn;
  auto next = [&](int Tn) __attribute__((always_inline)) {
    stream_slab(Tn * SPT + b / NTURN, p.rank, tn, kn);
    load_job(p, tn, kn, jw, lane, R, X);
    unsigned vo = (lane >> 4) * 32 * N_ + 4 * (lane & 15);
    const unsigned char* src = p.w + (size_t)(kn * 128 + (jw & 1) * 16) * N_ + ((unsigned)(tn - 30) < 10u ? SHC + (tn - 30 + NMIX * jw) % 10 : 0) * 256 + (jw >> 1) * 64;
#define LN1(r) "global_load_dword %" #r ", %16, %17\n\tv_add_u32 %16, %19, %16\n\t"
    asm volatile("s_cmp_ge_u32 %18, 10\n\ts_cbranch_scc1 LN_%=\n\t" LN1(0) LN1(1) LN1(2) LN1(3) LN1(4) LN1(5) LN1(6) LN1(7) LN1(8) LN1(9) LN1(10) LN1(11) LN1(12) LN1(13) LN1(14) LN1(15) "\nLN_%=:"
                 : "=&v"(R2[0]), "=&v"(R2[1]), "=&v"(R2[2]), "=&v"(R2[3]), "=&v"(R2[4]), "=&v"(R2[5]), "=&v"(R2[6]), "=&v"(R2[7]),
                   "=&v"(R2[8]), "=&v"(R2[9]), "=&v"(R2[10]), "=&v"(R2[11]), "=&v"(R2[12]), "=&v"(R2[13]), "=&v"(R2[14]), "=&v"(R2[15]), "+v"(vo)
                 : "s"(src), "s"((unsigned)(tn - 30)), "n"(N_) : "memory", "scc");
  };
  next(tau + NTURN);
  for (int i = 0; i < tau; ++i) BAR;
  for (int T = tau; T < NT; T += NTURN) {
    const unsigned sid = tile * 32 + p.rank * 4 + ks, vb = imgo + jo + sid * SLAB;
    const unsigned fo = flo + sid * 64u + jw * 4u + (cod ? 32u : 0u), fv = cod ? call << 6 | cd : call;
    const bool nar = tile >= 30 && tile < 40;
    unsigned long long fn = 0;
    if (nar) {
      const int nq = (tile - 30 + NMIX * jw) % 10, ra = nq * 16 / NSH, rb = (nq * 16 + 15) / NSH;
      const unsigned ds = (unsigned)(70 + nq - tile) * 32u;
      unsigned char* const da = ra != p.rank ? p.ws[ra & 7] : myws + (cwo - imgo);
      unsigned char* const db = rb != p.rank && rb != ra ? p.ws[rb & 7] : myws + (cwo - imgo);
      fn = (unsigned long long)(lane ? p.ws[ra & 7] : p.ws[rb & 7]) + fo + ds * 64u;
#pragma unroll
      for (int c = 0; c < 4; ++c) {
        const unsigned vc = vb + ds * SLAB + c * 2048u;
        const unsigned long long em_ = c == 3 ? em2 : ~0ull;
        asm volatile("s_mov_b64 exec, %4\n\t" ST1("%0", "%1", 2, "") ST1("%0", "%1", 3, "") "s_mov_b64 exec, -1\n\ts_nop 1" :: "v"(vc), "v"(D2[c]), "s"(da), "s"(db), "s"(em_) : "memory");
      }
    }
#pragma unroll
    for (int c = 0; c < 4; ++c) {
      const unsigned vc = vb + c * 2048u;
      const unsigned long long em_ = c == 3 ? em : ~0ull;
      asm volatile("s_mov_b64 exec, %9\n\t" ST7("%0", "%1", IMM) "s_mov_b64 exec, -1\n\ts_nop 1"
                   :: "v"(vc), "v"(D[c]), "s"(wsd[0]), "s"(wsd[1]), "s"(wsd[2]), "s"(wsd[3]), "s"(wsd[4]), "s"(wsd[5]), "s"(wsd[6]), "s"(em_) : "memory");
    }
    const int nb = NT - 1 - T < NTURN ? NT - 1 - T : NTURN;
    const int lag2 = __builtin_amdgcn_readfirstlane((int)(T * SPT >= LAG2));
    asm volatile("s_cmp_eq_u32 %20, 0\n\ts_cbranch_scc1 PB_%=\n\ts_barrier\nPB_%=:\n\ts_waitcnt vmcnt(36)\n\ts_cmp_lt_u32 %22, 10\n\ts_cbranch_scc1 PD_%=\n\ts_waitcnt vmcnt(28)\n"
                 "PD_%=:\n\ts_cmp_lg_u32 %21, 0\n\ts_cbranch_scc1 PC_%=\n\ts_waitcnt vmcnt(0)\nPC_%=:"
                 : TIE16(R), TIE4(X) : "s"(nb), "s"(lag2), "s"((unsigned)(tile - 30)) : "memory", "scc");
    asm volatile("" : TIE16(R2) :: "memory");
    const unsigned fv2 = call << 6 | cd2;
#define NFLAG if (nar) asm volatile("s_mov_b64 exec, 3\n\tglobal_store_dword %0, %1, off sc0 sc1\n\ts_mov_b64 exec, -1" :: "v"(fn), "v"(fv2) : "memory")
    if (!lag2) { store_flag7(fa + fo, fv); NFLAG; if (nb > 1 && NTURN > 2) BAR; }
    tile = tn; ks = kn;
    const bool nn = (unsigned)(tile - 30) < 10u;
    retile(tile < 16, R, X, D);
    cd = 32u; em = ~0ull;
    cod = tile >= 16 || (T + NTURN) * SPT + b / NTURN < XLIM;
    if (cod) cd = encode(D, (LAS unsigned char*)pk, lane, em, tile < 16);
    if (lag2) { if (nb > 1 && NTURN > 2) BAR; WAIT_V(0); store_flag7(fa + fo, fv); NFLAG; }
#define LDI 3
    if (nn) {
      retile(false, R2, X, D2);
      cd2 = encode(D2, (LAS unsigned char*)pk, lane, em2, false);
    }
    if (LDI == 3 && nb > 2) BAR;
    next(T + 2 * NTURN < NT ? T + 2 * NTURN : T);
    for (int i = LDI; i < nb; ++i) BAR;
  }
  const bool ok = wait_flags(myws, dno, call);
  load_job(p, 80, (int)pk_off(p.rank, job), jw, lane, R, X);
  asm volatile("s_waitcnt vmcnt(0)" : TIE16(R), TIE4(X) :: "memory");
  for (;;) {
    unsigned nx = 1u; const unsigned co = WCT_OFF + 64u + (call & 1u) * 128u;
    asm volatile("s_nop 1\n\ts_mov_b64 exec, 1\n\tglobal_atomic_add %0, %1, %0, %2 sc0\n\ts_mov_b64 exec, -1" : "+v"(nx) : "v"(co), "s"(myws) : "memory");
    unsigned cnt = 32u, hdr = 0u;
#pragma unroll
    for (int j = 0; j < 16; ++j) {
      const unsigned v = ok ? X[j >> 2][j & 3] : 0x7fc00000u;
      const unsigned long long m = __ballot(v != 0u);
      const unsigned lo = (unsigned)m, hi = (unsigned)(m >> 32);
      pk[v != 0u ? __builtin_amdgcn_mbcnt_hi(hi, __builtin_amdgcn_mbcnt_lo(lo, cnt)) : 1100u + (unsigned)lane] = v;
      hdr = lane == 2 * j ? lo : lane == 2 * j + 1 ? hi : hdr;
      cnt += (unsigned)__builtin_popcountll(m);
    }
    pk[lane < 32 ? (unsigned)lane : 1100u + (unsigned)lane] = hdr;
    const unsigned nd = (cnt + 31u) & ~31u, pj = p.rank * PK_JOBS + job;
#pragma unroll
    for (int c = 0; c < 5; ++c) {
      const int nl = (int)(nd >> 2) - c * 64;
      if (nl > 0) {
        const v4u d = *(const LAS v4u*)(pk + c * 256 + lane * 4);
        const unsigned vc = PK_OFF + (pj >> 5) * PK_BLK + (c < 3 ? (pj & 31u) * 3072u : 95232u + (pj & 31u) * 2048u) + c * 1024u + lane * 16u;
        const unsigned long long em = nl >= 64 ? ~0ull : (1ull << nl) - 1ull;
        asm volatile("s_mov_b64 exec, %9\n\t" ST7("%0", "%1", PKM) "s_mov_b64 exec, -1\n\ts_nop 1"
                     :: "v"(vc), "v"(d), "s"(wsd[0]), "s"(wsd[1]), "s"(wsd[2]), "s"(wsd[3]), "s"(wsd[4]), "s"(wsd[5]), "s"(wsd[6]), "s"(em) : "memory");
      }
    }
    asm volatile("s_waitcnt vmcnt(0)" : "+v"(nx) :: "memory");
    job = __builtin_amdgcn_readfirstlane(nx) + NPA * 8u;
    const bool more = job < PK_JOBS;
    {
      unsigned vo = lane * 16u; const unsigned char* src = p.out[p.rank] + pk_off(p.rank, more ? job : 0u);
      asm volatile("s_cmp_ge_u32 %6, %8\n\ts_cbranch_scc1 OE_%=\n\tglobal_load_dwordx4 %0, %4, %5\n\tv_add_u32 %4, %7, %4\n\tglobal_load_dwordx4 %1, %4, %5\n\tv_add_u32 %4, %7, %4\n\t"
                   "global_load_dwordx4 %2, %4, %5\n\tv_add_u32 %4, %7, %4\n\tglobal_load_dwordx4 %3, %4, %5\nOE_%=:"
                   : "=&v"(X[0]), "=&v"(X[1]), "=&v"(X[2]), "=&v"(X[3]), "+v"(vo) : "s"(src), "s"(job), "n"(N_ * 4), "n"(PK_JOBS) : "memory", "scc");
    }
    store_flag7(fa + OFLAG_OFF + pj * 4u, call);
    if (!more) break;
    asm volatile("s_waitcnt vmcnt(1)" : TIE4(X) :: "memory");
  }
  TS(1); TS(9);
}
static_assert(XLIM % 8 == 0 && XLIM >= 104 && XLIM <= 120, "whole units");
constexpr unsigned NUNIT = (XLIM + 168) * 14 / 16, UTAB = 120 * 1024;
#define UDLY 12
#define UDLX UDLY
#define UWIN 24u
__device__ __forceinline__ void unpacker(const Params& p, LAS unsigned char* lds, unsigned imgo, unsigned flo, unsigned call, int b, int wid, int lane) {
  unsigned char* const myws = p.ws[p.rank];
  LAS unsigned* lctl = (LAS unsigned*)(lds + LCTL);
  const unsigned long long t0 = __builtin_amdgcn_s_memrealtime();
  if (wid == 0) {
    unsigned i = lane, done = 0;
    for (;;) {
      const unsigned q = (i < NUNIT ? i : 0u) * 16u + (b - NPUSH), sp = q / 14u, r = q - sp * 14u, wp = sp < XLIM ? 0u : sp - XLIM + 56u;
      const int snd = (p.rank + 1 + (r >> 1)) & 7;
      const int k14 = wp / 14u, gp = k14 >> 2, q14 = wp - k14 * 14;
      int ks = k14 & 3;
      int qq = q14 + (gp == 1 ? snd : 2 * snd); qq -= qq >= (gp == 1 ? 12 : 14) ? (gp == 1 ? 12 : 14) : 0;
      int tile = gp == 1 && q14 >= 12 ? 58 + ((NSH * p.rank) >> 4) + q14 : (gp == 0 ? 16 : 14 * gp + (gp == 1 ? 16 : 14)) + qq;
      if (sp < XLIM) stream_slab((int)sp, snd, tile, ks);
      const unsigned hs = (tile * 32 + snd * 4 + ks) * 2 + (r & 1u), fo = flo + (hs >> 1) * 64u + 32u + (hs & 1u) * 16u, to = TMO_OFF;
      const unsigned long long pm = __ballot(i < NUNIT && i < done + UWIN);
      v4u v = {0u, 0u, 0u, 0u}; unsigned tm;
      asm volatile("s_mov_b64 exec, %5\n\tglobal_load_dwordx4 %0, %2, %4 sc0 sc1\n\ts_mov_b64 exec, -1\n\tglobal_load_dword %1, %3, %4 sc0 sc1\n\ts_waitcnt vmcnt(0)"
                   : "+v"(v), "=&v"(tm) : "v"(fo), "v"(to), "s"(myws), "s"(pm) : "memory");
      const unsigned tag = call & 0x3ffffffu;
      const bool ok = (pm >> lane & 1ull) && v[0] >> 6 == tag && v[1] >> 6 == tag && v[2] >> 6 == tag && v[3] >> 6 == tag;
      if (ok) {
        *(LAS unsigned long long*)(lds + UTAB + i * 8u) = (unsigned long long)((v[0] & 63u) | (v[1] & 63u) << 8 | (v[2] & 63u) << 16 | (v[3] & 63u) << 24) << 32 | hs | 0x80000000u;
        i += 64u;
      }
      const unsigned nk = (unsigned)__builtin_popcountll(__ballot(ok));
      done += nk;
      if (done >= NUNIT) break;
      if (__builtin_amdgcn_readfirstlane(tm) == call || __builtin_amdgcn_s_memrealtime() - t0 > DEADLINE_TICKS) {
        __hip_atomic_store((unsigned*)(myws + TMO_OFF), call, __ATOMIC_RELAXED, __HIP_MEMORY_SCOPE_AGENT);
        lctl[2] = 1u;
        break;
      }
      if (!nk) __builtin_amdgcn_s_sleep(4);
    }
    TS(3);
    return;
  }
  const unsigned cwo = CW_OFF, lb = (unsigned)(size_t)lds + (wid - 1) * 16640u, msk = 0x60606060u;
  LAS unsigned char* const buf = lds + (wid - 1) * 16640u;
  unsigned pfo = DONE_OFF + 4032u, pend = 0;
  const v4u cv = {call, call, call, call};
#define UFLAG asm volatile("s_mov_b64 exec, 1\n\tglobal_store_dwordx4 %0, %1, %2 sc0 sc1\n\ts_mov_b64 exec, -1\n\ts_nop 1" :: "v"(pfo), "v"(cv), "s"(myws) : "memory")
  for (unsigned i = wid - 1; i < NUNIT; i += 7u) {
    unsigned hs, cds;
    for (;;) {
      const unsigned long long e = __hip_atomic_load((LAS unsigned long long*)(lds + UTAB + i * 8u), __ATOMIC_RELAXED, __HIP_MEMORY_SCOPE_WORKGROUP);
      hs = __builtin_amdgcn_readfirstlane((unsigned)e); cds = __builtin_amdgcn_readfirstlane((unsigned)(e >> 32));
      if (hs) break;
      if (pend) { WAIT_V(0); UFLAG; pend = 0; continue; }
      if (__builtin_amdgcn_readfirstlane(__hip_atomic_load(lctl + 2, __ATOMIC_RELAXED, __HIP_MEMORY_SCOPE_WORKGROUP)) || __builtin_amdgcn_s_memrealtime() - t0 > 2 * DEADLINE_TICKS) return;
      __builtin_amdgcn_s_sleep(2);
    }
    hs &= 0x7fffffffu;
    const unsigned so = (hs >> 1) * SLAB + (hs & 1u) * 16384u;
    auto unit = [&](auto isx_) __attribute__((always_inline)) {
      constexpr bool isx = decltype(isx_)::value;
      v4u C[4][3]; unsigned E[4]; v2u P[4];
      unsigned va = imgo + so + lane * 16u + 4096u, ve = imgo + so + lane * 4u + 4096u, vp = imgo + so + lane * 8u + 4096u;
#define UL(j, o) "global_load_dwordx4 %" #j ", %16, %18 offset:" #o " sc0 sc1\n\t"
#define UE(j, o) "global_load_dword %" #j ", %17, %18 offset:" #o " sc0 sc1\n\t"
#define UOUT "=&v"(C[0][0]), "=&v"(C[0][1]), "=&v"(C[0][2]), "=&v"(C[1][0]), "=&v"(C[1][1]), "=&v"(C[1][2]), \
             "=&v"(C[2][0]), "=&v"(C[2][1]), "=&v"(C[2][2]), "=&v"(C[3][0]), "=&v"(C[3][1]), "=&v"(C[3][2]), \
             "=&v"(E[0]), "=&v"(E[1]), "=&v"(E[2]), "=&v"(E[3]), "+v"(va), "+v"(ve)
      if constexpr (isx) {
#define XL(j, o) "global_load_dwordx4 %" #j ", %16, %23 offset:" #o " sc0 sc1\n\t"
#define XE(j, o) "global_load_dword %" #j ", %17, %23 offset:" #o " sc0 sc1\n\t"
#define XP(j, o) "global_load_dwordx2 %" #j ", %22, %23 offset:" #o " sc0 sc1\n\t"
        asm volatile(XL(0, -4096) XL(1, -2048) XL(2, 0) XP(18, 2048) XE(12, 2560)
                     XL(3, -3072) XL(4, -1024) XL(5, 1024) XP(19, 3072) XE(13, 3584)
                     "v_add_u32 %16, 0x2000, %16\n\tv_add_u32 %17, 0x2000, %17\n\tv_add_u32 %22, 0x2000, %22\n\t"
                     XL(6, -4096) XL(7, -2048) XL(8, 0) XP(20, 2048) XE(14, 2560)
                     XL(9, -3072) XL(10, -1024) XL(11, 1024) XP(21, 3072) XE(15, 3584)
                     "s_waitcnt vmcnt(20)"
                     : UOUT, "=&v"(P[0]), "=&v"(P[1]), "=&v"(P[2]), "=&v"(P[3]), "+v"(vp) : "s"(myws) : "memory");
      } else {
        asm volatile(UL(0, -4096) UL(1, -2048) UL(2, 0) UE(12, 2048) UL(3, -3072) UL(4, -1024) UL(5, 1024) UE(13, 3072)
                     "v_add_u32 %16, 0x2000, %16\n\tv_add_u32 %17, 0x2000, %17\n\t"
                     UL(6, -4096) UL(7, -2048) UL(8, 0) UE(14, 2048) UL(9, -3072) UL(10, -1024) UL(11, 1024) UE(15, 3072)
                     "s_waitcnt vmcnt(16)"
                     : UOUT : "s"(myws) : "memory");
      }
      UFLAG;
#pragma unroll
      for (int j = 0; j < 4; ++j) {
        const unsigned cd = cds >> (8 * j) & 63u, c2 = (cd & 31u) * (isx ? 0x08080808u : 0x04040404u);
        v4u D[4];
        if constexpr (isx) {
          asm volatile("s_waitcnt vmcnt(%5)" : "+v"(C[j][0]), "+v"(C[j][1]), "+v"(C[j][2]), "+v"(E[j]), "+v"(P[j]) : "n"(16 - j) : "memory");
          unsigned z[2] = {c2, c2};
#pragma unroll
          for (int k = 0; k < 14; ++k) {
            const unsigned c = k < 12 ? C[j][k >> 2][k & 3] : P[j][k & 1], t = c & 0x40404040u;
            asm("v_xad_u32 %0, %1, %2, %3" : "=v"(D[k >> 2][k & 3]) : "v"(c), "v"(t), "s"(c2));
            z[k / 7] += k % 7 == 6 ? t << 1 : t >> (6 - k % 7);
          }
          D[3][2] = z[0]; D[3][3] = z[1];
        } else {
          asm volatile("s_waitcnt vmcnt(13)" : "+v"(C[j][0]), "+v"(C[j][1]), "+v"(C[j][2]), "+v"(E[j]) :: "memory");
#pragma unroll
          for (int d = 0; d < 4; ++d) {
            const unsigned t0_ = C[j][0][d] & msk, t1_ = C[j][1][d] & msk, t2_ = C[j][2][d] & msk;
            asm("v_xad_u32 %0, %1, %2, %3" : "=v"(D[0][d]) : "v"(C[j][0][d]), "v"(t0_), "s"(c2));
            asm("v_xad_u32 %0, %1, %2, %3" : "=v"(D[1][d]) : "v"(C[j][1][d]), "v"(t1_), "s"(c2));
            asm("v_xad_u32 %0, %1, %2, %3" : "=v"(D[2][d]) : "v"(C[j][2][d]), "v"(t2_), "s"(c2));
            D[3][d] = __builtin_amdgcn_alignbit(t2_, t2_, 30) + (t1_ >> 2) + (t0_ >> 4) + c2;
          }
        }
#pragma unroll
        for (int c = 0; c < 4; ++c) *(LAS v4u*)(buf + j * 4096 + c * 1024 + lane * 16) = D[c];
        const unsigned h = __builtin_amdgcn_readfirstlane(E[j]);
#pragma unroll
        for (int k = 0; k < 2; ++k) {
          const unsigned en = k ? E[j] >> 16 : E[j], ps = 2u * lane - 2u + k, y = en & 0x1fe0u, t = y | y << 6;
          const unsigned ch = (ps >= (h & 127u)) + (ps >= (h >> 8 & 127u)) + (ps >= (h >> 16 & 127u));
          const unsigned ad = lb + (ps < (h >> 24 & 127u) ? j * 4096u + ch * 1024u + (en & 31u) * 4u + (en >> 13 & 7u) * 128u : 16384u + lane * 4u), mv = (t | t << 12) & msk;
          asm volatile("ds_mskor_b32 %0, %1, %2" :: "v"(ad), "v"(msk), "v"(mv) : "memory");
        }
#pragma unroll
        for (int c = 0; c < 4; ++c) D[c] = *(const LAS v4u*)(buf + j * 4096 + c * 1024 + lane * 16);
        store_job(myws, (cd < 32u ? imgo : cwo + (isx ? 512u * SLAB : 0u)) + so + (j >> 1) * 8192u + (j & 1) * 1024u + lane * 16u + 4096u, D);
      }
    };
    if (hs < 1024u) unit(std::true_type{}); else unit(std::false_type{});
    pfo = flo + (hs >> 1) * 64u + (hs & 1u) * 16u; pend = 1;
  }
  WAIT_V(0); UFLAG;
}
#define NG 80u
#define N1 130u
#define XCNT 768u
__device__ __forceinline__ unsigned blk_job(int rank, unsigned q) { const unsigned bl = q / 7u; return ((rank + 1 + (q - bl * 7u)) & 7u) * PK_JOBS + bl * 32u; }
__device__ __forceinline__ void expand(const Params& p, LAS unsigned char* lds, unsigned call, int wid, unsigned q, unsigned mode) {
  unsigned char* const myws = p.ws[p.rank];
  LAS unsigned* lctl = (LAS unsigned*)(lds + LCTL);
  const int lane = fresh_lane();
  const unsigned lbase = (unsigned)(size_t)lds;
  for (unsigned r = 0;; ++r) {
    if (mode == 0u) {
      if (threadIdx.x == 0) lctl[6 + (r & 1u)] = __hip_atomic_fetch_add((unsigned*)(myws + WCT_OFF + (call & 1u) * 128u), 1u, __ATOMIC_RELAXED, __HIP_MEMORY_SCOPE_AGENT);
      __syncthreads();
      q = __builtin_amdgcn_readfirstlane(lctl[6 + (r & 1u)]); q += q < N1 ? 0u : NG;
    }
    if (q >= NBLK) break;
    const unsigned pj = blk_job(p.rank, q), snd = pj / PK_JOBS, vb = PK_OFF + (pj >> 5) * PK_BLK + lane * 16u;
    bool ok = true;
    if (mode != 2u) {
      ok = wait_flags(myws, OFLAG_OFF + (pj + wid + (lane & 24)) * 4u, call);
#pragma unroll
      for (int h = 0; h < 4; ++h) {
        const unsigned la = lbase + (wid + h * 8) * 3072u + (h >> 1) * 16384u, vo = vb + (wid + h * 8) * 3072u;
        asm volatile("s_mov_b32 m0, %0\n\ts_nop 0\n\tglobal_load_lds_dwordx4 %1, %2 sc0 sc1\n\tglobal_load_lds_dwordx4 %1, %2 offset:1024 sc0 sc1\n\t"
                     "global_load_lds_dwordx4 %1, %2 offset:2048 sc0 sc1" :: "s"(la), "v"(vo), "s"(myws) : "memory");
      }
      WAIT_V(0);
    }
    unsigned hd[4];
#pragma unroll
    for (int h = 0; h < 4; ++h) hd[h] = *(const LAS unsigned*)(lds + ((wid + h * 8) * 3072u + (h >> 1) * 16384u + (lane & 31) * 4u));
    const unsigned alt = ok ? 0u : 0x7fc00000u, okm = ok ? ~0u : 0u;
#pragma unroll
    for (int h = 0; h < 4; ++h) {
      const unsigned j = wid + h * 8, oo = pk_off(snd, pj - snd * PK_JOBS + j) + lane * 16u, so = j * 3072u + (h >> 1) * 16384u;
      const unsigned hdr = hd[h] & okm;
      unsigned pos[16], cnt = 32u; unsigned long long m[16];
#pragma unroll
      for (int i = 0; i < 16; ++i) {
        const unsigned lo = __builtin_amdgcn_readlane(hdr, 2 * i), hi = __builtin_amdgcn_readlane(hdr, 2 * i + 1);
        m[i] = (unsigned long long)hi << 32 | lo;
        pos[i] = __builtin_amdgcn_mbcnt_hi(hi, __builtin_amdgcn_mbcnt_lo(lo, cnt));
        cnt += (unsigned)__builtin_popcountll(m[i]);
      }
      if (cnt > XCNT) {
        const unsigned s2 = 49152u + wid * 2048u, la = lbase + s2, vo = vb + 98304u + j * 2048u, d2 = (s2 - so - 3072u) >> 2;
        asm volatile("s_waitcnt lgkmcnt(0)\n\ts_mov_b32 m0, %0\n\ts_nop 0\n\tglobal_load_lds_dwordx4 %1, %2 sc0 sc1\n\tglobal_load_lds_dwordx4 %1, %2 offset:1024 sc0 sc1\n\ts_waitcnt vmcnt(0)"
                     :: "s"(la), "v"(vo), "s"(myws) : "memory");
#pragma unroll
        for (int i = 0; i < 16; ++i) pos[i] = (pos[i] + (pos[i] >= 768u ? d2 : 0u)) & 0x3fffffffu;
      }
      unsigned x[16];
#pragma unroll
      for (int i = 0; i < 16; ++i) x[i] = *(const LAS unsigned*)(lds + (so + pos[i] * 4u));
      SCHED;
#pragma unroll
      for (int rr = 0; rr < 4; ++rr) {
        v4u v;
#pragma unroll
        for (int c = 0; c < 4; ++c) asm("v_cndmask_b32 %0, %1, %2, %3" : "=v"(v[c]) : "v"(alt), "v"(x[4 * rr + c]), "s"(m[4 * rr + c]));
        const unsigned vo_ = oo + rr * (N_ * 4u);
        asm volatile("global_store_dwordx4 %0, %1, %2" XSM "\n\ts_nop 1" :: "v"(vo_), "v"(v), "s"(p.out[p.rank]) : "memory");
      }
    }
    if (mode) break;
  }
}

__global__ __launch_bounds__(512, 2) void fused_kernel(const Params p) {
  extern __shared__ __attribute__((aligned(16))) unsigned char smem[];
  LAS unsigned char* lds = (LAS unsigned char*)smem;
  const int tid = threadIdx.x, wid = __builtin_amdgcn_readfirstlane(tid >> 6), wc = wid & 3;
  unsigned char* myws = p.ws[p.rank];
  const int b = blockIdx.x, rank = p.rank, g = b - NCOM, gl = b - NPUSH;
  const bool push = b < NPUSH;
  LAS unsigned* lctl = (LAS unsigned*)(lds + LCTL);
  const unsigned lbase = (unsigned)(size_t)lds;
  const int xc = b & 7, L = (xc >> 2) * 28 + (g >> 3), cn = L >> 2;
  const int pm = (xc & 3) * 4 + (L & 3), si = L * 4 + (xc & 3);
  const bool skip = cn < 2 && si >= NSH;
  const unsigned nvis = skip ? 96u : 128u;
  auto tile_of = [&](int u, int& tm, int& tn) __attribute__((always_inline)) {
    const int v = u + (skip && u > 0), idx = NSH * rank + si;
    const bool sh = v == 1 && cn < 2;
    tm = sh ? idx & 15 : pm; tn = sh ? SHC + (idx >> 4) : v == 0 ? cn : 14 * v - 2 + cn;
  };
  unsigned call, imgo, flo;
  {
    const int lane = tid & 63;
    unsigned R[16], R2[16]; v4u X[4], X2[4];
    int t1, k1, j1 = wid;
    if (push) { j1 = (b % NTURN) * WPT + wid % WPT; stream_slab(wid / WPT * SPT + b / NTURN, rank, t1, k1); } else { t1 = gl >> 2; k1 = gl & 3; }
    load_job(p, t1, k1, j1, lane, R, X);
    const bool two = !push && gl < 80;
    if (two) load_job(p, t1 + 60, k1, wid, lane, R2, X2);
    if (tid == 0) {
      unsigned* const cp = (unsigned*)(myws + CTB_OFF + b * CTS);
      const unsigned t = __hip_atomic_load(cp, __ATOMIC_RELAXED, __HIP_MEMORY_SCOPE_AGENT) * (unsigned)GRID;
      __hip_atomic_store(cp, t / (unsigned)GRID + 1u, __ATOMIC_RELAXED, __HIP_MEMORY_SCOPE_AGENT);
      lctl[0] = t / (unsigned)GRID + 1u; lctl[1] = INITK; lctl[2] = 0u; lctl[3] = 0u; lctl[5] = 0u;
      if (b == 0) {
        __hip_atomic_store((unsigned*)(myws + WCT_OFF + (t / (unsigned)GRID & 1u) * 128u), 0u, __ATOMIC_RELAXED, __HIP_MEMORY_SCOPE_AGENT);
        __hip_atomic_store((unsigned*)(myws + WCT_OFF + 64u + (t / (unsigned)GRID & 1u) * 128u), 0u, __ATOMIC_RELAXED, __HIP_MEMORY_SCOPE_AGENT);
        __hip_atomic_store((unsigned*)(myws + WCT_OFF + 256u + (t / (unsigned)GRID & 1u) * 128u), 0u, __ATOMIC_RELAXED, __HIP_MEMORY_SCOPE_AGENT);
      }
    }
    if (!push && g < 0 && tid < (int)NUNIT) *(LAS unsigned long long*)(lds + UTAB + tid * 8) = 0ull;
    if (wid >= 4 && g >= 0) {
      const int u = wid - 4, l = lane & 31, src = l >> 2, ks = l & 3;
      int tm, tn; tile_of(u, tm, tn);
      const int a0 = stream_idx(tm, ks, src), ax = a0 + (a0 < XLIM ? UDLX : 0), aw = stream_idx(16 + tn, ks, src) + UDLY;
#define CORD 9
      const int key = src == rank ? ks : u >= CORD ? 32 + ks * 8 + ((src - rank) & 7) : ((ax > aw ? ax : aw) + 1) * 32 + l;
      int rk = 0;
#pragma unroll
      for (int s = 0; s < 32; ++s) rk += __builtin_amdgcn_readlane(key, s) < key;
      const unsigned ent = (unsigned)((16 + tn) * 32 + l) << 15 | (unsigned)(tm * 32 + l);
      if (u * 32u < nvis) *(LAS unsigned*)(lds + LTAB + (u * 32 + rk) * 4) = ent;
      if (u == 0 && rk < 4) *(LAS unsigned*)(lds + LTAB + (nvis + rk) * 4) = ent;
    }
    asm volatile("s_waitcnt vmcnt(0)" : TIE16(R), TIE4(X) :: "memory");
    asm volatile("" : TIE16(R2) :: "memory");
    __syncthreads();
    call = __builtin_amdgcn_readfirstlane(lctl[0]);
    imgo = IMG_OFF + (call & 1u) * IMG_BYTES;
    flo = FLAG_OFF + (call & 1u) * FLAG_BYTES;
    TS(0);
    if (push) {
      if (b < NPA) pusher(p, lds, imgo, flo, call, b, wid, lane, R, X);
      WAIT_V(0);
      expand(p, lds, call, wid, 0u, 0u);
      TS(14);
      return;
    }
    v4u D[4];
    const unsigned s1 = t1 * 32 + rank * 4 + k1;
    retile(t1 < 16, R, X, D);
    store_job(myws, imgo + s1 * SLAB + job_off(wid, lane) + 4096, D);
    if (two) {
      retile(false, R2, X2, D);
      store_job(myws, imgo + (s1 + 60 * 32) * SLAB + job_off(wid, lane) + 4096, D);
    }
    WAIT_V(0);
    store_flag1(myws, flo + s1 * 64 + wid * 4, call);
    if (two) store_flag1(myws, flo + (s1 + 60 * 32) * 64 + wid * 4, call);
    TS(1);
    if (g < 0) {
      unpacker(p, lds, imgo, flo, call, b, wid, lane);
      __syncthreads();
      TS(4);
      expand(p, lds, call, wid, 0u, 0u);
      TS(14);
      return;
    }
  }
  const float scale = __builtin_bit_cast(float, __builtin_amdgcn_readfirstlane(__builtin_bit_cast(int, p.sx[0] * p.sw[0])));
  WAIT_V(0);

  auto run = [&](auto WRC) __attribute__((always_inline)) {
    constexpr int wr = decltype(WRC)::value;
    unsigned ppend = 0, pk = INITK, pw = 4u;
    auto peval = [&]() __attribute__((always_inline)) {
      const v4u f = *(const LAS v4u*)(lds + LCP + (unsigned)fresh_lane() * 16);
      unsigned long long bm = __ballot(f[0] == call && f[1] == call && f[2] == call && f[3] == call);
      bm &= bm >> 1; bm &= bm >> 2;
      bm = ~bm & 0x1111111111111111ull;
      unsigned c = bm ? (unsigned)__builtin_ctzll(bm) >> 2 : 16u;
      c = c < pw ? c : pw;
      pw = c == pw ? 16u : PWIN;
      const unsigned n = pk + c < nvis ? pk + c : nvis;
      if (n > pk) {
        pk = n; lctl[1] = n;
      }
      ppend = 0;
    };
    auto pissue = [&]() __attribute__((always_inline)) {
      const unsigned ln = (unsigned)fresh_lane();
      unsigned e = pk + ((ln >> 2) < pw ? ln >> 2 : 0u); e = e < nvis ? e : nvis - 1;
      const unsigned kk = *(const LAS unsigned*)(lds + LTAB + e * 4);
      const unsigned la = lbase + LCP, fo = flo + ((ln & 2) ? kk >> 15 : kk & 0x7fffu) * 64 + (ln & 1) * 16; unsigned char* const w_ = myws;
      asm volatile("s_mov_b32 m0, %0\n\ts_nop 0\n\tglobal_load_lds_dwordx4 %1, %2 sc0 sc1" :: "s"(la), "v"(fo), "s"(w_) : "memory");
      ppend = 1;
    };
    auto poll = [&](bool blocking, unsigned need) __attribute__((always_inline)) {
      if (!blocking) {
        if (ppend) {
          unsigned st; asm volatile("s_getreg_b32 %0, hwreg(HW_REG_IB_STS)" : "=s"(st) :: "memory");
          st = __builtin_amdgcn_readfirstlane(st);
          if ((st & 15u) | ((st >> 18) & 48u)) return;
          WAIT_V(0);
          peval();
        } else { pw = 16u; pissue(); }
        return;
      }
      WAIT_V(0);
      if (ppend) { peval(); if (pk < need) __builtin_amdgcn_s_sleep(PSLEEP); }
      if (pk < nvis) pissue();
      WAIT_L(0);
    };
    auto slow = [&](unsigned need) __attribute__((always_inline)) {
      const unsigned long long t0 = __builtin_amdgcn_s_memrealtime();
      for (;;) {
        if (wr == 1 && wc == 0) poll(true, need); else __builtin_amdgcn_s_sleep(1);
        const unsigned long long el = __builtin_amdgcn_s_memrealtime() - t0;
        if (wr == 1 && wc == 0 && el > 100000ull) {
          unsigned* tmo = (unsigned*)(myws + TMO_OFF);
          if (__builtin_amdgcn_readfirstlane(__hip_atomic_load(tmo, __ATOMIC_RELAXED, __HIP_MEMORY_SCOPE_AGENT)) == call || el > DEADLINE_TICKS) {
            __hip_atomic_store(tmo, call, __ATOMIC_RELAXED, __HIP_MEMORY_SCOPE_AGENT);
            lctl[2] = 1u; lctl[1] = DEADV; pk = nvis; ppend = 0;
          }
          WAIT_V(0);
        }
        if (__builtin_amdgcn_readfirstlane(__hip_atomic_load(lctl + 1, __ATOMIC_RELAXED, __HIP_MEMORY_SCOPE_WORKGROUP)) >= need || el > 2 * DEADLINE_TICKS) break;
      }
      asm volatile("" ::: "memory");
    };

#define GT0 26
    auto grab = [&](int t) __attribute__((always_inline)) {
      const unsigned ln = (unsigned)fresh_lane() & 31u, q = __builtin_amdgcn_readfirstlane(lctl[4]), pj = blk_job(rank, q);
      if (q >= NBLK) return;
      if (t == GT0) {
        const unsigned la = lbase + LCTL + 64u, fo = OFLAG_OFF + (pj + ln) * 4u; unsigned char* const w_ = myws;
        asm volatile("s_mov_b32 m0, %0\n\ts_mov_b32 exec_hi, 0\n\tglobal_load_lds_dword %1, %2 sc0 sc1\n\ts_mov_b32 exec_hi, -1" :: "s"(la), "v"(fo), "s"(w_) : "memory");
      } else {
        WAIT_V(0);
        lctl[5] = __ballot(lctl[16 + ln] != call) == 0ull ? PK_OFF + (pj >> 5) * PK_BLK - imgo : 0u;
      }
    };
#define TBURST 3u
#define TEND 12604u
    unsigned ti = (unsigned)(g * 3 + wc - 1);
    auto touch = [&]() __attribute__((always_inline)) {
      const unsigned k = (ti >= 960u) + (ti >= 1920u) + (ti >= 2880u), r = ti - k * 960u, i2 = ti - 9244u, bq = i2 / 12u;
      const unsigned o = ti < 3840u ? imgo + ((r >> 2) * 4u + k) * SLAB + (r & 3u) * 8192u
                       : ti < 3848u ? (ti - 3840u) * 8192u
                       : ti < 3868u ? flo + (ti - 3848u) * 8192u
                       : ti < 4124u ? imgo + (70u + ((NSH * rank) >> 4)) * 32u * SLAB + (ti - 3868u) * 8192u
                       : ti < 9244u ? imgo + 30u * 32u * SLAB + (ti - 4124u) * 8192u
                                    : PK_OFF + (blk_job(rank, bq) >> 5) * PK_BLK + (i2 - bq * 12u) * 8192u;
      const unsigned la = lbase + LTCH, fo = o + (unsigned)fresh_lane() * 128u; unsigned char* const w_ = myws;
      asm volatile("s_mov_b32 m0, %0\n\ts_nop 0\n\tglobal_load_lds_dword %1, %2 sc0 sc1" :: "s"(la), "v"(fo), "s"(w_) : "memory");
      ti += 672u;
    };
    const unsigned char* XI = myws + imgo;
    const unsigned voff = (unsigned)(wc * 4096 + (tid & 63) * 16);
    const unsigned ldsw = (unsigned)wc * 4096u;
    const int aoff = wr * 8192 + (tid & 63) * 16, boff = wc * 4096 + (tid & 63) * 16;
#define SA(b_, h) (((b_) * 2 + (h)) * HTB)
#define SB(b_, h) ((4 + (b_) * 2 + (h)) * HTB)
#define STG1(bufoff, va, o) __builtin_amdgcn_global_load_lds((const unsigned*)(XI + (va)), (LAS unsigned*)(lds + (bufoff) + ldsw), 16, o, 0)
#define STAGE(bufoff, goff) do { if (wr == 0) { const unsigned va_ = voff + (goff); STG1(bufoff, va_, 0); STG1(bufoff, va_, 1024); STG1(bufoff, va_, 2048); STG1(bufoff, va_, 3072); } } while (0)
#define LDA(dst, b_, h) do { _Pragma("unroll") for (int m = 0; m < 4; ++m) { \
    v4i lo = *(const LAS v4i*)(lds + SA(b_, h) + aoff + m * 2048); v4i hi = *(const LAS v4i*)(lds + SA(b_, h) + aoff + m * 2048 + 1024); dst[m] = cat8(lo, hi); } } while (0)
#define LDB(dst, b_, h) do { _Pragma("unroll") for (int n = 0; n < 2; ++n) { \
    v4i lo = *(const LAS v4i*)(lds + SB(b_, h) + boff + n * 2048); v4i hi = *(const LAS v4i*)(lds + SB(b_, h) + boff + n * 2048 + 1024); dst[n] = cat8(lo, hi); } } while (0)
#define MMA(ai, bj, At, Bt) do { __builtin_amdgcn_s_setprio(1); _Pragma("unroll") for (int m = 0; m < 4; ++m) _Pragma("unroll") for (int n = 0; n < 2; ++n) \
    acc[ai][bj][m][n] = __builtin_amdgcn_mfma_scale_f32_16x16x128_f8f6f4(Bt[n], At[m], acc[ai][bj][m][n], 0, 1, 0, 0, 0, 0); __builtin_amdgcn_s_setprio(0); } while (0)
#define MF1(ai, bj, At, Bt, q) acc[ai][bj][(q) >> 1][(q) & 1] = __builtin_amdgcn_mfma_scale_f32_16x16x128_f8f6f4(Bt[(q) & 1], At[(q) >> 1], acc[ai][bj][(q) >> 1][(q) & 1], 0, 1, 0, 0, 0, 0)
#define WV12 WAIT_V(12)
#define MEMSTEP do { if (wr == 0) WV12; else if (wc == 0 && (ppend || (pk < nvis && pk < need + PTH))) poll(false, 0u); } while (0)
#define KOFF(i) ((unsigned)__builtin_amdgcn_readlane(kv, (i)))
    f32x4 acc[2][2][4][2];
#pragma unroll
    for (int a = 0; a < 2; ++a)
#pragma unroll
      for (int c = 0; c < 2; ++c)
#pragma unroll
        for (int m = 0; m < 4; ++m)
#pragma unroll
          for (int n = 0; n < 2; ++n) acc[a][c][m][n] = (f32x4){0.f, 0.f, 0.f, 0.f};
    v8i At[4], B0[2], B1[2];
    constexpr unsigned hstep = HTB;
#define OFA(e_) ((e_) & ~0x7fffu)
#define OFB(e_) (((e_) & 0x7fffu) << 15)
    if (wr == 1 && wc > 0) for (unsigned i = 0; i < TBURST; ++i) touch();
    slow(4u);
    unsigned known = 4u, need = 4u;
    const unsigned e0 = __builtin_amdgcn_readfirstlane(*(const LAS unsigned*)(lds + LTAB));
    {
      const unsigned cA = OFA(e0), cB = OFB(e0);
      STAGE(SB(0, 0), cB); STAGE(SA(0, 0), cA); STAGE(SB(0, 1), cB + hstep); STAGE(SA(0, 1), cA + hstep);
      if (wr == 1) BAR;
      if (wr == 0) WAIT_V(8);
      BAR;
      STAGE(SB(1, 0), cB + SLAB); STAGE(SA(1, 0), cA + SLAB); STAGE(SB(1, 1), cB + SLAB + hstep);
      if (wr == 0) WAIT_V(12);
      BAR;
    }
    unsigned a1 = OFA(e0) + SLAB, a2 = a1 + SLAB, a3 = a2 + SLAB, b2 = OFB(e0) + 2 * SLAB, b3 = b2 + SLAB;
    for (int ui = 0;; ++ui) {
      const bool has_next = (unsigned)(ui * 32 + 32) < nvis;
      int kv = 0;
      if (wr == 0) { const unsigned ln = ui * 32u + (unsigned)fresh_lane(); kv = *(const LAS int*)(lds + LTAB + (ln < 131u ? ln : 131u) * 4); }
      for (int t = 0; t < NKS; t += 2) {
        if (known < need) { if (wr == 0) BAR; slow(need); if (wr == 1) BAR; known = need; }
        LDB(B0, 0, 0); SCHED; LDA(At, 0, 0); SCHED; STAGE(SA(1, 1), a1 + hstep);
        WAIT_L(8); BAR; WAIT_L(0); MMA(0, 0, At, B0); BAR; SCHED;
        LDB(B1, 0, 1); SCHED; STAGE(SB(0, 0), b2);
        BAR; WAIT_L(0); MMA(0, 1, At, B1); BAR;
        LDA(At, 0, 1); SCHED; STAGE(SA(0, 0), a2);
        BAR; WAIT_L(0); MMA(1, 0, At, B0); BAR; SCHED;
        STAGE(SB(0, 1), b2 + hstep); MEMSTEP;
        if (wr == 1 && wc == 1 && ui == 3 && (t == GT0 || t == NKS - 4)) grab(t);
        if (wr == 1 && wc > 0 && ti < TEND) touch();
        BAR; MMA(1, 1, At, B1); BAR;
        LDB(B0, 1, 0); SCHED; LDA(At, 1, 0); SCHED; STAGE(SA(0, 1), a2 + hstep);
        WAIT_L(8); BAR; WAIT_L(0); MMA(0, 0, At, B0); BAR; SCHED;
        LDB(B1, 1, 1); const unsigned kn = lctl[1]; SCHED; STAGE(SB(1, 0), b3);
        BAR; WAIT_L(0); MMA(0, 1, At, B1); BAR;
        LDA(At, 1, 1); SCHED; STAGE(SA(1, 0), a3);
        BAR; WAIT_L(0); MMA(1, 0, At, B0); BAR; SCHED;
        STAGE(SB(1, 1), b3 + hstep); if (wr == 1 || has_next || t < NKS - 2) MEMSTEP;
        BAR; SCHED;
        __builtin_amdgcn_s_setprio(1);
        MF1(1, 1, At, B1, 0); SCHED;
        unsigned o2 = KOFF(t + 4), o3 = KOFF(t + 5); known = __builtin_amdgcn_readfirstlane(kn);
        asm volatile("" : "+s"(o2), "+s"(o3), "+s"(known)); SCHED;
        MF1(1, 1, At, B1, 1); SCHED;
        a1 = a3; a2 = OFA(o2); b2 = OFB(o2);
        asm volatile("" : "+s"(a1), "+s"(a2), "+s"(b2)); SCHED;
        MF1(1, 1, At, B1, 2); SCHED;
        a3 = OFA(o3); b3 = OFB(o3);
        if (wr == 0 && !has_next && t == NKS - 4) {
          const unsigned ob_ = __builtin_amdgcn_readfirstlane(lctl[5]);
          a2 = ob_ ? ob_ : a1; a3 = ob_ ? ob_ + 2 * hstep : a1; b2 = ob_ ? ob_ + 3 * hstep : a1; b3 = ob_ ? ob_ + 5 * hstep : a1;
        }
        asm volatile("" : "+s"(a2), "+s"(b2), "+s"(a3), "+s"(b3)); SCHED;
        MF1(1, 1, At, B1, 3); SCHED;
        need = need + 2u < nvis ? need + 2u : nvis;
        asm volatile("" : "+s"(need)); SCHED;
        MF1(1, 1, At, B1, 4); MF1(1, 1, At, B1, 5); MF1(1, 1, At, B1, 6); MF1(1, 1, At, B1, 7);
        __builtin_amdgcn_s_setprio(0);
        BAR;
      }
      int tm, tn; tile_of(ui, tm, tn);
      const bool dead = __builtin_amdgcn_readfirstlane(lctl[2]) != 0u, sht = tn >= SHC;
      TS(2 + 2 * ui);
      unsigned gv = 1u;
      const int ng = __builtin_amdgcn_readfirstlane((int)!(wc == 1 && ui == 2 && has_next));
      {
        const int ln = fresh_lane();
        const unsigned vo = (unsigned)((tm * 256 + wc * 32 + 4 * (ln >> 4)) * N_ + tn * 256 + wr * 64 + 4 * (ln & 15)) * 4u;
        const float qnan = __builtin_nanf("");
        const float sc_ = dead ? qnan : scale, lo_ = dead ? qnan : 0.f;
        const int shs = __builtin_amdgcn_readfirstlane(sht ? 1 : ui < PLN ? 2 : 0);
        unsigned char* const ob = p.out[rank];
        if (wr == 1) {
          const unsigned co = WCT_OFF + 256u + (call & 1u) * 128u; unsigned char* const w_ = myws;
          asm volatile("s_cmp_lg_u32 %3, 0\n\ts_cbranch_scc1 GE_%=\n\ts_mov_b64 exec, 1\n\tglobal_atomic_add %0, %1, %0, %2 sc0\n\ts_mov_b64 exec, -1\nGE_%=:"
                       : "+v"(gv) : "v"(co), "s"(w_), "s"(ng) : "memory", "scc");
        }
#pragma unroll
        for (int bj = 0; bj < 2; ++bj)
#pragma unroll
          for (int n = 0; n < 2; ++n) {
#pragma unroll
            for (int r4 = 0; r4 < 4; ++r4) {
              unsigned char* rowp = ob + (size_t)(bj * 128 + n * 16 + r4) * N_ * 4;
#pragma unroll
              for (int ai = 0; ai < 2; ++ai) {
                f32x4 v;
#pragma unroll
                for (int c = 0; c < 4; ++c) {
                  const float x = acc[ai][bj][c][n][r4] * sc_;
                  asm("v_max_f32 %0, %1, %2" : "=v"(v[c]) : "s"(lo_), "v"(x));
                }
                asm volatile("s_cmp_lg_u32 %4, 0\n\ts_cbranch_scc1 SW_%=\n\tglobal_store_dwordx4 %0, %1, %2 offset:%3" OSM "\n\ts_branch SE_%=\n"
                             "SW_%=:\n\tglobal_store_dwordx4 %0, %1, %2 offset:%3 sc0 sc1" SHM "\nSE_%=:\n\ts_nop 1"
                             :: "v"(vo), "v"(v), "s"(rowp), "n"(ai * 512), "s"(shs) : "memory", "scc");
              }
            }
#pragma unroll
            for (int ai = 0; ai < 2; ++ai)
#pragma unroll
              for (int c = 0; c < 4; ++c) { acc[ai][bj][c][n] = (f32x4){0.f, 0.f, 0.f, 0.f}; asm volatile("" : "+v"(acc[ai][bj][c][n])); }
          }
      }
      if (!has_next && wr == 0) WAIT_V(32);
      if (wr == 1) {
        asm volatile("s_waitcnt vmcnt(32)" : "+v"(gv) :: "memory");
        if (!ng) { const unsigned q_ = __builtin_amdgcn_readfirstlane(gv); lctl[4] = q_ < NG ? q_ + N1 : NBLK; }
      }
      if (sht) {
        WAIT_V(0);
        unsigned o_ = 0;
        if (fresh_lane() == 0) o_ = __hip_atomic_fetch_add(lctl + 3, 1u, __ATOMIC_RELAXED, __HIP_MEMORY_SCOPE_WORKGROUP);
        if (__builtin_amdgcn_readfirstlane(o_) == 7u) store_flag1(myws, DONE_OFF + si * 64, call);
        if (wr == 0) WAIT_V(0);
      }
      TS(3 + 2 * ui);
      if (!has_next) break;
    }
    if (wr == 0) BAR;
  };
  if (wid < 4) run(std::integral_constant<int, 0>{}); else run(std::integral_constant<int, 1>{});
  expand(p, lds, call, wid, __builtin_amdgcn_readfirstlane(lctl[4]), skip ? 0u : __builtin_amdgcn_readfirstlane(lctl[5]) ? 2u : 1u);
  TS(14);
}
extern "C" void kernel_launch(int rank, int nranks, ncclComm_t comm,
                              hipStream_t stream,
                              void* const* d_in, const int* in_sizes, int n_in,
                              void* d_out, int out_size,
                              const int* in_dtypes, int out_dtype,
                              void* const* d_out_all,
                              void* const* d_ws_all, size_t ws_bytes) {
  if (nranks != NR || (unsigned)rank >= (unsigned)NR || ws_bytes < WS_NEED || n_in < 4) {
    fprintf(stderr, "kernel_launch: unexpected configuration\n"); return;
  }
  static bool attr_done[64] = {};
  int dev = 0; HIP_CHECK(hipGetDevice(&dev));
  if (!attr_done[dev & 63]) {
    HIP_CHECK(hipFuncSetAttribute((const void*)fused_kernel, hipFuncAttributeMaxDynamicSharedMemorySize, LDS_BYTES));
    attr_done[dev & 63] = true;
  }
  Params p;
  p.x = (const unsigned char*)d_in[0]; p.w = (const unsigned char*)d_in[1];
  p.sx = (const float*)d_in[2]; p.sw = (const float*)d_in[3];
  for (int i = 0; i < NR; ++i) { p.ws[i] = (unsigned char*)d_ws_all[i]; p.out[i] = (unsigned char*)(i == rank ? d_out : d_out_all[i]); }
  p.rank = rank;
  fused_kernel<<<GRID, 512, LDS_BYTES, stream>>>(p);
}
```
